```python
import numpy as np

kernel_name = "gpu_ag_gemm_m8192_k8192_n8192_f32_none"


ABSMAX_RTOL = 0.001

COMPUTE_DTYPE = "float32"
EPILOGUE = "none"
IN_QUANT = None
OUT_QUANT = None
IN_DTYPES = {"x": "float32", "w_mat": "float32"}
WORLD_SIZE = 8

def setup_inputs():

    x = np.random.randn(8192, 8192).astype(np.float32)
    w_mat = (np.random.randn(8192, 8192) / np.sqrt(8192)).astype(np.float32)
    return {"x": x, "w_mat": w_mat}

def _epilogue(y):
    return y

def reference(x, w_mat):
    y = (x @ w_mat).astype(np.float32)
    return _epilogue(y)
```

```cpp
#include <hip/hip_runtime.h>
#include <rccl/rccl.h>
#include <cstdio>

#define HIP_CHECK(expr) do { hipError_t _e = (expr); if (_e != hipSuccess) { \
    fprintf(stderr, "HIP error %s at %s:%d\n", hipGetErrorString(_e), __FILE__, __LINE__); \
    return; } } while (0)

namespace agk {
typedef unsigned u32;
typedef unsigned long long u64;
typedef u32 u32x4 __attribute__((ext_vector_type(4)));
typedef int i32x4 __attribute__((ext_vector_type(4)));
typedef float f32x4 __attribute__((ext_vector_type(4)));
typedef _Float16 f16x8 __attribute__((ext_vector_type(8)));
#define LAS __attribute__((address_space(3)))
typedef __amdgpu_buffer_rsrc_t rsrc_t;

constexpr int KDIM = 8192, NLOC = 1024, NKT = 128;
constexpr u32 IMG = 16384u;
constexpr u32 PANEL = NKT * IMG;
constexpr int ASLOTS = 34, BSLOTS = 16;
constexpr u32 CTRL_BYTES = 1u << 20;
constexpr u32 PAR_BYTES = (ASLOTS + BSLOTS) * PANEL;
constexpr u32 WS_NEED = CTRL_BYTES + 2u * PAR_BYTES;
constexpr u32 IN_BYTES = 1024u * 8192u * 4u;
constexpr u32 OFF_CTR = 0;
constexpr u32 OFF_TMO = 16384;
constexpr u32 OFF_LF = 20480;
constexpr u32 OFF_YF = 24576;
constexpr u32 OFF_XR = 28672;
constexpr u32 OFF_WR = 32768;
constexpr u32 OFF_WX = 36864;
constexpr u32 OFF_SA = 131072;
constexpr u32 OFF_SB = 196608;
constexpr u32 OFF_DBG = 262144;
constexpr int NWG = 256, NCOMM = 32;
constexpr u64 DEADLINE = 30000000ull;
constexpr u32 STAGE = 32768u;
constexpr u32 L_SNAP = 4u * STAGE;
constexpr u32 L_SC = L_SNAP + 512u;
constexpr u32 L_RED = L_SC + 1024u;
constexpr u32 LDS_BYTES = L_RED + 2048u;

struct Params {
    const float* x; const float* w;
    char* ws[8]; float* out[8];
    int rank;
};

#define AUX_SC1 16
#define AUX_SYS 17
#define WAIT_VM(n) asm volatile("s_waitcnt vmcnt(" #n ")" ::: "memory")
#define WAIT_LGKM0() asm volatile("s_waitcnt lgkmcnt(0)" ::: "memory")
#define BAR() do { asm volatile("" ::: "memory"); __builtin_amdgcn_s_barrier(); asm volatile("" ::: "memory"); } while (0)

__device__ __forceinline__ rsrc_t mk_rsrc(const void* p, u32 bytes) { return __builtin_amdgcn_make_buffer_rsrc((void*)p, 0, bytes, 0x00020000); }
__device__ __forceinline__ u32 ld_agent(const void* p) { return __hip_atomic_load((const u32*)p, __ATOMIC_RELAXED, __HIP_MEMORY_SCOPE_AGENT); }
__device__ __forceinline__ u32 ld_sys(const void* p) { return __hip_atomic_load((const u32*)p, __ATOMIC_RELAXED, __HIP_MEMORY_SCOPE_SYSTEM); }
__device__ __forceinline__ void st_agent(void* p, u32 v) { __hip_atomic_store((u32*)p, v, __ATOMIC_RELAXED, __HIP_MEMORY_SCOPE_AGENT); }
__device__ __forceinline__ void st_sys(void* p, u32 v) { __hip_atomic_store((u32*)p, v, __ATOMIC_RELAXED, __HIP_MEMORY_SCOPE_SYSTEM); }

#define STAMP(ws, idx) do { } while (0)

__device__ __forceinline__ void poll_ge(const char* p, u32 need, char* ws, u32 call) {
    const u64 t0 = __builtin_amdgcn_s_memrealtime();
    u32 n = 0;
    for (;;) {
        const u32 v = ld_sys(p);
        if (__all((int)(v - need) >= 0)) return;
        if ((++n & 15u) == 0u) {
            const u32 t = __builtin_amdgcn_readfirstlane(ld_agent(ws + OFF_TMO));
            if (t == call || __builtin_amdgcn_s_memrealtime() - t0 > DEADLINE) { st_agent(ws + OFF_TMO, call); return; }
        }
        __builtin_amdgcn_s_sleep(2);
    }
}

__device__ __forceinline__ u32 umax(u32 a, u32 b) { return a > b ? a : b; }
__device__ __forceinline__ u32 absbits(float f) { return __float_as_uint(f) & 0x7fffffffu; }
__device__ __forceinline__ int exp_of(u32 amaxbits) { int e = (int)(amaxbits >> 23); return e < 15 ? 15 : e; }
__device__ __forceinline__ float scale_of(int e) { return __uint_as_float((u32)(268 - e) << 23); }
__device__ __forceinline__ u32x4 cvt8(f32x4 a, f32x4 b, float s) {
    f16x8 h;
    h[0] = (_Float16)(a[0] * s); h[1] = (_Float16)(a[1] * s); h[2] = (_Float16)(a[2] * s); h[3] = (_Float16)(a[3] * s);
    h[4] = (_Float16)(b[0] * s); h[5] = (_Float16)(b[1] * s); h[6] = (_Float16)(b[2] * s); h[7] = (_Float16)(b[3] * s);
    return __builtin_bit_cast(u32x4, h);
}

template <int NR>
__device__ __forceinline__ void x_rows(const Params& p, LAS char* lds, int row0, u32 par, int a, int tid, int wave) {
    const rsrc_t rx = mk_rsrc(p.x, IN_BYTES), rw = mk_rsrc(p.ws[p.rank], WS_NEED);
    f32x4 v[NR][4];
#pragma unroll
    for (int r = 0; r < NR; ++r)
#pragma unroll
        for (int u = 0; u < 2; ++u) {
            const u32 off = (u32)(row0 + r) * (KDIM * 4u) + (u32)(u * 512 + tid) * 32u;
            v[r][2 * u] = __builtin_bit_cast(f32x4, __builtin_amdgcn_raw_buffer_load_b128(rx, off, 0, 0));
            v[r][2 * u + 1] = __builtin_bit_cast(f32x4, __builtin_amdgcn_raw_buffer_load_b128(rx, off + 16u, 0, 0));
        }
    LAS u32* red = (LAS u32*)(lds + L_RED);
#pragma unroll
    for (int r = 0; r < NR; ++r) {
        u32 m = 0;
#pragma unroll
        for (int i = 0; i < 4; ++i)
#pragma unroll
            for (int j = 0; j < 4; ++j) m = umax(m, absbits(v[r][i][j]));
#pragma unroll
        for (int o = 32; o >= 1; o >>= 1) m = umax(m, (u32)__shfl_xor((int)m, o));
        if ((tid & 63) == 0) red[r * 8 + wave] = m;
    }
    WAIT_LGKM0(); BAR();
#pragma unroll
    for (int r = 0; r < NR; ++r) {
        const u32x4 m0 = *(LAS u32x4*)(red + r * 8), m1 = *(LAS u32x4*)(red + r * 8 + 4);
        const u32 m = umax(umax(umax(m0[0], m0[1]), umax(m0[2], m0[3])), umax(umax(m1[0], m1[1]), umax(m1[2], m1[3])));
        const int e = exp_of(m); const float s = scale_of(e);
        const int row = row0 + r, rb = row >> 7, rin = row & 127;
        const u32 slot = rb < 6 ? (u32)(a * 6 + rb) : (u32)(32 + rb - 6);
        const u32 base = CTRL_BYTES + par * PAR_BYTES + slot * PANEL + (u32)rin * 128u;
#pragma unroll
        for (int u = 0; u < 2; ++u) {
            const u32 pi = (u32)(u * 512 + tid);
            const u32 off = base + (pi >> 3) * IMG + (((pi & 7u) ^ (((u32)rin >> 1) & 7u)) << 4);
            __builtin_amdgcn_raw_buffer_store_b128(cvt8(v[r][2 * u], v[r][2 * u + 1], s), rw, off, 0, AUX_SC1);
        }
        const u32 so = tid == 0 ? OFF_SA + ((par * ASLOTS + slot) * 128u + (u32)rin) * 4u : 0x80000000u;
        __builtin_amdgcn_raw_buffer_store_b32((u32)(e - 141), rw, so, 0, AUX_SC1);
    }
}

struct Tile {
    u32 aoff, boff;
    u32 saoff, sboff;
    int a_src, b_src;
    u32 a_base, b_base;
    int a_rb, b_ct;
    int dst;
    u32 ooff;
    u32 yidx;
};

template <int KG>
__device__ __forceinline__ void epilogue(const Params& p, LAS char* lds, const Tile& T, f32x4 (&acc)[4][4], u32 call, int lane, int wave) {
    char* ws = p.ws[p.rank];
    const u32 tmo = ld_agent(ws + OFF_TMO);
    const int wm = (wave >> 1) & 1, wn = wave & 1;
#pragma unroll
    for (int i = 0; i < 2; ++i)
#pragma unroll
        for (int n = 0; n < 4; ++n) *(LAS f32x4*)(lds + (u32)wave * 8192u + (u32)(i * 4 + n) * 1024u + (u32)lane * 16u) = acc[(1 - KG) * 2 + i][n];
    WAIT_LGKM0(); BAR();
    const LAS int* sc = (const LAS int*)(lds + L_SC);
    i32x4 eb[4];
#pragma unroll
    for (int n = 0; n < 4; ++n) eb[n] = *(const LAS i32x4*)(sc + 128 + wn * 64 + n * 16 + (lane >> 4) * 4);
    const rsrc_t ro = mk_rsrc(p.out[T.dst], IN_BYTES);
    const bool bad = tmo == call;
    const bool remote = T.dst != p.rank;
#pragma unroll
    for (int i = 0; i < 2; ++i) {
        const int r = wm * 64 + (KG * 2 + i) * 16 + (lane & 15);
        const int ea = sc[r];
#pragma unroll
        for (int n = 0; n < 4; ++n) {
            const f32x4 o = *(const LAS f32x4*)(lds + (u32)(wave ^ 4) * 8192u + (u32)(i * 4 + n) * 1024u + (u32)lane * 16u);
            f32x4 s = acc[KG * 2 + i][n] + o;
#pragma unroll
            for (int j = 0; j < 4; ++j) s[j] = bad ? __builtin_nanf("") : __builtin_amdgcn_ldexpf(s[j], ea + eb[n][j]);
            const u32 off = T.ooff + (u32)r * (NLOC * 4u) + (u32)(wn * 64 + n * 16 + (lane >> 4) * 4) * 4u;
            if (remote) __builtin_amdgcn_raw_buffer_store_b128(__builtin_bit_cast(u32x4, s), ro, off, 0, AUX_SYS);
            else __builtin_amdgcn_raw_buffer_store_b128(__builtin_bit_cast(u32x4, s), ro, off, 0, 0);
        }
    }
}

__device__ __forceinline__ void gemm_tile(const Params& p, LAS char* lds, const Tile& T, u32 call, int tid, int lane, int wave, int stamp0) {
    char* ws = p.ws[p.rank];
    const int src = lane < 32 ? T.a_src : T.b_src;
    const u32 gstep = src >= 0 ? 1u : 0u;
    const u32 gbase = src >= 0 ? call * 128u + (lane < 32 ? T.a_base : T.b_base) + 1u : call;
    const char* gptr = ws + (src >= 0 ? OFF_LF + (u32)src * 128u + (u32)(lane & 31) * 4u : OFF_XR + blockIdx.x * 4u);
    if (wave == 0) {
        if (T.a_src < 0) { poll_ge(ws + OFF_XR + (u32)(T.a_rb * 128 + lane) * 4u, call, ws, call); poll_ge(ws + OFF_XR + (u32)(T.a_rb * 128 + 64 + lane) * 4u, call, ws, call); }
        if (T.b_src < 0) { const int rg = (lane & 7) < 6 ? (lane & 7) : 6; poll_ge(ws + OFF_WR + (u32)((T.b_ct * 4 + ((lane >> 3) & 3)) * 8 + rg) * 4u, call, ws, call); }
        poll_ge(gptr, gbase, ws, call);
        __builtin_amdgcn_fence(__ATOMIC_ACQUIRE, "");
        WAIT_VM(0);
    }
    BAR();
    if (wave == 0) STAMP(ws, stamp0);
    const u32 voff = (u32)wave * 2048u + (u32)lane * 16u;
    const char* gA = ws + T.aoff + voff; const char* gB = ws + T.boff + voff;
    const u32 ldsw = (u32)wave * 2048u;
#define GLDS16(g, l) __builtin_amdgcn_global_load_lds((const u32*)(g), (LAS u32*)(lds + (l)), 16, 0, AUX_SC1)
#define LOADT(t, st) do { const u32 _o = (u32)(t) * IMG; \
        GLDS16(gA + _o, (st) * STAGE + ldsw); GLDS16(gA + _o + 1024u, (st) * STAGE + ldsw + 1024u); \
        GLDS16(gB + _o, (st) * STAGE + IMG + ldsw); GLDS16(gB + _o + 1024u, (st) * STAGE + IMG + ldsw + 1024u); } while (0)
#define SNAP(c) __builtin_amdgcn_global_load_lds((const u32*)gptr, (LAS u32*)(lds + L_SNAP + ((u32)(c) & 1u) * 256u), 4, 0, AUX_SYS)
    const int kg = wave >> 2, wm = (wave >> 1) & 1, wn = wave & 1;
    const u32 swz = (u32)(((kg * 4 + (lane >> 4)) ^ ((lane & 15) >> 1)) << 4);
    const u32 ard = (u32)(wm * 64 + (lane & 15)) * 128u + swz, brd = IMG + (u32)(wn * 64 + (lane & 15)) * 128u + swz;
#define READF(fa, fb, st) do { _Pragma("unroll") for (int _i = 0; _i < 4; ++_i) { \
        fb[_i] = *(const LAS f16x8*)(lds + (st) * STAGE + brd + _i * 2048u); fa[_i] = *(const LAS f16x8*)(lds + (st) * STAGE + ard + _i * 2048u); } } while (0)
#define MMA(fa, fb) do { _Pragma("unroll") for (int _m = 0; _m < 4; ++_m) _Pragma("unroll") for (int _n = 0; _n < 4; ++_n) \
        acc[_m][_n] = __builtin_amdgcn_mfma_f32_16x16x32_f16(fb[_n], fa[_m], acc[_m][_n], 0, 0, 0); } while (0)
    f32x4 acc[4][4];
#pragma unroll
    for (int m = 0; m < 4; ++m)
#pragma unroll
        for (int n = 0; n < 4; ++n) acc[m][n] = (f32x4){0.f, 0.f, 0.f, 0.f};
    f16x8 a0[4], b0[4], a1[4], b1[4];
    if (wave >= 4) {
        const u32 so = (wave < 6 ? T.saoff : T.sboff) + (u32)((wave & 1) * 64 + lane) * 4u;
        __builtin_amdgcn_global_load_lds((const u32*)(ws + so), (LAS u32*)(lds + L_SC + (u32)(wave - 4) * 256u), 4, 0, AUX_SYS);
    }
    LOADT(0, 0u);
    if (wave == 0) SNAP(1);
    LOADT(1, 1u); LOADT(2, 2u); LOADT(3, 3u);
    if (wave == 0) WAIT_VM(13); else WAIT_VM(12);
    BAR();
    READF(a0, b0, 0u);
#define GATE(c) do { const u32 _v = *(const LAS u32*)(lds + L_SNAP + ((u32)(c) & 1u) * 256u + (u32)lane * 4u); const u32 _need = gbase + gstep * (u32)(c); \
        if (!__all((int)(_v - _need) >= 0)) { if (wave == 0) { poll_ge(gptr, _need, ws, call); WAIT_VM(0); } BAR(); } } while (0)
    for (int c = 0; c < 31; ++c) {
        const int t = 4 * c;
        WAIT_VM(8); WAIT_LGKM0(); BAR();
        GATE(c + 1);
        LOADT(t + 4, 0u); READF(a1, b1, 1u); MMA(a0, b0);
        WAIT_VM(8); WAIT_LGKM0(); BAR();
        if (wave == 0) SNAP(c + 2);
        LOADT(t + 5, 1u); READF(a0, b0, 2u); MMA(a1, b1);
        if (wave == 0) WAIT_VM(9); else WAIT_VM(8);
        WAIT_LGKM0(); BAR();
        LOADT(t + 6, 2u); READF(a1, b1, 3u); MMA(a0, b0);
        if (wave == 0) WAIT_VM(9); else WAIT_VM(8);
        WAIT_LGKM0(); BAR();
        LOADT(t + 7, 3u); READF(a0, b0, 0u); MMA(a1, b1);
    }
    WAIT_VM(8); WAIT_LGKM0(); BAR();
    READF(a1, b1, 1u); MMA(a0, b0);
    WAIT_VM(4); WAIT_LGKM0(); BAR();
    READF(a0, b0, 2u); MMA(a1, b1);
    WAIT_VM(0); WAIT_LGKM0(); BAR();
    READF(a1, b1, 3u); MMA(a0, b0);
    WAIT_LGKM0(); BAR();
    MMA(a1, b1);
    if (wave == 0) STAMP(ws, stamp0 + 1);
    if (kg == 0) epilogue<0>(p, lds, T, acc, call, lane, wave); else epilogue<1>(p, lds, T, acc, call, lane, wave);
    if (T.dst != p.rank) {
        WAIT_VM(0); BAR();
        if (tid == 0) st_sys(p.ws[T.dst] + OFF_YF + ((u32)p.rank * 64u + T.yidx) * 4u, call);
    }
    if (wave == 0) STAMP(ws, stamp0 + 2);
#undef GLDS16
#undef LOADT
#undef SNAP
#undef READF
#undef MMA
#undef GATE
}

__device__ __forceinline__ void make_tile(Tile& T, int rank, int slot, int ct, u32 par) {
    const int g = rank >> 2, a = rank & 3;
    T.aoff = CTRL_BYTES + par * PAR_BYTES + (u32)slot * PANEL;
    T.boff = CTRL_BYTES + par * PAR_BYTES + (u32)(ASLOTS + ct) * PANEL;
    T.saoff = OFF_SA + (par * ASLOTS + (u32)slot) * 512u;
    T.sboff = OFF_SB + (par * BSLOTS + (u32)ct) * 512u;
    int grow;
    if (slot < 24) { const int i = slot / 6, rb = slot - i * 6; grow = (g * 4 + i) * 1024 + rb * 128;
        T.a_src = i == a ? -1 : g * 4 + i; T.a_base = (u32)(rb >> 1) * 32u; T.a_rb = rb; }
    else { const int j = (slot - 24) >> 1, rb = (slot - 24) & 1; grow = ((1 - g) * 4 + j) * 1024 + 768 + rb * 128;
        T.a_src = (1 - g) * 4 + j; T.a_base = 32u; T.a_rb = 0; }
    T.b_base = 0u; T.b_ct = ct & 7;
    int col;
    if (ct < 8) { T.b_src = -1; T.dst = rank; col = ct * 128; T.yidx = 0; }
    else { const int j = (ct - 8) >> 1, h = ct & 1; T.b_src = (1 - g) * 4 + j; T.dst = T.b_src; col = a * 256 + h * 128; T.yidx = (u32)(slot * 2 + h); }
    T.ooff = ((u32)grow * NLOC + (u32)col) * 4u;
}

__device__ __forceinline__ void push_pair(rsrc_t rs, rsrc_t rd, u32 soff, u32 doff, u32 flagoff, u32 fval, int cw, int lane, u32 ssc, u32 dsc) {
    const u32 po = ((u32)cw >> 4) * PANEL + ((u32)cw & 15u) * 4096u + (u32)lane * 16u;
    const u32 so = soff + po, dof = doff + po;
    const u32 fo = lane == 0 ? flagoff : 0x80000000u;
    if (cw == 0) {
        const u32x4 e = __builtin_amdgcn_raw_buffer_load_b128(rs, ssc + (u32)lane * 16u, 0, AUX_SC1);
        __builtin_amdgcn_raw_buffer_store_b128(e, rd, dsc + (u32)lane * 16u, 0, AUX_SYS);
    }
    u32x4 va[4], vb[4];
#pragma unroll
    for (int u = 0; u < 4; ++u) va[u] = __builtin_amdgcn_raw_buffer_load_b128(rs, so + u * 1024u, 0, AUX_SC1);
#pragma unroll
    for (int u = 0; u < 4; ++u) vb[u] = __builtin_amdgcn_raw_buffer_load_b128(rs, so + 65536u + u * 1024u, 0, AUX_SC1);
    for (u32 kc = 0; kc < 32u; kc += 2u) {
        const u32 o = kc * 65536u;
        const u32 n0 = kc + 2u < 32u ? so + o + 131072u : 0x80000000u;
#pragma unroll
        for (int u = 0; u < 4; ++u) __builtin_amdgcn_raw_buffer_store_b128(va[u], rd, dof + o + u * 1024u, 0, AUX_SYS);
#pragma unroll
        for (int u = 0; u < 4; ++u) va[u] = __builtin_amdgcn_raw_buffer_load_b128(rs, n0 + u * 1024u, 0, AUX_SC1);
        WAIT_VM(4);
        __builtin_amdgcn_raw_buffer_store_b32(fval + kc + 1u, rd, fo, 0, AUX_SYS);
#pragma unroll
        for (int u = 0; u < 4; ++u) __builtin_amdgcn_raw_buffer_store_b128(vb[u], rd, dof + o + 65536u + u * 1024u, 0, AUX_SYS);
#pragma unroll
        for (int u = 0; u < 4; ++u) vb[u] = __builtin_amdgcn_raw_buffer_load_b128(rs, n0 + 65536u + u * 1024u, 0, AUX_SC1);
        WAIT_VM(4);
        __builtin_amdgcn_raw_buffer_store_b32(fval + kc + 2u, rd, fo, 0, AUX_SYS);
    }
}

__global__ void __launch_bounds__(512, 2) ag_gemm_kernel(Params p) {
    extern __shared__ __attribute__((aligned(16))) char smem[];
    LAS char* lds = (LAS char*)smem;
    const int tid = threadIdx.x, lane = tid & 63, wave = __builtin_amdgcn_readfirstlane(tid >> 6);
    const int b = blockIdx.x, rank = p.rank, g = rank >> 2, a = rank & 3;
    char* ws = p.ws[rank];
    if (wave == 0) STAMP(ws, 0);
    const u32 call = (u32)__builtin_amdgcn_readfirstlane(ld_agent(ws + OFF_CTR + (u32)b * 64u)) + 1u;
    const u32 par = call & 1u;
    const rsrc_t rws = mk_rsrc(ws, WS_NEED);

    x_rows<1>(p, lds, b, par, a, tid, wave);
    if (b < NCOMM) {
        WAIT_VM(0); BAR();
        if (tid == 0) { st_agent(ws + OFF_XR + (u32)b * 4u, call); st_agent(ws + OFF_CTR + (u32)b * 64u, call); }
        if (wave == 0) STAMP(ws, 1);
        const u32 pb = CTRL_BYTES + par * PAR_BYTES;
        if (wave < 3) {
            const int dst = g * 4 + ((a + 1 + wave) & 3);
            const rsrc_t rd = mk_rsrc(p.ws[dst], WS_NEED);
            for (int blk = 0; blk < 3; ++blk) {
#pragma unroll
                for (int i = 0; i < 4; ++i) poll_ge(ws + OFF_XR + (u32)(blk * 256 + i * 64 + lane) * 4u, call, ws, call);
                STAMP(ws, 2 + wave * 4 + blk);
                const u32 slot = (u32)(a * 6 + blk * 2);
                const u32 sc = OFF_SA + (par * ASLOTS + slot) * 512u;
                push_pair(rws, rd, pb + slot * PANEL, pb + slot * PANEL, OFF_LF + (u32)rank * 128u + (u32)b * 4u, call * 128u + (u32)blk * 32u, b, lane, sc, sc);
            }
            STAMP(ws, 2 + wave * 4 + 3);
        } else if (wave < 7) {
            const int j = (a + wave - 3) & 3, dst = (1 - g) * 4 + j;
            const rsrc_t rd = mk_rsrc(p.ws[dst], WS_NEED);
            const u32 fo = OFF_LF + (u32)rank * 128u + (u32)b * 4u;
            { const int rg = (lane & 7) < 6 ? (lane & 7) : 6;
              poll_ge(ws + OFF_WR + (u32)((j * 8 + (lane >> 3)) * 8 + rg) * 4u, call, ws, call); }
            STAMP(ws, 2 + wave * 4);
            push_pair(rws, rd, pb + (u32)(ASLOTS + 2 * j) * PANEL, pb + (u32)(ASLOTS + 8 + 2 * a) * PANEL, fo, call * 128u, b, lane,
                      OFF_SB + (par * BSLOTS + (u32)(2 * j)) * 512u, OFF_SB + (par * BSLOTS + (u32)(8 + 2 * a)) * 512u);
#pragma unroll
            for (int i = 0; i < 4; ++i) poll_ge(ws + OFF_XR + (u32)(768 + i * 64 + lane) * 4u, call, ws, call);
            STAMP(ws, 2 + wave * 4 + 1);
            push_pair(rws, rd, pb + 32u * PANEL, pb + (u32)(24 + 2 * a) * PANEL, fo, call * 128u + 32u, b, lane,
                      OFF_SA + (par * ASLOTS + 32u) * 512u, OFF_SA + (par * ASLOTS + (u32)(24 + 2 * a)) * 512u);
            STAMP(ws, 2 + wave * 4 + 3);
        } else if (b == 0) {
            for (int j = 0; j < 4; ++j) poll_ge(ws + OFF_YF + (u32)(((1 - g) * 4 + j) * 64 + lane) * 4u, call, ws, call);
            __builtin_amdgcn_fence(__ATOMIC_ACQUIRE, "");
            if (ld_agent(ws + OFF_TMO) == call && lane == 0) p.out[rank][(size_t)rank * 1024 * NLOC] = __builtin_nanf("");
            STAMP(ws, 30);
        }
        return;
    }

    {
        const int c = b - NCOMM, strip = c / 7, rg = c - strip * 7;
        const int kt0 = (rg * 128) / 7, kt1 = ((rg + 1) * 128) / 7;
        const rsrc_t rw = mk_rsrc(p.w, IN_BYTES);
        const int q = lane & 7, n4 = lane >> 3;
        f32x4 v[3][8];
#pragma unroll
        for (int i = 0; i < 3; ++i) {
            const int kt = kt0 + wave + 8 * i;
            const u32 base = kt < kt1 ? ((u32)(kt * 64 + q * 8) * NLOC + (u32)(strip * 32 + n4 * 4)) * 4u : 0x80000000u;
#pragma unroll
            for (int jj = 0; jj < 8; ++jj) v[i][jj] = __builtin_bit_cast(f32x4, __builtin_amdgcn_raw_buffer_load_b128(rw, base + (u32)jj * (NLOC * 4u), 0, 0));
        }
        LAS u32* cm = (LAS u32*)(lds + L_RED + 256u);
        if (tid < 64) cm[tid] = 0u;
        WAIT_VM(24); WAIT_LGKM0(); BAR();
        if (tid == 0) { st_agent(ws + OFF_XR + (u32)b * 4u, call); st_agent(ws + OFF_CTR + (u32)b * 64u, call); }
        if (wave == 0) STAMP(ws, 1);
        u32 m[4] = {0u, 0u, 0u, 0u};
#pragma unroll
        for (int i = 0; i < 3; ++i)
#pragma unroll
            for (int jj = 0; jj < 8; ++jj)
#pragma unroll
                for (int k = 0; k < 4; ++k) m[k] = umax(m[k], absbits(v[i][jj][k]));
#pragma unroll
        for (int k = 0; k < 4; ++k) {
            m[k] = umax(m[k], (u32)__shfl_xor((int)m[k], 1)); m[k] = umax(m[k], (u32)__shfl_xor((int)m[k], 2)); m[k] = umax(m[k], (u32)__shfl_xor((int)m[k], 4));
        }
        if (q < 4) __hip_atomic_fetch_max(cm + n4 * 4 + q, q == 0 ? m[0] : q == 1 ? m[1] : q == 2 ? m[2] : m[3], __ATOMIC_RELAXED, __HIP_MEMORY_SCOPE_WORKGROUP);
        WAIT_LGKM0(); BAR();
        u64* wx = (u64*)(ws + OFF_WX) + (u32)strip * 256u;
        if (tid < 32) __hip_atomic_store(wx + rg * 32 + tid, ((u64)call << 32) | cm[tid], __ATOMIC_RELAXED, __HIP_MEMORY_SCOPE_AGENT);
        if (wave < 4) {
            const int gi = tid < 224 ? tid : 0;
            const u64 t0 = __builtin_amdgcn_s_memrealtime();
            u64 gv; u32 n = 0;
            for (;;) {
                gv = __hip_atomic_load(wx + gi, __ATOMIC_RELAXED, __HIP_MEMORY_SCOPE_AGENT);
                if (__all((u32)(gv >> 32) == call)) break;
                if ((++n & 15u) == 0u) {
                    const u32 t = __builtin_amdgcn_readfirstlane(ld_agent(ws + OFF_TMO));
                    if (t == call || __builtin_amdgcn_s_memrealtime() - t0 > DEADLINE) { st_agent(ws + OFF_TMO, call); break; }
                }
                __builtin_amdgcn_s_sleep(1);
            }
            __hip_atomic_fetch_max(cm + 32 + (gi & 31), (u32)gv, __ATOMIC_RELAXED, __HIP_MEMORY_SCOPE_WORKGROUP);
        }
        WAIT_LGKM0(); BAR();
        const u32x4 mx = *(LAS u32x4*)(cm + 32 + n4 * 4);
        const int ct = strip >> 2;
        const u32 pbase = CTRL_BYTES + par * PAR_BYTES + (u32)(ASLOTS + ct) * PANEL;
#pragma unroll
        for (int k = 0; k < 4; ++k) {
            const int e = exp_of(mx[k]); const float s = scale_of(e);
            const u32 nin = (u32)((strip & 3) * 32 + n4 * 4 + k);
            const u32 ro = nin * 128u + (((u32)q ^ ((nin >> 1) & 7u)) << 4);
#pragma unroll
            for (int i = 0; i < 3; ++i) {
                const int kt = kt0 + wave + 8 * i;
                const u32 off = kt < kt1 ? pbase + (u32)kt * IMG + ro : 0x80000000u;
                const f32x4 lo = {v[i][0][k], v[i][1][k], v[i][2][k], v[i][3][k]}, hi = {v[i][4][k], v[i][5][k], v[i][6][k], v[i][7][k]};
                __builtin_amdgcn_raw_buffer_store_b128(cvt8(lo, hi, s), rws, off, 0, AUX_SC1);
            }
            const u32 so = (rg == 0 && wave == 0 && q == 0) ? OFF_SB + ((par * BSLOTS + (u32)ct) * 128u + nin) * 4u : 0x80000000u;
            __builtin_amdgcn_raw_buffer_store_b32((u32)(e - 141), rws, so, 0, AUX_SC1);
        }
        WAIT_VM(0); BAR();
        if (tid == 0) st_agent(ws + OFF_WR + (u32)(strip * 8 + rg) * 4u, call);
        if (wave == 0) STAMP(ws, 2);
    }
    const int xcd = b & 7, j = b >> 3;
    int slots[3], nt;
    int ct;
    if (j < 16) {
        const int pj = j - 4, i = (a + 1 + (pj >> 2)) & 3, h = (pj >> 1) & 1;
        ct = 2 * xcd + (pj & 1); nt = 3;
        slots[0] = i * 6 + h; slots[1] = slots[0] + 2; slots[2] = slots[0] + 4;
    } else {
        const int qj = j - 16, r8 = qj >> 1, qi = qj * 8 + xcd;
        x_rows<6>(p, lds, 256 + qi * 6, par, a, tid, wave);
        WAIT_VM(0); BAR();
        if (tid < 6) st_agent(ws + OFF_XR + (u32)(256 + qi * 6 + tid) * 4u, call);
        if (wave == 0) STAMP(ws, 3);
        ct = 2 * xcd + (qj & 1);
        if (r8 < 6) { nt = 2; slots[0] = a * 6 + r8; slots[1] = 24 + r8; } else { nt = 1; slots[0] = 24 + r8; slots[1] = 0; }
        slots[2] = 0;
    }
    for (int it = 0; it < nt; ++it) {
        Tile T;
        make_tile(T, rank, it == 0 ? slots[0] : it == 1 ? slots[1] : slots[2], ct, par);
        gemm_tile(p, lds, T, call, tid, lane, wave, 4 + it * 3);
    }
}
}

extern "C" void kernel_launch(int rank, int nranks, ncclComm_t comm,
                              hipStream_t stream,
                              void* const* d_in, const int* in_sizes, int n_in,
                              void* d_out, int out_size,
                              const int* in_dtypes, int out_dtype,
                              void* const* d_out_all,
                              void* const* d_ws_all, size_t ws_bytes) {
    (void)comm; (void)in_dtypes; (void)out_dtype; (void)d_out;
    if (nranks != 8 || (unsigned)rank >= 8u || n_in < 2 || ws_bytes < (size_t)agk::WS_NEED || in_sizes[0] != 1024 * 8192 ||
        in_sizes[1] != 8192 * 1024 || out_size != 8192 * 1024) {
        fprintf(stderr, "kernel_launch: unexpected problem shape\n");
        return;
    }
    static bool ready[8] = {false, false, false, false, false, false, false, false};
    if (!ready[rank]) {
        HIP_CHECK(hipFuncSetAttribute((const void*)agk::ag_gemm_kernel, hipFuncAttributeMaxDynamicSharedMemorySize, (int)agk::LDS_BYTES));
        ready[rank] = true;
    }
    agk::Params p;
    p.x = (const float*)d_in[0]; p.w = (const float*)d_in[1]; p.rank = rank;
    for (int i = 0; i < 8; ++i) { p.ws[i] = (char*)d_ws_all[i]; p.out[i] = (float*)d_out_all[i]; }
    hipLaunchKernelGGL(agk::ag_gemm_kernel, dim3(agk::NWG), dim3(512), agk::LDS_BYTES, stream, p);
    HIP_CHECK(hipGetLastError());
}
```

```cpp
#include <hip/hip_runtime.h>
#include <rccl/rccl.h>
#include <cstdio>

#define HIP_CHECK(expr) do { hipError_t _e = (expr); if (_e != hipSuccess) { \
    fprintf(stderr, "HIP error %s at %s:%d\n", hipGetErrorString(_e), __FILE__, __LINE__); \
    return; } } while (0)

namespace agk {
typedef unsigned u32;
typedef unsigned long long u64;
typedef u32 u32x4 __attribute__((ext_vector_type(4)));
typedef int i32x4 __attribute__((ext_vector_type(4)));
typedef float f32x4 __attribute__((ext_vector_type(4)));
typedef _Float16 f16x8 __attribute__((ext_vector_type(8)));
#define LAS __attribute__((address_space(3)))
typedef __amdgpu_buffer_rsrc_t rsrc_t;
typedef __attribute__((address_space(1))) char gchar;

constexpr int KDIM = 8192, NLOC = 1024, NKT = 128;
constexpr u32 IMG = 16384u;
constexpr u32 PANEL = NKT * IMG;
constexpr u32 UNIT = 131072u;
constexpr int ASLOTS = 34, BSLOTS = 16;
constexpr u32 CTRL_BYTES = 1u << 20;
constexpr u32 RELAY = (ASLOTS + BSLOTS) * PANEL;
constexpr u32 PAR_BYTES = RELAY + 2u * UNIT;
constexpr u32 OFF_YST = CTRL_BYTES + 2u * PAR_BYTES;
constexpr u32 WS_NEED = OFF_YST + 4u * 32u * UNIT;
constexpr u32 IN_BYTES = 1024u * 8192u * 4u;
constexpr u32 OFF_CTR = 0;
constexpr u32 OFF_TMO = 16384;
constexpr u32 OFF_CL = 20480;
constexpr u32 OFF_LF = 524288;
constexpr u32 OFF_RD = 655360;
constexpr u32 OFF_YR = 24576;
constexpr u32 OFF_XR = 28672;
constexpr u32 OFF_WR = 32768;
constexpr u32 OFF_WX = 36864;
constexpr u32 OFF_SA = 131072;
constexpr u32 OFF_SB = 196608;
constexpr u32 OFF_DBG = 262144;
constexpr int NWG = 256, NCOMM = 32;
constexpr u32 NUNITS = 98u;
constexpr u32 OOB = 0x80000000u, NOFLAG = 0x40000000u;
constexpr u64 DEADLINE = 30000000ull;
constexpr u32 STAGE = 32768u;
constexpr u32 L_SNAP = 4u * STAGE;
constexpr u32 L_SC = L_SNAP + 512u;
constexpr u32 L_RED = L_SC + 1024u;
constexpr u32 LDS_BYTES = L_RED + 2048u;

struct Params {
    const float* x; const float* w;
    char* ws[8]; float* out[8];
    int rank;
};

#define AUX_SC1 16
#define AUX_SYS 17
#define GL_AUX 16
#define W_LD 3
#define W_ST 2
#define X_LD 4
#define X_ST 2
#define WAIT_VM(n) asm volatile("s_waitcnt vmcnt(%0)" :: "n"(n) : "memory")
#define WAIT_LGKM0() asm volatile("s_waitcnt lgkmcnt(0)" ::: "memory")
#define BAR() do { asm volatile("" ::: "memory"); __builtin_amdgcn_s_barrier(); asm volatile("" ::: "memory"); } while (0)
#define WAIT_VM_TRACKED(n) do { asm volatile("" ::: "memory"); __builtin_amdgcn_s_waitcnt(((n) & 15) | (7 << 4) | (15 << 8) | (((n) >> 4) << 14)); asm volatile("" ::: "memory"); } while (0)

__device__ __forceinline__ rsrc_t mk_rsrc(const void* p, u32 bytes) { return __builtin_amdgcn_make_buffer_rsrc((void*)p, 0, bytes, 0x00020000); }
__device__ __forceinline__ u32 ld_agent(const void* p) { return __hip_atomic_load((const u32*)p, __ATOMIC_RELAXED, __HIP_MEMORY_SCOPE_AGENT); }
__device__ __forceinline__ u32 ld_sys(const void* p) { return __hip_atomic_load((const u32*)p, __ATOMIC_RELAXED, __HIP_MEMORY_SCOPE_SYSTEM); }
__device__ __forceinline__ void st_agent(void* p, u32 v) { __hip_atomic_store((u32*)p, v, __ATOMIC_RELAXED, __HIP_MEMORY_SCOPE_AGENT); }
__device__ __forceinline__ void st_sys_g(gchar* p, u32 v) { __hip_atomic_store((__attribute__((address_space(1))) u32*)p, v, __ATOMIC_RELAXED, __HIP_MEMORY_SCOPE_SYSTEM); }
__device__ __forceinline__ void st_sys(void* p, u32 v) { __hip_atomic_store((u32*)p, v, __ATOMIC_RELAXED, __HIP_MEMORY_SCOPE_SYSTEM); }

#define STAMP(ws, idx) do { } while (0)

__device__ __forceinline__ bool give_up(char* ws, u32 call, u64 t0) {
    const u32 t = __builtin_amdgcn_readfirstlane(ld_agent(ws + OFF_TMO));
    if (t == call || __builtin_amdgcn_s_memrealtime() - t0 > DEADLINE) { st_agent(ws + OFF_TMO, call); return true; }
    return false;
}
__device__ __forceinline__ void poll_ge(const char* p, u32 need, char* ws, u32 call) {
    const u64 t0 = __builtin_amdgcn_s_memrealtime();
    u32 n = 0;
    for (;;) {
        const u32 v = ld_sys(p);
        if (__all((int)(v - need) >= 0)) return;
        if ((++n & 15u) == 0u && give_up(ws, call, t0)) return;
        __builtin_amdgcn_s_sleep(2);
    }
}
__device__ __forceinline__ bool all4_ge(u32x4 v, u32 need) {
    return __all((int)(v[0] - need) >= 0 && (int)(v[1] - need) >= 0 && (int)(v[2] - need) >= 0 && (int)(v[3] - need) >= 0);
}
__device__ __forceinline__ void poll4_ge(rsrc_t rs, u32 off, u32 need, char* ws, u32 call) {
    const u64 t0 = __builtin_amdgcn_s_memrealtime();
    u32 n = 0;
    for (;;) {
        asm volatile("" ::: "memory");
        const u32x4 v = __builtin_amdgcn_raw_buffer_load_b128(rs, off, 0, AUX_SYS);
        if (all4_ge(v, need)) return;
        if ((++n & 15u) == 0u && give_up(ws, call, t0)) return;
        __builtin_amdgcn_s_sleep(1);
    }
}

__device__ __forceinline__ u32 umax(u32 a, u32 b) { return a > b ? a : b; }
__device__ __forceinline__ u32 absbits(float f) { return __float_as_uint(f) & 0x7fffffffu; }
__device__ __forceinline__ int exp_of(u32 amaxbits) { int e = (int)(amaxbits >> 23); return e < 15 ? 15 : e; }
__device__ __forceinline__ float scale_of(int e) { return __uint_as_float((u32)(268 - e) << 23); }
__device__ __forceinline__ u32x4 cvt8(f32x4 a, f32x4 b, float s) {
    f16x8 h;
    h[0] = (_Float16)(a[0] * s); h[1] = (_Float16)(a[1] * s); h[2] = (_Float16)(a[2] * s); h[3] = (_Float16)(a[3] * s);
    h[4] = (_Float16)(b[0] * s); h[5] = (_Float16)(b[1] * s); h[6] = (_Float16)(b[2] * s); h[7] = (_Float16)(b[3] * s);
    return __builtin_bit_cast(u32x4, h);
}

template <int NR, bool HEAD>
__device__ __forceinline__ gchar* x_rows(const Params& p, LAS char* lds, int row0, u32 par, u32 call, int tid, int wave) {
    const int g = p.rank >> 2, a = p.rank & 3;
    char* ws = p.ws[p.rank];
    const rsrc_t rx = mk_rsrc(p.x, IN_BYTES), rw = mk_rsrc(ws, WS_NEED);
    const u32 pi0 = (u32)((7 - wave) * 128 + (tid & 63));
    f32x4 v[NR][4];
#pragma unroll
    for (int r = 0; r < NR; ++r)
#pragma unroll
        for (int u = 0; u < 2; ++u) {
            const u32 off = (u32)(row0 + r) * (KDIM * 4u) + (pi0 + (u32)u * 64u) * 32u;
            v[r][2 * u] = __builtin_bit_cast(f32x4, __builtin_amdgcn_raw_buffer_load_b128(rx, off, 0, 0));
            v[r][2 * u + 1] = __builtin_bit_cast(f32x4, __builtin_amdgcn_raw_buffer_load_b128(rx, off + 16u, 0, 0));
            if (!HEAD) WAIT_VM(X_LD - 2);
        }
    const int rb = row0 >> 7;
    const u32 slot = rb < 6 ? (u32)(a * 6 + rb) : (u32)(32 + rb - 6);
    const u32 rslot = rb < 6 ? slot : (u32)(24 + 2 * a + rb - 6);
    const int ug = rb < 6 ? g : 1 - g;
    gchar* user = (gchar*)((tid & 2) ? ((tid & 1) ? p.ws[ug * 4 + 3] : p.ws[ug * 4 + 2]) : ((tid & 1) ? p.ws[ug * 4 + 1] : p.ws[ug * 4]));
    LAS u32* red = (LAS u32*)(lds + L_RED);
#pragma unroll
    for (int r = 0; r < NR; ++r) {
        u32 m = 0;
#pragma unroll
        for (int i = 0; i < 4; ++i)
#pragma unroll
            for (int j = 0; j < 4; ++j) m = umax(m, absbits(v[r][i][j]));
#pragma unroll
        for (int o = 32; o >= 1; o >>= 1) m = umax(m, (u32)__shfl_xor((int)m, o));
        if ((tid & 63) == 0) red[r * 8 + wave] = m;
    }
    asm volatile("" : "+v"(user));
    WAIT_LGKM0(); BAR();
#pragma unroll
    for (int r = 0; r < NR; ++r) {
        const u32x4 m0 = *(LAS u32x4*)(red + r * 8), m1 = *(LAS u32x4*)(red + r * 8 + 4);
        const u32 m = umax(umax(umax(m0[0], m0[1]), umax(m0[2], m0[3])), umax(umax(m1[0], m1[1]), umax(m1[2], m1[3])));
        const int e = exp_of(m); const float s = scale_of(e);
        const u32 rin = (u32)((row0 + r) & 127);
        const u32 eoff = OFF_SA + ((par * 32u + rslot) * 128u + rin) * 4u;
        if (!HEAD) { if (tid < 4) st_sys_g(user + eoff, (u32)(e - 141)); }
        else if (tid == 0 && rb < 6) st_sys(ws + eoff, (u32)(e - 141));
        const u32 off = CTRL_BYTES + par * PAR_BYTES + rin * 128u + (pi0 >> 3) * IMG + (((pi0 & 7u) ^ ((rin >> 1) & 7u)) << 4);
        const u32x4 h0 = cvt8(v[r][0], v[r][1], s), h1 = cvt8(v[r][2], v[r][3], s);
        __builtin_amdgcn_raw_buffer_store_b128(h0, rw, off + slot * PANEL, 0, AUX_SC1);
        __builtin_amdgcn_raw_buffer_store_b128(h1, rw, off + slot * PANEL + 8u * IMG, 0, AUX_SC1);
        if (!HEAD) WAIT_VM(X_ST - 2);
        if (HEAD) {
#define HROUND(k) do { _Pragma("unroll") for (int i = 0; i < 4; ++i) { const int d = ug * 4 + ((a + 1 + i) & 3); \
                __builtin_amdgcn_raw_buffer_store_b128((k) < 2 ? h0 : h1, mk_rsrc(p.ws[d], WS_NEED), \
                    (d == p.rank || ((tid >> 5) & 1) != ((k) & 1)) ? OOB : off + rslot * PANEL + ((k) < 2 ? 0u : 8u * IMG), 0, AUX_SYS); } } while (0)
            if (wave == 7) { HROUND(0); WAIT_VM(4); } else WAIT_VM(0);
            BAR();
            if (tid == 0) st_agent(ws + OFF_XR + (u32)row0 * 4u, call);
            if (wave == 7) {
                WAIT_VM(0); HROUND(1); WAIT_VM(0); HROUND(2); WAIT_VM(0); HROUND(3);
                if ((tid & 63) < 4) st_sys_g(user + eoff, (u32)(e - 141));
            }
#undef HROUND
        }
    }
    if (!HEAD) {
        WAIT_VM(0); BAR();
        if (tid < NR) st_agent(ws + OFF_XR + (u32)(row0 + tid) * 4u, call);
    }
    return user + OFF_RD + ((u32)p.rank * 128u + (u32)(row0 & 127)) * 4u;
}

__device__ __forceinline__ int slot_row(int g, int slot) {
    if (slot < 24) { const int i = slot / 6; return (g * 4 + i) * 1024 + (slot - i * 6) * 128; }
    return ((1 - g) * 4 + ((slot - 24) >> 1)) * 1024 + 768 + (slot & 1) * 128;
}

template <bool BIG>
__device__ __forceinline__ void gemm_tile(const Params& p, LAS char* lds, int slot, int ct, int rh, int ch, u32 par, u32 call, int tid, int lane, int wave, int stamp0) {
    constexpr int NB = BIG ? 4 : 2, NC = BIG ? 32 : 16, BT = BIG ? 128 : 64;
    const int rank = p.rank, g = rank >> 2, a = rank & 3;
    char* ws = p.ws[rank];
    const rsrc_t rws = mk_rsrc(ws, WS_NEED);
    int a_src, a_rb; u32 a_base, a_sh;
    if (slot < 24) { const int i = slot / 6; a_rb = slot - i * 6; a_src = i == a ? -1 : g * 4 + i;
        a_base = a_rb == 0 ? 0u : a_rb < 3 ? 16u : a_rb < 5 ? 48u : 80u; a_sh = (a_rb == 0 || a_rb == 5) ? 1u : 0u; }
    else { a_rb = 0; a_src = (1 - g) * 4 + ((slot - 24) >> 1); a_base = (slot & 1) ? 48u : 0u; a_sh = 1u; }
    const int b_src = ct < 8 ? -1 : (1 - g) * 4 + ((ct - 8) >> 1);
    const int src = lane < 32 ? a_src : (BIG ? b_src : -1);
    const u32 gsh = src < 0 ? 31u : lane < 32 && BIG ? a_sh : 0u;
    const char* gptr = ws + (src >= 0 ? OFF_LF + ((u32)src * NUNITS + (lane < 32 ? a_base : 16u)) * 128u + (u32)(lane & 31) * 4u : OFF_CTR + blockIdx.x * 64u);
    const u32 gmin = src >= 0 && lane < 32 && a_base == 0u ? 2u : 0u;
#define GPTR(c) (gptr + (umax((u32)(c) >> gsh, gmin) << 7))
    if (wave == 0) {
        if (a_src < 0) poll4_ge(rws, OFF_XR + (u32)a_rb * 512u + (u32)(lane & 31) * 16u, call, ws, call);
        if (b_src < 0) poll4_ge(rws, OFF_WR + (u32)ct * 128u + (u32)(lane & 7) * 16u, call, ws, call);
        else if (!BIG) for (u32 i = 0; i < 4u; ++i) poll4_ge(rws, OFF_LF + ((u32)b_src * NUNITS + 16u) * 128u + i * 1024u + (u32)lane * 16u, call, ws, call);
        if (a_src >= 0 && a_base == 0u) poll4_ge(rws, OFF_RD + (u32)a_src * 512u + (u32)(lane & 31) * 16u, call, ws, call);
        poll_ge(GPTR(0), call, ws, call);
        WAIT_VM(0);
    }
    BAR();
    if (wave == 0) STAMP(ws, stamp0);
    const u32 pb = CTRL_BYTES + par * PAR_BYTES;
    const char* gA = ws + pb + (u32)slot * PANEL + (BIG ? (u32)wave * 2048u : (u32)rh * 8192u + (u32)wave * 1024u) + (u32)lane * 16u;
    const char* gB = ws + pb + (u32)(ASLOTS + ct) * PANEL + (BIG ? (u32)wave * 2048u : (u32)ch * 8192u + (u32)wave * 1024u) + (u32)lane * 16u;
    const u32 ldsw = (u32)wave * (BIG ? 2048u : 1024u);
    constexpr u32 GSTEP = BIG ? IMG : 2u * IMG, G2 = BIG ? 1024u : IMG, L2O = BIG ? 1024u : 8192u;
#define GLDS16(g, l) __builtin_amdgcn_global_load_lds((const u32*)(g), (LAS u32*)(lds + (l)), 16, 0, GL_AUX)
#define LOADT(t, st) do { const u32 _o = (u32)(t) * GSTEP; \
        GLDS16(gA + _o, (st) * STAGE + ldsw); GLDS16(gA + _o + G2, (st) * STAGE + ldsw + L2O); \
        GLDS16(gB + _o, (st) * STAGE + IMG + ldsw); GLDS16(gB + _o + G2, (st) * STAGE + IMG + ldsw + L2O); } while (0)
#define SNAP(c) __builtin_amdgcn_global_load_lds((const u32*)GPTR(c), (LAS u32*)(lds + L_SNAP + ((u32)(c) & 1u) * 256u), 4, 0, AUX_SYS)
    const int ks = BIG ? wave >> 2 : wave >> 1, wm = BIG ? (wave >> 1) & 1 : 0, wn = wave & 1;
    const u32 swz = (u32)((((ks & 1) * 4 + (lane >> 4)) ^ ((lane & 15) >> 1)) << 4) + (BIG ? 0u : (u32)(ks >> 1) * 8192u);
    const u32 ard = (u32)(wm * 64 + (lane & 15)) * 128u + swz, brd = IMG + (u32)(wn * (BT / 2) + (lane & 15)) * 128u + swz;
#define READF(fa, fb, st) do { _Pragma("unroll") for (int _i = 0; _i < 4; ++_i) { \
        if (_i < NB) fb[_i] = *(const LAS f16x8*)(lds + (st) * STAGE + brd + _i * 2048u); \
        fa[_i] = *(const LAS f16x8*)(lds + (st) * STAGE + ard + _i * 2048u); } } while (0)
#define MMA(fa, fb) do { _Pragma("unroll") for (int _m = 0; _m < 4; ++_m) _Pragma("unroll") for (int _n = 0; _n < NB; ++_n) \
        acc[_m][_n] = __builtin_amdgcn_mfma_f32_16x16x32_f16(fb[_n], fa[_m], acc[_m][_n], 0, 0, 0); } while (0)
    f32x4 acc[4][NB];
#pragma unroll
    for (int m = 0; m < 4; ++m)
#pragma unroll
        for (int n = 0; n < NB; ++n) acc[m][n] = (f32x4){0.f, 0.f, 0.f, 0.f};
    f16x8 a0[4], b0[4], a1[4], b1[4];
    if (wave >= 4) {
        const u32 so = (wave < 6 ? OFF_SA + (par * 32u + (u32)slot) * 512u : OFF_SB + (par * BSLOTS + (u32)ct) * 512u) + (u32)((wave & 1) * 64 + lane) * 4u;
        __builtin_amdgcn_global_load_lds((const u32*)(ws + so), (LAS u32*)(lds + L_SC + (u32)(wave - 4) * 256u), 4, 0, AUX_SYS);
    }
    LOADT(0, 0u);
    if (wave == 0) SNAP(1);
    LOADT(1, 1u); LOADT(2, 2u); LOADT(3, 3u);
    if (wave == 0) WAIT_VM(13); else WAIT_VM(12);
    BAR();
    READF(a0, b0, 0u);
#define GATE(c) do { const u32 _v = *(const LAS u32*)(lds + L_SNAP + ((u32)(c) & 1u) * 256u + (u32)lane * 4u); \
        if (!__all((int)(_v - call) >= 0)) { if (wave == 0) { poll_ge(GPTR(c), call, ws, call); WAIT_VM(0); } BAR(); } } while (0)
    for (int c = 0; c < NC - 1; ++c) {
        const int t = 4 * c;
        WAIT_VM(8); WAIT_LGKM0(); BAR();
        GATE(c + 1);
        LOADT(t + 4, 0u); READF(a1, b1, 1u); MMA(a0, b0);
        WAIT_VM(8); WAIT_LGKM0(); BAR();
        if (wave == 0) SNAP(c + 2 < NC ? c + 2 : c);
        LOADT(t + 5, 1u); READF(a0, b0, 2u); MMA(a1, b1);
        if (wave == 0) WAIT_VM(9); else WAIT_VM(8);
        WAIT_LGKM0(); BAR();
        LOADT(t + 6, 2u); READF(a1, b1, 3u); MMA(a0, b0);
        if (wave == 0) WAIT_VM(9); else WAIT_VM(8);
        WAIT_LGKM0(); BAR();
        LOADT(t + 7, 3u); READF(a0, b0, 0u); MMA(a1, b1);
    }
    WAIT_VM(8); WAIT_LGKM0(); BAR();
    READF(a1, b1, 1u); MMA(a0, b0);
    WAIT_VM(4); WAIT_LGKM0(); BAR();
    READF(a0, b0, 2u); MMA(a1, b1);
    WAIT_VM(0); WAIT_LGKM0(); BAR();
    READF(a1, b1, 3u); MMA(a0, b0);
    const u32 tmo = ld_agent(ws + OFF_TMO);
    WAIT_LGKM0(); BAR();
    MMA(a1, b1);
    if (wave == 0) STAMP(ws, stamp0 + 1);
    constexpr u32 RB = BT * 4u, PPR = BT / 4u, AREA = BT * RB;
#pragma unroll
    for (int m = 0; m < 4; ++m)
#pragma unroll
        for (int n = 0; n < NB; ++n) {
            const u32 r = (u32)(wm * 64 + m * 16 + (lane & 15)), pc = (u32)(wn * (BT / 8) + n * 4 + (lane >> 4));
            *(LAS f32x4*)(lds + (u32)ks * AREA + r * RB + ((pc ^ (r & 7u)) << 4)) = acc[m][n];
        }
    WAIT_LGKM0(); BAR();
    const LAS int* sc = (const LAS int*)(lds + L_SC);
    const u32 pc = (u32)lane & (PPR - 1u);
    const u32 col0 = BIG ? 0u : (u32)ch * 64u, row0 = BIG ? 0u : (u32)rh * 64u;
    const i32x4 eb = *(const LAS i32x4*)(sc + 128 + col0 + pc * 4u);
    const bool bad = tmo == call;
    const bool remote = ct >= 8, staged = remote && BIG;
    const rsrc_t ro = mk_rsrc(staged ? (const void*)ws : (const void*)p.out[remote ? b_src : rank], staged ? WS_NEED : IN_BYTES);
    const u32 yu = (u32)(((ct - 8) >> 1) * 32 + slot);
    const u32 ocol = (remote ? (u32)(a * 256 + (ct & 1) * 128) : (u32)ct * 128u) + col0;
    const u32 obase = staged ? OFF_YST + yu * UNIT + (u32)(ct & 1) * 512u : ((u32)slot_row(g, slot) * NLOC + ocol) * 4u;
    const u32 ostr = staged ? 1024u : NLOC * 4u;
#pragma unroll
    for (u32 it = 0; it < (BIG ? 8u : 2u); ++it) {
        const u32 r = (u32)wave * (BT / 8u) + (it * 64u + (u32)lane) / PPR;
        const u32 lo = r * RB + ((pc ^ (r & 7u)) << 4);
        f32x4 s = *(const LAS f32x4*)(lds + lo) + *(const LAS f32x4*)(lds + AREA + lo);
        if (!BIG) s += *(const LAS f32x4*)(lds + 2u * AREA + lo) + *(const LAS f32x4*)(lds + 3u * AREA + lo);
        const int ea = sc[row0 + r];
#pragma unroll
        for (int j = 0; j < 4; ++j) s[j] = bad ? __builtin_nanf("") : __builtin_amdgcn_ldexpf(s[j], ea + eb[j]);
        __builtin_amdgcn_raw_buffer_store_b128(__builtin_bit_cast(u32x4, s), ro, obase + (row0 + r) * ostr + pc * 16u, 0, BIG ? AUX_SC1 : AUX_SYS);
    }
    if (remote) {
        WAIT_VM(0); BAR();
        if (BIG) { if (tid < 2) st_agent(ws + OFF_YR + yu * 16u + (u32)(ct & 1) * 8u + (u32)tid * 4u, call); }
        else if (tid < 4) st_sys(p.ws[b_src] + OFF_LF + ((u32)rank * NUNITS + 93u + (u32)((slot / 6 - a - 1) & 3)) * 128u + (u32)(rh * 4 + (ct & 1) * 2 + ch) * 16u + (u32)tid * 4u, call);
    }
    if (wave == 0) STAMP(ws, stamp0 + 2);
#undef GLDS16
#undef LOADT
#undef SNAP
#undef READF
#undef MMA
#undef GATE
#undef GPTR
}

constexpr u32 NPOS_INTRA = 92u, NPOS_CROSS = 93u;
struct Desc { u32 src, dst, gate, flag; };
struct Stream {
    bool cross; int rank, g, a, j; u32 pb;
    __device__ __forceinline__ int yslot(u32 k) const {
        if (k < 6u) return a * 6 + (int)k;
        if (k < 9u) return ((a + (int)k - 5) & 3) * 6;
        if (k < 13u) return 24 + 2 * ((int)k - 9);
        if (k < 19u) return ((a + 1 + (((int)k - 13) >> 1)) & 3) * 6 + 1 + (((int)k - 13) & 1);
        if (k < 23u) return 25 + 2 * ((int)k - 19);
        return ((a + 1 + (((int)k - 23) >> 1)) & 3) * 6 + 3 + (((int)k - 23) & 1);
    }
    __device__ __forceinline__ u32 single(u32 slot, u32 k) const { return pb + slot * PANEL + k * UNIT; }
    __device__ __forceinline__ u32 pair(u32 slot, u32 k) const { return pb + slot * PANEL + k * 65536u; }
    __device__ __forceinline__ Desc at(u32 pos) const {
        Desc d;
        const u32 s = (u32)a * 6u, M31 = 7u << 20, M63 = 7u << 20, PAIR = 1u << 28;
        u32 u, fsrc = (u32)rank;
        if (!cross) {
            if (pos >= NPOS_INTRA) pos = NPOS_INTRA - 1u;
            u = pos < 78u ? pos + 2u : pos + 4u;
            d.src = u < 16u ? single(s, u) : u < 48u ? pair(s + 1u, u - 16u) : u < 80u ? pair(s + 3u, u - 48u) : single(s + 5u, u - 80u);
            d.dst = d.src;
            d.gate = u < 16u ? OFF_XR + M31 : u < 48u ? OFF_XR + 512u + M63 + PAIR : u < 80u ? OFF_XR + 1536u + M63 + PAIR : OFF_XR + 2560u + M31;
        } else {
            if (pos >= NPOS_CROSS) pos = NPOS_CROSS - 1u;
            const u32 hop = j == a ? 62u : 68u;
            if (pos < 62u) {
                u = pos + 2u;
                d.src = u < 16u ? single(32u, u) : u < 48u ? pair((u32)(ASLOTS + 2 * j), u - 16u) : single(33u, u - 48u);
                d.dst = u < 16u ? single((u32)(24 + 2 * a), u) : u < 48u ? pair((u32)(ASLOTS + 8 + 2 * a), u - 16u) : single((u32)(25 + 2 * a), u - 48u);
                d.gate = u < 16u ? OFF_XR + 3072u + M31 : u < 48u ? OFF_WR + (u32)j * 256u + (1u << 20) + (3u << 28) : OFF_XR + 3584u + M31;
            } else if (pos - hop < 2u) {
                const u32 k = pos - hop;
                if (j == a) { u = 96u + k; d.src = single(s + 5u, k); d.dst = pb + RELAY + k * UNIT; d.gate = OFF_XR + 2560u + M31; }
                else { u = 80u + k; fsrc = (u32)(rank ^ 4); d.src = pb + RELAY + k * UNIT; d.dst = single(s + 5u, k);
                       d.gate = OFF_LF + (fsrc * NUNITS + 96u + k) * 128u + (4u << 28); }
            } else {
                const u32 k = pos < hop ? pos - 62u : pos - 64u;
                const int ys = yslot(k);
                u = 64u + k;
                d.src = OFF_YST + (u32)(j * 32 + ys) * UNIT;
                d.dst = (u32)slot_row(g, ys) * (NLOC * 4u) + (u32)a * 1024u;
                d.gate = OFF_YR + (u32)(j * 32 + ys) * 16u + (2u << 28);
            }
        }
        d.flag = OFF_LF + (fsrc * NUNITS + u) * 128u;
        return d;
    }
};
struct Tables { Desc lo, hi; };
__device__ __forceinline__ u32 tab(u32 t0, u32 t1, u32 i) {
    const u32 x = (u32)__builtin_amdgcn_readlane((int)t0, (int)(i & 63u)), y = (u32)__builtin_amdgcn_readlane((int)t1, (int)(i & 63u));
    return i < 64u ? x : y;
}

#define PW 2
#define NACT 32
__device__ __forceinline__ void push_stream(const Params& p, const Tables& T, bool cross, u32 cw, int dstrank, u32 call, int lane, int wave) {
    char* ws = p.ws[p.rank];
    const rsrc_t rs = mk_rsrc(ws, WS_NEED), rdw = mk_rsrc(p.ws[dstrank], WS_NEED), rdo = mk_rsrc(p.out[dstrank], IN_BYTES);
    const u32 l16 = (u32)lane * 16u;
    const u32 l0 = lane == 0 ? 0u : OOB;
    const u32 co = OFF_CL + (u32)wave * 128u + l0;
    const u32 NPIECES = (cross ? NPOS_CROSS : NPOS_INTRA) * 32u;
    const u32 cbase = 2u * NACT - (call - 1u) * NPIECES;
    u32x4 va[4], vb[4];
    u32 qa = OOB, qb = OOB;
    u32 c0 = cw, c1 = cw + NACT;
    u32 pf = NOFLAG;
    bool claimed = false, dead = false;
    int cl = 0; u32 idle = 0; u64 t0 = 0;
#pragma unroll
    for (int u = 0; u < 4; ++u) { va[u] = (u32x4){0u, 0u, 0u, 0u}; vb[u] = va[u]; }
#define KIND(q) (tab(T.lo.gate, T.hi.gate, (q) >> 5) >> 28)
#define POFF(q, kd) (((kd) & 1u) ? (((q) >> 4) & 1u) * PANEL + ((q) & 15u) * 4096u : ((q) & 31u) * 4096u)
#define SOFF(q) (tab(T.lo.src, T.hi.src, (q) >> 5) + POFF(q, KIND(q)) + l16)
#define GOFF(q) ({ const u32 _g = tab(T.lo.gate, T.hi.gate, (q) >> 5), _k = _g >> 28; \
        const u32 _ix = (((_k & 1u) ? ((q) >> 2) & 4u : 0u) + ((q) & 3u)) << (_k == 3u ? 5 : 7); \
        (_g & 0xfffffu) + (_k == 2u ? 0u : _k == 4u ? ((q) & 28u) * 4u : _ix) + (l16 & (_g >> 16)); })
#define PFLAG() do { __builtin_amdgcn_raw_buffer_store_b32(call, rdw, l0 + pf, 0, AUX_SYS); } while (0)
#define PST(v, u) __builtin_amdgcn_raw_buffer_store_b128(v[u], _rd, _d + u * _ds, 0, AUX_SYS)
#define PSTORES(v) do { WAIT_VM_TRACKED(7); PST(v, 0); WAIT_VM_TRACKED(7); PFLAG(); PST(v, 1); WAIT_VM_TRACKED(2); PST(v, 2); WAIT_VM_TRACKED(1); PST(v, 3); } while (0)
    u32x4 gv = __builtin_amdgcn_raw_buffer_load_b128(rs, GOFF(c0), 0, AUX_SYS);
#define PSTEP(v, q) do { \
        if (q != OOB) { \
            PSTAMP(q >> 5); \
            const u32 _kd = KIND(q); const bool _y = _kd == 2u; \
            const u32 _d = tab(T.lo.dst, T.hi.dst, q >> 5) + (_y ? (q & 31u) * 16384u : POFF(q, _kd)) + l16, _ds = _y ? NLOC * 4u : 1024u; \
            const rsrc_t _rd = _y ? rdo : rdw; \
            PSTORES(v); pf = tab(T.lo.flag, T.hi.flag, q >> 5) + (q & 31u) * 4u; idle = 0u; \
        } else { \
            WAIT_VM_TRACKED(0); PFLAG(); pf = NOFLAG; \
            if (idle == 0u) t0 = __builtin_amdgcn_s_memrealtime(); \
            if ((++idle & 15u) == 0u && give_up(ws, call, t0)) dead = true; \
        } \
        if (claimed) { c1 = (u32)__builtin_amdgcn_readfirstlane(cl) + cbase; if (c1 > NPIECES + 127u) c1 = NPIECES + 127u; } \
        claimed = (all4_ge(gv, call) || dead) && c0 < NPIECES; \
        q = claimed ? c0 : OOB; if (claimed) c0 = c1; \
        cl = __builtin_amdgcn_raw_ptr_buffer_atomic_add_i32(1, rs, claimed ? co : OOB, 0, AUX_SC1); \
        gv = __builtin_amdgcn_raw_buffer_load_b128(rs, GOFF(c0), 0, AUX_SYS); \
        const u32 _n = claimed ? SOFF(q) : OOB; \
        _Pragma("unroll") for (int u = 0; u < 4; ++u) v[u] = __builtin_amdgcn_raw_buffer_load_b128(rs, _n + u * 1024u, 0, AUX_SC1); \
        } while (0)
#define PSTAMP(i) do { } while (0)
    do {
        PSTEP(va, qa);
        PSTEP(vb, qb);
    } while (c0 < NPIECES || qa != OOB || qb != OOB);
    WAIT_VM_TRACKED(0); PFLAG();
    STAMP(ws, 2 + wave * 8 + 7);
#undef PSTEP
#undef PSTAMP
#undef GOFF
#undef SOFF
#undef POFF
#undef KIND
#undef PST
#undef PSTORES
#undef PFLAG
}

__global__ void __launch_bounds__(512, 2) ag_gemm_kernel(Params p) {
    extern __shared__ __attribute__((aligned(16))) char smem[];
    LAS char* lds = (LAS char*)smem;
    const int tid = threadIdx.x, lane = tid & 63, wave = __builtin_amdgcn_readfirstlane(tid >> 6);
    const int b = blockIdx.x, rank = p.rank, g = rank >> 2, a = rank & 3;
    char* ws = p.ws[rank];
    if (wave == 0) STAMP(ws, 0);
    const u32 call = (u32)__builtin_amdgcn_readfirstlane(ld_agent(ws + OFF_CTR + (u32)b * 64u)) + 1u;
    const u32 par = call & 1u;
    const rsrc_t rws = mk_rsrc(ws, WS_NEED);

    gchar* rdflag = x_rows<1, true>(p, lds, b < 128 ? b : 640 + b, par, call, tid, wave);
    if (tid == 0) st_agent(ws + OFF_CTR + (u32)b * 64u, call);
    if (wave == 0) STAMP(ws, 1);
    if (b < NCOMM) {
        if (wave < 7) {
            if (b >= NACT) return;
            Stream S;
            S.cross = wave >= 3; S.rank = rank; S.g = g; S.a = a; S.j = (a + wave - 3) & 3; S.pb = CTRL_BYTES + par * PAR_BYTES;
            Tables T;
            T.lo = S.at((u32)lane); T.hi = S.at((u32)lane + 64u);
            push_stream(p, T, S.cross, (u32)b, wave < 3 ? g * 4 + ((a + 1 + wave) & 3) : (1 - g) * 4 + S.j, call, lane, wave);
        } else {
            WAIT_VM(0);
            if (lane < 4) st_sys_g(rdflag, call);
            if (b >= 16) return;
            poll4_ge(rws, OFF_LF + ((u32)((1 - g) * 4 + (b >> 2)) * NUNITS + 64u) * 128u + (u32)(b & 3) * 1024u + (u32)lane * 16u, call, ws, call);
            if (ld_agent(ws + OFF_TMO) == call && lane == 0) p.out[rank][(size_t)rank * 1024 * NLOC] = __builtin_nanf("");
            STAMP(ws, 62);
        }
        return;
    }

    {
        const int c = b - NCOMM, strip = c / 7, rg = c - strip * 7;
        const int kt0 = (rg * 128) / 7, kt1 = ((rg + 1) * 128) / 7;
        const rsrc_t rw = mk_rsrc(p.w, IN_BYTES);
        const int q = lane & 7, n4 = lane >> 3;
        if (wave == 7) { WAIT_VM(0); if (lane < 4) st_sys_g(rdflag, call); }
        f32x4 v[3][8];
#pragma unroll
        for (int i = 0; i < 3; ++i) {
            const int kt = kt0 + wave + 8 * i;
            const u32 base = kt < kt1 ? ((u32)(kt * 64 + q * 8) * NLOC + (u32)(strip * 32 + n4 * 4)) * 4u : OOB;
#pragma unroll
            for (int jj = 0; jj < 8; ++jj) {
                v[i][jj] = __builtin_bit_cast(f32x4, __builtin_amdgcn_raw_buffer_load_b128(rw, base + (u32)jj * (NLOC * 4u), 0, 0));
                WAIT_VM(W_LD - 1);
            }
        }
        LAS u32* cm = (LAS u32*)(lds + L_RED + 256u);
        if (tid < 64) cm[tid] = 0u;
        WAIT_LGKM0(); BAR();
        u32 m[4] = {0u, 0u, 0u, 0u};
#pragma unroll
        for (int i = 0; i < 3; ++i)
#pragma unroll
            for (int jj = 0; jj < 8; ++jj)
#pragma unroll
                for (int k = 0; k < 4; ++k) m[k] = umax(m[k], absbits(v[i][jj][k]));
#pragma unroll
        for (int k = 0; k < 4; ++k) {
            m[k] = umax(m[k], (u32)__shfl_xor((int)m[k], 1)); m[k] = umax(m[k], (u32)__shfl_xor((int)m[k], 2)); m[k] = umax(m[k], (u32)__shfl_xor((int)m[k], 4));
        }
        if (q < 4) __hip_atomic_fetch_max(cm + n4 * 4 + q, q == 0 ? m[0] : q == 1 ? m[1] : q == 2 ? m[2] : m[3], __ATOMIC_RELAXED, __HIP_MEMORY_SCOPE_WORKGROUP);
        WAIT_LGKM0(); BAR();
        u64* wx = (u64*)(ws + OFF_WX) + (u32)strip * 256u;
        if (tid < 32) __hip_atomic_store(wx + rg * 32 + tid, ((u64)call << 32) | cm[tid], __ATOMIC_RELAXED, __HIP_MEMORY_SCOPE_AGENT);
        if (wave < 4) {
            const int gi = tid < 224 ? tid : 0;
            const u64 t0 = __builtin_amdgcn_s_memrealtime();
            u64 gv; u32 n = 0;
            for (;;) {
                gv = __hip_atomic_load(wx + gi, __ATOMIC_RELAXED, __HIP_MEMORY_SCOPE_AGENT);
                if (__all((u32)(gv >> 32) == call)) break;
                if ((++n & 15u) == 0u && give_up(ws, call, t0)) break;
                __builtin_amdgcn_s_sleep(1);
            }
            __hip_atomic_fetch_max(cm + 32 + (gi & 31), (u32)gv, __ATOMIC_RELAXED, __HIP_MEMORY_SCOPE_WORKGROUP);
        }
        WAIT_LGKM0(); BAR();
        const int ct = strip >> 2;
        if (rg == 0 && tid < 64) {
            const u32 nin = (u32)((strip & 3) * 32 + (tid & 31));
            const u32 ev = (u32)(exp_of(cm[32 + (tid & 31)]) - 141);
            char* t = tid < 32 ? ws + OFF_SB + (par * BSLOTS + (u32)ct) * 512u : p.ws[(1 - g) * 4 + (strip >> 3)] + OFF_SB + (par * BSLOTS + (u32)(8 + 2 * a + (ct & 1))) * 512u;
            st_sys(t + nin * 4u, ev);
        }
        const u32x4 mx = *(LAS u32x4*)(cm + 32 + n4 * 4);
        const u32 pbase = CTRL_BYTES + par * PAR_BYTES + (u32)(ASLOTS + ct) * PANEL;
#pragma unroll
        for (int k = 0; k < 4; ++k) {
            const float s = scale_of(exp_of(mx[k]));
            const u32 nin = (u32)((strip & 3) * 32 + n4 * 4 + k);
            const u32 ro = nin * 128u + (((u32)q ^ ((nin >> 1) & 7u)) << 4);
#pragma unroll
            for (int i = 0; i < 3; ++i) {
                const int kt = kt0 + wave + 8 * i;
                const u32 off = kt < kt1 ? pbase + (u32)kt * IMG + ro : OOB;
                const f32x4 lo = {v[i][0][k], v[i][1][k], v[i][2][k], v[i][3][k]}, hi = {v[i][4][k], v[i][5][k], v[i][6][k], v[i][7][k]};
                __builtin_amdgcn_raw_buffer_store_b128(cvt8(lo, hi, s), rws, off, 0, AUX_SC1);
                WAIT_VM(W_ST - 1);
            }
        }
        WAIT_VM(0); BAR();
        if (tid == 0 || (tid == 1 && rg == 6)) st_agent(ws + OFF_WR + (u32)(strip * 8 + rg + tid) * 4u, call);
        if (wave == 0) STAMP(ws, 2);
    }
    const int xcd = b & 7, q = (b >> 3) - 4;
    if (xcd >= 4 && q < 24) {
        const int xi = q * 4 + xcd - 4, r0 = 128 + xi * 8, row = r0 < 768 ? r0 : r0 + 128;
        x_rows<4, false>(p, lds, row, par, call, tid, wave);
        x_rows<4, false>(p, lds, row + 4, par, call, tid, wave);
        if (wave == 0) STAMP(ws, 3);
    }
    const int qr = q >> 2, qc = q & 3, x4 = xcd & 3;
    const int early = (x4 >> 1) == 0 ? (qr < 6 ? a * 6 + qr : -1) : (qr < 3 ? ((a + 1 + qr) & 3) * 6 : 24 + 2 * (qr - 3));
    const int peer = ((a + 1 + (qr >> 1)) & 3) * 6;
    for (int it = 0; it < 4; ++it) {
        int slot = -1, ct = 4 * x4 + qc;
        if (it == 0) { if (xcd < 4) { slot = early; ct = 4 * (x4 & 1) + qc; } }
        else if (it == 1) { if (xcd < 4) { if (qr < 6) slot = peer + 1 + (qr & 1); } else { slot = early; ct = 8 + 4 * (x4 & 1) + qc; } }
        else if (it == 2) { if (xcd >= 4) { if (qr < 6) slot = peer + 3 + (qr & 1); } else if (qr < 4) slot = 25 + 2 * qr; }
        else if (qr < 6) { slot = peer + 5; ct = (xcd + 8 * qc) >> 1; }
        if (slot < 0) continue;
        if (it < 3) gemm_tile<true>(p, lds, slot, ct, 0, 0, par, call, tid, lane, wave, 4 + it * 3);
        else gemm_tile<false>(p, lds, slot, ct, qr & 1, xcd & 1, par, call, tid, lane, wave, 4 + it * 3);
    }
}
}

extern "C" void kernel_launch(int rank, int nranks, ncclComm_t comm,
                              hipStream_t stream,
                              void* const* d_in, const int* in_sizes, int n_in,
                              void* d_out, int out_size,
                              const int* in_dtypes, int out_dtype,
                              void* const* d_out_all,
                              void* const* d_ws_all, size_t ws_bytes) {
    (void)comm; (void)in_dtypes; (void)out_dtype; (void)d_out;
    if (nranks != 8 || (unsigned)rank >= 8u || n_in < 2 || ws_bytes < (size_t)agk::WS_NEED || in_sizes[0] != 1024 * 8192 ||
        in_sizes[1] != 8192 * 1024 || out_size != 8192 * 1024) {
        fprintf(stderr, "kernel_launch: unexpected problem shape\n");
        return;
    }
    static bool ready[8] = {false, false, false, false, false, false, false, false};
    if (!ready[rank]) {
        HIP_CHECK(hipFuncSetAttribute((const void*)agk::ag_gemm_kernel, hipFuncAttributeMaxDynamicSharedMemorySize, (int)agk::LDS_BYTES));
        ready[rank] = true;
    }
    agk::Params p;
    p.x = (const float*)d_in[0]; p.w = (const float*)d_in[1]; p.rank = rank;
    for (int i = 0; i < 8; ++i) { p.ws[i] = (char*)d_ws_all[i]; p.out[i] = (float*)d_out_all[i]; }
    hipLaunchKernelGGL(agk::ag_gemm_kernel, dim3(agk::NWG), dim3(512), agk::LDS_BYTES, stream, p);
    HIP_CHECK(hipGetLastError());
}
```

```cpp
#include <hip/hip_runtime.h>
#include <rccl/rccl.h>
#include <cstdio>

#define HIP_CHECK(expr) do { hipError_t _e = (expr); if (_e != hipSuccess) { \
    fprintf(stderr, "HIP error %s at %s:%d\n", hipGetErrorString(_e), __FILE__, __LINE__); \
    return; } } while (0)

namespace agk {
typedef unsigned u32;
typedef unsigned long long u64;
typedef u32 u32x4 __attribute__((ext_vector_type(4)));
typedef int i32x4 __attribute__((ext_vector_type(4)));
typedef float f32x4 __attribute__((ext_vector_type(4)));
typedef _Float16 f16x8 __attribute__((ext_vector_type(8)));
#define LAS __attribute__((address_space(3)))
typedef __amdgpu_buffer_rsrc_t rsrc_t;
typedef __attribute__((address_space(1))) char gchar;

constexpr int KDIM = 8192, NLOC = 1024, NKT = 128;
constexpr u32 IMG = 16384u;
constexpr u32 PANEL = NKT * IMG;
constexpr u32 UNIT = 131072u;
constexpr int ASLOTS = 34, BSLOTS = 16;
#define DELTA 3
constexpr u32 CTRL_BYTES = 1u << 20;
constexpr u32 RELAY = (ASLOTS + BSLOTS) * PANEL;
constexpr u32 PAR_BYTES = RELAY + 8u * UNIT;
constexpr u32 OFF_YST = CTRL_BYTES + 2u * PAR_BYTES;
constexpr u32 WS_NEED = OFF_YST + 4u * 32u * UNIT;
constexpr u32 IN_BYTES = 1024u * 8192u * 4u;
constexpr u32 OFF_CTR = 0;
constexpr u32 OFF_TMO = 16384;
constexpr u32 OFF_CL = 20480;
constexpr u32 OFF_LF = 524288;
constexpr u32 OFF_RD = 655360;
constexpr u32 OFF_YR = 24576;
constexpr u32 OFF_RF = 28672;
constexpr u32 OFF_WR = 32768;
constexpr u32 OFF_WS = 33792;
constexpr u32 OFF_WX = 36864;
constexpr u32 OFF_SA = 131072;
constexpr u32 OFF_SB = 196608;
constexpr u32 OFF_DBG = 262144;
constexpr int NWG = 256, NCOMM = 32;
constexpr u32 NUNITS = 96u;
constexpr u32 OOB = 0x80000000u, NOFLAG = 0x40000000u;
constexpr u64 DEADLINE = 30000000ull;
constexpr u32 STAGE = 32768u;
constexpr u32 L_SNAP = 4u * STAGE;
constexpr u32 L_SC = L_SNAP + 512u;
constexpr u32 L_RED = L_SC + 1024u;
constexpr u32 LDS_BYTES = L_RED + 2048u;

struct Params {
    const float* x; const float* w;
    char* ws[8]; float* out[8];
    int rank;
};

#define AUX_SC1 16
#define AUX_SYS 17
#define GL_AUX 16
#define W_LD 3
#define W_ST 2
#define X_LD 4
#define X_ST 2
#define WAIT_VM(n) asm volatile("s_waitcnt vmcnt(%0)" :: "n"(n) : "memory")
#define WAIT_LGKM0() asm volatile("s_waitcnt lgkmcnt(0)" ::: "memory")
#define BAR() do { asm volatile("" ::: "memory"); __builtin_amdgcn_s_barrier(); asm volatile("" ::: "memory"); } while (0)
#define WAIT_VM_TRACKED(n) do { asm volatile("" ::: "memory"); __builtin_amdgcn_s_waitcnt(((n) & 15) | (7 << 4) | (15 << 8) | (((n) >> 4) << 14)); asm volatile("" ::: "memory"); } while (0)

__device__ __forceinline__ rsrc_t mk_rsrc(const void* p, u32 bytes) { return __builtin_amdgcn_make_buffer_rsrc((void*)p, 0, bytes, 0x00020000); }
__device__ __forceinline__ u32 ld_agent(const void* p) { return __hip_atomic_load((const u32*)p, __ATOMIC_RELAXED, __HIP_MEMORY_SCOPE_AGENT); }
__device__ __forceinline__ u32 ld_sys(const void* p) { return __hip_atomic_load((const u32*)p, __ATOMIC_RELAXED, __HIP_MEMORY_SCOPE_SYSTEM); }
__device__ __forceinline__ void st_agent(void* p, u32 v) { __hip_atomic_store((u32*)p, v, __ATOMIC_RELAXED, __HIP_MEMORY_SCOPE_AGENT); }
__device__ __forceinline__ void st_sys_g(gchar* p, u32 v) { __hip_atomic_store((__attribute__((address_space(1))) u32*)p, v, __ATOMIC_RELAXED, __HIP_MEMORY_SCOPE_SYSTEM); }
__device__ __forceinline__ void st_sys(void* p, u32 v) { __hip_atomic_store((u32*)p, v, __ATOMIC_RELAXED, __HIP_MEMORY_SCOPE_SYSTEM); }

#define STAMP(ws, idx) do { } while (0)

__device__ __forceinline__ bool give_up(char* ws, u32 call, u64 t0) {
    const u32 t = __builtin_amdgcn_readfirstlane(ld_agent(ws + OFF_TMO));
    if (t == call || __builtin_amdgcn_s_memrealtime() - t0 > DEADLINE) { st_agent(ws + OFF_TMO, call); return true; }
    return false;
}
__device__ __forceinline__ void poll_ge(const char* p, u32 need, char* ws, u32 call) {
    const u64 t0 = __builtin_amdgcn_s_memrealtime();
    u32 n = 0;
    for (;;) {
        const u32 v = ld_sys(p);
        if (__all((int)(v - need) >= 0)) return;
        if ((++n & 15u) == 0u && give_up(ws, call, t0)) return;
        __builtin_amdgcn_s_sleep(2);
    }
}
__device__ __forceinline__ bool all4_ge(u32x4 v, u32 need) {
    return __all((int)(v[0] - need) >= 0 && (int)(v[1] - need) >= 0 && (int)(v[2] - need) >= 0 && (int)(v[3] - need) >= 0);
}
__device__ __forceinline__ void poll4_ge(rsrc_t rs, u32 off, u32 need, char* ws, u32 call) {
    const u64 t0 = __builtin_amdgcn_s_memrealtime();
    u32 n = 0;
    for (;;) {
        asm volatile("" ::: "memory");
        const u32x4 v = __builtin_amdgcn_raw_buffer_load_b128(rs, off, 0, AUX_SYS);
        if (all4_ge(v, need)) return;
        if ((++n & 15u) == 0u && give_up(ws, call, t0)) return;
        __builtin_amdgcn_s_sleep(1);
    }
}

__device__ __forceinline__ u32 umax(u32 a, u32 b) { return a > b ? a : b; }
__device__ __forceinline__ u32 absbits(float f) { return __float_as_uint(f) & 0x7fffffffu; }
__device__ __forceinline__ int exp_of(u32 amaxbits) { int e = (int)(amaxbits >> 23); return e < 15 ? 15 : e; }
__device__ __forceinline__ float scale_of(int e) { return __uint_as_float((u32)(268 - e) << 23); }
__device__ __forceinline__ u32x4 cvt8(f32x4 a, f32x4 b, float s) {
    f16x8 h;
    h[0] = (_Float16)(a[0] * s); h[1] = (_Float16)(a[1] * s); h[2] = (_Float16)(a[2] * s); h[3] = (_Float16)(a[3] * s);
    h[4] = (_Float16)(b[0] * s); h[5] = (_Float16)(b[1] * s); h[6] = (_Float16)(b[2] * s); h[7] = (_Float16)(b[3] * s);
    return __builtin_bit_cast(u32x4, h);
}

template <int NR, bool HEAD>
__device__ __forceinline__ gchar* x_rows(const Params& p, LAS char* lds, int row0, u32 par, u32 call, int tid, int wave) {
    const int g = p.rank >> 2, a = p.rank & 3;
    char* ws = p.ws[p.rank];
    const rsrc_t rx = mk_rsrc(p.x, IN_BYTES), rw = mk_rsrc(ws, WS_NEED);
    const u32 pi0 = (u32)((7 - wave) * 128 + (tid & 63));
    f32x4 v[NR][4];
#pragma unroll
    for (int r = 0; r < NR; ++r)
#pragma unroll
        for (int u = 0; u < 2; ++u) {
            const u32 off = (u32)(row0 + r) * (KDIM * 4u) + (pi0 + (u32)u * 64u) * 32u;
            v[r][2 * u] = __builtin_bit_cast(f32x4, __builtin_amdgcn_raw_buffer_load_b128(rx, off, 0, 0));
            v[r][2 * u + 1] = __builtin_bit_cast(f32x4, __builtin_amdgcn_raw_buffer_load_b128(rx, off + 16u, 0, 0));
            if (!HEAD) WAIT_VM(X_LD - 2);
        }
    const int rb = row0 >> 7;
    const u32 slot = rb < 6 ? (u32)(a * 6 + rb) : (u32)(32 + rb - 6);
    const u32 rslot = rb < 6 ? slot : (u32)(24 + 2 * a + rb - 6);
    const int ug = rb < 6 ? g : 1 - g;
    gchar* user = (gchar*)((tid & 4) ? p.ws[rb < 6 ? p.rank ^ 4 : p.rank]
                                     : (tid & 2) ? ((tid & 1) ? p.ws[ug * 4 + 3] : p.ws[ug * 4 + 2]) : ((tid & 1) ? p.ws[ug * 4 + 1] : p.ws[ug * 4]));
    const u32 wr = (u32)(tid & 63) >> 3;
    const bool wsend = wr < (u32)NR && ((tid & 7) < 4 || ((tid & 7) == 4 && rb >= 5));
    const u32 woff = OFF_SA + ((par * 64u + ((tid & 4) ? (rb < 6 ? 34u : slot) : rslot)) * 128u + (((u32)row0 + wr) & 127u)) * 4u;
    u32 word = 0;
    LAS u32* red = (LAS u32*)(lds + L_RED);
#pragma unroll
    for (int r = 0; r < NR; ++r) {
        u32 m = 0;
#pragma unroll
        for (int i = 0; i < 4; ++i)
#pragma unroll
            for (int j = 0; j < 4; ++j) m = umax(m, absbits(v[r][i][j]));
#pragma unroll
        for (int o = 32; o >= 1; o >>= 1) m = umax(m, (u32)__shfl_xor((int)m, o));
        if ((tid & 63) == 0) red[r * 8 + wave] = m;
    }
    asm volatile("" : "+v"(user));
    WAIT_LGKM0(); BAR();
#pragma unroll
    for (int r = 0; r < NR; ++r) {
        const u32x4 m0 = *(LAS u32x4*)(red + r * 8), m1 = *(LAS u32x4*)(red + r * 8 + 4);
        const u32 m = umax(umax(umax(m0[0], m0[1]), umax(m0[2], m0[3])), umax(umax(m1[0], m1[1]), umax(m1[2], m1[3])));
        const int e = exp_of(m); const float s = scale_of(e);
        const u32 rin = (u32)((row0 + r) & 127);
        if (wr == (u32)r) word = call | ((u32)(e - 141) & 255u);
        const u32 off = CTRL_BYTES + par * PAR_BYTES + rin * 128u + (pi0 >> 3) * IMG + (((pi0 & 7u) ^ ((rin >> 1) & 7u)) << 4);
        const u32x4 h0 = cvt8(v[r][0], v[r][1], s), h1 = cvt8(v[r][2], v[r][3], s);
        __builtin_amdgcn_raw_buffer_store_b128(h0, rw, off + slot * PANEL, 0, AUX_SC1);
        __builtin_amdgcn_raw_buffer_store_b128(h1, rw, off + slot * PANEL + 8u * IMG, 0, AUX_SC1);
        if (!HEAD) WAIT_VM(X_ST - 2);
        if (HEAD) {
#define HROUND(k) do { _Pragma("unroll") for (int i = 0; i < 4; ++i) { const int d = ug * 4 + ((a + 1 + i) & 3); \
                __builtin_amdgcn_raw_buffer_store_b128((k) < 2 ? h0 : h1, mk_rsrc(p.ws[d], WS_NEED), \
                    (d == p.rank || ((tid >> 5) & 1) != ((k) & 1)) ? OOB : off + rslot * PANEL + ((k) < 2 ? 0u : 8u * IMG), 0, AUX_SYS); } } while (0)
            if (wave == 7) { HROUND(0); WAIT_VM(4); } else WAIT_VM(0);
            BAR();
            if (wave == 7) {
                if (wsend) st_sys_g(user + woff, word);
                WAIT_VM(1); HROUND(1); WAIT_VM(0); HROUND(2); WAIT_VM(0); HROUND(3);
            }
#undef HROUND
        }
    }
    if (!HEAD) {
        WAIT_VM(0); BAR();
        if (wave == 0 && wsend) st_sys_g(user + woff, word);
    }
    return user + OFF_RD + ((u32)p.rank * 128u + (u32)(row0 & 127)) * 4u;
}

__device__ __forceinline__ int slot_row(int rank, int slot) {
    const int g = rank >> 2;
    if (slot < 24) { const int i = slot / 6; return (g * 4 + i) * 1024 + (slot - i * 6) * 128; }
    return (slot < 32 ? (1 - g) * 4 + ((slot - 24) >> 1) : rank) * 1024 + 768 + (slot & 1) * 128;
}

template <bool BIG>
__device__ __forceinline__ void gemm_tile(const Params& p, LAS char* lds, int slot, int ct, int rh, int ch, u32 par, u32 call, int tid, int lane, int wave, int stamp0) {
    constexpr int NB = BIG ? 4 : 2, NC = BIG ? 32 : 16, BT = BIG ? 128 : 64;
    const int rank = p.rank, g = rank >> 2, a = rank & 3;
    char* ws = p.ws[rank];
    const rsrc_t rws = mk_rsrc(ws, WS_NEED);
    int a_src; u32 a_base, a_sh;
    if (slot < 24) { const int i = slot / 6, a_rb = slot - i * 6; a_src = i == a ? -1 : g * 4 + i;
        a_base = a_rb == 0 ? 0u : a_rb < 3 ? 16u : a_rb < 5 ? 48u : 80u + (u32)rh * 8u; a_sh = a_rb == 0 ? 1u : 0u; }
    else { a_src = slot < 32 ? (1 - g) * 4 + ((slot - 24) >> 1) : -1; a_base = (slot & 1) ? 48u : 0u; a_sh = 1u; }
    const int b_src = ct < 8 ? -1 : (1 - g) * 4 + ((ct - 8) >> 1);
    const int src = lane < 32 ? a_src : (BIG ? b_src : -1);
    const u32 gsh = src < 0 ? 31u : lane < 32 && BIG ? a_sh : 0u;
    const char* gptr = ws + (src >= 0 ? OFF_LF + ((u32)src * NUNITS + (lane < 32 ? a_base : 16u)) * 128u + (u32)(lane & (BIG ? 31 : 15)) * 4u : OFF_CTR + blockIdx.x * 64u + 4u);
    const u32 gmin = src >= 0 && lane < 32 && a_base == 0u ? 2u : 0u;
#define GPTR(c) (gptr + (umax((u32)(c) >> gsh, gmin) << (BIG ? 7 : 6)))
    if (wave == 0) {
        if (a_src < 0 || (!BIG && rh == 0)) poll4_ge(rws, OFF_SA + (par * 64u + (u32)slot) * 512u + (u32)(lane & (BIG ? 31 : 15)) * 16u, call, ws, call);
        if (b_src < 0) poll4_ge(rws, OFF_WR + (u32)ct * 128u + (u32)(lane & 7) * 16u, call, ws, call);
        else if (!BIG) for (u32 i = 0; i < 4u; ++i) poll4_ge(rws, OFF_LF + ((u32)b_src * NUNITS + 16u) * 128u + i * 1024u + (u32)lane * 16u, call, ws, call);
        if (a_src >= 0 && a_base == 0u) poll4_ge(rws, OFF_RD + (u32)a_src * 512u + (u32)(lane & 31) * 16u, call, ws, call);
        poll_ge(GPTR(0), call, ws, call);
        WAIT_VM(0);
    }
    BAR();
    if (wave == 0) STAMP(ws, stamp0);
    const u32 pb = CTRL_BYTES + par * PAR_BYTES;
    const char* gA = ws + pb + (u32)slot * PANEL + (BIG ? (u32)wave * 2048u : (u32)rh * 8192u + (u32)wave * 1024u) + (u32)lane * 16u;
    const char* gB = ws + pb + (u32)(ASLOTS + ct) * PANEL + (BIG ? (u32)wave * 2048u : (u32)ch * 8192u + (u32)wave * 1024u) + (u32)lane * 16u;
    const u32 ldsw = (u32)wave * (BIG ? 2048u : 1024u);
    constexpr u32 GSTEP = BIG ? IMG : 2u * IMG, G2 = BIG ? 1024u : IMG, L2O = BIG ? 1024u : 8192u;
#define GLDS16(g, l) __builtin_amdgcn_global_load_lds((const u32*)(g), (LAS u32*)(lds + (l)), 16, 0, GL_AUX)
#define LOADT(t, st) do { const u32 _o = (u32)(t) * GSTEP; \
        GLDS16(gA + _o, (st) * STAGE + ldsw); GLDS16(gA + _o + G2, (st) * STAGE + ldsw + L2O); \
        GLDS16(gB + _o, (st) * STAGE + IMG + ldsw); GLDS16(gB + _o + G2, (st) * STAGE + IMG + ldsw + L2O); } while (0)
#define SNAP(c) __builtin_amdgcn_global_load_lds((const u32*)GPTR(c), (LAS u32*)(lds + L_SNAP + ((u32)(c) & 1u) * 256u), 4, 0, AUX_SYS)
    const int ks = BIG ? wave >> 2 : wave >> 1, wm = BIG ? (wave >> 1) & 1 : 0, wn = wave & 1;
    const u32 swz = (u32)((((ks & 1) * 4 + (lane >> 4)) ^ ((lane & 15) >> 1)) << 4) + (BIG ? 0u : (u32)(ks >> 1) * 8192u);
    const u32 ard = (u32)(wm * 64 + (lane & 15)) * 128u + swz, brd = IMG + (u32)(wn * (BT / 2) + (lane & 15)) * 128u + swz;
#define READF(fa, fb, st) do { _Pragma("unroll") for (int _i = 0; _i < 4; ++_i) { \
        if (_i < NB) fb[_i] = *(const LAS f16x8*)(lds + (st) * STAGE + brd + _i * 2048u); \
        fa[_i] = *(const LAS f16x8*)(lds + (st) * STAGE + ard + _i * 2048u); } } while (0)
#define MMA(fa, fb) do { _Pragma("unroll") for (int _m = 0; _m < 4; ++_m) _Pragma("unroll") for (int _n = 0; _n < NB; ++_n) \
        acc[_m][_n] = __builtin_amdgcn_mfma_f32_16x16x32_f16(fb[_n], fa[_m], acc[_m][_n], 0, 0, 0); } while (0)
    f32x4 acc[4][NB];
#pragma unroll
    for (int m = 0; m < 4; ++m)
#pragma unroll
        for (int n = 0; n < NB; ++n) acc[m][n] = (f32x4){0.f, 0.f, 0.f, 0.f};
    f16x8 a0[4], b0[4], a1[4], b1[4];
    if (wave >= 4) {
        const u32 so = (wave < 6 ? OFF_SA + (par * 64u + (u32)slot) * 512u : OFF_SB + (par * BSLOTS + (u32)ct) * 512u) + (u32)((wave & 1) * 64 + lane) * 4u;
        __builtin_amdgcn_global_load_lds((const u32*)(ws + so), (LAS u32*)(lds + L_SC + (u32)(wave - 4) * 256u), 4, 0, AUX_SYS);
    }
    LOADT(0, 0u);
    if (wave == 0) SNAP(1);
    LOADT(1, 1u); LOADT(2, 2u); LOADT(3, 3u);
    if (wave == 0) WAIT_VM(13); else WAIT_VM(12);
    BAR();
    READF(a0, b0, 0u);
#define GATE(c) do { const u32 _v = *(const LAS u32*)(lds + L_SNAP + ((u32)(c) & 1u) * 256u + (u32)lane * 4u); \
        if (!__all((int)(_v - call) >= 0)) { if (wave == 0) { poll_ge(GPTR(c), call, ws, call); WAIT_VM(0); } BAR(); } } while (0)
    for (int c = 0; c < NC - 1; ++c) {
        const int t = 4 * c;
        WAIT_VM(8); WAIT_LGKM0(); BAR();
        GATE(c + 1);
        LOADT(t + 4, 0u); READF(a1, b1, 1u); MMA(a0, b0);
        WAIT_VM(8); WAIT_LGKM0(); BAR();
        if (wave == 0) SNAP(c + 2 < NC ? c + 2 : c);
        LOADT(t + 5, 1u); READF(a0, b0, 2u); MMA(a1, b1);
        if (wave == 0) WAIT_VM(9); else WAIT_VM(8);
        WAIT_LGKM0(); BAR();
        LOADT(t + 6, 2u); READF(a1, b1, 3u); MMA(a0, b0);
        if (wave == 0) WAIT_VM(9); else WAIT_VM(8);
        WAIT_LGKM0(); BAR();
        LOADT(t + 7, 3u); READF(a0, b0, 0u); MMA(a1, b1);
    }
    WAIT_VM(8); WAIT_LGKM0(); BAR();
    READF(a1, b1, 1u); MMA(a0, b0);
    WAIT_VM(4); WAIT_LGKM0(); BAR();
    READF(a0, b0, 2u); MMA(a1, b1);
    WAIT_VM(0); WAIT_LGKM0(); BAR();
    READF(a1, b1, 3u); MMA(a0, b0);
    const u32 tmo = ld_agent(ws + OFF_TMO);
    WAIT_LGKM0(); BAR();
    MMA(a1, b1);
    if (wave == 0) STAMP(ws, stamp0 + 1);
    constexpr u32 RB = BT * 4u, PPR = BT / 4u, AREA = BT * RB;
#pragma unroll
    for (int m = 0; m < 4; ++m)
#pragma unroll
        for (int n = 0; n < NB; ++n) {
            const u32 r = (u32)(wm * 64 + m * 16 + (lane & 15)), pc = (u32)(wn * (BT / 8) + n * 4 + (lane >> 4));
            *(LAS f32x4*)(lds + (u32)ks * AREA + r * RB + ((pc ^ (r & 7u)) << 4)) = acc[m][n];
        }
    WAIT_LGKM0(); BAR();
    const LAS int* sc = (const LAS int*)(lds + L_SC);
    const u32 pc = (u32)lane & (PPR - 1u);
    const u32 col0 = BIG ? 0u : (u32)ch * 64u, row0 = BIG ? 0u : (u32)rh * 64u;
    const i32x4 eb = *(const LAS i32x4*)(sc + 128 + col0 + pc * 4u);
    const bool bad = tmo == call;
    const bool remote = ct >= 8;
    const rsrc_t ro = mk_rsrc(remote ? (const void*)ws : (const void*)p.out[rank], remote ? WS_NEED : IN_BYTES);
    const u32 obase = (remote ? OFF_YST + (u32)(((ct - 8) >> 1) * 32 + slot) * UNIT + (u32)(ct & 1) * 512u : ((u32)slot_row(rank, slot) * NLOC + (u32)ct * 128u) * 4u) + col0 * 4u;
    const u32 ostr = remote ? 1024u : NLOC * 4u;
#pragma unroll
    for (u32 it = 0; it < (BIG ? 8u : 2u); ++it) {
        const u32 r = (u32)wave * (BT / 8u) + (it * 64u + (u32)lane) / PPR;
        const u32 lo = r * RB + ((pc ^ (r & 7u)) << 4);
        f32x4 s = *(const LAS f32x4*)(lds + lo) + *(const LAS f32x4*)(lds + AREA + lo);
        if (!BIG) s += *(const LAS f32x4*)(lds + 2u * AREA + lo) + *(const LAS f32x4*)(lds + 3u * AREA + lo);
        const int ea = (int)(signed char)sc[row0 + r];
#pragma unroll
        for (int j = 0; j < 4; ++j) s[j] = bad ? __builtin_nanf("") : __builtin_amdgcn_ldexpf(s[j], ea + eb[j]);
        __builtin_amdgcn_raw_buffer_store_b128(__builtin_bit_cast(u32x4, s), ro, obase + (row0 + r) * ostr + pc * 16u, 0, AUX_SC1);
    }
    if (remote) {
        WAIT_VM(0); BAR();
        if (tid < (BIG ? 2 : 1)) st_sys(p.ws[b_src] + OFF_YR + (u32)(a * 64 + rh * 32 + slot) * 16u + (u32)(ct & 1) * 8u + (u32)(tid + ch) * 4u, call);
    }
    if (wave == 0) STAMP(ws, stamp0 + 2);
#undef GLDS16
#undef LOADT
#undef SNAP
#undef READF
#undef MMA
#undef GATE
#undef GPTR
}

constexpr u32 NPOS_INTRA = 94u - DELTA, NPOS_CROSS = 95u + DELTA;
constexpr u32 FAN = 0x20000000u;
struct Desc { u32 src, dst, gate, flag; };
struct Stream {
    bool cross; int rank, g, a, j; u32 pb, sa;
    __device__ __forceinline__ int yslot(u32 k) const {
        if (k < 6u) return j * 6 + (int)k;
        if (k < 9u) return ((j + (int)k - 5) & 3) * 6;
        if (k < 12u) return 24 + 2 * ((a + (int)k - 8) & 3);
        if (k < 18u) return ((j + 1 + (((int)k - 12) >> 1)) & 3) * 6 + 1 + ((int)k & 1);
        if (k < 21u) return 25 + 2 * ((a + (int)k - 17) & 3);
        if (k < 27u) return ((j + 1 + (((int)k - 21) >> 1)) & 3) * 6 + 3 + (((int)k - 21) & 1);
        return ((j + 1 + (int)(k % 3u)) & 3) * 6 + 5;
    }
    __device__ __forceinline__ u32 single(u32 slot, u32 k) const { return pb + slot * PANEL + k * UNIT; }
    __device__ __forceinline__ u32 pair(u32 slot, u32 k) const { return pb + slot * PANEL + k * 65536u; }
    __device__ __forceinline__ u32 half(u32 slot, u32 h, u32 k) const { return pb + slot * PANEL + k * 2u * UNIT + h * 8192u; }
    __device__ __forceinline__ Desc at(u32 pos) const {
        Desc d;
        const u32 s = (u32)j * 6u, M8 = 7u << 20, PAIR = 1u << 28, HALF = 4u << 28;
        u32 u, fsrc = (u32)((cross ? 1 - g : g) * 4 + j);
        if (!cross) {
            if (pos >= NPOS_INTRA) pos = NPOS_INTRA - 1u;
            if (pos < 78u) {
                u = pos + 2u;
                d.src = u < 16u ? single(s, u) : u < 48u ? pair(s + 1u, u - 16u) : pair(s + 3u, u - 48u);
                d.gate = sa + s * 512u + M8 + (u < 16u ? 0u : u < 48u ? 512u + PAIR : 1536u + PAIR);
            } else {
                const u32 e = NPOS_INTRA - 1u - pos, n0 = 8u - DELTA;
                const u32 h = e < 2u || e - 2u >= 2u * n0 || (e & 1u) ? 1u : 0u;
                const u32 k = e < 2u ? 7u - e : e - 2u >= 2u * n0 ? 5u - n0 - (e - 2u - 2u * n0) : h ? 5u - ((e - 2u) >> 1) : 7u - ((e - 2u) >> 1);
                u = 80u + h * 8u + k;
                d.src = half(s + 5u, h, k);
                d.gate = sa + (s + 5u) * 512u + h * 256u + M8 + HALF;
            }
            d.dst = d.src;
            d.flag = OFF_LF + (fsrc * NUNITS + u) * 128u;
        } else {
            if (pos >= NPOS_CROSS) pos = NPOS_CROSS - 1u;
            const u32 hop = j == a ? 62u : 66u + DELTA;
            if (pos < 62u) {
                u = pos + 2u;
                d.src = u < 16u ? single(32u, u) : u < 48u ? pair((u32)(ASLOTS + 2 * a), u - 16u) : single(33u, u - 48u);
                d.dst = u < 16u ? single((u32)(24 + 2 * j), u) : u < 48u ? pair((u32)(ASLOTS + 8 + 2 * j), u - 16u) : single((u32)(25 + 2 * j), u - 48u);
                d.gate = u < 16u ? sa + (u32)(24 + 2 * j) * 512u + M8 : u < 48u ? OFF_WS + (u32)j * 256u + (1u << 20) + (3u << 28) : sa + (u32)(25 + 2 * j) * 512u + M8;
                d.flag = OFF_LF + (fsrc * NUNITS + u) * 128u;
            } else if (pos - hop < DELTA) {
                const u32 k = pos - hop;
                if (j == a) { d.src = half(s + 5u, 0u, k); d.dst = pb + RELAY + k * 2u * UNIT; d.gate = sa + 34u * 512u + M8 + HALF;
                              d.flag = FAN + OFF_RF + ((u32)a * 4u + k) * 128u; }
                else { d.src = pb + RELAY + k * 2u * UNIT; d.dst = half(s + 5u, 0u, k); d.gate = OFF_RF + ((u32)j * 4u + k) * 128u + (6u << 28);
                       d.flag = OFF_LF + ((fsrc ^ 4u) * NUNITS + 80u + k) * 128u; }
            } else {
                const u32 k = pos < hop ? pos - 62u : pos - 62u - DELTA;
                const int ys = yslot(k);
                const u32 h = k >= 30u ? 1u : 0u;
                d.src = OFF_YST + (u32)(a * 32 + ys) * UNIT + h * 65536u;
                d.dst = ((u32)slot_row((int)fsrc, ys) + h * 64u) * (NLOC * 4u) + (u32)j * 1024u;
                d.gate = OFF_YR + ((u32)(j * 64 + ys) + h * 32u) * 16u + ((k >= 27u ? 10u : 2u) << 28);
                d.flag = NOFLAG;
            }
        }
        return d;
    }
};
struct Tables { Desc lo, hi; };
__device__ __forceinline__ u32 tab(u32 t0, u32 t1, u32 i) {
    const u32 x = (u32)__builtin_amdgcn_readlane((int)t0, (int)(i & 63u)), y = (u32)__builtin_amdgcn_readlane((int)t1, (int)(i & 63u));
    return i < 64u ? x : y;
}

#define NACT 32
__device__ __forceinline__ void pull_stream(const Params& p, const Tables& T, bool cross, u32 cw, int srcrank, u32 calln, u32 call, int lane, int wave) {
    char* ws = p.ws[p.rank];
    const rsrc_t rl = mk_rsrc(ws, WS_NEED), rs = mk_rsrc(p.ws[srcrank], WS_NEED), rout = mk_rsrc(p.out[p.rank], IN_BYTES);
    const int og = (p.rank ^ 4) & 4, a = p.rank & 3;
    gchar* fan = (gchar*)(lane == 0 ? p.ws[og + ((a + 1) & 3)] : lane == 1 ? p.ws[og + ((a + 2) & 3)] : p.ws[og + ((a + 3) & 3)]);
    const u32 l16 = (u32)lane * 16u;
    const u32 l0 = lane == 0 ? 0u : OOB;
    const u32 co = OFF_CL + (u32)wave * 128u + l0;
    const u32 NPIECES = (cross ? NPOS_CROSS : NPOS_INTRA) * 32u;
    const u32 cbase = 2u * NACT - (calln - 1u) * NPIECES;
    u32x4 v0 = {0u, 0u, 0u, 0u}, v1 = v0, v2, v3;
    u32 q = OOB, so = OOB;
    u32 c0 = cw, c1 = cw + NACT;
    u32 pf = NOFLAG;
    bool claimed = false, dead = false;
    int cl = 0; u32 idle = 0; u64 t0 = 0;
#define KIND(q) (tab(T.lo.gate, T.hi.gate, (q) >> 5) >> 28)
#define POFF(q, kd) (((kd) & 1u) ? (((q) >> 4) & 1u) * PANEL + ((q) & 15u) * 4096u : ((kd) & 4u) ? (((q) >> 1) & 15u) * IMG + ((q) & 1u) * 4096u : ((q) & 31u) * (((kd) & 8u) ? 2048u : 4096u))
#define GOFF(q) ({ const u32 _g = tab(T.lo.gate, T.hi.gate, (q) >> 5), _k = _g >> 28; \
        const u32 _ix = (((_k & 1u) ? ((q) >> 2) & 4u : 0u) + ((q) & (_k == 4u ? 1u : 3u))) << (_k == 3u ? 5 : 7); \
        (_g & 0xfffffu) + ((_k & 7u) == 2u ? 0u : _k == 6u ? ((q) & 28u) * 4u : _ix) + (l16 & (_g >> 16)); })
#define PFLAG() st_sys_g((pf & FAN) ? fan + (pf - FAN) : (gchar*)ws + (pf >= NOFLAG ? OFF_CTR + cw * 64u + 8u + (u32)wave * 4u : pf), call)
#define PLD(u, o) v##u = __builtin_amdgcn_raw_buffer_load_b128(rs, (o) + (u) * 1024u, 0, AUX_SYS)
#define PST(u) do { asm volatile("" : "+v"(v##u)); \
        __builtin_amdgcn_raw_buffer_store_b128(v##u | (dead ? 0x7fc00000u : 0u), rd, d + (u) * ds, 0, AUX_SC1); } while (0)
#define PSTAMP(i) do { } while (0)
    u32x4 gv = __builtin_amdgcn_raw_buffer_load_b128(rl, GOFF(c0), 0, AUX_SYS);
    do {
        const u32 qq = q != OOB ? q : 0u;
        const u32 kd = KIND(qq); const bool y = (kd & 7u) == 2u;
        u32 d = q != OOB ? tab(T.lo.dst, T.hi.dst, qq >> 5) + (y ? POFF(qq, kd) * 4u : POFF(qq, kd)) + l16 : OOB;
        const u32 ds = y ? NLOC * 4u : 1024u;
        const rsrc_t rd = y ? rout : rl;
        if (kd & 8u) so = OOB;
        WAIT_VM_TRACKED(5); PLD(2, so); PST(0);
        WAIT_VM_TRACKED(5); PLD(3, so); PST(1);
        WAIT_VM_TRACKED(3);
        if (kd & 8u) d = OOB;
        if (q != OOB) idle = 0u;
        else {
            WAIT_VM_TRACKED(0);
            if (idle == 0u) t0 = __builtin_amdgcn_s_memrealtime();
            if ((++idle & 15u) == 0u && give_up(ws, call, t0)) dead = true;
        }
        PFLAG();
        pf = q != OOB ? tab(T.lo.flag, T.hi.flag, qq >> 5) + (qq & 31u) * 4u : NOFLAG;
        if (claimed) { c1 = (u32)__builtin_amdgcn_readfirstlane(cl) + cbase; if (c1 > NPIECES + 127u) c1 = NPIECES + 127u; }
        claimed = (all4_ge(gv, call) || dead) && c0 < NPIECES;
        q = claimed ? c0 : OOB; if (claimed) c0 = c1;
        so = claimed ? tab(T.lo.src, T.hi.src, q >> 5) + POFF(q, KIND(q)) + l16 : OOB;
        PLD(0, so); PST(2);
        WAIT_VM_TRACKED(4); PLD(1, so); PST(3);
        cl = __builtin_amdgcn_raw_ptr_buffer_atomic_add_i32(1, rl, claimed ? co : OOB, 0, AUX_SC1);
        gv = __builtin_amdgcn_raw_buffer_load_b128(rl, GOFF(c0), 0, AUX_SYS);
        if (claimed) PSTAMP(q >> 5);
    } while (c0 < NPIECES || q != OOB);
    WAIT_VM_TRACKED(0); PFLAG();
    STAMP(ws, 2 + wave * 8 + 7);
#undef PSTAMP
#undef GOFF
#undef POFF
#undef KIND
#undef PST
#undef PLD
#undef PFLAG
}

__global__ void __launch_bounds__(512, 2) ag_gemm_kernel(Params p) {
    extern __shared__ __attribute__((aligned(16))) char smem[];
    LAS char* lds = (LAS char*)smem;
    const int tid = threadIdx.x, lane = tid & 63, wave = __builtin_amdgcn_readfirstlane(tid >> 6);
    const int b = blockIdx.x, rank = p.rank, g = rank >> 2, a = rank & 3;
    char* ws = p.ws[rank];
    if (wave == 0) STAMP(ws, 0);
    const u32 calln = (u32)__builtin_amdgcn_readfirstlane(ld_agent(ws + OFF_CTR + (u32)b * 64u)) + 1u;
    const u32 call = calln << 8, par = calln & 1u;
    const rsrc_t rws = mk_rsrc(ws, WS_NEED);

    gchar* rdflag = x_rows<1, true>(p, lds, b < 128 ? b : 640 + b, par, call, tid, wave);
    if (tid < 2) st_agent(ws + OFF_CTR + (u32)b * 64u + (u32)tid * 4u, tid ? call : calln);
    if (wave == 0) STAMP(ws, 1);
    if (b < NCOMM) {
        if (wave < 7) {
            if (b >= NACT) return;
            Stream S;
            S.cross = wave >= 3; S.rank = rank; S.g = g; S.a = a; S.j = (a + wave + 1) & 3; S.pb = CTRL_BYTES + par * PAR_BYTES; S.sa = OFF_SA + par * 32768u;
            Tables T;
            T.lo = S.at((u32)lane); T.hi = S.at((u32)lane + 64u);
            pull_stream(p, T, S.cross, (u32)b, (wave < 3 ? g : 1 - g) * 4 + S.j, calln, call, lane, wave);
        } else {
            WAIT_VM(0);
            if (lane < 4) st_sys_g(rdflag, call);
        }
        return;
    }

    {
        const int c = b - NCOMM, strip = c / 7, rg = c - strip * 7;
        const int kt0 = (rg * 128) / 7, kt1 = ((rg + 1) * 128) / 7;
        const rsrc_t rw = mk_rsrc(p.w, IN_BYTES);
        const int q = lane & 7, n4 = lane >> 3;
        if (wave == 7) { WAIT_VM(0); if (lane < 4) st_sys_g(rdflag, call); }
        f32x4 v[3][8];
#pragma unroll
        for (int i = 0; i < 3; ++i) {
            const int kt = kt0 + wave + 8 * i;
            const u32 base = kt < kt1 ? ((u32)(kt * 64 + q * 8) * NLOC + (u32)(strip * 32 + n4 * 4)) * 4u : OOB;
#pragma unroll
            for (int jj = 0; jj < 8; ++jj) {
                v[i][jj] = __builtin_bit_cast(f32x4, __builtin_amdgcn_raw_buffer_load_b128(rw, base + (u32)jj * (NLOC * 4u), 0, 0));
                WAIT_VM(W_LD - 1);
            }
        }
        LAS u32* cm = (LAS u32*)(lds + L_RED + 256u);
        if (tid < 64) cm[tid] = 0u;
        WAIT_LGKM0(); BAR();
        u32 m[4] = {0u, 0u, 0u, 0u};
#pragma unroll
        for (int i = 0; i < 3; ++i)
#pragma unroll
            for (int jj = 0; jj < 8; ++jj)
#pragma unroll
                for (int k = 0; k < 4; ++k) m[k] = umax(m[k], absbits(v[i][jj][k]));
#pragma unroll
        for (int k = 0; k < 4; ++k) {
            m[k] = umax(m[k], (u32)__shfl_xor((int)m[k], 1)); m[k] = umax(m[k], (u32)__shfl_xor((int)m[k], 2)); m[k] = umax(m[k], (u32)__shfl_xor((int)m[k], 4));
        }
        if (q < 4) __hip_atomic_fetch_max(cm + n4 * 4 + q, q == 0 ? m[0] : q == 1 ? m[1] : q == 2 ? m[2] : m[3], __ATOMIC_RELAXED, __HIP_MEMORY_SCOPE_WORKGROUP);
        WAIT_LGKM0(); BAR();
        u64* wx = (u64*)(ws + OFF_WX) + (u32)strip * 256u;
        if (tid < 32) __hip_atomic_store(wx + rg * 32 + tid, ((u64)call << 32) | cm[tid], __ATOMIC_RELAXED, __HIP_MEMORY_SCOPE_AGENT);
        if (wave < 4) {
            const int gi = tid < 224 ? tid : 0;
            const u64 t0 = __builtin_amdgcn_s_memrealtime();
            u64 gv; u32 n = 0;
            for (;;) {
                gv = __hip_atomic_load(wx + gi, __ATOMIC_RELAXED, __HIP_MEMORY_SCOPE_AGENT);
                if (__all((u32)(gv >> 32) == call)) break;
                if ((++n & 15u) == 0u && give_up(ws, call, t0)) break;
                __builtin_amdgcn_s_sleep(1);
            }
            __hip_atomic_fetch_max(cm + 32 + (gi & 31), (u32)gv, __ATOMIC_RELAXED, __HIP_MEMORY_SCOPE_WORKGROUP);
        }
        WAIT_LGKM0(); BAR();
        const int ct = strip >> 2;
        if (rg == 0 && tid < 64) {
            const u32 nin = (u32)((strip & 3) * 32 + (tid & 31));
            const u32 ev = (u32)(exp_of(cm[32 + (tid & 31)]) - 141);
            char* t = tid < 32 ? ws + OFF_SB + (par * BSLOTS + (u32)ct) * 512u : p.ws[(1 - g) * 4 + (strip >> 3)] + OFF_SB + (par * BSLOTS + (u32)(8 + 2 * a + (ct & 1))) * 512u;
            st_sys(t + nin * 4u, ev);
        }
        const u32x4 mx = *(LAS u32x4*)(cm + 32 + n4 * 4);
        const u32 pbase = CTRL_BYTES + par * PAR_BYTES + (u32)(ASLOTS + ct) * PANEL;
#pragma unroll
        for (int k = 0; k < 4; ++k) {
            const float s = scale_of(exp_of(mx[k]));
            const u32 nin = (u32)((strip & 3) * 32 + n4 * 4 + k);
            const u32 ro = nin * 128u + (((u32)q ^ ((nin >> 1) & 7u)) << 4);
#pragma unroll
            for (int i = 0; i < 3; ++i) {
                const int kt = kt0 + wave + 8 * i;
                const u32 off = kt < kt1 ? pbase + (u32)kt * IMG + ro : OOB;
                const f32x4 lo = {v[i][0][k], v[i][1][k], v[i][2][k], v[i][3][k]}, hi = {v[i][4][k], v[i][5][k], v[i][6][k], v[i][7][k]};
                __builtin_amdgcn_raw_buffer_store_b128(cvt8(lo, hi, s), rws, off, 0, AUX_SC1);
                WAIT_VM(W_ST - 1);
            }
        }
        WAIT_VM(0); BAR();
        if (tid < 4 && (!(tid & 1) || rg == 6))
            st_sys((tid < 2 ? ws + OFF_WR + (u32)strip * 32u : p.ws[(1 - g) * 4 + (strip >> 3)] + OFF_WS + (u32)(a * 8 + (strip & 7)) * 32u) + (u32)(rg + (tid & 1)) * 4u, call);
        if (wave == 0) STAMP(ws, 2);
    }
    const int xcd = b & 7, q = (b >> 3) - 4;
    if (xcd >= 4 && q < 24) {
        const int r0 = 128 + (q * 4 + xcd - 4) * 4;
        x_rows<4, false>(p, lds, r0, par, call, tid, wave);
        x_rows<4, false>(p, lds, r0 + (r0 < 384 ? 384 : 512), par, call, tid, wave);
        if (wave == 0) STAMP(ws, 3);
    }
    const int qr = q >> 2, qc = q & 3, x4 = xcd & 3;
    const int early = (x4 >> 1) == 0 ? (qr < 6 ? a * 6 + qr : -1) : (qr < 3 ? ((a + 1 + qr) & 3) * 6 : 24 + 2 * (qr - 3));
    const int peer = ((a + 1 + (qr >> 1)) & 3) * 6;
    for (int it = 0; it < 4; ++it) {
        int slot = -1, ct = 4 * x4 + qc, rh = 0, ch = 0;
        if (it == 0) { if (xcd < 4) { slot = early; ct = 4 * (x4 & 1) + qc; } }
        else if (it == 1) { if (xcd < 4) { if (qr < 6) slot = peer + 1 + (qr & 1); } else { slot = early; ct = 8 + 4 * (x4 & 1) + qc; } }
        else if (it == 2) {
            if (qr == 6) { slot = 25 + (xcd & 6); ct = 8 + 4 * (xcd & 1) + qc; }
            else if (xcd >= 4) slot = peer + 3 + (qr & 1);
            else if (xcd < 2 && qr < 4) slot = 25 + 2 * qr;
        } else {
            int pi = -1; ch = qc & 1;
            if (xcd < 2) { pi = qr - 4; rh = 1; ct = 8 + 2 * xcd + (qc >> 1); }
            else if (qr == 6) { pi = (xcd - 2) >> 1; rh = 1; ct = 12 + 2 * (xcd & 1) + (qc >> 1); }
            else if (xcd < 4) { pi = qr >> 1; ct = 4 * xcd + 2 * (qr & 1) + (qc >> 1); }
            else { pi = qr >> 1; rh = qr & 1; ct = 2 * x4 + (qc >> 1); }
            if (pi >= 0) slot = ((a + 1 + pi) & 3) * 6 + 5;
        }
        if (slot < 0) continue;
        if (slot >= 24 && ct >= 8 && (slot - 24) >> 1 == (ct - 8) >> 1) { slot = 32 + (slot & 1); ct -= 8; }
        if (it < 3) gemm_tile<true>(p, lds, slot, ct, 0, 0, par, call, tid, lane, wave, 4 + it * 3);
        else gemm_tile<false>(p, lds, slot, ct, rh, ch, par, call, tid, lane, wave, 4 + it * 3);
    }
}
}

extern "C" void kernel_launch(int rank, int nranks, ncclComm_t comm,
                              hipStream_t stream,
                              void* const* d_in, const int* in_sizes, int n_in,
                              void* d_out, int out_size,
                              const int* in_dtypes, int out_dtype,
                              void* const* d_out_all,
                              void* const* d_ws_all, size_t ws_bytes) {
    (void)comm; (void)in_dtypes; (void)out_dtype; (void)d_out;
    if (nranks != 8 || (unsigned)rank >= 8u || n_in < 2 || ws_bytes < (size_t)agk::WS_NEED || in_sizes[0] != 1024 * 8192 ||
        in_sizes[1] != 8192 * 1024 || out_size != 8192 * 1024) {
        fprintf(stderr, "kernel_launch: unexpected problem shape\n");
        return;
    }
    static bool ready[8] = {false, false, false, false, false, false, false, false};
    if (!ready[rank]) {
        HIP_CHECK(hipFuncSetAttribute((const void*)agk::ag_gemm_kernel, hipFuncAttributeMaxDynamicSharedMemorySize, (int)agk::LDS_BYTES));
        ready[rank] = true;
    }
    agk::Params p;
    p.x = (const float*)d_in[0]; p.w = (const float*)d_in[1]; p.rank = rank;
    for (int i = 0; i < 8; ++i) { p.ws[i] = (char*)d_ws_all[i]; p.out[i] = (float*)d_out_all[i]; }
    hipLaunchKernelGGL(agk::ag_gemm_kernel, dim3(agk::NWG), dim3(512), agk::LDS_BYTES, stream, p);
    HIP_CHECK(hipGetLastError());
}
```

```cpp
#include <hip/hip_runtime.h>
#include <rccl/rccl.h>
#include <cstdio>

#define HIP_CHECK(expr) do { hipError_t _e = (expr); if (_e != hipSuccess) { \
    fprintf(stderr, "HIP error %s at %s:%d\n", hipGetErrorString(_e), __FILE__, __LINE__); \
    return; } } while (0)

namespace agk {
typedef unsigned u32;
typedef unsigned long long u64;
typedef u32 u32x4 __attribute__((ext_vector_type(4)));
typedef int i32x4 __attribute__((ext_vector_type(4)));
typedef float f32x4 __attribute__((ext_vector_type(4)));
typedef _Float16 f16x8 __attribute__((ext_vector_type(8)));
#define LAS __attribute__((address_space(3)))
typedef __amdgpu_buffer_rsrc_t rsrc_t;
typedef __attribute__((address_space(1))) char gchar;

constexpr int KDIM = 8192, NLOC = 1024, NKT = 128;
constexpr u32 IMG = 16384u;
constexpr u32 PANEL = NKT * IMG;
constexpr u32 UNIT = 131072u;
constexpr int ASLOTS = 34, BSLOTS = 16;
#define HU 2
#define HW 2
#define DELTA 3
constexpr u32 CTRL_BYTES = 1u << 20;
constexpr u32 RELAY = (ASLOTS + BSLOTS) * PANEL;
constexpr u32 PAR_BYTES = RELAY + 8u * UNIT;
constexpr u32 OFF_YST = CTRL_BYTES + 2u * PAR_BYTES;
constexpr u32 WS_NEED = OFF_YST + 4u * 32u * UNIT;
constexpr u32 IN_BYTES = 1024u * 8192u * 4u;
constexpr u32 OFF_CTR = 0;
constexpr u32 OFF_TMO = 16384;
constexpr u32 OFF_CL = 20480;
constexpr u32 OFF_LF = 524288;
constexpr u32 OFF_RD = 655360;
constexpr u32 OFF_YR = 24576;
constexpr u32 OFF_RF = 28672;
constexpr u32 OFF_WR = 32768;
constexpr u32 OFF_WS = 33792;
constexpr u32 OFF_WX = 36864;
constexpr u32 OFF_SA = 131072;
constexpr u32 OFF_SB = 196608;
constexpr u32 OFF_DBG = 262144;
constexpr int NWG = 256, NCOMM = 32;
constexpr u32 NUNITS = 96u;
constexpr u32 OOB = 0x80000000u, NOFLAG = 0x40000000u;
constexpr u64 DEADLINE = 30000000ull;
constexpr u32 STAGE = 32768u;
constexpr u32 L_SNAP = 4u * STAGE;
constexpr u32 L_SC = L_SNAP + 512u;
constexpr u32 L_RED = L_SC + 1024u;
constexpr u32 LDS_BYTES = L_RED + 2048u;

struct Params {
    const float* x; char* me; int rank, pad; const float* w; float* out;
    char* ws[8];
    char* peer[8];
};

#define AUX_SC1 16
#define AUX_SYS 17
#define GL_AUX 16
#define W_LD 3
#define W_ST 2
#define X_LD 4
#define X_ST 2
#define WAIT_VM(n) asm volatile("s_waitcnt vmcnt(%0)" :: "n"(n) : "memory")
#define WAIT_LGKM0() asm volatile("s_waitcnt lgkmcnt(0)" ::: "memory")
#define BAR() do { asm volatile("" ::: "memory"); __builtin_amdgcn_s_barrier(); asm volatile("" ::: "memory"); } while (0)
#define WAIT_VM_TRACKED(n) do { asm volatile("" ::: "memory"); __builtin_amdgcn_s_waitcnt(((n) & 15) | (7 << 4) | (15 << 8) | (((n) >> 4) << 14)); asm volatile("" ::: "memory"); } while (0)

__device__ __forceinline__ rsrc_t mk_rsrc(const void* p, u32 bytes) { return __builtin_amdgcn_make_buffer_rsrc((void*)p, 0, bytes, 0x00020000); }
__device__ __forceinline__ u32 ld_agent(const void* p) { return __hip_atomic_load((const u32*)p, __ATOMIC_RELAXED, __HIP_MEMORY_SCOPE_AGENT); }
__device__ __forceinline__ u32 ld_sys(const void* p) { return __hip_atomic_load((const u32*)p, __ATOMIC_RELAXED, __HIP_MEMORY_SCOPE_SYSTEM); }
__device__ __forceinline__ void st_agent(void* p, u32 v) { __hip_atomic_store((u32*)p, v, __ATOMIC_RELAXED, __HIP_MEMORY_SCOPE_AGENT); }
__device__ __forceinline__ void st_sys_g(gchar* p, u32 v) { __hip_atomic_store((__attribute__((address_space(1))) u32*)p, v, __ATOMIC_RELAXED, __HIP_MEMORY_SCOPE_SYSTEM); }
__device__ __forceinline__ void st_sys(void* p, u32 v) { __hip_atomic_store((u32*)p, v, __ATOMIC_RELAXED, __HIP_MEMORY_SCOPE_SYSTEM); }

#define STAMP(ws, idx) do { } while (0)

__device__ __forceinline__ bool give_up(char* ws, u32 call, u64 t0) {
    const u32 t = __builtin_amdgcn_readfirstlane(ld_agent(ws + OFF_TMO));
    if (t == call || __builtin_amdgcn_s_memrealtime() - t0 > DEADLINE) { st_agent(ws + OFF_TMO, call); return true; }
    return false;
}
__device__ __forceinline__ void poll_ge(const char* p, u32 need, char* ws, u32 call) {
    const u64 t0 = __builtin_amdgcn_s_memrealtime();
    u32 n = 0;
    for (;;) {
        const u32 v = ld_sys(p);
        if (__all((int)(v - need) >= 0)) return;
        if ((++n & 15u) == 0u && give_up(ws, call, t0)) return;
        __builtin_amdgcn_s_sleep(2);
    }
}
__device__ __forceinline__ bool all4_ge(u32x4 v, u32 need) {
    return __all((int)(v[0] - need) >= 0 && (int)(v[1] - need) >= 0 && (int)(v[2] - need) >= 0 && (int)(v[3] - need) >= 0);
}
__device__ __forceinline__ void poll4_ge(rsrc_t rs, u32 off, u32 need, char* ws, u32 call) {
    const u64 t0 = __builtin_amdgcn_s_memrealtime();
    u32 n = 0;
    for (;;) {
        asm volatile("" ::: "memory");
        const u32x4 v = __builtin_amdgcn_raw_buffer_load_b128(rs, off, 0, AUX_SYS);
        if (all4_ge(v, need)) return;
        if ((++n & 15u) == 0u && give_up(ws, call, t0)) return;
        __builtin_amdgcn_s_sleep(1);
    }
}

__device__ __forceinline__ u32 umax(u32 a, u32 b) { return a > b ? a : b; }
__device__ __forceinline__ u32 absbits(float f) { return __float_as_uint(f) & 0x7fffffffu; }
__device__ __forceinline__ int exp_of(u32 amaxbits) { int e = (int)(amaxbits >> 23); return e < 15 ? 15 : e; }
__device__ __forceinline__ float scale_of(int e) { return __uint_as_float((u32)(268 - e) << 23); }
__device__ __forceinline__ u32x4 cvt8(f32x4 a, f32x4 b, float s) {
    f16x8 h;
    h[0] = (_Float16)(a[0] * s); h[1] = (_Float16)(a[1] * s); h[2] = (_Float16)(a[2] * s); h[3] = (_Float16)(a[3] * s);
    h[4] = (_Float16)(b[0] * s); h[5] = (_Float16)(b[1] * s); h[6] = (_Float16)(b[2] * s); h[7] = (_Float16)(b[3] * s);
    return __builtin_bit_cast(u32x4, h);
}

template <int NR, bool HEAD, class F>
__device__ __forceinline__ gchar* x_rows(const Params& p, LAS char* lds, int row0, const u32& par, const u32& call, int tid, int wave, F mid) {
    const int a = p.rank & 3;
    char* ws = p.me;
    const rsrc_t rx = mk_rsrc(p.x, IN_BYTES), rw = mk_rsrc(ws, WS_NEED);
    const u32 pi0 = (u32)((7 - wave) * 128 + (tid & 63));
    f32x4 v[NR][4];
#pragma unroll
    for (int r = 0; r < NR; ++r)
#pragma unroll
        for (int u = 0; u < 2; ++u) {
            const u32 off = (u32)(row0 + r) * (KDIM * 4u) + (pi0 + (u32)u * 64u) * 32u;
            v[r][2 * u] = __builtin_bit_cast(f32x4, __builtin_amdgcn_raw_buffer_load_b128(rx, off, 0, 0));
            v[r][2 * u + 1] = __builtin_bit_cast(f32x4, __builtin_amdgcn_raw_buffer_load_b128(rx, off + 16u, 0, 0));
            if (!HEAD) WAIT_VM(X_LD - 2);
        }
    mid();
    const int rb = row0 >> 7;
    const u32 slot = rb < 6 ? (u32)(a * 6 + rb) : (u32)(32 + rb - 6);
    const u32 rslot = rb < 6 ? slot : (u32)(24 + 2 * a + rb - 6);
    char* hp[4];
#pragma unroll
    for (int i = 0; i < 4; ++i) hp[i] = rb < 6 ? p.peer[i] : p.peer[4 + i];
    char* fifth = rb < 6 ? p.peer[7] : ws;
    asm volatile("" : "+s"(hp[0]), "+s"(hp[1]), "+s"(hp[2]), "+s"(hp[3]), "+s"(fifth));
    gchar* user = (gchar*)((tid & 4) ? fifth : (tid & 2) ? ((tid & 1) ? hp[3] : hp[2]) : ((tid & 1) ? hp[1] : hp[0]));
    const u32 wr = (u32)(tid & 63) >> 3;
    const bool wsend = wr < (u32)NR && ((tid & 7) < 4 || ((tid & 7) == 4 && rb >= 5));
    const u32 woff = OFF_SA + ((par * 64u + ((tid & 4) ? (rb < 6 ? 34u : slot) : rslot)) * 128u + (((u32)row0 + wr) & 127u)) * 4u;
    u32 word = 0;
    LAS u32* red = (LAS u32*)(lds + L_RED);
#pragma unroll
    for (int r = 0; r < NR; ++r) {
        u32 m = 0;
#pragma unroll
        for (int i = 0; i < 4; ++i)
#pragma unroll
            for (int j = 0; j < 4; ++j) m = umax(m, absbits(v[r][i][j]));
        m = umax(m, (u32)__builtin_amdgcn_update_dpp(0, (int)m, 0xb1, 15, 15, true));
        m = umax(m, (u32)__builtin_amdgcn_update_dpp(0, (int)m, 0x4e, 15, 15, true));
        m = umax(m, (u32)__builtin_amdgcn_update_dpp(0, (int)m, 0x141, 15, 15, true));
        m = umax(m, (u32)__builtin_amdgcn_update_dpp(0, (int)m, 0x140, 15, 15, true));
        m = umax(umax((u32)__builtin_amdgcn_readlane((int)m, 0), (u32)__builtin_amdgcn_readlane((int)m, 16)),
                 umax((u32)__builtin_amdgcn_readlane((int)m, 32), (u32)__builtin_amdgcn_readlane((int)m, 48)));
        if ((tid & 63) == 0) red[r * 8 + wave] = m;
    }
    asm volatile("" : "+v"(user));
    WAIT_LGKM0(); BAR();
#pragma unroll
    for (int r = 0; r < NR; ++r) {
        const u32x4 m0 = *(LAS u32x4*)(red + r * 8), m1 = *(LAS u32x4*)(red + r * 8 + 4);
        const u32 m = umax(umax(umax(m0[0], m0[1]), umax(m0[2], m0[3])), umax(umax(m1[0], m1[1]), umax(m1[2], m1[3])));
        const int e = exp_of(m); const float s = scale_of(e);
        const u32 rin = (u32)((row0 + r) & 127);
        if (wr == (u32)r) word = call | ((u32)(e - 141) & 255u);
        const u32 off = CTRL_BYTES + par * PAR_BYTES + rin * 128u + (pi0 >> 3) * IMG + (((pi0 & 7u) ^ ((rin >> 1) & 7u)) << 4);
        const u32x4 h0 = cvt8(v[r][0], v[r][1], s), h1 = cvt8(v[r][2], v[r][3], s);
        __builtin_amdgcn_raw_buffer_store_b128(h0, rw, off + slot * PANEL, 0, AUX_SC1);
        __builtin_amdgcn_raw_buffer_store_b128(h1, rw, off + slot * PANEL + 8u * IMG, 0, AUX_SC1);
        if (!HEAD) WAIT_VM(X_ST - 2);
        if (HEAD) {
            u32x4 h2 = h0;
#define HSUB(k) do { _Pragma("unroll") for (int i = 0; i < 4; ++i) { \
                __builtin_amdgcn_raw_buffer_store_b128((k) < 4 ? h0 : (k) < 8 ? h1 : h2, mk_rsrc(hp[i], WS_NEED), \
                    ((i == 3 && rb < 6) || ((tid >> 4) & 3) != ((k) & 3)) ? OOB : off + rslot * PANEL + (u32)((k) >> 2) * 8u * IMG, 0, AUX_SYS); } } while (0)
            if (wave == 7) {
#pragma unroll
                for (int k = 0; k < HW; ++k) HSUB(k);
                WAIT_VM(4 * HW);
            } else {
                if (HU > 2 && wave == 6) { ((LAS u32x4*)lds)[448 + (tid & 63)] = h0; WAIT_LGKM0(); }
                WAIT_VM(0);
            }
            BAR();
            if (wave == 7) {
                if (wsend) st_sys_g(user + woff, word);
                if (HU > 2) h2 = ((LAS u32x4*)lds)[448 + (tid & 63)];
#pragma unroll
                for (int k = HW; k < 4 * HU; ++k) {
                    if (k < 2 * HW) WAIT_VM(4 * HW - 3); else WAIT_VM(4 * HW - 4);
                    HSUB(k);
                }
            }
#undef HSUB
        }
    }
    if (!HEAD) {
        WAIT_VM(0); BAR();
        if (wave == 0 && wsend) st_sys_g(user + woff, word);
    }
    return user + OFF_RD + ((u32)p.rank * 128u + (u32)(row0 & 127)) * 4u;
}

__device__ __forceinline__ int slot_row(int rank, int slot) {
    const int g = rank >> 2;
    if (slot < 24) { const int i = slot / 6; return (g * 4 + i) * 1024 + (slot - i * 6) * 128; }
    return (slot < 32 ? (1 - g) * 4 + ((slot - 24) >> 1) : rank) * 1024 + 768 + (slot & 1) * 128;
}

template <bool BIG>
__device__ __forceinline__ void gemm_tile(const Params& p, LAS char* lds, int slot, int ct, int rh, int ch, u32 par, u32 call, int tid, int lane, int wave, int stamp0) {
    constexpr int NB = BIG ? 4 : 2, NC = BIG ? 32 : 16, BT = BIG ? 128 : 64;
    const int rank = p.rank, g = rank >> 2, a = rank & 3;
    char* ws = p.me;
    const rsrc_t rws = mk_rsrc(ws, WS_NEED);
    int a_src; u32 a_base, a_sh;
    if (slot < 24) { const int i = slot / 6, a_rb = slot - i * 6; a_src = i == a ? -1 : g * 4 + i;
        a_base = a_rb == 0 ? 0u : a_rb < 3 ? 16u : a_rb < 5 ? 48u : 80u + (u32)rh * 8u; a_sh = a_rb == 0 ? 1u : 0u; }
    else { a_src = slot < 32 ? (1 - g) * 4 + ((slot - 24) >> 1) : -1; a_base = (slot & 1) ? 48u : 0u; a_sh = 1u; }
    const int b_src = ct < 8 ? -1 : (1 - g) * 4 + ((ct - 8) >> 1);
    const int src = lane < 32 ? a_src : (BIG ? b_src : -1);
    const u32 gsh = src < 0 ? 31u : lane < 32 && BIG ? a_sh : 0u;
    const char* gptr = ws + (src >= 0 ? OFF_LF + ((u32)src * NUNITS + (lane < 32 ? a_base : 16u)) * 128u + (u32)(lane & (BIG ? 31 : 15)) * 4u : OFF_CTR + blockIdx.x * 64u + 4u);
    const u32 gmin = src >= 0 && lane < 32 && a_base == 0u ? (u32)HU : 0u;
#define GPTR(c) (gptr + (umax((u32)(c) >> gsh, gmin) << (BIG ? 7 : 6)))
    if (wave == 0) {
        if (a_src < 0 || (!BIG && rh == 0)) poll4_ge(rws, OFF_SA + (par * 64u + (u32)slot) * 512u + (u32)(lane & (BIG ? 31 : 15)) * 16u, call, ws, call);
        if (b_src < 0) poll4_ge(rws, OFF_WR + (u32)ct * 128u + (u32)(lane & 7) * 16u, call, ws, call);
        else if (!BIG) for (u32 i = 0; i < 4u; ++i) poll4_ge(rws, OFF_LF + ((u32)b_src * NUNITS + 16u) * 128u + i * 1024u + (u32)lane * 16u, call, ws, call);
        if (a_src >= 0 && a_base == 0u) poll4_ge(rws, OFF_RD + (u32)a_src * 512u + (u32)(lane & 31) * 16u, call, ws, call);
        poll_ge(GPTR(0), call, ws, call);
        WAIT_VM(0);
    }
    BAR();
    if (wave == 0) STAMP(ws, stamp0);
    const u32 pb = CTRL_BYTES + par * PAR_BYTES;
    const char* gA = ws + pb + (u32)slot * PANEL + (BIG ? (u32)wave * 2048u : (u32)rh * 8192u + (u32)wave * 1024u) + (u32)lane * 16u;
    const char* gB = ws + pb + (u32)(ASLOTS + ct) * PANEL + (BIG ? (u32)wave * 2048u : (u32)ch * 8192u + (u32)wave * 1024u) + (u32)lane * 16u;
    const u32 ldsw = (u32)wave * (BIG ? 2048u : 1024u);
    constexpr u32 GSTEP = BIG ? IMG : 2u * IMG, G2 = BIG ? 1024u : IMG, L2O = BIG ? 1024u : 8192u;
#define GLDS16(g, l) __builtin_amdgcn_global_load_lds((const u32*)(g), (LAS u32*)(lds + (l)), 16, 0, GL_AUX)
#define LOADT(t, st) do { const u32 _o = (u32)(t) * GSTEP; \
        GLDS16(gA + _o, (st) * STAGE + ldsw); GLDS16(gA + _o + G2, (st) * STAGE + ldsw + L2O); \
        GLDS16(gB + _o, (st) * STAGE + IMG + ldsw); GLDS16(gB + _o + G2, (st) * STAGE + IMG + ldsw + L2O); } while (0)
#define SNAP(c) __builtin_amdgcn_global_load_lds((const u32*)GPTR(c), (LAS u32*)(lds + L_SNAP + ((u32)(c) & 1u) * 256u), 4, 0, AUX_SYS)
    const int ks = BIG ? wave >> 2 : wave >> 1, wm = BIG ? (wave >> 1) & 1 : 0, wn = wave & 1;
    const u32 swz = (u32)((((ks & 1) * 4 + (lane >> 4)) ^ ((lane & 15) >> 1)) << 4) + (BIG ? 0u : (u32)(ks >> 1) * 8192u);
    const u32 ard = (u32)(wm * 64 + (lane & 15)) * 128u + swz, brd = IMG + (u32)(wn * (BT / 2) + (lane & 15)) * 128u + swz;
#define READF(fa, fb, st) do { _Pragma("unroll") for (int _i = 0; _i < 4; ++_i) { \
        if (_i < NB) fb[_i] = *(const LAS f16x8*)(lds + (st) * STAGE + brd + _i * 2048u); \
        fa[_i] = *(const LAS f16x8*)(lds + (st) * STAGE + ard + _i * 2048u); } } while (0)
#define MMA(fa, fb) do { _Pragma("unroll") for (int _m = 0; _m < 4; ++_m) _Pragma("unroll") for (int _n = 0; _n < NB; ++_n) \
        acc[_m][_n] = __builtin_amdgcn_mfma_f32_16x16x32_f16(fb[_n], fa[_m], acc[_m][_n], 0, 0, 0); } while (0)
    f32x4 acc[4][NB];
#pragma unroll
    for (int m = 0; m < 4; ++m)
#pragma unroll
        for (int n = 0; n < NB; ++n) acc[m][n] = (f32x4){0.f, 0.f, 0.f, 0.f};
    f16x8 a0[4], b0[4], a1[4], b1[4];
    if (wave >= 4) {
        const u32 so = (wave < 6 ? OFF_SA + (par * 64u + (u32)slot) * 512u : OFF_SB + (par * BSLOTS + (u32)ct) * 512u) + (u32)((wave & 1) * 64 + lane) * 4u;
        __builtin_amdgcn_global_load_lds((const u32*)(ws + so), (LAS u32*)(lds + L_SC + (u32)(wave - 4) * 256u), 4, 0, AUX_SYS);
    }
    LOADT(0, 0u);
    if (wave == 0) SNAP(1);
    LOADT(1, 1u); LOADT(2, 2u); LOADT(3, 3u);
    if (wave == 0) WAIT_VM(13); else WAIT_VM(12);
    BAR();
    READF(a0, b0, 0u);
#define GATE(c) do { const u32 _v = *(const LAS u32*)(lds + L_SNAP + ((u32)(c) & 1u) * 256u + (u32)lane * 4u); \
        if (!__all((int)(_v - call) >= 0)) { if (wave == 0) { poll_ge(GPTR(c), call, ws, call); WAIT_VM(0); } BAR(); } } while (0)
    for (int c = 0; c < NC - 1; ++c) {
        const int t = 4 * c;
        WAIT_VM(8); WAIT_LGKM0(); BAR();
        GATE(c + 1);
        LOADT(t + 4, 0u); READF(a1, b1, 1u); MMA(a0, b0);
        WAIT_VM(8); WAIT_LGKM0(); BAR();
        if (wave == 0) SNAP(c + 2 < NC ? c + 2 : c);
        LOADT(t + 5, 1u); READF(a0, b0, 2u); MMA(a1, b1);
        if (wave == 0) WAIT_VM(9); else WAIT_VM(8);
        WAIT_LGKM0(); BAR();
        LOADT(t + 6, 2u); READF(a1, b1, 3u); MMA(a0, b0);
        if (wave == 0) WAIT_VM(9); else WAIT_VM(8);
        WAIT_LGKM0(); BAR();
        LOADT(t + 7, 3u); READF(a0, b0, 0u); MMA(a1, b1);
    }
    WAIT_VM(8); WAIT_LGKM0(); BAR();
    READF(a1, b1, 1u); MMA(a0, b0);
    WAIT_VM(4); WAIT_LGKM0(); BAR();
    READF(a0, b0, 2u); MMA(a1, b1);
    WAIT_VM(0); WAIT_LGKM0(); BAR();
    READF(a1, b1, 3u); MMA(a0, b0);
    const u32 tmo = ld_agent(ws + OFF_TMO);
    WAIT_LGKM0(); BAR();
    MMA(a1, b1);
    if (wave == 0) STAMP(ws, stamp0 + 1);
    constexpr u32 RB = BT * 4u, PPR = BT / 4u, AREA = BT * RB;
#pragma unroll
    for (int m = 0; m < 4; ++m)
#pragma unroll
        for (int n = 0; n < NB; ++n) {
            const u32 r = (u32)(wm * 64 + m * 16 + (lane & 15)), pc = (u32)(wn * (BT / 8) + n * 4 + (lane >> 4));
            *(LAS f32x4*)(lds + (u32)ks * AREA + r * RB + ((pc ^ (r & 7u)) << 4)) = acc[m][n];
        }
    WAIT_LGKM0(); BAR();
    const LAS int* sc = (const LAS int*)(lds + L_SC);
    const u32 pc = (u32)lane & (PPR - 1u);
    const u32 col0 = BIG ? 0u : (u32)ch * 64u, row0 = BIG ? 0u : (u32)rh * 64u;
    const i32x4 eb = *(const LAS i32x4*)(sc + 128 + col0 + pc * 4u);
    const bool bad = tmo == call;
    const bool remote = ct >= 8;
    const rsrc_t ro = mk_rsrc(remote ? (const void*)ws : (const void*)p.out, remote ? WS_NEED : IN_BYTES);
    const u32 obase = (remote ? OFF_YST + (u32)(((ct - 8) >> 1) * 32 + slot) * UNIT + (u32)(ct & 1) * 512u : ((u32)slot_row(rank, slot) * NLOC + (u32)ct * 128u) * 4u) + col0 * 4u;
    const u32 ostr = remote ? 1024u : NLOC * 4u;
#pragma unroll
    for (u32 it = 0; it < (BIG ? 8u : 2u); ++it) {
        const u32 r = (u32)wave * (BT / 8u) + (it * 64u + (u32)lane) / PPR;
        const u32 lo = r * RB + ((pc ^ (r & 7u)) << 4);
        f32x4 s = *(const LAS f32x4*)(lds + lo) + *(const LAS f32x4*)(lds + AREA + lo);
        if (!BIG) s += *(const LAS f32x4*)(lds + 2u * AREA + lo) + *(const LAS f32x4*)(lds + 3u * AREA + lo);
        const int ea = (int)(signed char)sc[row0 + r];
#pragma unroll
        for (int j = 0; j < 4; ++j) s[j] = bad ? __builtin_nanf("") : __builtin_amdgcn_ldexpf(s[j], ea + eb[j]);
        __builtin_amdgcn_raw_buffer_store_b128(__builtin_bit_cast(u32x4, s), ro, obase + (row0 + r) * ostr + pc * 16u, 0, AUX_SC1);
    }
    if (remote) {
        WAIT_VM(0); BAR();
        if (tid < (BIG ? 2 : 1)) st_sys(p.ws[b_src] + OFF_YR + (u32)(a * 64 + rh * 32 + slot) * 16u + (u32)(ct & 1) * 8u + (u32)(tid + ch) * 4u, call);
    }
    if (wave == 0) STAMP(ws, stamp0 + 2);
#undef GLDS16
#undef LOADT
#undef SNAP
#undef READF
#undef MMA
#undef GATE
#undef GPTR
}

constexpr u32 NPOS_INTRA = 96u - HU - DELTA, NPOS_CROSS = 97u - HU + DELTA;
constexpr u32 FAN = 0x20000000u;
struct Desc { u32 src, dst, gate, flag; };
struct Stream {
    bool cross; int rank, g, a, j; u32 pb, sa;
    __device__ __forceinline__ int yslot(u32 k) const {
        if (k < 6u) return j * 6 + (int)k;
        if (k < 9u) return ((j + (int)k - 5) & 3) * 6;
        if (k < 12u) return 24 + 2 * ((a + (int)k - 8) & 3);
        if (k < 18u) return ((j + 1 + (((int)k - 12) >> 1)) & 3) * 6 + 1 + ((int)k & 1);
        if (k < 21u) return 25 + 2 * ((a + (int)k - 17) & 3);
        if (k < 27u) return ((j + 1 + (((int)k - 21) >> 1)) & 3) * 6 + 3 + (((int)k - 21) & 1);
        return ((j + 1 + (int)(k % 3u)) & 3) * 6 + 5;
    }
    __device__ __forceinline__ u32 single(u32 slot, u32 k) const { return pb + slot * PANEL + k * UNIT; }
    __device__ __forceinline__ u32 pair(u32 slot, u32 k) const { return pb + slot * PANEL + k * 65536u; }
    __device__ __forceinline__ u32 half(u32 slot, u32 h, u32 k) const { return pb + slot * PANEL + k * 2u * UNIT + h * 8192u; }
    __device__ __forceinline__ Desc at(u32 pos) const {
        Desc d;
        const u32 s = (u32)j * 6u, M8 = 7u << 20, PAIR = 1u << 28, HALF = 4u << 28;
        u32 u, fsrc = (u32)((cross ? 1 - g : g) * 4 + j);
        if (!cross) {
            if (pos >= NPOS_INTRA) pos = NPOS_INTRA - 1u;
            if (pos < 80u - HU) {
                u = pos + HU;
                d.src = u < 16u ? single(s, u) : u < 48u ? pair(s + 1u, u - 16u) : pair(s + 3u, u - 48u);
                d.gate = sa + s * 512u + (u < 16u ? 0u : u < 48u ? 512u + PAIR + M8 : 1536u + PAIR + M8);
            } else {
                const u32 e = NPOS_INTRA - 1u - pos, n0 = 8u - DELTA;
                const u32 h = e < 2u || e - 2u >= 2u * n0 || (e & 1u) ? 1u : 0u;
                const u32 k = e < 2u ? 7u - e : e - 2u >= 2u * n0 ? 5u - n0 - (e - 2u - 2u * n0) : h ? 5u - ((e - 2u) >> 1) : 7u - ((e - 2u) >> 1);
                u = 80u + h * 8u + k;
                d.src = half(s + 5u, h, k);
                d.gate = sa + (s + 5u) * 512u + h * 256u + M8 + HALF;
            }
            d.dst = d.src;
            d.flag = OFF_LF + (fsrc * NUNITS + u) * 128u;
        } else {
            if (pos >= NPOS_CROSS) pos = NPOS_CROSS - 1u;
            pos += HU;
            const u32 hop = j == a ? 64u : 68u + DELTA;
            if (pos < 64u) {
                u = pos;
                d.src = u < 16u ? single(32u, u) : u < 48u ? pair((u32)(ASLOTS + 2 * a), u - 16u) : single(33u, u - 48u);
                d.dst = u < 16u ? single((u32)(24 + 2 * j), u) : u < 48u ? pair((u32)(ASLOTS + 8 + 2 * j), u - 16u) : single((u32)(25 + 2 * j), u - 48u);
                d.gate = u < 16u ? sa + (u32)(24 + 2 * j) * 512u : u < 48u ? OFF_WS + (u32)j * 256u + (1u << 20) + (3u << 28) : sa + (u32)(25 + 2 * j) * 512u;
                d.flag = OFF_LF + (fsrc * NUNITS + u) * 128u;
            } else if (pos - hop < DELTA) {
                const u32 k = pos - hop;
                if (j == a) { d.src = half(s + 5u, 0u, k); d.dst = pb + RELAY + k * 2u * UNIT; d.gate = sa + 34u * 512u + M8 + HALF;
                              d.flag = FAN + OFF_RF + ((u32)a * 4u + k) * 128u; }
                else { d.src = pb + RELAY + k * 2u * UNIT; d.dst = half(s + 5u, 0u, k); d.gate = OFF_RF + ((u32)j * 4u + k) * 128u + (6u << 28);
                       d.flag = OFF_LF + ((fsrc ^ 4u) * NUNITS + 80u + k) * 128u; }
            } else {
                const u32 k = pos < hop ? pos - 64u : pos - 64u - DELTA;
                const int ys = yslot(k);
                const u32 h = k >= 30u ? 1u : 0u;
                d.src = OFF_YST + (u32)(a * 32 + ys) * UNIT + h * 65536u;
                d.dst = ((u32)slot_row((int)fsrc, ys) + h * 64u) * (NLOC * 4u) + (u32)j * 1024u;
                d.gate = OFF_YR + ((u32)(j * 64 + ys) + h * 32u) * 16u + ((k >= 27u ? 10u : 2u) << 28);
                d.flag = NOFLAG;
            }
        }
        return d;
    }
};
struct Tables { Desc lo, hi; };
__device__ __forceinline__ u32 tab(u32 t0, u32 t1, u32 i) {
    const u32 x = (u32)__builtin_amdgcn_readlane((int)t0, (int)(i & 63u)), y = (u32)__builtin_amdgcn_readlane((int)t1, (int)(i & 63u));
    return i < 64u ? x : y;
}

#define PSLEEP0 10
#define PSLEEP 4
#define NACT 32
__device__ __forceinline__ void pull_stream(const Params& p, LAS char* lds, const Tables& T, bool cross, u32 cw, char* src, gchar* fan, u32 calln, u32 call, int lane, int wave) {
    char* ws = p.me;
    const rsrc_t rl = mk_rsrc(ws, WS_NEED), rs = mk_rsrc(src, WS_NEED), rout = mk_rsrc(p.out, IN_BYTES);
    LAS char* pl = lds + (u32)wave * 1024u;
    const u32 pla = (u32)(size_t)pl + (u32)lane * 4u;
    const u32 l16 = (u32)lane * 16u, l4r = (u32)(lane >> 5) * IMG + (l16 & 511u);
    const u32 l0 = lane == 0 ? 0u : OOB;
    const u32 co = OFF_CL + (u32)wave * 128u + l0;
    const u32 NPIECES = (cross ? NPOS_CROSS : NPOS_INTRA) * 32u;
    const u32 cbase = 2u * NACT - (calln - 1u) * NPIECES;
    u32x4 v0 = {0u, 0u, 0u, 0u}, v1 = v0, v2 = v0, v3;
    u32 q = OOB, so = OOB, ss = 1024u;
    u32 c0 = cw, c1 = cw + NACT;
    u32 pf = NOFLAG;
    bool claimed = false, dead = false;
    int cl = 0;
#define KIND(q) (tab(T.lo.gate, T.hi.gate, (q) >> 5) >> 28)
#define POFF(q, kd) (((kd) & 1u) ? (((q) >> 4) & 1u) * PANEL + ((q) & 15u) * 4096u : ((kd) & 4u) ? (((q) >> 1) & 15u) * IMG + ((q) & 1u) * 4096u : ((q) & 31u) * (((kd) & 8u) ? 2048u : (kd) ? 4096u : 512u))
#define GBASE(q) ({ const u32 _g = tab(T.lo.gate, T.hi.gate, (q) >> 5), _k = _g >> 28; \
        const u32 _ix = (((_k & 1u) ? ((q) >> 2) & 4u : 0u) + ((q) & (_k == 4u ? 1u : 3u))) << (_k == 3u ? 5 : 7); \
        (_g & 0xfffffu) + ((_k & 7u) == 2u ? 0u : _k == 6u ? ((q) & 28u) * 4u : _k == 0u ? ((q) & 31u) * 16u : _ix); })
#define GOFF(q) (GBASE(q) + (l16 & (tab(T.lo.gate, T.hi.gate, (q) >> 5) >> 16)))
#define PFLAG() st_sys_g((pf & FAN) ? fan + (pf - FAN) : (gchar*)ws + (pf >= NOFLAG ? OFF_CTR + cw * 64u + 8u + (u32)wave * 4u : pf), call)
#define PLD(u, o) v##u = __builtin_amdgcn_raw_buffer_load_b128(rs, (o) + (u) * ss, 0, AUX_SYS)
#define PST(u) do { asm volatile("" : "+v"(v##u)); \
        __builtin_amdgcn_raw_buffer_store_b128(v##u | (dead ? 0x7fc00000u : 0u), rd, d + (u) * ds, 0, AUX_SC1); } while (0)
#define PSTAMP(i) do { } while (0)
#define GPOLL(i, sl) do { __builtin_amdgcn_global_load_lds((const u32*)gp, (LAS u32*)(pl + (i) * 256u), 4, 0, AUX_SYS); __builtin_amdgcn_s_sleep(sl); } while (0)
#define GSTEP(i, sl) { GPOLL(((i) + 3) & 3, sl); WAIT_VM_TRACKED(3); u32 _w; \
            asm volatile("ds_read_b32 %0, %1 offset:%2\n\ts_waitcnt lgkmcnt(0)" : "=v"(_w) : "v"(pla), "n"((i) * 256) : "memory"); \
            if (__all((int)(_w - call) >= 0)) break; }
#define WATCH(sl) do { \
        const char* gp = ws + GBASE(c0) + ((u32)lane * 4u & ((tab(T.lo.gate, T.hi.gate, c0 >> 5) >> 16 & 0x3f0u) | 12u)); \
        const u64 t0 = __builtin_amdgcn_s_memrealtime(); \
        GPOLL(0, sl); GPOLL(1, sl); GPOLL(2, sl); \
        for (u32 n = 1;; ++n) { \
            GSTEP(0, sl) GSTEP(1, sl) GSTEP(2, sl) GSTEP(3, sl) \
            if ((n & 15u) == 0u && give_up(ws, call, t0)) { dead = true; break; } \
        } } while (0)
    WATCH(PSLEEP0);
    u32x4 gv = {call, call, call, call};
    do {
        const u32 qq = q != OOB ? q : 0u;
        const u32 kd = KIND(qq); const bool y = (kd & 7u) == 2u;
        u32 d = q != OOB ? tab(T.lo.dst, T.hi.dst, qq >> 5) + (y ? POFF(qq, kd) * 4u : POFF(qq, kd)) + (kd ? l16 : l4r) : OOB;
        const u32 ds = y ? NLOC * 4u : ss;
        const rsrc_t rd = y ? rout : rl;
        if (kd & 8u) so = OOB;
        WAIT_VM_TRACKED(5); PLD(2, so); PST(0);
        WAIT_VM_TRACKED(4); PFLAG(); PLD(3, so); PST(1);
        WAIT_VM_TRACKED(4);
        if (kd & 8u) d = OOB;
        bool open = dead || all4_ge(gv, call);
        if (q == OOB && !open) { WATCH(PSLEEP); open = true; }
        pf = q != OOB ? tab(T.lo.flag, T.hi.flag, qq >> 5) + (qq & 31u) * 4u : NOFLAG;
        if (claimed) { c1 = (u32)__builtin_amdgcn_readfirstlane(cl) + cbase; if (c1 > NPIECES + 127u) c1 = NPIECES + 127u; }
        claimed = open && c0 < NPIECES;
        q = claimed ? c0 : OOB; if (claimed) c0 = c1;
        const u32 kn = claimed ? KIND(q) : 1u;
        so = claimed ? tab(T.lo.src, T.hi.src, q >> 5) + POFF(q, kn) + (kn ? l16 : l4r) : OOB;
        ss = kn ? 1024u : 2u * IMG;
        PLD(0, so); PST(2);
        WAIT_VM_TRACKED(3); PST(3); PLD(1, so);
        cl = __builtin_amdgcn_raw_ptr_buffer_atomic_add_i32(1, rl, claimed ? co : OOB, 0, AUX_SC1);
        gv = __builtin_amdgcn_raw_buffer_load_b128(rl, GOFF(c0), 0, AUX_SYS);
        if (claimed) PSTAMP(q >> 5);
    } while (c0 < NPIECES || q != OOB);
    WAIT_VM_TRACKED(0); PFLAG();
    asm volatile(".Lagk_end:");
    STAMP(ws, 2 + wave * 8 + 7);
#undef PSTAMP
#undef GOFF
#undef GBASE
#undef WATCH
#undef GSTEP
#undef GPOLL
#undef POFF
#undef KIND
#undef PST
#undef PLD
#undef PFLAG
}

__global__ void __launch_bounds__(512, 2) ag_gemm_kernel(Params p) {
    extern __shared__ __attribute__((aligned(16))) char smem[];
    LAS char* lds = (LAS char*)smem;
    const int tid = threadIdx.x, lane = tid & 63, wave = __builtin_amdgcn_readfirstlane(tid >> 6);
    const int b = blockIdx.x, rank = p.rank, g = rank >> 2, a = rank & 3;
    char* ws = p.me;
    if (wave == 0) STAMP(ws, 0);
    const u32 prev = ld_agent(ws + OFF_CTR + (u32)b * 64u);
    u32 calln, call, par;
    const rsrc_t rws = mk_rsrc(ws, WS_NEED);

    Tables T = {};
    u32 code = 0;
    char* src = ws; gchar* fan = (gchar*)ws;
    gchar* rdflag = x_rows<1, true>(p, lds, b < 128 ? b : 640 + b, par, call, tid, wave, [&]() {
        u64 pc; u32 len;
        asm volatile("s_getpc_b64 %0\n.Lagk_pc:\n\ts_mov_b32 %1, .Lagk_end-.Lagk_pc" : "=s"(pc), "=s"(len));
        code = __builtin_amdgcn_raw_buffer_load_b32(mk_rsrc((const void*)pc, (int)len > 0 ? len : 0u), (u32)(b >> 3) * 2048u + (u32)tid * 4u, 0, 0);
        asm volatile("" ::: "memory");
        calln = (u32)__builtin_amdgcn_readfirstlane(prev) + 1u; call = calln << 8; par = calln & 1u;
        if (b < NCOMM) {
            Stream S;
            S.cross = wave >= 3; S.rank = rank; S.g = g; S.a = a; S.j = (a + wave + 1) & 3; S.pb = CTRL_BYTES + par * PAR_BYTES; S.sa = OFF_SA + par * 32768u;
            T.lo = S.at((u32)lane); T.hi = S.at((u32)lane + 64u);
            src = p.peer[wave == 3 ? 7 : wave];
            char *f0 = p.peer[4], *f1 = p.peer[5], *f2 = p.peer[6];
            asm volatile("" : "+s"(src), "+s"(f0), "+s"(f1), "+s"(f2));
            fan = (gchar*)(lane == 0 ? f0 : lane == 1 ? f1 : f2);
        }
    });
    asm volatile("" :: "v"(code));
    if (tid < 2) st_agent(ws + OFF_CTR + (u32)b * 64u + (u32)tid * 4u, tid ? call : calln);
    if (wave == 0) STAMP(ws, 1);
    if (b < NCOMM) {
        if (wave < 7) {
            if (b >= NACT) return;
            pull_stream(p, lds, T, wave >= 3, (u32)b, src, fan, calln, call, lane, wave);
        } else {
            WAIT_VM(0);
            if (lane < 4) st_sys_g(rdflag, call);
        }
        return;
    }

    {
        const int c = b - NCOMM, strip = c / 7, rg = c - strip * 7;
        const int kt0 = (rg * 128) / 7, kt1 = ((rg + 1) * 128) / 7;
        const rsrc_t rw = mk_rsrc(p.w, IN_BYTES);
        const int q = lane & 7, n4 = lane >> 3;
        if (wave == 7) { WAIT_VM(0); if (lane < 4) st_sys_g(rdflag, call); }
        f32x4 v[3][8];
#pragma unroll
        for (int i = 0; i < 3; ++i) {
            const int kt = kt0 + wave + 8 * i;
            const u32 base = kt < kt1 ? ((u32)(kt * 64 + q * 8) * NLOC + (u32)(strip * 32 + n4 * 4)) * 4u : OOB;
#pragma unroll
            for (int jj = 0; jj < 8; ++jj) {
                v[i][jj] = __builtin_bit_cast(f32x4, __builtin_amdgcn_raw_buffer_load_b128(rw, base + (u32)jj * (NLOC * 4u), 0, 0));
                WAIT_VM(W_LD - 1);
            }
        }
        LAS u32* cm = (LAS u32*)(lds + L_RED + 256u);
        if (tid < 64) cm[tid] = 0u;
        WAIT_LGKM0(); BAR();
        u32 m[4] = {0u, 0u, 0u, 0u};
#pragma unroll
        for (int i = 0; i < 3; ++i)
#pragma unroll
            for (int jj = 0; jj < 8; ++jj)
#pragma unroll
                for (int k = 0; k < 4; ++k) m[k] = umax(m[k], absbits(v[i][jj][k]));
#pragma unroll
        for (int k = 0; k < 4; ++k) {
            m[k] = umax(m[k], (u32)__shfl_xor((int)m[k], 1)); m[k] = umax(m[k], (u32)__shfl_xor((int)m[k], 2)); m[k] = umax(m[k], (u32)__shfl_xor((int)m[k], 4));
        }
        if (q < 4) __hip_atomic_fetch_max(cm + n4 * 4 + q, q == 0 ? m[0] : q == 1 ? m[1] : q == 2 ? m[2] : m[3], __ATOMIC_RELAXED, __HIP_MEMORY_SCOPE_WORKGROUP);
        WAIT_LGKM0(); BAR();
        u64* wx = (u64*)(ws + OFF_WX) + (u32)strip * 256u;
        if (tid < 32) __hip_atomic_store(wx + rg * 32 + tid, ((u64)call << 32) | cm[tid], __ATOMIC_RELAXED, __HIP_MEMORY_SCOPE_AGENT);
        if (wave < 4) {
            const int gi = tid < 224 ? tid : 0;
            const u64 t0 = __builtin_amdgcn_s_memrealtime();
            u64 gv; u32 n = 0;
            for (;;) {
                gv = __hip_atomic_load(wx + gi, __ATOMIC_RELAXED, __HIP_MEMORY_SCOPE_AGENT);
                if (__all((u32)(gv >> 32) == call)) break;
                if ((++n & 15u) == 0u && give_up(ws, call, t0)) break;
                __builtin_amdgcn_s_sleep(1);
            }
            __hip_atomic_fetch_max(cm + 32 + (gi & 31), (u32)gv, __ATOMIC_RELAXED, __HIP_MEMORY_SCOPE_WORKGROUP);
        }
        WAIT_LGKM0(); BAR();
        const int ct = strip >> 2;
        if (rg == 0 && tid < 64) {
            const u32 nin = (u32)((strip & 3) * 32 + (tid & 31));
            const u32 ev = (u32)(exp_of(cm[32 + (tid & 31)]) - 141);
            char* t = tid < 32 ? ws + OFF_SB + (par * BSLOTS + (u32)ct) * 512u : p.ws[(1 - g) * 4 + (strip >> 3)] + OFF_SB + (par * BSLOTS + (u32)(8 + 2 * a + (ct & 1))) * 512u;
            st_sys(t + nin * 4u, ev);
        }
        const u32x4 mx = *(LAS u32x4*)(cm + 32 + n4 * 4);
        const u32 pbase = CTRL_BYTES + par * PAR_BYTES + (u32)(ASLOTS + ct) * PANEL;
#pragma unroll
        for (int k = 0; k < 4; ++k) {
            const float s = scale_of(exp_of(mx[k]));
            const u32 nin = (u32)((strip & 3) * 32 + n4 * 4 + k);
            const u32 ro = nin * 128u + (((u32)q ^ ((nin >> 1) & 7u)) << 4);
#pragma unroll
            for (int i = 0; i < 3; ++i) {
                const int kt = kt0 + wave + 8 * i;
                const u32 off = kt < kt1 ? pbase + (u32)kt * IMG + ro : OOB;
                const f32x4 lo = {v[i][0][k], v[i][1][k], v[i][2][k], v[i][3][k]}, hi = {v[i][4][k], v[i][5][k], v[i][6][k], v[i][7][k]};
                __builtin_amdgcn_raw_buffer_store_b128(cvt8(lo, hi, s), rws, off, 0, AUX_SC1);
                WAIT_VM(W_ST - 1);
            }
        }
        WAIT_VM(0); BAR();
        if (tid < 4 && (!(tid & 1) || rg == 6))
            st_sys((tid < 2 ? ws + OFF_WR + (u32)strip * 32u : p.ws[(1 - g) * 4 + (strip >> 3)] + OFF_WS + (u32)(a * 8 + (strip & 7)) * 32u) + (u32)(rg + (tid & 1)) * 4u, call);
        if (wave == 0) STAMP(ws, 2);
    }
    const int xcd = b & 7, q = (b >> 3) - 4;
    if (xcd >= 4 && q < 24) {
        const int r0 = 128 + (q * 4 + xcd - 4) * 4;
        x_rows<4, false>(p, lds, r0, par, call, tid, wave, []() {});
        x_rows<4, false>(p, lds, r0 + (r0 < 384 ? 384 : 512), par, call, tid, wave, []() {});
        if (wave == 0) STAMP(ws, 3);
    }
    const int qr = q >> 2, qc = q & 3, x4 = xcd & 3;
    const int early = (x4 >> 1) == 0 ? (qr < 6 ? a * 6 + qr : -1) : (qr < 3 ? ((a + 1 + qr) & 3) * 6 : 24 + 2 * (qr - 3));
    const int peer = ((a + 1 + (qr >> 1)) & 3) * 6;
    for (int it = 0; it < 4; ++it) {
        int slot = -1, ct = 4 * x4 + qc, rh = 0, ch = 0;
        if (it == 0) { if (xcd < 4) { slot = early; ct = 4 * (x4 & 1) + qc; } }
        else if (it == 1) { if (xcd < 4) { if (qr < 6) slot = peer + 1 + (qr & 1); } else { slot = early; ct = 8 + 4 * (x4 & 1) + qc; } }
        else if (it == 2) {
            if (qr == 6) { slot = 25 + (xcd & 6); ct = 8 + 4 * (xcd & 1) + qc; }
            else if (xcd >= 4) slot = peer + 3 + (qr & 1);
            else if (xcd < 2 && qr < 4) slot = 25 + 2 * qr;
        } else {
            int pi = -1; ch = qc & 1;
            if (xcd < 2) { pi = qr - 4; rh = 1; ct = 8 + 2 * xcd + (qc >> 1); }
            else if (qr == 6) { pi = (xcd - 2) >> 1; rh = 1; ct = 12 + 2 * (xcd & 1) + (qc >> 1); }
            else if (xcd < 4) { pi = qr >> 1; ct = 4 * xcd + 2 * (qr & 1) + (qc >> 1); }
            else { pi = qr >> 1; rh = qr & 1; ct = 2 * x4 + (qc >> 1); }
            if (pi >= 0) slot = ((a + 1 + pi) & 3) * 6 + 5;
        }
        if (slot < 0) continue;
        if (slot >= 24 && ct >= 8 && (slot - 24) >> 1 == (ct - 8) >> 1) { slot = 32 + (slot & 1); ct -= 8; }
        if (it < 3) gemm_tile<true>(p, lds, slot, ct, 0, 0, par, call, tid, lane, wave, 4 + it * 3);
        else gemm_tile<false>(p, lds, slot, ct, rh, ch, par, call, tid, lane, wave, 4 + it * 3);
    }
}
}

extern "C" void kernel_launch(int rank, int nranks, ncclComm_t comm,
                              hipStream_t stream,
                              void* const* d_in, const int* in_sizes, int n_in,
                              void* d_out, int out_size,
                              const int* in_dtypes, int out_dtype,
                              void* const* d_out_all,
                              void* const* d_ws_all, size_t ws_bytes) {
    (void)comm; (void)in_dtypes; (void)out_dtype; (void)d_out_all;
    if (nranks != 8 || (unsigned)rank >= 8u || n_in < 2 || ws_bytes < (size_t)agk::WS_NEED || in_sizes[0] != 1024 * 8192 ||
        in_sizes[1] != 8192 * 1024 || out_size != 8192 * 1024) {
        fprintf(stderr, "kernel_launch: unexpected problem shape\n");
        return;
    }
    static bool ready[8] = {false, false, false, false, false, false, false, false};
    if (!ready[rank]) {
        HIP_CHECK(hipFuncSetAttribute((const void*)agk::ag_gemm_kernel, hipFuncAttributeMaxDynamicSharedMemorySize, (int)agk::LDS_BYTES));
        ready[rank] = true;
    }
    agk::Params p;
    p.x = (const float*)d_in[0]; p.w = (const float*)d_in[1]; p.rank = rank; p.pad = 0;
    p.me = (char*)d_ws_all[rank]; p.out = (float*)d_out;
    for (int i = 0; i < 8; ++i) { p.ws[i] = (char*)d_ws_all[i]; p.peer[i] = (char*)d_ws_all[((rank ^ i) & 4) + ((rank + 1 + i) & 3)]; }
    hipLaunchKernelGGL(agk::ag_gemm_kernel, dim3(agk::NWG), dim3(512), agk::LDS_BYTES, stream, p);
    HIP_CHECK(hipGetLastError());
}
```

```cpp
#include <hip/hip_runtime.h>
#include <rccl/rccl.h>
#include <cstdio>

#define HIP_CHECK(expr) do { hipError_t _e = (expr); if (_e != hipSuccess) { \
    fprintf(stderr, "HIP error %s at %s:%d\n", hipGetErrorString(_e), __FILE__, __LINE__); \
    return; } } while (0)

namespace agk {
typedef unsigned u32;
typedef unsigned long long u64;
typedef u32 u32x4 __attribute__((ext_vector_type(4)));
typedef int i32x4 __attribute__((ext_vector_type(4)));
typedef float f32x4 __attribute__((ext_vector_type(4)));
typedef _Float16 f16x8 __attribute__((ext_vector_type(8)));
#define LAS __attribute__((address_space(3)))
typedef __amdgpu_buffer_rsrc_t rsrc_t;
typedef __attribute__((address_space(1))) char gchar;

constexpr int KDIM = 8192, NLOC = 1024, NKT = 128;
constexpr u32 IMG = 16384u;
constexpr u32 PANEL = NKT * IMG;
constexpr u32 UNIT = 131072u;
constexpr int ASLOTS = 34, BSLOTS = 16;
#define HU 2
#define HW 2
#define HINT 1
#define DCUT 16
#define DELTA 4
constexpr u32 CTRL_BYTES = 1u << 20;
constexpr u32 RELAY = (ASLOTS + BSLOTS) * PANEL;
constexpr u32 PAR_BYTES = RELAY + 8u * UNIT;
constexpr u32 OFF_YST = CTRL_BYTES + 2u * PAR_BYTES;
constexpr u32 WS_NEED = OFF_YST + 4u * 32u * UNIT;
constexpr u32 IN_BYTES = 1024u * 8192u * 4u;
constexpr u32 OFF_CTR = 0;
constexpr u32 OFF_TMO = 16384;
constexpr u32 OFF_CL = 20480;
constexpr u32 OFF_LF = 524288;
constexpr u32 OFF_RD = 655360;
constexpr u32 OFF_YR = 24576;
constexpr u32 OFF_RF = 28672;
constexpr u32 OFF_WR = 32768;
constexpr u32 OFF_WS = 33792;
constexpr u32 OFF_WX = 36864;
constexpr u32 OFF_SA = 131072;
constexpr u32 OFF_SB = 196608;
constexpr u32 OFF_DBG = 262144;
constexpr int NWG = 256, NCOMM = 32;
constexpr u32 NUNITS = 96u;
constexpr u32 OOB = 0x80000000u, NOFLAG = 0x40000000u;
constexpr u64 DEADLINE = 30000000ull;
constexpr u32 STAGE = 32768u;
constexpr u32 L_SNAP = 4u * STAGE;
constexpr u32 L_SC = L_SNAP + 512u;
constexpr u32 L_RED = L_SC + 1024u;
constexpr u32 LDS_BYTES = L_RED + 2048u;

struct Params {
    const float* x; char* me; int rank, pad; const float* w; float* out;
    char* ws[8];
    char* peer[8];
};

#define AUX_SC1 16
#define AUX_SYS 17
#define GL_AUX 16
#define W_LD 3
#define W_ST 2
#define X_LD 4
#define X_ST 2
#define WAIT_VM(n) asm volatile("s_waitcnt vmcnt(%0)" :: "n"(n) : "memory")
#define WAIT_LGKM0() asm volatile("s_waitcnt lgkmcnt(0)" ::: "memory")
#define BAR() do { asm volatile("" ::: "memory"); __builtin_amdgcn_s_barrier(); asm volatile("" ::: "memory"); } while (0)
#define WAIT_VM_TRACKED(n) do { asm volatile("" ::: "memory"); __builtin_amdgcn_s_waitcnt(((n) & 15) | (7 << 4) | (15 << 8) | (((n) >> 4) << 14)); asm volatile("" ::: "memory"); } while (0)

__device__ __forceinline__ rsrc_t mk_rsrc(const void* p, u32 bytes) { return __builtin_amdgcn_make_buffer_rsrc((void*)p, 0, bytes, 0x00020000); }
__device__ __forceinline__ u32 ld_agent(const void* p) { return __hip_atomic_load((const u32*)p, __ATOMIC_RELAXED, __HIP_MEMORY_SCOPE_AGENT); }
__device__ __forceinline__ u32 ld_sys(const void* p) { return __hip_atomic_load((const u32*)p, __ATOMIC_RELAXED, __HIP_MEMORY_SCOPE_SYSTEM); }
__device__ __forceinline__ void st_agent(void* p, u32 v) { __hip_atomic_store((u32*)p, v, __ATOMIC_RELAXED, __HIP_MEMORY_SCOPE_AGENT); }
__device__ __forceinline__ void st_sys_g(gchar* p, u32 v) { __hip_atomic_store((__attribute__((address_space(1))) u32*)p, v, __ATOMIC_RELAXED, __HIP_MEMORY_SCOPE_SYSTEM); }
__device__ __forceinline__ void st_sys(void* p, u32 v) { __hip_atomic_store((u32*)p, v, __ATOMIC_RELAXED, __HIP_MEMORY_SCOPE_SYSTEM); }

#define STAMP(ws, idx) do { } while (0)

__device__ __forceinline__ bool give_up(char* ws, u32 call, u64 t0) {
    const u32 t = __builtin_amdgcn_readfirstlane(ld_agent(ws + OFF_TMO));
    if (t == call || __builtin_amdgcn_s_memrealtime() - t0 > DEADLINE) { st_agent(ws + OFF_TMO, call); return true; }
    return false;
}
__device__ __forceinline__ void poll_ge(const char* p, u32 need, char* ws, u32 call) {
    const u64 t0 = __builtin_amdgcn_s_memrealtime();
    u32 n = 0;
    for (;;) {
        const u32 v = ld_sys(p);
        if (__all((int)(v - need) >= 0)) return;
        if ((++n & 15u) == 0u && give_up(ws, call, t0)) return;
        __builtin_amdgcn_s_sleep(2);
    }
}
__device__ __forceinline__ bool all4_ge(u32x4 v, u32 need) {
    return __all((int)(v[0] - need) >= 0 && (int)(v[1] - need) >= 0 && (int)(v[2] - need) >= 0 && (int)(v[3] - need) >= 0);
}
__device__ __forceinline__ void poll4_ge(rsrc_t rs, u32 off, u32 need, char* ws, u32 call) {
    const u64 t0 = __builtin_amdgcn_s_memrealtime();
    u32 n = 0;
    for (;;) {
        asm volatile("" ::: "memory");
        const u32x4 v = __builtin_amdgcn_raw_buffer_load_b128(rs, off, 0, AUX_SYS);
        if (all4_ge(v, need)) return;
        if ((++n & 15u) == 0u && give_up(ws, call, t0)) return;
        __builtin_amdgcn_s_sleep(1);
    }
}

__device__ __forceinline__ u32 umax(u32 a, u32 b) { return a > b ? a : b; }
__device__ __forceinline__ u32 absbits(float f) { return __float_as_uint(f) & 0x7fffffffu; }
__device__ __forceinline__ int exp_of(u32 amaxbits) { int e = (int)(amaxbits >> 23); return e < 15 ? 15 : e; }
__device__ __forceinline__ float scale_of(int e) { return __uint_as_float((u32)(268 - e) << 23); }
__device__ __forceinline__ u32x4 cvt8(f32x4 a, f32x4 b, float s) {
    f16x8 h;
    h[0] = (_Float16)(a[0] * s); h[1] = (_Float16)(a[1] * s); h[2] = (_Float16)(a[2] * s); h[3] = (_Float16)(a[3] * s);
    h[4] = (_Float16)(b[0] * s); h[5] = (_Float16)(b[1] * s); h[6] = (_Float16)(b[2] * s); h[7] = (_Float16)(b[3] * s);
    return __builtin_bit_cast(u32x4, h);
}

template <int NR, bool HEAD, class F>
__device__ __forceinline__ gchar* x_rows(const Params& p, LAS char* lds, int row0, const u32& par, const u32& call, int tid, int wave, F mid) {
    const int a = p.rank & 3;
    char* ws = p.me;
    const rsrc_t rx = mk_rsrc(p.x, IN_BYTES), rw = mk_rsrc(ws, WS_NEED);
    const u32 pi0 = (u32)((7 - wave) * 128 + (tid & 63));
    f32x4 v[NR][4];
#pragma unroll
    for (int r = 0; r < NR; ++r)
#pragma unroll
        for (int u = 0; u < 2; ++u) {
            const u32 off = (u32)(row0 + r) * (KDIM * 4u) + (pi0 + (u32)u * 64u) * 32u;
            v[r][2 * u] = __builtin_bit_cast(f32x4, __builtin_amdgcn_raw_buffer_load_b128(rx, off, 0, 0));
            v[r][2 * u + 1] = __builtin_bit_cast(f32x4, __builtin_amdgcn_raw_buffer_load_b128(rx, off + 16u, 0, 0));
            if (!HEAD) WAIT_VM(X_LD - 2);
        }
    mid();
    const int rb = row0 >> 7;
    const u32 slot = rb < 6 ? (u32)(a * 6 + rb) : (u32)(32 + rb - 6);
    const u32 rslot = rb < 6 ? slot : (u32)(24 + 2 * a + rb - 6);
    char* hp[4];
#pragma unroll
    for (int i = 0; i < 4; ++i) hp[i] = rb < 6 ? p.peer[i] : p.peer[4 + i];
    char* fifth = rb < 6 ? p.peer[7] : ws;
    asm volatile("" : "+s"(hp[0]), "+s"(hp[1]), "+s"(hp[2]), "+s"(hp[3]), "+s"(fifth));
    gchar* user = (gchar*)((tid & 4) ? fifth : (tid & 2) ? ((tid & 1) ? hp[3] : hp[2]) : ((tid & 1) ? hp[1] : hp[0]));
    const u32 wr = (u32)(tid & 63) >> 3;
    const bool wsend = wr < (u32)NR && ((tid & 7) < 4 || ((tid & 7) == 4 && rb >= 5));
    const u32 woff = OFF_SA + ((par * 64u + ((tid & 4) ? (rb < 6 ? 34u : slot) : rslot)) * 128u + (((u32)row0 + wr) & 127u)) * 4u;
    u32 word = 0;
    LAS u32* red = (LAS u32*)(lds + L_RED);
#pragma unroll
    for (int r = 0; r < NR; ++r) {
        u32 m = 0;
#pragma unroll
        for (int i = 0; i < 4; ++i)
#pragma unroll
            for (int j = 0; j < 4; ++j) m = umax(m, absbits(v[r][i][j]));
        m = umax(m, (u32)__builtin_amdgcn_update_dpp(0, (int)m, 0xb1, 15, 15, true));
        m = umax(m, (u32)__builtin_amdgcn_update_dpp(0, (int)m, 0x4e, 15, 15, true));
        m = umax(m, (u32)__builtin_amdgcn_update_dpp(0, (int)m, 0x141, 15, 15, true));
        m = umax(m, (u32)__builtin_amdgcn_update_dpp(0, (int)m, 0x140, 15, 15, true));
        m = umax(umax((u32)__builtin_amdgcn_readlane((int)m, 0), (u32)__builtin_amdgcn_readlane((int)m, 16)),
                 umax((u32)__builtin_amdgcn_readlane((int)m, 32), (u32)__builtin_amdgcn_readlane((int)m, 48)));
        if ((tid & 63) == 0) red[r * 8 + wave] = m;
    }
    asm volatile("" : "+v"(user));
    WAIT_LGKM0(); BAR();
#pragma unroll
    for (int r = 0; r < NR; ++r) {
        const u32x4 m0 = *(LAS u32x4*)(red + r * 8), m1 = *(LAS u32x4*)(red + r * 8 + 4);
        const u32 m = umax(umax(umax(m0[0], m0[1]), umax(m0[2], m0[3])), umax(umax(m1[0], m1[1]), umax(m1[2], m1[3])));
        const int e = exp_of(m); const float s = scale_of(e);
        const u32 rin = (u32)((row0 + r) & 127);
        if (wr == (u32)r) word = call | ((u32)(e - 141) & 255u);
        const u32 off = CTRL_BYTES + par * PAR_BYTES + rin * 128u + (pi0 >> 3) * IMG + (((pi0 & 7u) ^ ((rin >> 1) & 7u)) << 4);
        const u32x4 h0 = cvt8(v[r][0], v[r][1], s), h1 = cvt8(v[r][2], v[r][3], s);
        __builtin_amdgcn_raw_buffer_store_b128(h0, rw, off + slot * PANEL, 0, AUX_SC1);
        __builtin_amdgcn_raw_buffer_store_b128(h1, rw, off + slot * PANEL + 8u * IMG, 0, AUX_SC1);
        if (!HEAD) WAIT_VM(X_ST - 2);
        if (HEAD) {
            u32x4 h2 = h0;
#define HSUB(k) do { _Pragma("unroll") for (int i = 0; i < 4; ++i) { \
                __builtin_amdgcn_raw_buffer_store_b128((k) < 4 ? h0 : (k) < 8 ? h1 : h2, mk_rsrc(hp[i], WS_NEED), \
                    ((i == 3 && rb < 6) || ((tid >> 4) & 3) != ((k) & 3)) ? OOB : off + rslot * PANEL + (u32)((k) >> 2) * 8u * IMG, 0, AUX_SYS); } } while (0)
            if (wave == 7) {
#pragma unroll
                for (int k = 0; k < HW; ++k) HSUB(k);
                WAIT_VM(4 * HW);
            } else {
                if (HU > 2 && wave == 6) { ((LAS u32x4*)lds)[448 + (tid & 63)] = h0; WAIT_LGKM0(); }
                WAIT_VM(0);
            }
            BAR();
            if (wave == 7) {
                if (wsend) st_sys_g(user + woff, word);
                if (HU > 2) h2 = ((LAS u32x4*)lds)[448 + (tid & 63)];
#pragma unroll
                for (int k = HW; k < 4 * HU; ++k) {
                    if (k < 2 * HW) WAIT_VM(4 * HW - 3); else WAIT_VM(4 * HW - 4);
                    HSUB(k);
                }
            }
#undef HSUB
        }
    }
    if (!HEAD) {
        WAIT_VM(0); BAR();
        if (wave == 0 && wsend) st_sys_g(user + woff, word);
    }
    return user + OFF_RD + ((u32)p.rank * 128u + (u32)(row0 & 127)) * 4u;
}

__device__ __forceinline__ int slot_row(int rank, int slot) {
    const int g = rank >> 2;
    if (slot < 24) { const int i = slot / 6; return (g * 4 + i) * 1024 + (slot - i * 6) * 128; }
    return (slot < 32 ? (1 - g) * 4 + ((slot - 24) >> 1) : rank) * 1024 + 768 + (slot & 1) * 128;
}

template <bool BIG>
__device__ __forceinline__ void gemm_tile(const Params& p, LAS char* lds, int slot, int ct, int rh, int ch, u32 par, u32 call, int tid, int lane, int wave, int stamp0) {
    constexpr int NB = BIG ? 4 : 2, NC = BIG ? 32 : 16, BT = BIG ? 128 : 64;
    const int rank = p.rank, g = rank >> 2, a = rank & 3;
    char* ws = p.me;
    const rsrc_t rws = mk_rsrc(ws, WS_NEED);
    int a_src; u32 a_base, a_sh;
    if (slot < 24) { const int i = slot / 6, a_rb = slot - i * 6; a_src = i == a ? -1 : g * 4 + i;
        a_base = a_rb == 0 ? 0u : a_rb < 3 ? 16u : a_rb < 5 ? 48u : 80u + (u32)rh * 8u; a_sh = a_rb == 0 ? 1u : 0u; }
    else { a_src = slot < 32 ? (1 - g) * 4 + ((slot - 24) >> 1) : -1; a_base = (slot & 1) ? 48u : 0u; a_sh = 1u; }
    const int b_src = ct < 8 ? -1 : (1 - g) * 4 + ((ct - 8) >> 1);
    const int src = lane < 32 ? a_src : (BIG ? b_src : -1);
    const u32 gsh = src < 0 ? 31u : lane < 32 && BIG ? a_sh : 0u;
    const char* gptr = ws + (src >= 0 ? OFF_LF + ((u32)src * NUNITS + (lane < 32 ? a_base : 16u)) * 128u + (u32)(lane & (BIG ? 31 : 15)) * 4u : OFF_CTR + blockIdx.x * 64u + 4u);
    const u32 gmin = src >= 0 && lane < 32 && a_base == 0u ? (u32)HU : 0u;
#define GPTR(c) (gptr + (umax((u32)(c) >> gsh, gmin) << (BIG ? 7 : 6)))
    if (wave == 0) {
        if (a_src < 0 || (!BIG && rh == 0)) poll4_ge(rws, OFF_SA + (par * 64u + (u32)slot) * 512u + (u32)(lane & (BIG ? 31 : 15)) * 16u, call, ws, call);
        if (b_src < 0) poll4_ge(rws, OFF_WR + (u32)ct * 128u + (u32)(lane & 7) * 16u, call, ws, call);
        else if (!BIG) for (u32 i = 0; i < 4u; ++i) poll4_ge(rws, OFF_LF + ((u32)b_src * NUNITS + 16u) * 128u + i * 1024u + (u32)lane * 16u, call, ws, call);
        if (a_src >= 0 && a_base == 0u) poll4_ge(rws, OFF_RD + (u32)a_src * 512u + (u32)(lane & 31) * 16u, call, ws, call);
        poll_ge(GPTR(0), call, ws, call);
        WAIT_VM(0);
    }
    BAR();
    if (wave == 0) STAMP(ws, stamp0);
    constexpr u32 RB = BT * 4u, PPR = BT / 4u, AREA = BT * RB;
    const u32 pc = (u32)lane & (PPR - 1u);
    const u32 col0 = BIG ? 0u : (u32)ch * 64u, row0 = BIG ? 0u : (u32)rh * 64u;
    const bool remote = ct >= 8;
    const rsrc_t ro = mk_rsrc(remote ? (const void*)ws : (const void*)p.out, remote ? WS_NEED : IN_BYTES);
    const u32 ostr = remote ? 1024u : NLOC * 4u;
    const u32 obase = (remote ? OFF_YST + (u32)(((ct - 8) >> 1) * 32 + slot) * UNIT + (u32)(ct & 1) * 512u : ((u32)slot_row(rank, slot) * NLOC + (u32)ct * 128u) * 4u) + col0 * 4u + row0 * ostr + pc * 16u;
#define YFLAG() do { if (tid < (BIG ? 2 : 1)) st_sys(p.ws[b_src] + OFF_YR + (u32)(a * 64 + rh * 32 + slot) * 16u + (u32)(ct & 1) * 8u + (u32)(tid + ch) * 4u, call); } while (0)
    const bool hint = HINT >= 0 && !BIG && remote;
    if (hint) {
        const u32x4 zero = {0u, 0u, 0u, 0u};
#pragma unroll
        for (u32 it = 0; it < 2u; ++it) __builtin_amdgcn_raw_buffer_store_b128(zero, ro, obase + ((u32)wave * (BT / 8u) + (it * 64u + (u32)lane) / PPR) * ostr, 0, AUX_SC1);
    }
    const u32 pb = CTRL_BYTES + par * PAR_BYTES;
    const char* gA = ws + pb + (u32)slot * PANEL + (BIG ? (u32)wave * 2048u : (u32)rh * 8192u + (u32)wave * 1024u) + (u32)lane * 16u;
    const char* gB = ws + pb + (u32)(ASLOTS + ct) * PANEL + (BIG ? (u32)wave * 2048u : (u32)ch * 8192u + (u32)wave * 1024u) + (u32)lane * 16u;
    const u32 ldsw = (u32)wave * (BIG ? 2048u : 1024u);
    constexpr u32 GSTEP = BIG ? IMG : 2u * IMG, G2 = BIG ? 1024u : IMG, L2O = BIG ? 1024u : 8192u;
#define GLDS16(g, l) __builtin_amdgcn_global_load_lds((const u32*)(g), (LAS u32*)(lds + (l)), 16, 0, GL_AUX)
#define LOADT(t, st) do { const u32 _o = (u32)(t) * GSTEP; \
        GLDS16(gA + _o, (st) * STAGE + ldsw); GLDS16(gA + _o + G2, (st) * STAGE + ldsw + L2O); \
        GLDS16(gB + _o, (st) * STAGE + IMG + ldsw); GLDS16(gB + _o + G2, (st) * STAGE + IMG + ldsw + L2O); } while (0)
#define SNAP(c) __builtin_amdgcn_global_load_lds((const u32*)((c) < NC ? GPTR(c) : ws + OFF_TMO), (LAS u32*)(lds + L_SNAP + ((u32)(c) & 1u) * 256u), 4, 0, AUX_SYS)
    const int ks = BIG ? wave >> 2 : wave >> 1, wm = BIG ? (wave >> 1) & 1 : 0, wn = wave & 1;
    const u32 swz = (u32)((((ks & 1) * 4 + (lane >> 4)) ^ ((lane & 15) >> 1)) << 4) + (BIG ? 0u : (u32)(ks >> 1) * 8192u);
    const u32 ard = (u32)(wm * 64 + (lane & 15)) * 128u + swz, brd = IMG + (u32)(wn * (BT / 2) + (lane & 15)) * 128u + swz;
#define READF(fa, fb, st) do { _Pragma("unroll") for (int _i = 0; _i < 4; ++_i) { \
        if (_i < NB) fb[_i] = *(const LAS f16x8*)(lds + (st) * STAGE + brd + _i * 2048u); \
        fa[_i] = *(const LAS f16x8*)(lds + (st) * STAGE + ard + _i * 2048u); } } while (0)
#define MMA(fa, fb) do { _Pragma("unroll") for (int _m = 0; _m < 4; ++_m) _Pragma("unroll") for (int _n = 0; _n < NB; ++_n) \
        acc[_m][_n] = __builtin_amdgcn_mfma_f32_16x16x32_f16(fb[_n], fa[_m], acc[_m][_n], 0, 0, 0); } while (0)
    f32x4 acc[4][NB];
#pragma unroll
    for (int m = 0; m < 4; ++m)
#pragma unroll
        for (int n = 0; n < NB; ++n) acc[m][n] = (f32x4){0.f, 0.f, 0.f, 0.f};
    f16x8 a0[4], b0[4], a1[4], b1[4];
    if (wave >= 4) {
        const u32 so = (wave < 6 ? OFF_SA + (par * 64u + (u32)slot) * 512u : OFF_SB + (par * BSLOTS + (u32)ct) * 512u) + (u32)((wave & 1) * 64 + lane) * 4u;
        __builtin_amdgcn_global_load_lds((const u32*)(ws + so), (LAS u32*)(lds + L_SC + (u32)(wave - 4) * 256u), 4, 0, AUX_SYS);
    }
    LOADT(0, 0u);
    if (wave == 0) SNAP(1);
    LOADT(1, 1u); LOADT(2, 2u); LOADT(3, 3u);
    if (wave == 0) WAIT_VM(13); else WAIT_VM(12);
    BAR();
    READF(a0, b0, 0u);
#define GATE(c) do { const u32 _v = *(const LAS u32*)(lds + L_SNAP + ((u32)(c) & 1u) * 256u + (u32)lane * 4u); \
        if (!__all((int)(_v - call) >= 0)) { if (wave == 0) { poll_ge(GPTR(c), call, ws, call); WAIT_VM(0); } BAR(); } } while (0)
    for (int c = 0; c < NC - 1; ++c) {
        const int t = 4 * c;
        WAIT_VM(8); WAIT_LGKM0(); BAR();
        GATE(c + 1);
        LOADT(t + 4, 0u); READF(a1, b1, 1u); MMA(a0, b0);
        WAIT_VM(8); WAIT_LGKM0(); BAR();
        if (wave == 0) SNAP(c + 2);
        LOADT(t + 5, 1u); READF(a0, b0, 2u); MMA(a1, b1);
        if (wave == 0) WAIT_VM(9); else WAIT_VM(8);
        WAIT_LGKM0(); BAR();
        LOADT(t + 6, 2u); READF(a1, b1, 3u); MMA(a0, b0);
        if (wave == 0) WAIT_VM(9); else WAIT_VM(8);
        WAIT_LGKM0(); BAR();
        LOADT(t + 7, 3u); READF(a0, b0, 0u); MMA(a1, b1);
    }
#define DSTEP(k, n) do { if ((k) > HINT && hint && wave == 0) WAIT_VM((n) + 1); else WAIT_VM(n); WAIT_LGKM0(); BAR(); if ((k) == HINT && hint) YFLAG(); } while (0)
    DSTEP(0, 8);
    READF(a1, b1, 1u); MMA(a0, b0);
    DSTEP(1, 4);
    READF(a0, b0, 2u); MMA(a1, b1);
    DSTEP(2, 0);
    READF(a1, b1, 3u); MMA(a0, b0);
    WAIT_LGKM0(); BAR();
    if (HINT == 3 && hint) YFLAG();
    MMA(a1, b1);
#undef DSTEP
    if (wave == 0) STAMP(ws, stamp0 + 1);
#pragma unroll
    for (int m = 0; m < 4; ++m)
#pragma unroll
        for (int n = 0; n < NB; ++n) {
            const u32 r = (u32)(wm * 64 + m * 16 + (lane & 15)), pc = (u32)(wn * (BT / 8) + n * 4 + (lane >> 4));
            *(LAS f32x4*)(lds + (u32)ks * AREA + r * RB + ((pc ^ (r & 7u)) << 4)) = acc[m][n];
        }
    WAIT_LGKM0(); BAR();
    const LAS int* sc = (const LAS int*)(lds + L_SC);
    const i32x4 eb = *(const LAS i32x4*)(sc + 128 + col0 + pc * 4u);
    const bool bad = *(const LAS u32*)(lds + L_SNAP) == call;
#pragma unroll
    for (u32 it = 0; it < (BIG ? 8u : 2u); ++it) {
        const u32 r = (u32)wave * (BT / 8u) + (it * 64u + (u32)lane) / PPR;
        const u32 lo = r * RB + ((pc ^ (r & 7u)) << 4);
        f32x4 s = *(const LAS f32x4*)(lds + lo) + *(const LAS f32x4*)(lds + AREA + lo);
        if (!BIG) s += *(const LAS f32x4*)(lds + 2u * AREA + lo) + *(const LAS f32x4*)(lds + 3u * AREA + lo);
        const int ea = (int)(signed char)sc[row0 + r];
#pragma unroll
        for (int j = 0; j < 4; ++j) s[j] = bad ? __builtin_nanf("") : __builtin_amdgcn_ldexpf(s[j], ea + eb[j]);
        u32x4 o = __builtin_bit_cast(u32x4, s);
        if (HINT >= 0 && !BIG) {
#pragma unroll
            for (int j = 0; j < 4; ++j) o[j] = o[j] ? o[j] : 0x80000000u;
        }
        __builtin_amdgcn_raw_buffer_store_b128(o, ro, obase + r * ostr, 0, AUX_SC1);
    }
    if (remote && (BIG || HINT < 0)) {
        WAIT_VM(0); BAR();
        YFLAG();
    }
    if (wave == 0) STAMP(ws, stamp0 + 2);
#undef GLDS16
#undef LOADT
#undef SNAP
#undef READF
#undef MMA
#undef GATE
#undef YFLAG
#undef GPTR
}

constexpr u32 N0 = 8u - DELTA + (DCUT ? 1u : 0u);
constexpr u32 NPOS_INTRA = 88u - HU + N0, NPOS_CROSS = 97u - HU + DELTA;
constexpr u32 FAN = 0x20000000u;
struct Desc { u32 src, dst, gate, flag; };
struct Stream {
    bool cross; int rank, g, a, j; u32 pb, sa;
    __device__ __forceinline__ int yslot(u32 k) const {
        if (k < 6u) return j * 6 + (int)k;
        if (k < 9u) return ((j + (int)k - 5) & 3) * 6;
        if (k < 12u) return 24 + 2 * ((a + (int)k - 8) & 3);
        if (k < 18u) return ((j + 1 + (((int)k - 12) >> 1)) & 3) * 6 + 1 + ((int)k & 1);
        if (k < 21u) return 25 + 2 * ((a + (int)k - 17) & 3);
        if (k < 27u) return ((j + 1 + (((int)k - 21) >> 1)) & 3) * 6 + 3 + (((int)k - 21) & 1);
        return ((j + 1 + (int)(k % 3u)) & 3) * 6 + 5;
    }
    __device__ __forceinline__ u32 single(u32 slot, u32 k) const { return pb + slot * PANEL + k * UNIT; }
    __device__ __forceinline__ u32 pair(u32 slot, u32 k) const { return pb + slot * PANEL + k * 65536u; }
    __device__ __forceinline__ u32 half(u32 slot, u32 h, u32 k) const { return pb + slot * PANEL + k * 2u * UNIT + h * 8192u; }
    __device__ __forceinline__ Desc at(u32 pos) const {
        Desc d;
        const u32 s = (u32)j * 6u, M8 = 7u << 20, PAIR = 1u << 28, HALF = 4u << 28;
        u32 u, fsrc = (u32)((cross ? 1 - g : g) * 4 + j);
        if (!cross) {
            if (pos >= NPOS_INTRA) pos = NPOS_INTRA - 1u;
            if (pos < 80u - HU) {
                u = pos + HU;
                d.src = u < 16u ? single(s, u) : u < 48u ? pair(s + 1u, u - 16u) : pair(s + 3u, u - 48u);
                d.gate = sa + s * 512u + (u < 16u ? 0u : u < 48u ? 512u + PAIR + M8 : 1536u + PAIR + M8);
            } else {
                const u32 e = NPOS_INTRA - 1u - pos, n0 = N0;
                const u32 h = e < 2u || e - 2u >= 2u * n0 || (e & 1u) ? 1u : 0u;
                const u32 k = e < 2u ? 7u - e : e - 2u >= 2u * n0 ? 5u - n0 - (e - 2u - 2u * n0) : h ? 5u - ((e - 2u) >> 1) : 7u - ((e - 2u) >> 1);
                u = 80u + h * 8u + k;
                d.src = half(s + 5u, h, k);
                d.gate = sa + (s + 5u) * 512u + h * 256u + M8 + HALF;
            }
            d.dst = d.src;
            d.flag = OFF_LF + (fsrc * NUNITS + u) * 128u;
        } else {
            if (pos >= NPOS_CROSS) pos = NPOS_CROSS - 1u;
            pos += HU;
            const u32 hop = j == a ? 64u : 68u + DELTA;
            if (pos < 64u) {
                u = pos;
                d.src = u < 16u ? single(32u, u) : u < 48u ? pair((u32)(ASLOTS + 2 * a), u - 16u) : single(33u, u - 48u);
                d.dst = u < 16u ? single((u32)(24 + 2 * j), u) : u < 48u ? pair((u32)(ASLOTS + 8 + 2 * j), u - 16u) : single((u32)(25 + 2 * j), u - 48u);
                d.gate = u < 16u ? sa + (u32)(24 + 2 * j) * 512u : u < 48u ? OFF_WS + (u32)j * 256u + (1u << 20) + (3u << 28) : sa + (u32)(25 + 2 * j) * 512u;
                d.flag = OFF_LF + (fsrc * NUNITS + u) * 128u;
            } else if (pos - hop < DELTA) {
                const u32 k = pos - hop;
                if (j == a) { d.src = half(s + 5u, 0u, k); d.dst = pb + RELAY + k * 2u * UNIT; d.gate = sa + 34u * 512u + M8 + HALF;
                              d.flag = FAN + OFF_RF + ((u32)a * 4u + k) * 128u; }
                else { d.src = pb + RELAY + k * 2u * UNIT; d.dst = half(s + 5u, 0u, k); d.gate = OFF_RF + ((u32)j * 4u + k) * 128u + (6u << 28);
                       d.flag = OFF_LF + ((fsrc ^ 4u) * NUNITS + 80u + k) * 128u; }
            } else {
                const u32 k = pos < hop ? pos - 64u : pos - 64u - DELTA;
                const int ys = yslot(k);
                const u32 h = k >= 30u ? 1u : 0u;
                d.src = OFF_YST + (u32)(a * 32 + ys) * UNIT + h * 65536u;
                d.dst = ((u32)slot_row((int)fsrc, ys) + h * 64u) * (NLOC * 4u) + (u32)j * 1024u;
                d.gate = OFF_YR + ((u32)(j * 64 + ys) + h * 32u) * 16u + ((k >= 27u ? 10u : 2u) << 28);
                d.flag = NOFLAG;
            }
        }
        return d;
    }
};
struct Tables { Desc lo, hi; };
__device__ __forceinline__ u32 tab(u32 t0, u32 t1, u32 i) {
    const u32 x = (u32)__builtin_amdgcn_readlane((int)t0, (int)(i & 63u)), y = (u32)__builtin_amdgcn_readlane((int)t1, (int)(i & 63u));
    return i < 64u ? x : y;
}

#define PSLEEP0 10
#define PSLEEP 4
#define NACT 32
__device__ __forceinline__ void pull_stream(const Params& p, LAS char* lds, const Tables& T, bool cross, u32 cw, char* src, gchar* fan, u32 calln, u32 call, int lane, int wave) {
    const int hw = wave;
    char* ws = p.me;
    const rsrc_t rl = mk_rsrc(ws, WS_NEED), rs = mk_rsrc(src, WS_NEED), rout = mk_rsrc(p.out, IN_BYTES);
    LAS char* pl = lds + (u32)hw * 1024u;
    const u32 pla = (u32)(size_t)pl + (u32)lane * 4u;
    const u32 l16 = (u32)lane * 16u, l4r = (u32)(lane >> 5) * IMG + (l16 & 511u);
    const u32 l0 = lane == 0 ? 0u : OOB;
    const u32 co = OFF_CL + (u32)wave * 128u + l0;
    const u32 NPIECES = (cross ? NPOS_CROSS : NPOS_INTRA) * 32u;
    const u32 skip = !DCUT ? 0u : cross ? (u32)DCUT : 32u - DCUT;
    const u32 qcut = cross ? ((wave == 3 ? 64u : 68u + DELTA) + DELTA - HU) * 32u - DCUT : (NPOS_INTRA - 1u - 2u * N0) * 32u;
    const u32 cbase = 2u * NACT - (calln - 1u) * (NPIECES - skip);
    u32x4 v0 = {0u, 0u, 0u, 0u}, v1 = v0, v2 = v0, v3;
    u32 q = OOB, so = OOB, ss = 1024u;
    u32 c0 = cw, c1 = cw + NACT;
    u32 pf = NOFLAG;
    bool claimed = false, dead = false;
    int cl = 0;
#define KIND(q) (tab(T.lo.gate, T.hi.gate, (q) >> 5) >> 28)
#define POFF(q, kd) (((kd) & 1u) ? (((q) >> 4) & 1u) * PANEL + ((q) & 15u) * 4096u : ((kd) & 4u) ? (((q) >> 1) & 15u) * IMG + ((q) & 1u) * 4096u : ((q) & 31u) * (((kd) & 8u) ? 2048u : (kd) ? 4096u : 512u))
#define GBASE(q) ({ const u32 _g = tab(T.lo.gate, T.hi.gate, (q) >> 5), _k = _g >> 28; \
        const u32 _ix = (((_k & 1u) ? ((q) >> 2) & 4u : 0u) + ((q) & (_k == 4u ? 1u : 3u))) << (_k == 3u ? 5 : 7); \
        (_g & 0xfffffu) + ((_k & 7u) == 2u ? 0u : _k == 6u ? ((q) & 28u) * 4u : _k == 0u ? ((q) & 31u) * 16u : _ix); })
#define GOFF(q) (GBASE(q) + (l16 & (tab(T.lo.gate, T.hi.gate, (q) >> 5) >> 16)))
#define PFLAG() st_sys_g((pf & FAN) ? fan + (pf - FAN) : (gchar*)ws + (pf >= NOFLAG ? OFF_CTR + cw * 64u + 8u + (u32)hw * 4u : pf), call)
#define PLD(u, o) v##u = __builtin_amdgcn_raw_buffer_load_b128(rs, (o) + (u) * ss, 0, AUX_SYS)
#define PSTA(u, o) do { asm volatile("" : "+v"(v##u)); \
        __builtin_amdgcn_raw_buffer_store_b128(v##u | (dead ? 0x7fc00000u : 0u), rd, o, 0, AUX_SC1); } while (0)
#define PST(u) PSTA(u, d + (u) * ds)
#define PSTAMP(i) do { } while (0)
#define GPOLL(i, sl) do { __builtin_amdgcn_global_load_lds((const u32*)gp, (LAS u32*)(pl + (i) * 256u), 4, 0, AUX_SYS); __builtin_amdgcn_s_sleep(sl); } while (0)
#define GSTEP(i, sl) { GPOLL(((i) + 3) & 3, sl); WAIT_VM_TRACKED(3); u32 _w; \
            asm volatile("ds_read_b32 %0, %1 offset:%2\n\ts_waitcnt lgkmcnt(0)" : "=v"(_w) : "v"(pla), "n"((i) * 256) : "memory"); \
            if (__all((int)(_w - call) >= 0)) break; }
#define WATCH(sl) do { \
        const char* gp = ws + GBASE(c0) + ((u32)lane * 4u & ((tab(T.lo.gate, T.hi.gate, c0 >> 5) >> 16 & 0x3f0u) | 12u)); \
        const u64 t0 = __builtin_amdgcn_s_memrealtime(); \
        GPOLL(0, sl); GPOLL(1, sl); GPOLL(2, sl); \
        for (u32 n = 1;; ++n) { \
            GSTEP(0, sl) GSTEP(1, sl) GSTEP(2, sl) GSTEP(3, sl) \
            if ((n & 15u) == 0u && give_up(ws, call, t0)) { dead = true; break; } \
        } } while (0)
    WATCH(PSLEEP0);
    u32x4 gv = {call, call, call, call};
    u32 d = OOB, ds = 1024u, so2 = OOB, d2 = OOB; bool y = false, val = false;
#define VALID(u) __all(v##u[0] != 0u && v##u[1] != 0u && v##u[2] != 0u && v##u[3] != 0u)
#define RETRY(u) do { if (HINT >= 0 && val && !dead && !VALID(u)) { \
        const u64 t0 = __builtin_amdgcn_s_memrealtime(); u32 n = 0; \
        do { PLD(u, so); NRETRY; \
            if ((++n & 15u) == 0u && give_up(ws, call, t0)) dead = true; \
            WAIT_VM_TRACKED(0); \
        } while (!dead && !VALID(u)); } } while (0)
#define NRETRY do { } while (0)
    do {
        const rsrc_t rd = y ? rout : rl;
        WAIT_VM_TRACKED(5); PLD(2, so2); RETRY(0); PST(0);
        WAIT_VM_TRACKED(4); PLD(3, so2); PFLAG(); RETRY(1); PST(1);
        WAIT_VM_TRACKED(5);
        bool open = dead || all4_ge(gv, call);
        if (q == OOB && !open) { WATCH(PSLEEP); open = true; }
        pf = q != OOB ? tab(T.lo.flag, T.hi.flag, q >> 5) + (q & 31u) * 4u : NOFLAG;
        if (claimed) { c1 = (u32)__builtin_amdgcn_readfirstlane(cl) + cbase; if (c1 >= qcut) c1 += skip; if (c1 > NPIECES + 127u) c1 = NPIECES + 127u; }
        claimed = open && c0 < NPIECES;
        q = claimed ? c0 : OOB; if (claimed) c0 = c1;
        const u32 qq = claimed ? q : 0u;
        const u32 kn = claimed ? KIND(qq) : 1u, po = POFF(qq, kn), lo = kn ? l16 : l4r;
        so = claimed ? tab(T.lo.src, T.hi.src, qq >> 5) + po + lo : OOB;
        ss = kn ? 1024u : 2u * IMG;
        y = (kn & 7u) == 2u;
        u32 dn = claimed ? tab(T.lo.dst, T.hi.dst, qq >> 5) + (y ? po * 4u : po) + lo : OOB;
        u32 go = GOFF(c0), ao = claimed ? co : OOB, so1 = so + ss, dx2 = d2 + 2u * ds, dx3 = d2 + 3u * ds;
        if (claimed) PSTAMP(q >> 5);
        asm volatile("" : "+v"(so), "+v"(dn), "+v"(go), "+v"(ao), "+v"(so1), "+v"(dx2), "+v"(dx3));
        WAIT_VM_TRACKED(4);
        v0 = __builtin_amdgcn_raw_buffer_load_b128(rs, so, 0, AUX_SYS);
        PSTA(2, dx2);
        WAIT_VM_TRACKED(4);
        PSTA(3, dx3);
        v1 = __builtin_amdgcn_raw_buffer_load_b128(rs, so1, 0, AUX_SYS);
        cl = __builtin_amdgcn_raw_ptr_buffer_atomic_add_i32(1, rl, ao, 0, AUX_SC1);
        gv = __builtin_amdgcn_raw_buffer_load_b128(rl, go, 0, AUX_SYS);
        d = dn; ds = y ? NLOC * 4u : ss;
        val = (kn & 8u) != 0u;
        so2 = val ? OOB : so; d2 = val ? OOB : dn;
    } while (c0 < NPIECES || q != OOB);
    WAIT_VM_TRACKED(0); PFLAG();
    asm volatile(".Lagk_end:");
    if (hw < 7) STAMP(ws, 2 + wave * 8 + 7);
#undef PSTAMP
#undef GOFF
#undef GBASE
#undef WATCH
#undef GSTEP
#undef GPOLL
#undef POFF
#undef KIND
#undef PST
#undef PSTA
#undef RETRY
#undef VALID
#undef NRETRY
#undef PLD
#undef PFLAG
}

__global__ void __launch_bounds__(512, 2) ag_gemm_kernel(Params p) {
    extern __shared__ __attribute__((aligned(16))) char smem[];
    LAS char* lds = (LAS char*)smem;
    const int tid = threadIdx.x, lane = tid & 63, wave = __builtin_amdgcn_readfirstlane(tid >> 6);
    const int b = blockIdx.x, rank = p.rank, g = rank >> 2, a = rank & 3;
    char* ws = p.me;
    if (wave == 0) STAMP(ws, 0);
    const u32 prev = ld_agent(ws + OFF_CTR + (u32)b * 64u);
    u32 calln, call, par;
    const int lw = wave;
    const rsrc_t rws = mk_rsrc(ws, WS_NEED);

    Tables T = {};
    u32 code = 0;
    char* src = ws; gchar* fan = (gchar*)ws;
    gchar* rdflag = x_rows<1, true>(p, lds, b < 128 ? b : 640 + b, par, call, tid, wave, [&]() {
        u64 pc; u32 len;
        asm volatile("s_getpc_b64 %0\n.Lagk_pc:\n\ts_mov_b32 %1, .Lagk_end-.Lagk_pc" : "=s"(pc), "=s"(len));
        code = __builtin_amdgcn_raw_buffer_load_b32(mk_rsrc((const void*)pc, (int)len > 0 ? len : 0u), (u32)(b >> 3) * 2048u + (u32)tid * 4u, 0, 0);
        asm volatile("" ::: "memory");
        calln = (u32)__builtin_amdgcn_readfirstlane(prev) + 1u; call = calln << 8; par = calln & 1u;
        if (b < NCOMM) {
            Stream S;
            S.cross = lw >= 3; S.rank = rank; S.g = g; S.a = a; S.j = (a + lw + 1) & 3; S.pb = CTRL_BYTES + par * PAR_BYTES; S.sa = OFF_SA + par * 32768u;
            T.lo = S.at((u32)lane); T.hi = S.at((u32)lane + 64u);
            src = p.peer[lw == 3 ? 7 : lw];
            char *f0 = p.peer[4], *f1 = p.peer[5], *f2 = p.peer[6];
            asm volatile("" : "+s"(src), "+s"(f0), "+s"(f1), "+s"(f2));
            fan = (gchar*)(lane == 0 ? f0 : lane == 1 ? f1 : f2);
        }
    });
    asm volatile("" :: "v"(code));
    if (tid < 2) st_agent(ws + OFF_CTR + (u32)b * 64u + (u32)tid * 4u, tid ? call : calln);
    if (wave == 0) STAMP(ws, 1);
    if (b < NCOMM) {
        if (wave == 7) {
            WAIT_VM(0);
            if (lane < 4) st_sys_g(rdflag, call);
        }
        else if (b < NACT) pull_stream(p, lds, T, wave >= 3, (u32)b, src, fan, calln, call, lane, wave);
        return;
    }

    {
        const int c = b - NCOMM, strip = c / 7, rg = c - strip * 7;
        const int kt0 = (rg * 128) / 7, kt1 = ((rg + 1) * 128) / 7;
        const rsrc_t rw = mk_rsrc(p.w, IN_BYTES);
        const int q = lane & 7, n4 = lane >> 3;
        if (wave == 7) { WAIT_VM(0); if (lane < 4) st_sys_g(rdflag, call); }
        f32x4 v[3][8];
#pragma unroll
        for (int i = 0; i < 3; ++i) {
            const int kt = kt0 + wave + 8 * i;
            const u32 base = kt < kt1 ? ((u32)(kt * 64 + q * 8) * NLOC + (u32)(strip * 32 + n4 * 4)) * 4u : OOB;
#pragma unroll
            for (int jj = 0; jj < 8; ++jj) {
                v[i][jj] = __builtin_bit_cast(f32x4, __builtin_amdgcn_raw_buffer_load_b128(rw, base + (u32)jj * (NLOC * 4u), 0, 0));
                WAIT_VM(W_LD - 1);
            }
        }
        LAS u32* cm = (LAS u32*)(lds + L_RED + 256u);
        if (tid < 64) cm[tid] = 0u;
        WAIT_LGKM0(); BAR();
        u32 m[4] = {0u, 0u, 0u, 0u};
#pragma unroll
        for (int i = 0; i < 3; ++i)
#pragma unroll
            for (int jj = 0; jj < 8; ++jj)
#pragma unroll
                for (int k = 0; k < 4; ++k) m[k] = umax(m[k], absbits(v[i][jj][k]));
#pragma unroll
        for (int k = 0; k < 4; ++k) {
            m[k] = umax(m[k], (u32)__shfl_xor((int)m[k], 1)); m[k] = umax(m[k], (u32)__shfl_xor((int)m[k], 2)); m[k] = umax(m[k], (u32)__shfl_xor((int)m[k], 4));
        }
        if (q < 4) __hip_atomic_fetch_max(cm + n4 * 4 + q, q == 0 ? m[0] : q == 1 ? m[1] : q == 2 ? m[2] : m[3], __ATOMIC_RELAXED, __HIP_MEMORY_SCOPE_WORKGROUP);
        WAIT_LGKM0(); BAR();
        u64* wx = (u64*)(ws + OFF_WX) + (u32)strip * 256u;
        if (tid < 32) __hip_atomic_store(wx + rg * 32 + tid, ((u64)call << 32) | cm[tid], __ATOMIC_RELAXED, __HIP_MEMORY_SCOPE_AGENT);
        if (wave < 4) {
            const int gi = tid < 224 ? tid : 0;
            const u64 t0 = __builtin_amdgcn_s_memrealtime();
            u64 gv; u32 n = 0;
            for (;;) {
                gv = __hip_atomic_load(wx + gi, __ATOMIC_RELAXED, __HIP_MEMORY_SCOPE_AGENT);
                if (__all((u32)(gv >> 32) == call)) break;
                if ((++n & 15u) == 0u && give_up(ws, call, t0)) break;
                __builtin_amdgcn_s_sleep(1);
            }
            __hip_atomic_fetch_max(cm + 32 + (gi & 31), (u32)gv, __ATOMIC_RELAXED, __HIP_MEMORY_SCOPE_WORKGROUP);
        }
        WAIT_LGKM0(); BAR();
        const int ct = strip >> 2;
        if (rg == 0 && tid < 64) {
            const u32 nin = (u32)((strip & 3) * 32 + (tid & 31));
            const u32 ev = (u32)(exp_of(cm[32 + (tid & 31)]) - 141);
            char* t = tid < 32 ? ws + OFF_SB + (par * BSLOTS + (u32)ct) * 512u : p.ws[(1 - g) * 4 + (strip >> 3)] + OFF_SB + (par * BSLOTS + (u32)(8 + 2 * a + (ct & 1))) * 512u;
            st_sys(t + nin * 4u, ev);
        }
        const u32x4 mx = *(LAS u32x4*)(cm + 32 + n4 * 4);
        const u32 pbase = CTRL_BYTES + par * PAR_BYTES + (u32)(ASLOTS + ct) * PANEL;
#pragma unroll
        for (int k = 0; k < 4; ++k) {
            const float s = scale_of(exp_of(mx[k]));
            const u32 nin = (u32)((strip & 3) * 32 + n4 * 4 + k);
            const u32 ro = nin * 128u + (((u32)q ^ ((nin >> 1) & 7u)) << 4);
#pragma unroll
            for (int i = 0; i < 3; ++i) {
                const int kt = kt0 + wave + 8 * i;
                const u32 off = kt < kt1 ? pbase + (u32)kt * IMG + ro : OOB;
                const f32x4 lo = {v[i][0][k], v[i][1][k], v[i][2][k], v[i][3][k]}, hi = {v[i][4][k], v[i][5][k], v[i][6][k], v[i][7][k]};
                __builtin_amdgcn_raw_buffer_store_b128(cvt8(lo, hi, s), rws, off, 0, AUX_SC1);
                WAIT_VM(W_ST - 1);
            }
        }
        WAIT_VM(0); BAR();
        if (tid < 4 && (!(tid & 1) || rg == 6))
            st_sys((tid < 2 ? ws + OFF_WR + (u32)strip * 32u : p.ws[(1 - g) * 4 + (strip >> 3)] + OFF_WS + (u32)(a * 8 + (strip & 7)) * 32u) + (u32)(rg + (tid & 1)) * 4u, call);
        if (wave == 0) STAMP(ws, 2);
    }
    const int xcd = b & 7, q = (b >> 3) - 4;
    if (xcd >= 4 && q < 24) {
        const int r0 = 128 + (q * 4 + xcd - 4) * 4;
        x_rows<4, false>(p, lds, r0, par, call, tid, wave, []() {});
        x_rows<4, false>(p, lds, r0 + (r0 < 384 ? 384 : 512), par, call, tid, wave, []() {});
        if (wave == 0) STAMP(ws, 3);
    }
    const int qr = q >> 2, qc = q & 3, x4 = xcd & 3;
    const int early = (x4 >> 1) == 0 ? (qr < 6 ? a * 6 + qr : -1) : (qr < 3 ? ((a + 1 + qr) & 3) * 6 : 24 + 2 * (qr - 3));
    const int peer = ((a + 1 + (qr >> 1)) & 3) * 6;
    for (int it = 0; it < 4; ++it) {
        int slot = -1, ct = 4 * x4 + qc, rh = 0, ch = 0;
        if (it == 0) { if (xcd < 4) { slot = early; ct = 4 * (x4 & 1) + qc; } }
        else if (it == 1) { if (xcd < 4) { if (qr < 6) slot = peer + 1 + (qr & 1); } else { slot = early; ct = 8 + 4 * (x4 & 1) + qc; } }
        else if (it == 2) {
            if (qr == 6) { slot = 25 + (xcd & 6); ct = 8 + 4 * (xcd & 1) + qc; }
            else if (xcd >= 4) slot = peer + 3 + (qr & 1);
            else if (xcd < 2 && qr < 4) slot = 25 + 2 * qr;
        } else {
            int pi = -1; ch = qc & 1;
            if (xcd < 2) { pi = qr - 4; rh = 1; ct = 8 + 2 * xcd + (qc >> 1); }
            else if (qr == 6) { pi = (xcd - 2) >> 1; rh = 1; ct = 12 + 2 * (xcd & 1) + (qc >> 1); }
            else if (xcd < 4) { pi = qr >> 1; ct = 4 * xcd + 2 * (qr & 1) + (qc >> 1); }
            else { pi = qr >> 1; rh = qr & 1; ct = 2 * x4 + (qc >> 1); }
            if (pi >= 0) slot = ((a + 1 + pi) & 3) * 6 + 5;
        }
        if (slot < 0) continue;
        if (slot >= 24 && ct >= 8 && (slot - 24) >> 1 == (ct - 8) >> 1) { slot = 32 + (slot & 1); ct -= 8; }
        if (it < 3) gemm_tile<true>(p, lds, slot, ct, 0, 0, par, call, tid, lane, wave, 4 + it * 3);
        else gemm_tile<false>(p, lds, slot, ct, rh, ch, par, call, tid, lane, wave, 4 + it * 3);
    }
}
}

extern "C" void kernel_launch(int rank, int nranks, ncclComm_t comm,
                              hipStream_t stream,
                              void* const* d_in, const int* in_sizes, int n_in,
                              void* d_out, int out_size,
                              const int* in_dtypes, int out_dtype,
                              void* const* d_out_all,
                              void* const* d_ws_all, size_t ws_bytes) {
    (void)comm; (void)in_dtypes; (void)out_dtype; (void)d_out_all;
    if (nranks != 8 || (unsigned)rank >= 8u || n_in < 2 || ws_bytes < (size_t)agk::WS_NEED || in_sizes[0] != 1024 * 8192 ||
        in_sizes[1] != 8192 * 1024 || out_size != 8192 * 1024) {
        fprintf(stderr, "kernel_launch: unexpected problem shape\n");
        return;
    }
    static bool ready[8] = {false, false, false, false, false, false, false, false};
    if (!ready[rank]) {
        HIP_CHECK(hipFuncSetAttribute((const void*)agk::ag_gemm_kernel, hipFuncAttributeMaxDynamicSharedMemorySize, (int)agk::LDS_BYTES));
        ready[rank] = true;
    }
    agk::Params p;
    p.x = (const float*)d_in[0]; p.w = (const float*)d_in[1]; p.rank = rank; p.pad = 0;
    p.me = (char*)d_ws_all[rank]; p.out = (float*)d_out;
    for (int i = 0; i < 8; ++i) { p.ws[i] = (char*)d_ws_all[i]; p.peer[i] = (char*)d_ws_all[((rank ^ i) & 4) + ((rank + 1 + i) & 3)]; }
    hipLaunchKernelGGL(agk::ag_gemm_kernel, dim3(agk::NWG), dim3(512), agk::LDS_BYTES, stream, p);
    HIP_CHECK(hipGetLastError());
}
```

```cpp
#include <hip/hip_runtime.h>
#include <rccl/rccl.h>
#include <cstdio>

#define HIP_CHECK(expr) do { hipError_t _e = (expr); if (_e != hipSuccess) { \
    fprintf(stderr, "HIP error %s at %s:%d\n", hipGetErrorString(_e), __FILE__, __LINE__); \
    return; } } while (0)

namespace agk {
typedef unsigned u32;
typedef unsigned long long u64;
typedef u32 u32x4 __attribute__((ext_vector_type(4)));
typedef int i32x4 __attribute__((ext_vector_type(4)));
typedef float f32x4 __attribute__((ext_vector_type(4)));
typedef float f32x2 __attribute__((ext_vector_type(2)));
typedef _Float16 f16x2 __attribute__((ext_vector_type(2)));
typedef _Float16 f16x8 __attribute__((ext_vector_type(8)));
#define LAS __attribute__((address_space(3)))
typedef __amdgpu_buffer_rsrc_t rsrc_t;
typedef __attribute__((address_space(1))) char gchar;

constexpr int KDIM = 8192, NLOC = 1024, NKT = 128;
constexpr u32 IMG = 16384u;
constexpr u32 PANEL = NKT * IMG;
constexpr u32 UNIT = 131072u;
constexpr int ASLOTS = 34, BSLOTS = 16;
#define HU 3
#define HWIN 8
#define SCH 2
#define ESWAP 1
#define HSTEP 1
#define HK 6
#define XDLY 30
#define HINT 1
#define DCUT 16
#define DELTA 4
constexpr u32 CTRL_BYTES = 1u << 20;
constexpr u32 RELAY = (ASLOTS + BSLOTS) * PANEL;
constexpr u32 PAR_BYTES = RELAY + 8u * UNIT;
constexpr u32 OFF_YST = CTRL_BYTES + 2u * PAR_BYTES;
constexpr u32 WS_NEED = OFF_YST + 4u * 32u * UNIT;
constexpr u32 IN_BYTES = 1024u * 8192u * 4u;
constexpr u32 OFF_CTR = 0;
constexpr u32 OFF_TMO = 16384;
constexpr u32 OFF_CL = 20480;
constexpr u32 OFF_LF = 524288;
constexpr u32 OFF_RD = 655360;
constexpr u32 OFF_YR = 24576;
constexpr u32 OFF_RF = 28672;
constexpr u32 OFF_WR = 32768;
constexpr u32 OFF_WS = 33792;
constexpr u32 OFF_WX = 36864;
constexpr u32 OFF_SA = 131072;
constexpr u32 OFF_SB = 196608;
constexpr u32 OFF_DBG = 262144;
constexpr int NWG = 256, NCOMM = 32;
constexpr u32 NUNITS = 96u;
constexpr u32 OOB = 0x80000000u, NOFLAG = 0x40000000u;
constexpr u64 DEADLINE = 30000000ull;
constexpr u32 STAGE = 32768u;
constexpr u32 L_SNAP = 4u * STAGE;
constexpr u32 L_SC = L_SNAP + 512u;
constexpr u32 L_RED = L_SC + 1024u;
constexpr u32 LDS_BYTES = L_RED + 2048u;

struct Params {
    const float* x; char* me; int rank, pad;
    char* peer[8];
    const float* w; float* out;
    char* ws[8];
};

#define AUX_SC1 16
#define AUX_SYS 17
#define GL_AUX 16
#define W_LD 3
#define W_ST 2
#define X_LD 4
#define X_ST 2
#define WAIT_VM(n) asm volatile("s_waitcnt vmcnt(%0)" :: "n"(n) : "memory")
#define WAIT_LGKM0() do { asm volatile("" ::: "memory"); __builtin_amdgcn_s_waitcnt(15 | (7 << 4) | (0 << 8) | (3 << 14)); asm volatile("" ::: "memory"); } while (0)
#define BAR() do { asm volatile("" ::: "memory"); __builtin_amdgcn_s_barrier(); asm volatile("" ::: "memory"); } while (0)
#define WAIT_VM_TRACKED(n) do { asm volatile("" ::: "memory"); __builtin_amdgcn_s_waitcnt(((n) & 15) | (7 << 4) | (15 << 8) | (((n) >> 4) << 14)); asm volatile("" ::: "memory"); } while (0)

__device__ __forceinline__ rsrc_t mk_rsrc(const void* p, u32 bytes) { return __builtin_amdgcn_make_buffer_rsrc((void*)p, 0, bytes, 0x00020000); }
__device__ __forceinline__ u32 ld_agent(const void* p) { return __hip_atomic_load((const u32*)p, __ATOMIC_RELAXED, __HIP_MEMORY_SCOPE_AGENT); }
__device__ __forceinline__ u32 ld_sys(const void* p) { return __hip_atomic_load((const u32*)p, __ATOMIC_RELAXED, __HIP_MEMORY_SCOPE_SYSTEM); }
__device__ __forceinline__ void st_agent(void* p, u32 v) { __hip_atomic_store((u32*)p, v, __ATOMIC_RELAXED, __HIP_MEMORY_SCOPE_AGENT); }
__device__ __forceinline__ void st_sys_g(gchar* p, u32 v) { __hip_atomic_store((__attribute__((address_space(1))) u32*)p, v, __ATOMIC_RELAXED, __HIP_MEMORY_SCOPE_SYSTEM); }
__device__ __forceinline__ void st_sys(void* p, u32 v) { __hip_atomic_store((u32*)p, v, __ATOMIC_RELAXED, __HIP_MEMORY_SCOPE_SYSTEM); }

#define STAMP(ws, idx) do { } while (0)

__device__ __forceinline__ bool give_up(char* ws, u32 call, u64 t0) {
    const u32 t = __builtin_amdgcn_readfirstlane(ld_agent(ws + OFF_TMO));
    if (t == call || __builtin_amdgcn_s_memrealtime() - t0 > DEADLINE) { st_agent(ws + OFF_TMO, call); return true; }
    return false;
}
__device__ __forceinline__ void poll_ge(const char* p, u32 need, char* ws, u32 call) {
    const u64 t0 = __builtin_amdgcn_s_memrealtime();
    u32 n = 0;
    for (;;) {
        const u32 v = ld_sys(p);
        if (__all((int)(v - need) >= 0)) return;
        if ((++n & 15u) == 0u && give_up(ws, call, t0)) return;
        __builtin_amdgcn_s_sleep(2);
    }
}
__device__ __forceinline__ bool all4_ge(u32x4 v, u32 need) {
    return __all((int)(v[0] - need) >= 0 && (int)(v[1] - need) >= 0 && (int)(v[2] - need) >= 0 && (int)(v[3] - need) >= 0);
}
__device__ __forceinline__ void poll4_ge(rsrc_t rs, u32 off, u32 need, char* ws, u32 call) {
    const u64 t0 = __builtin_amdgcn_s_memrealtime();
    u32 n = 0;
    for (;;) {
        asm volatile("" ::: "memory");
        const u32x4 v = __builtin_amdgcn_raw_buffer_load_b128(rs, off, 0, AUX_SYS);
        if (all4_ge(v, need)) return;
        if ((++n & 15u) == 0u && give_up(ws, call, t0)) return;
        __builtin_amdgcn_s_sleep(1);
    }
}

__device__ __forceinline__ u32 umax(u32 a, u32 b) { return a > b ? a : b; }
__device__ __forceinline__ u32 absbits(float f) { return __float_as_uint(f) & 0x7fffffffu; }
__device__ __forceinline__ int exp_of(u32 amaxbits) { int e = (int)(amaxbits >> 23); return e < 15 ? 15 : e; }
__device__ __forceinline__ float scale_of(int e) { return __uint_as_float((u32)(268 - e) << 23); }
__device__ __forceinline__ u32 cvt2(float a, float b, float s) {
    const f32x2 t = (f32x2){a, b} * s;
    return __builtin_bit_cast(u32, __builtin_convertvector(t, f16x2));
}
__device__ __forceinline__ u32x4 cvt8(f32x4 a, f32x4 b, float s) {
    return (u32x4){cvt2(a[0], a[1], s), cvt2(a[2], a[3], s), cvt2(b[0], b[1], s), cvt2(b[2], b[3], s)};
}
__device__ __forceinline__ u32 amax4(u32 m, f32x4 v) {
    asm("v_max3_f32 %0, |%1|, |%2|, %0\n\tv_max3_f32 %0, |%3|, |%4|, %0" : "+v"(m) : "v"(v[0]), "v"(v[1]), "v"(v[2]), "v"(v[3]));
    return m;
}

__device__ __forceinline__ u32 wave_max(u32 m) {
    m = umax(m, (u32)__builtin_amdgcn_update_dpp(0, (int)m, 0xb1, 15, 15, true));
    m = umax(m, (u32)__builtin_amdgcn_update_dpp(0, (int)m, 0x4e, 15, 15, true));
    m = umax(m, (u32)__builtin_amdgcn_update_dpp(0, (int)m, 0x141, 15, 15, true));
    m = umax(m, (u32)__builtin_amdgcn_update_dpp(0, (int)m, 0x140, 15, 15, true));
    return umax(umax((u32)__builtin_amdgcn_readlane((int)m, 0), (u32)__builtin_amdgcn_readlane((int)m, 16)),
                umax((u32)__builtin_amdgcn_readlane((int)m, 32), (u32)__builtin_amdgcn_readlane((int)m, 48)));
}
__device__ __forceinline__ u32 row_word(u32 call, int e) { return call | ((u32)(e - 141) & 255u); }

__device__ __forceinline__ void x_rows(const Params& p, LAS char* lds, int row0, u32 par, u32 call, int tid, int wave) {
    constexpr int NR = 4;
    const int a = p.rank & 3;
    char* ws = p.me;
    const rsrc_t rx = mk_rsrc(p.x, IN_BYTES), rw = mk_rsrc(ws, WS_NEED);
    const u32 pi0 = (u32)(wave * 128 + (tid & 63));
    f32x4 v[NR][4];
#pragma unroll
    for (int r = 0; r < NR; ++r)
#pragma unroll
        for (int u = 0; u < 2; ++u) {
            const u32 off = (u32)(row0 + r) * (KDIM * 4u) + (pi0 + (u32)u * 64u) * 32u;
            v[r][2 * u] = __builtin_bit_cast(f32x4, __builtin_amdgcn_raw_buffer_load_b128(rx, off, 0, 0));
            v[r][2 * u + 1] = __builtin_bit_cast(f32x4, __builtin_amdgcn_raw_buffer_load_b128(rx, off + 16u, 0, 0));
            WAIT_VM(X_LD - 2);
        }
    const int rb = row0 >> 7;
    const u32 slot = rb < 6 ? (u32)(a * 6 + rb) : (u32)(32 + rb - 6);
    const u32 rslot = rb < 6 ? slot : (u32)(24 + 2 * a + rb - 6);
    char* hp[4];
#pragma unroll
    for (int i = 0; i < 4; ++i) hp[i] = rb < 6 ? p.peer[i] : p.peer[4 + i];
    char* fifth = rb < 6 ? p.peer[7] : ws;
    asm volatile("" : "+s"(hp[0]), "+s"(hp[1]), "+s"(hp[2]), "+s"(hp[3]), "+s"(fifth));
    gchar* user = (gchar*)((tid & 4) ? fifth : (tid & 2) ? ((tid & 1) ? hp[3] : hp[2]) : ((tid & 1) ? hp[1] : hp[0]));
    const u32 wr = (u32)(tid & 63) >> 3;
    const bool wsend = wr < (u32)NR && ((tid & 7) < 4 || ((tid & 7) == 4 && rb >= 5));
    const u32 woff = OFF_SA + ((par * 64u + ((tid & 4) ? (rb < 6 ? 34u : slot) : rslot)) * 128u + (((u32)row0 + wr) & 127u)) * 4u;
    u32 word = 0;
    LAS u32* red = (LAS u32*)(lds + L_RED);
#pragma unroll
    for (int r = 0; r < NR; ++r) {
        u32 m = 0;
#pragma unroll
        for (int i = 0; i < 4; ++i) m = amax4(m, v[r][i]);
        m = wave_max(m);
        if ((tid & 63) == 0) red[r * 8 + wave] = m;
    }
    WAIT_LGKM0(); BAR();
#pragma unroll
    for (int r = 0; r < NR; ++r) {
        const u32x4 m0 = *(LAS u32x4*)(red + r * 8), m1 = *(LAS u32x4*)(red + r * 8 + 4);
        const u32 m = umax(umax(umax(m0[0], m0[1]), umax(m0[2], m0[3])), umax(umax(m1[0], m1[1]), umax(m1[2], m1[3])));
        const int e = exp_of(m); const float s = scale_of(e);
        const u32 rin = (u32)((row0 + r) & 127);
        if (wr == (u32)r) word = row_word(call, e);
        const u32 off = CTRL_BYTES + par * PAR_BYTES + rin * 128u + (pi0 >> 3) * IMG + (((pi0 & 7u) ^ ((rin >> 1) & 7u)) << 4);
        __builtin_amdgcn_raw_buffer_store_b128(cvt8(v[r][0], v[r][1], s), rw, off + slot * PANEL, 0, AUX_SC1);
        __builtin_amdgcn_raw_buffer_store_b128(cvt8(v[r][2], v[r][3], s), rw, off + slot * PANEL + 8u * IMG, 0, AUX_SC1);
        WAIT_VM(X_ST - 2);
    }
    WAIT_VM(0); BAR();
    if (wave == 0 && wsend) st_sys_g(user + woff, word);
}

template <class F>
__device__ __forceinline__ gchar* x_head(const Params& p, LAS char* lds, int row, u32 prev, u32& calln, u32& hword, int tid, int wave, F mid) {
    constexpr int NS = HU > 3 ? HU : 3;
    const int lane = tid & 63;
    char* ws = p.me;
    const rsrc_t rx = mk_rsrc(p.x, IN_BYTES), rw = mk_rsrc(ws, WS_NEED);
    LAS u32* red = (LAS u32*)(lds + L_RED);
    if (wave != 7) {
        if (tid < 4) red[tid] = 0u;
        WAIT_LGKM0(); BAR();
        __builtin_amdgcn_s_sleep(XDLY);
    }
    const u32 s0 = wave == 7 ? 0u : (u32)wave, sd = wave == 7 ? 1u : 7u, se = wave == 7 ? (u32)HU : 16u;
    f32x4 v[NS][2];
#pragma unroll
    for (int u = 0; u < NS; ++u) {
        const u32 set = s0 + (u32)u * sd;
        const u32 off = set < se ? (u32)row * (KDIM * 4u) + (set * 64u + (u32)lane) * 32u : OOB;
        v[u][0] = __builtin_bit_cast(f32x4, __builtin_amdgcn_raw_buffer_load_b128(rx, off, 0, 0));
        v[u][1] = __builtin_bit_cast(f32x4, __builtin_amdgcn_raw_buffer_load_b128(rx, off + 16u, 0, 0));
    }
    __builtin_amdgcn_sched_barrier(0);
    if (wave == 7) BAR();
    const bool major = row < 128;
    const int a = p.rank & 3;
    char* hp[4];
#pragma unroll
    for (int i = 0; i < 4; ++i) hp[i] = major ? p.peer[i] : p.peer[4 + i];
    const u32 slot = major ? (u32)(a * 6) : 32u, rslot = major ? slot : (u32)(24 + 2 * a), rin = (u32)row & 127u;
    u32 off = CTRL_BYTES + rin * 128u + ((u32)lane >> 3) * IMG + ((((u32)lane & 7u) ^ ((rin >> 1) & 7u)) << 4);
    u32 m = 0;
    gchar* user = (gchar*)((tid & 4) ? ws : (tid & 2) ? ((tid & 1) ? hp[3] : hp[2]) : ((tid & 1) ? hp[1] : hp[0]));
#define WORD() do { u32 t = 0; \
        if (lane == 0) t = __hip_atomic_fetch_add(red + 2, 1u, __ATOMIC_RELAXED, __HIP_MEMORY_SCOPE_WORKGROUP); \
        if (__builtin_amdgcn_readfirstlane((int)t) == 7 && lane < (major ? 4 : 5)) \
            st_sys_g(user + OFF_SA + (((calln & 1u) * 64u + (lane == 4 ? slot : rslot)) * 128u + rin) * 4u, \
                     row_word(calln << 8, exp_of(__hip_atomic_load(red, __ATOMIC_RELAXED, __HIP_MEMORY_SCOPE_WORKGROUP)))); } while (0)
    if (wave == 7) {
        rsrc_t hr[4];
#pragma unroll
        for (int i = 0; i < 4; ++i) hr[i] = mk_rsrc(hp[i], i == 3 && major ? 0u : WS_NEED);
        u32 o4[4];
#pragma unroll
        for (int i = 0; i < 4; ++i) o4[i] = (lane >> 4) == i ? off + rslot * PANEL : OOB;
        asm volatile("" : "+s"(hr[0]), "+s"(hr[1]), "+s"(hr[2]), "+s"(hr[3]), "+v"(o4[0]), "+v"(o4[1]), "+v"(o4[2]), "+v"(o4[3]), "+v"(m), "+v"(prev));
        calln = (u32)__builtin_amdgcn_readfirstlane(prev) + 1u;
        const u32 so = (calln & 1u) * PAR_BYTES;
#pragma unroll
        for (int u = 0; u < HU; ++u) m = amax4(amax4(m, v[u][0]), v[u][1]);
        m = wave_max(m);
        const int e = exp_of(m); const float s = scale_of(e);
        u32x4 h;
#pragma unroll
        for (int k = 0; k < 4 * HU; ++k) {
            if ((k & 3) == 0) h = cvt8(v[k >> 2][0], v[k >> 2][1], s);
            if (k == 2 && lane == 0) __hip_atomic_fetch_add(red + 1, 1u, __ATOMIC_RELAXED, __HIP_MEMORY_SCOPE_WORKGROUP);
#pragma unroll
            for (int i = 0; i < 4; ++i) {
                if (HSTEP ? i == 0 && 4 * k >= HWIN : 4 * k + i >= HWIN) WAIT_VM(HWIN - (HSTEP ? 4 : 1));
                __builtin_amdgcn_raw_buffer_store_b128(h, hr[i], o4[k & 3], so + (u32)(k >> 2) * 8u * IMG, AUX_SYS);
            }
            asm volatile("s_nop 1" : "+v"(h) :: "memory");
            if (k == HK) WORD();
        }
        hword = row_word(calln << 8, e);
    }
    else calln = (u32)__builtin_amdgcn_readfirstlane(prev) + 1u;
    const u32 par = calln & 1u;
    if (wave != 7) {
        mid();
        off += par * PAR_BYTES + slot * PANEL;
#pragma unroll
        for (int u = 0; u < 3; ++u) m = amax4(amax4(m, v[u][0]), v[u][1]);
        m = wave_max(m);
        if (lane == 0) {
            __hip_atomic_fetch_max(red, m, __ATOMIC_RELAXED, __HIP_MEMORY_SCOPE_WORKGROUP);
            __hip_atomic_fetch_add(red + 1, 1u, __ATOMIC_RELAXED, __HIP_MEMORY_SCOPE_WORKGROUP);
        }
        u64 mc;
        for (u32 n = 0; n < (1u << 24); ++n) {
            mc = __hip_atomic_load((LAS u64*)red, __ATOMIC_RELAXED, __HIP_MEMORY_SCOPE_WORKGROUP);
            if ((u32)__builtin_amdgcn_readfirstlane((int)(mc >> 32)) == 8u) break;
        }
        const int e = exp_of((u32)mc); const float s = scale_of(e);
#pragma unroll
        for (int u = 0; u < 3; ++u) {
            const u32 set = s0 + (u32)u * 7u;
            __builtin_amdgcn_raw_buffer_store_b128(cvt8(v[u][0], v[u][1], s), rw, set < 16u ? off + set * 8u * IMG : OOB, 0, AUX_SC1);
        }
        WAIT_VM(0);
        WORD();
    }
#undef WORD
    return user + OFF_RD + ((par * 8u + (u32)p.rank) * 128u + rin) * 4u;
}

__device__ __forceinline__ int slot_row(int rank, int slot) {
    const int g = rank >> 2;
    if (slot < 24) { const int i = slot / 6; return (g * 4 + i) * 1024 + (slot - i * 6) * 128; }
    return (slot < 32 ? (1 - g) * 4 + ((slot - 24) >> 1) : rank) * 1024 + 768 + (slot & 1) * 128;
}

struct Tile { int slot, ct, rh, ch; };
__device__ __forceinline__ void a_stream(int rank, int slot, int rh, int& a_src, u32& a_base, u32& a_sh) {
    const int g = rank >> 2, a = rank & 3;
    if (slot < 24) { const int i = slot / 6, a_rb = slot - i * 6; a_src = i == a ? -1 : g * 4 + i;
        a_base = a_rb == 0 ? 0u : a_rb < 3 ? 16u : a_rb < 5 ? 48u : 80u + (u32)rh * 8u; a_sh = a_rb == 0 ? 1u : 0u; }
    else { a_src = slot < 32 ? (1 - g) * 4 + ((slot - 24) >> 1) : -1; a_base = (slot & 1) ? 48u : 0u; a_sh = 1u; }
}
__device__ __forceinline__ u32 start_off(int rank, const Tile& t, bool big, u32 par, int lane, int wave) {
    int a_src; u32 a_base, a_sh;
    a_stream(rank, t.slot, t.rh, a_src, a_base, a_sh);
    const int b_src = t.ct < 8 ? -1 : (1 - (rank >> 2)) * 4 + ((t.ct - 8) >> 1);
    u32 o = OOB, m = 63u;
    if (wave == 0) { if (a_src >= 0) { o = OFF_LF + ((u32)a_src * NUNITS + (a_base ? a_base : (u32)HU)) * 128u; m = big ? 7u : 3u; } }
    else if (wave == 1) { if (a_src < 0 || (!big && t.rh == 0)) { o = OFF_SA + (par * 64u + (u32)t.slot) * 512u; m = big ? 31u : 15u; } }
    else if (wave == 2) { if (b_src < 0) { o = OFF_WR + (u32)t.ct * 128u; m = 7u; } }
    else if (wave == 3) { if (a_src >= 0 && a_base == 0u) { o = OFF_RD + (par * 8u + (u32)a_src) * 512u; m = 31u; } }
    else if (b_src >= 0 && (!big || wave == 4)) { o = OFF_LF + ((u32)b_src * NUNITS + 16u) * 128u + (u32)(wave - 4) * 1024u; m = big ? 7u : 63u; }
    return o + ((u32)lane & m) * 16u;
}

template <bool BIG>
__device__ __forceinline__ void gemm_tile(const Params& p, LAS char* lds, const Tile& tl, bool first, const Tile& nx, bool nbig, u32 par, u32 call, int tid, int lane, int wave, int stamp0, char*& pend) {
    constexpr int NB = BIG ? 4 : 2, NC = BIG ? 32 : 16, BT = BIG ? 128 : 64;
    const int rank = p.rank, g = rank >> 2, a = rank & 3;
    const int slot = tl.slot, ct = tl.ct, rh = tl.rh, ch = tl.ch;
    asm volatile("" : "+v"(lane), "+v"(tid));
    char* ws = p.me;
    const rsrc_t rws = mk_rsrc(ws, WS_NEED);
    int a_src; u32 a_base, a_sh;
    a_stream(rank, slot, rh, a_src, a_base, a_sh);
    const int b_src = ct < 8 ? -1 : (1 - g) * 4 + ((ct - 8) >> 1);
    const int src = lane < 32 ? a_src : (BIG ? b_src : -1);
    const u32 gsh = src < 0 ? 31u : lane < 32 && BIG ? a_sh : 0u;
    const char* gptr = ws + (src >= 0 ? OFF_LF + ((u32)src * NUNITS + (lane < 32 ? a_base : 16u)) * 128u + (u32)(lane & (BIG ? 31 : 15)) * 4u : OFF_CTR + blockIdx.x * 64u + 4u);
    const u32 gmin = src >= 0 && lane < 32 && a_base == 0u ? (u32)HU : 0u;
    const bool headt = BIG && a_src >= 0 && a_base == 0u;
#define GPTR(c) (gptr + (umax((u32)(c) >> gsh, gmin) << (BIG ? 7 : 6)))
#define PEND() do { if (pend && wave == 1 && lane < 2) st_sys(pend + (u32)lane * 4u, call); pend = nullptr; } while (0)
    LAS u32* okw = (LAS u32*)(lds + L_RED + 1024u);
    WAIT_LGKM0(); BAR();
    {
        const u32x4 k0 = *(const LAS u32x4*)okw, k1 = *(const LAS u32x4*)(okw + 4);
        if (first || __builtin_amdgcn_readfirstlane((int)(k0[0] | k0[1] | k0[2] | k0[3] | k1[0] | k1[1] | k1[2] | k1[3]))) {
            if (pend) { WAIT_VM(0); BAR(); PEND(); }
            const u32 po = start_off(rank, tl, BIG, par, lane, wave);
            if (__builtin_amdgcn_readfirstlane((int)po) >= 0) poll4_ge(rws, po, call, ws, call);
            WAIT_VM_TRACKED(0); BAR();
        }
    }
    if (wave == 0) STAMP(ws, stamp0);
    constexpr u32 RB = BT * 4u, PPR = BT / 4u, AREA = BT * RB;
    const u32 pc = (u32)lane & (PPR - 1u);
    const u32 col0 = BIG ? 0u : (u32)ch * 64u, row0 = BIG ? 0u : (u32)rh * 64u;
    const bool remote = ct >= 8;
    const rsrc_t ro = mk_rsrc(remote ? (const void*)ws : (const void*)p.out, remote ? WS_NEED : IN_BYTES);
    const u32 ostr = remote ? 1024u : NLOC * 4u;
    const u32 obase = (remote ? OFF_YST + (u32)(((ct - 8) >> 1) * 32 + slot) * UNIT + (u32)(ct & 1) * 512u : ((u32)slot_row(rank, slot) * NLOC + (u32)ct * 128u) * 4u) + col0 * 4u + row0 * ostr + pc * 16u;
#define YFLAG() do { if (tid == 0) st_sys(p.ws[b_src] + OFF_YR + (u32)(a * 64 + rh * 32 + slot) * 16u + (u32)(ct & 1) * 8u + (u32)ch * 4u, call); } while (0)
    const bool hint = !BIG && remote;
    if (hint) {
        const u32x4 zero = {0u, 0u, 0u, 0u};
#pragma unroll
        for (u32 it = 0; it < 2u; ++it) __builtin_amdgcn_raw_buffer_store_b128(zero, ro, obase + ((u32)wave * (BT / 8u) + (it * 64u + (u32)lane) / PPR) * ostr, 0, AUX_SC1);
    }
    const u32 pb = CTRL_BYTES + par * PAR_BYTES;
    const char* gA = ws + pb + (u32)slot * PANEL + (BIG ? (u32)wave * 2048u : (u32)rh * 8192u + (u32)wave * 1024u) + (u32)lane * 16u;
    const char* gB = ws + pb + (u32)(ASLOTS + ct) * PANEL + (BIG ? (u32)wave * 2048u : (u32)ch * 8192u + (u32)wave * 1024u) + (u32)lane * 16u;
    const u32 ldsw = (u32)wave * (BIG ? 2048u : 1024u);
    constexpr u32 GSTEP = BIG ? IMG : 2u * IMG, G2 = BIG ? 1024u : IMG, L2O = BIG ? 1024u : 8192u;
#define SCHED() __builtin_amdgcn_sched_barrier(0)
#define GLDS16(g, l) __builtin_amdgcn_global_load_lds((const u32*)(g), (LAS u32*)(lds + (l)), 16, 0, GL_AUX)
#define LOADT(t, st) do { const u32 _o = (u32)(t) * GSTEP; \
        GLDS16(gA + _o, (st) * STAGE + ldsw); GLDS16(gA + _o + G2, (st) * STAGE + ldsw + L2O); \
        GLDS16(gB + _o, (st) * STAGE + IMG + ldsw); GLDS16(gB + _o + G2, (st) * STAGE + IMG + ldsw + L2O); } while (0)
#define SNAP(c) __builtin_amdgcn_global_load_lds((const u32*)((c) < NC ? GPTR(c) : ws + OFF_TMO), (LAS u32*)(lds + L_SNAP + ((u32)(c) & 1u) * 256u), 4, 0, AUX_SYS)
    const int ks = BIG ? wave >> 2 : wave >> 1, wm = BIG ? (wave >> 1) & 1 : 0, wn = wave & 1;
    const u32 swz = (u32)((((ks & 1) * 4 + (lane >> 4)) ^ ((lane & 15) >> 1)) << 4) + (BIG ? 0u : (u32)(ks >> 1) * 8192u);
    const u32 ard = (u32)(wm * 64 + (lane & 15)) * 128u + swz, brd = IMG + (u32)(wn * (BT / 2) + (lane & 15)) * 128u + swz;
#define READF(fa, fb, st) do { _Pragma("unroll") for (int _i = 0; _i < 4; ++_i) { \
        if (_i < NB) fb[_i] = *(const LAS f16x8*)(lds + (st) * STAGE + brd + _i * 2048u); \
        fa[_i] = *(const LAS f16x8*)(lds + (st) * STAGE + ard + _i * 2048u); } } while (0)
#define MMA(fa, fb) do { SCHED(); _Pragma("unroll") for (int _m = 0; _m < 4; ++_m) _Pragma("unroll") for (int _n = 0; _n < NB; ++_n) \
        acc[_m][_n] = __builtin_amdgcn_mfma_f32_16x16x32_f16(fb[_n], fa[_m], acc[_m][_n], 0, 0, 0); SCHED(); } while (0)
#define SB() __builtin_amdgcn_sched_barrier(0)
#define STEP(t, sl, na, nb, sr, fa, fb) do { const u32 _o = (u32)(t) * GSTEP; \
        _Pragma("unroll") for (int _m = 0; _m < 4; ++_m) { \
            _Pragma("unroll") for (int _n = 0; _n < NB / 2; ++_n) acc[_m][_n] = __builtin_amdgcn_mfma_f32_16x16x32_f16(fb[_n], fa[_m], acc[_m][_n], 0, 0, 0); \
            SB(); \
            if (_m < 2) { _Pragma("unroll") for (int _i = 0; _i < 2; ++_i) { \
                if (_i < NB / 2) nb[_m * (NB / 2) + _i] = *(const LAS f16x8*)(lds + (sr) * STAGE + brd + (_m * (NB / 2) + _i) * 2048u); \
                na[_m * 2 + _i] = *(const LAS f16x8*)(lds + (sr) * STAGE + ard + (_m * 2 + _i) * 2048u); } } \
            GLDS16((_m < 2 ? gA : gB) + _o + (_m & 1) * G2, (sl) * STAGE + (_m >> 1) * IMG + ldsw + (_m & 1) * L2O); \
            SB(); \
            _Pragma("unroll") for (int _n = NB / 2; _n < NB; ++_n) acc[_m][_n] = __builtin_amdgcn_mfma_f32_16x16x32_f16(fb[_n], fa[_m], acc[_m][_n], 0, 0, 0); \
        } SB(); } while (0)
    f32x4 acc[4][NB];
#pragma unroll
    for (int m = 0; m < 4; ++m)
#pragma unroll
        for (int n = 0; n < NB; ++n) acc[m][n] = (f32x4){0.f, 0.f, 0.f, 0.f};
    f16x8 a0[4], b0[4], a1[4], b1[4];
    if (wave >= 4) {
        const u32 so = (wave < 6 ? OFF_SA + (par * 64u + (u32)slot) * 512u : OFF_SB + (par * BSLOTS + (u32)ct) * 512u) + (u32)((wave & 1) * 64 + lane) * 4u;
        __builtin_amdgcn_global_load_lds((const u32*)(ws + so), (LAS u32*)(lds + L_SC + (u32)(wave - 4) * 256u), 4, 0, AUX_SYS);
    }
    else if (wave >= 2 && headt)
        __builtin_amdgcn_global_load_lds((const u32*)(ws + OFF_RD + (par * 8u + (u32)a_src) * 512u + (u32)((wave & 1) * 64 + lane) * 4u), (LAS u32*)(lds + L_RED + (u32)(wave - 2) * 256u), 4, 0, AUX_SYS);
    LOADT(0, 0u);
    if (wave == 0) SNAP(1);
    LOADT(1, 1u); LOADT(2, 2u); LOADT(3, 3u);
    if (wave == 0) WAIT_VM(13); else WAIT_VM(12);
    BAR();
    PEND();
    READF(a0, b0, 0u);
#define GATE(c) do { const u32 _v = *(const LAS u32*)(lds + L_SNAP + ((u32)(c) & 1u) * 256u + (u32)lane * 4u); \
        if (!__all((int)(_v - call) >= 0)) { if (wave == 0) { poll_ge(GPTR(c), call, ws, call); WAIT_VM(0); } BAR(); } } while (0)
    for (int c = 0; c < NC - 1; ++c) {
        const int t = 4 * c;
        WAIT_VM(8); WAIT_LGKM0(); BAR();
        GATE(c + 1);
        STEP(t + 4, 0u, a1, b1, 1u, a0, b0);
        WAIT_VM(8); WAIT_LGKM0(); BAR();
        if (wave == 0) SNAP(c + 2);
        STEP(t + 5, 1u, a0, b0, 2u, a1, b1);
        if (wave == 0) WAIT_VM(9); else WAIT_VM(8);
        WAIT_LGKM0(); BAR();
        STEP(t + 6, 2u, a1, b1, 3u, a0, b0);
        if (wave == 0) WAIT_VM(9); else WAIT_VM(8);
        WAIT_LGKM0(); BAR();
        STEP(t + 7, 3u, a0, b0, 0u, a1, b1);
        if (BIG && c == 2 * HU - 1 && headt) {
#pragma unroll
            for (int m = 0; m < 4; ++m) {
                const u32 r = (u32)(wm * 64 + m * 16 + (lane & 15)) * 4u;
                const int d = (int)*(const LAS signed char*)(lds + L_RED + r) - (int)*(const LAS signed char*)(lds + L_SC + r);
#pragma unroll
                for (int n = 0; n < NB; ++n)
#pragma unroll
                    for (int j = 0; j < 4; ++j) acc[m][n][j] = __builtin_amdgcn_ldexpf(acc[m][n][j], d);
            }
        }
    }
#define DSTEP(k, n) do { if ((k) > HINT && hint && wave == 0) WAIT_VM((n) + 1); else WAIT_VM(n); WAIT_LGKM0(); BAR(); if ((k) == HINT && hint) YFLAG(); } while (0)
    DSTEP(0, 8);
    READF(a1, b1, 1u); MMA(a0, b0);
    DSTEP(1, 4);
    READF(a0, b0, 2u); MMA(a1, b1);
    DSTEP(2, 0);
    READF(a1, b1, 3u); MMA(a0, b0);
    WAIT_LGKM0(); BAR();
    if (HINT == 3 && hint) YFLAG();
    u32 npo = OOB; u32x4 nv = {0u, 0u, 0u, 0u};
    if (BIG && nx.slot >= 0) { npo = start_off(rank, nx, nbig, par, lane, wave); nv = __builtin_amdgcn_raw_buffer_load_b128(rws, npo, 0, AUX_SYS); }
    MMA(a1, b1);
#undef DSTEP
    if (wave == 0) STAMP(ws, stamp0 + 1);
#pragma unroll
    for (int m = 0; m < 4; ++m)
#pragma unroll
        for (int n = 0; n < NB; ++n) {
            const u32 r = (u32)(wm * 64 + m * 16 + (lane & 15)), pc = (u32)(wn * (BT / 8) + n * 4 + (lane >> 4));
            *(LAS f32x4*)(lds + (u32)ks * AREA + r * RB + ((pc ^ (r & 7u)) << 4)) = acc[m][n];
        }
    WAIT_LGKM0(); BAR();
    const LAS int* sc = (const LAS int*)(lds + L_SC);
    const i32x4 eb = *(const LAS i32x4*)(sc + 128 + col0 + pc * 4u);
    const bool bad = *(const LAS u32*)(lds + L_SNAP) == call;
    if (BIG) { WAIT_VM_TRACKED(0); const bool ok = (int)npo < 0 || all4_ge(nv, call); if (lane == 0) okw[wave] = ok ? 0u : 1u; }
#pragma unroll
    for (u32 it = 0; it < (BIG ? 8u : 2u); ++it) {
        const u32 r = (u32)wave * (BT / 8u) + (it * 64u + (u32)lane) / PPR;
        const u32 lo = r * RB + ((pc ^ (r & 7u)) << 4);
        f32x4 s = *(const LAS f32x4*)(lds + lo) + *(const LAS f32x4*)(lds + AREA + lo);
        if (!BIG) s += *(const LAS f32x4*)(lds + 2u * AREA + lo) + *(const LAS f32x4*)(lds + 3u * AREA + lo);
        const int ea = (int)(signed char)sc[row0 + r];
#pragma unroll
        for (int j = 0; j < 4; ++j) s[j] = bad ? __builtin_nanf("") : __builtin_amdgcn_ldexpf(s[j], ea + eb[j]);
        u32x4 o = __builtin_bit_cast(u32x4, s);
        if (HINT >= 0 && !BIG) {
#pragma unroll
            for (int j = 0; j < 4; ++j) o[j] = o[j] ? o[j] : 0x80000000u;
        }
        __builtin_amdgcn_raw_buffer_store_b128(o, ro, obase + r * ostr, 0, AUX_SC1);
    }
    if (remote && BIG) pend = p.ws[b_src] + OFF_YR + (u32)(a * 64 + slot) * 16u + (u32)(ct & 1) * 8u;
    if (wave == 0) STAMP(ws, stamp0 + 2);
#undef GLDS16
#undef SCHED
#undef SB
#undef STEP
#undef LOADT
#undef SNAP
#undef READF
#undef MMA
#undef GATE
#undef YFLAG
#undef PEND
#undef GPTR
}

constexpr u32 N0 = 8u - DELTA + (DCUT ? 1u : 0u);
constexpr u32 NPOS_INTRA = 88u - HU + N0, NPOS_CROSS = 97u - HU + DELTA;
constexpr u32 FAN = 0x20000000u;
struct Desc { u32 src, dst, gate, flag; };
struct Stream {
    bool cross; int rank, g, a, j; u32 pb, sa;
    __device__ __forceinline__ int yslot(u32 k) const {
        if (k < 6u) return j * 6 + (int)k;
        if (k < 9u) return ((j + (int)k - 5) & 3) * 6;
        if (k < 12u) return 24 + 2 * ((a + (int)k - 8) & 3);
        if (k < 18u) return ((j + 1 + (((int)k - 12) >> 1)) & 3) * 6 + 1 + ((int)k & 1);
        if (k < 21u) return 25 + 2 * ((a + (int)k - 17) & 3);
        if (k < 27u) return ((j + 1 + (((int)k - 21) >> 1)) & 3) * 6 + 3 + (((int)k - 21) & 1);
        return ((j + 1 + (int)(k % 3u)) & 3) * 6 + 5;
    }
    __device__ __forceinline__ u32 single(u32 slot, u32 k) const { return pb + slot * PANEL + k * UNIT; }
    __device__ __forceinline__ u32 pair(u32 slot, u32 k) const { return pb + slot * PANEL + k * 65536u; }
    __device__ __forceinline__ u32 half(u32 slot, u32 h, u32 k) const { return pb + slot * PANEL + k * 2u * UNIT + h * 8192u; }
    __device__ __forceinline__ Desc at(u32 pos) const {
        Desc d;
        const u32 s = (u32)j * 6u, M8 = 7u << 20, PAIR = 1u << 28, HALF = 4u << 28;
        u32 u, fsrc = (u32)((cross ? 1 - g : g) * 4 + j);
        if (!cross) {
            if (pos >= NPOS_INTRA) pos = NPOS_INTRA - 1u;
            if (pos < 80u - HU) {
                u = pos + HU;
                d.src = u < 16u ? single(s, u) : u < 48u ? pair(s + 1u, u - 16u) : pair(s + 3u, u - 48u);
                d.gate = sa + s * 512u + (u < 16u ? 0u : u < 48u ? 512u + PAIR + M8 : 1536u + PAIR + M8);
            } else {
                const u32 e = NPOS_INTRA - 1u - pos, n0 = N0;
                const u32 h = e < 2u || e - 2u >= 2u * n0 || (e & 1u) ? 1u : 0u;
                const u32 k = e < 2u ? 7u - e : e - 2u >= 2u * n0 ? 5u - n0 - (e - 2u - 2u * n0) : h ? 5u - ((e - 2u) >> 1) : 7u - ((e - 2u) >> 1);
                u = 80u + h * 8u + k;
                d.src = half(s + 5u, h, k);
                d.gate = sa + (s + 5u) * 512u + h * 256u + M8 + HALF;
            }
            d.dst = d.src;
            d.flag = OFF_LF + (fsrc * NUNITS + u) * 128u;
        } else {
            if (pos >= NPOS_CROSS) pos = NPOS_CROSS - 1u;
            pos += HU;
            const u32 hop = j == a ? 64u : 68u + DELTA;
            if (pos < 64u) {
                u = pos;
                d.src = u < 16u ? single(32u, u) : u < 48u ? pair((u32)(ASLOTS + 2 * a), u - 16u) : single(33u, u - 48u);
                d.dst = u < 16u ? single((u32)(24 + 2 * j), u) : u < 48u ? pair((u32)(ASLOTS + 8 + 2 * j), u - 16u) : single((u32)(25 + 2 * j), u - 48u);
                d.gate = u < 16u ? sa + (u32)(24 + 2 * j) * 512u : u < 48u ? OFF_WS + (u32)j * 256u + (1u << 20) + (3u << 28) : sa + (u32)(25 + 2 * j) * 512u;
                d.flag = OFF_LF + (fsrc * NUNITS + u) * 128u;
            } else if (pos - hop < DELTA) {
                const u32 k = pos - hop;
                if (j == a) { d.src = half(s + 5u, 0u, k); d.dst = pb + RELAY + k * 2u * UNIT; d.gate = sa + 34u * 512u + M8 + HALF;
                              d.flag = FAN + OFF_RF + ((u32)a * 4u + k) * 128u; }
                else { d.src = pb + RELAY + k * 2u * UNIT; d.dst = half(s + 5u, 0u, k); d.gate = OFF_RF + ((u32)j * 4u + k) * 128u + (6u << 28);
                       d.flag = OFF_LF + ((fsrc ^ 4u) * NUNITS + 80u + k) * 128u; }
            } else {
                const u32 k = pos < hop ? pos - 64u : pos - 64u - DELTA;
                const int ys = yslot(k);
                const u32 h = k >= 30u ? 1u : 0u;
                d.src = OFF_YST + (u32)(a * 32 + ys) * UNIT + h * 65536u;
                d.dst = ((u32)slot_row((int)fsrc, ys) + h * 64u) * (NLOC * 4u) + (u32)j * 1024u;
                d.gate = OFF_YR + ((u32)(j * 64 + ys) + h * 32u) * 16u + ((k >= 27u ? 10u : 2u) << 28);
                d.flag = NOFLAG;
            }
        }
        return d;
    }
};
struct Tables { Desc lo, hi; };
__device__ __forceinline__ u32 tab(u32 t0, u32 t1, u32 i) {
    const u32 x = (u32)__builtin_amdgcn_readlane((int)t0, (int)(i & 63u)), y = (u32)__builtin_amdgcn_readlane((int)t1, (int)(i & 63u));
    return i < 64u ? x : y;
}

#define PSLEEP0 10
#define PSLEEP 4
#define NACT 32
__device__ __forceinline__ void pull_stream(const Params& p, LAS char* lds, const Tables& T, bool cross, u32 cw, char* src, gchar* fan, u32 calln, u32 call, int lane, int wave) {
    const int hw = wave;
    char* ws = p.me;
    const rsrc_t rl = mk_rsrc(ws, WS_NEED), rs = mk_rsrc(src, WS_NEED), rout = mk_rsrc(p.out, IN_BYTES);
    LAS char* pl = lds + (u32)hw * 1024u;
    const u32 pla = (u32)(size_t)pl + (u32)lane * 4u;
    const u32 l16 = (u32)lane * 16u, l4r = (u32)(lane >> 5) * IMG + (l16 & 511u);
    const u32 l0 = lane == 0 ? 0u : OOB;
    const u32 co = OFF_CL + (u32)wave * 128u + l0;
    const u32 NPIECES = (cross ? NPOS_CROSS : NPOS_INTRA) * 32u;
    const u32 skip = !DCUT ? 0u : cross ? (u32)DCUT : 32u - DCUT;
    const u32 qcut = cross ? ((wave == 3 ? 64u : 68u + DELTA) + DELTA - HU) * 32u - DCUT : (NPOS_INTRA - 1u - 2u * N0) * 32u;
    const u32 cbase = 2u * NACT - (calln - 1u) * (NPIECES - skip);
    u32x4 v0 = {0u, 0u, 0u, 0u}, v1 = v0, v2 = v0, v3;
    u32 q = OOB, so = OOB, ss = 1024u;
    u32 c0 = cw, c1 = cw + NACT;
    u32 pf = NOFLAG;
    bool claimed = false, dead = false;
    int cl = 0;
#define KIND(q) (tab(T.lo.gate, T.hi.gate, (q) >> 5) >> 28)
#define POFF(q, kd) (((kd) & 1u) ? (((q) >> 4) & 1u) * PANEL + ((q) & 15u) * 4096u : ((kd) & 4u) ? (((q) >> 1) & 15u) * IMG + ((q) & 1u) * 4096u : ((q) & 31u) * (((kd) & 8u) ? 2048u : (kd) ? 4096u : 512u))
#define GBASE(q) ({ const u32 _g = tab(T.lo.gate, T.hi.gate, (q) >> 5), _k = _g >> 28; \
        const u32 _ix = (((_k & 1u) ? ((q) >> 2) & 4u : 0u) + ((q) & (_k == 4u ? 1u : 3u))) << (_k == 3u ? 5 : 7); \
        (_g & 0xfffffu) + ((_k & 7u) == 2u ? 0u : _k == 6u ? ((q) & 28u) * 4u : _k == 0u ? ((q) & 31u) * 16u : _ix); })
#define GOFF(q) (GBASE(q) + (l16 & (tab(T.lo.gate, T.hi.gate, (q) >> 5) >> 16)))
#define PFLAG() st_sys_g((pf & FAN) ? fan + (pf - FAN) : (gchar*)ws + (pf >= NOFLAG ? OFF_CTR + cw * 64u + 8u + (u32)hw * 4u : pf), call)
#define PLD(u, o) v##u = __builtin_amdgcn_raw_buffer_load_b128(rs, (o) + (u) * ss, 0, AUX_SYS)
#define PSTA(u, o) do { asm volatile("" : "+v"(v##u)); \
        __builtin_amdgcn_raw_buffer_store_b128(v##u | (dead ? 0x7fc00000u : 0u), rd, o, 0, AUX_SC1); } while (0)
#define PST(u) PSTA(u, d + (u) * ds)
#define PSTAMP(i) do { } while (0)
#define GPOLL(i, sl) do { __builtin_amdgcn_global_load_lds((const u32*)gp, (LAS u32*)(pl + (i) * 256u), 4, 0, AUX_SYS); __builtin_amdgcn_s_sleep(sl); } while (0)
#define GSTEP(i, sl) { GPOLL(((i) + 3) & 3, sl); WAIT_VM_TRACKED(3); u32 _w; \
            asm volatile("ds_read_b32 %0, %1 offset:%2\n\ts_waitcnt lgkmcnt(0)" : "=v"(_w) : "v"(pla), "n"((i) * 256) : "memory"); \
            if (__all((int)(_w - call) >= 0)) break; }
#define WATCH(sl) do { \
        const char* gp = ws + GBASE(c0) + ((u32)lane * 4u & ((tab(T.lo.gate, T.hi.gate, c0 >> 5) >> 16 & 0x3f0u) | 12u)); \
        const u64 t0 = __builtin_amdgcn_s_memrealtime(); \
        GPOLL(0, sl); GPOLL(1, sl); GPOLL(2, sl); \
        for (u32 n = 1;; ++n) { \
            GSTEP(0, sl) GSTEP(1, sl) GSTEP(2, sl) GSTEP(3, sl) \
            if ((n & 15u) == 0u && give_up(ws, call, t0)) { dead = true; break; } \
        } } while (0)
    WATCH(PSLEEP0);
    u32x4 gv = {call, call, call, call};
    u32 d = OOB, ds = 1024u, so2 = OOB, d2 = OOB; bool y = false, val = false;
#define VALID(u) __all(v##u[0] != 0u && v##u[1] != 0u && v##u[2] != 0u && v##u[3] != 0u)
#define RETRY(u) do { if (HINT >= 0 && val && !dead && !VALID(u)) { \
        const u64 t0 = __builtin_amdgcn_s_memrealtime(); u32 n = 0; \
        do { PLD(u, so); NRETRY; \
            if ((++n & 15u) == 0u && give_up(ws, call, t0)) dead = true; \
            WAIT_VM_TRACKED(0); \
        } while (!dead && !VALID(u)); } } while (0)
#define NRETRY do { } while (0)
    do {
        const rsrc_t rd = y ? rout : rl;
        WAIT_VM_TRACKED(5); PLD(2, so2); RETRY(0); PST(0);
        WAIT_VM_TRACKED(4); PLD(3, so2); PFLAG(); RETRY(1); PST(1);
        WAIT_VM_TRACKED(5);
        bool open = dead || all4_ge(gv, call);
        if (q == OOB && !open) { WATCH(PSLEEP); open = true; }
        pf = q != OOB ? tab(T.lo.flag, T.hi.flag, q >> 5) + (q & 31u) * 4u : NOFLAG;
        if (claimed) { c1 = (u32)__builtin_amdgcn_readfirstlane(cl) + cbase; if (c1 >= qcut) c1 += skip; if (c1 > NPIECES + 127u) c1 = NPIECES + 127u; }
        claimed = open && c0 < NPIECES;
        q = claimed ? c0 : OOB; if (claimed) c0 = c1;
        const u32 qq = claimed ? q : 0u;
        const u32 kn = claimed ? KIND(qq) : 1u, po = POFF(qq, kn), lo = kn ? l16 : l4r;
        so = claimed ? tab(T.lo.src, T.hi.src, qq >> 5) + po + lo : OOB;
        ss = kn ? 1024u : 2u * IMG;
        y = (kn & 7u) == 2u;
        u32 dn = claimed ? tab(T.lo.dst, T.hi.dst, qq >> 5) + (y ? po * 4u : po) + lo : OOB;
        u32 go = GOFF(c0), ao = claimed ? co : OOB, so1 = so + ss, dx2 = d2 + 2u * ds, dx3 = d2 + 3u * ds;
        if (claimed) PSTAMP(q >> 5);
        asm volatile("" : "+v"(so), "+v"(dn), "+v"(go), "+v"(ao), "+v"(so1), "+v"(dx2), "+v"(dx3));
        WAIT_VM_TRACKED(4);
        v0 = __builtin_amdgcn_raw_buffer_load_b128(rs, so, 0, AUX_SYS);
        PSTA(2, dx2);
        WAIT_VM_TRACKED(4);
        PSTA(3, dx3);
        v1 = __builtin_amdgcn_raw_buffer_load_b128(rs, so1, 0, AUX_SYS);
        cl = __builtin_amdgcn_raw_ptr_buffer_atomic_add_i32(1, rl, ao, 0, AUX_SC1);
        gv = __builtin_amdgcn_raw_buffer_load_b128(rl, go, 0, AUX_SYS);
        d = dn; ds = y ? NLOC * 4u : ss;
        val = (kn & 8u) != 0u;
        so2 = val ? OOB : so; d2 = val ? OOB : dn;
    } while (c0 < NPIECES || q != OOB);
    WAIT_VM_TRACKED(0); PFLAG();
    asm volatile(".Lagk_end:");
    if (hw < 7) STAMP(ws, 2 + wave * 8 + 7);
#undef PSTAMP
#undef GOFF
#undef GBASE
#undef WATCH
#undef GSTEP
#undef GPOLL
#undef POFF
#undef KIND
#undef PST
#undef PSTA
#undef RETRY
#undef VALID
#undef NRETRY
#undef PLD
#undef PFLAG
}

__global__ void __launch_bounds__(512, 2) ag_gemm_kernel(Params p) {
    extern __shared__ __attribute__((aligned(16))) char smem[];
    LAS char* lds = (LAS char*)smem;
    const int tid = threadIdx.x, lane = tid & 63, wave = __builtin_amdgcn_readfirstlane(tid >> 6);
    const int b = blockIdx.x, rank = p.rank, g = rank >> 2, a = rank & 3;
    u64 pc; u32 len;
    asm volatile("s_getpc_b64 %0\n.Lagk_pc:\n\ts_mov_b32 %1, .Lagk_end-.Lagk_pc" : "=s"(pc), "=s"(len));
    const u32 code = __builtin_amdgcn_raw_buffer_load_b32(mk_rsrc((const void*)pc, (int)len > 0 ? len : 0u), (u32)(b >> 3) * 2048u + (u32)tid * 4u, 0, 0);
    char* ws = p.me;
    if (wave == 0) STAMP(ws, 0);
    const u32 prev = ld_agent(ws + OFF_CTR + (u32)b * 64u);
    u32 calln;
    const int lw = wave;
    const rsrc_t rws = mk_rsrc(ws, WS_NEED);

    Tables T = {};
    char* src = ws; gchar* fan = (gchar*)ws;
    u32 hword = 0;
    gchar* rdflag = x_head(p, lds, b < 128 ? b : 640 + b, prev, calln, hword, tid, wave, [&]() {
        if (b < NCOMM) {
            const u32 par = calln & 1u;
            Stream S;
            S.cross = lw >= 3; S.rank = rank; S.g = g; S.a = a; S.j = (a + lw + 1) & 3; S.pb = CTRL_BYTES + par * PAR_BYTES; S.sa = OFF_SA + par * 32768u;
            T.lo = S.at((u32)lane); T.hi = S.at((u32)lane + 64u);
            src = p.peer[lw == 3 ? 7 : lw];
            char *f0 = p.peer[4], *f1 = p.peer[5], *f2 = p.peer[6];
            asm volatile("" : "+s"(src), "+s"(f0), "+s"(f1), "+s"(f2));
            fan = (gchar*)(lane == 0 ? f0 : lane == 1 ? f1 : f2);
        }
    });
    asm volatile("" :: "v"(code));
    const u32 call = calln << 8, par = calln & 1u;
    if (tid < 2) st_agent(ws + OFF_CTR + (u32)b * 64u + (u32)tid * 4u, tid ? call : calln);
    if (wave == 0) STAMP(ws, 1);
    if (b < NCOMM) {
        if (wave == 7) {
            WAIT_VM(0);
            if (lane < 4) st_sys_g(rdflag, hword);
        }
        else if (b < NACT) pull_stream(p, lds, T, wave >= 3, (u32)b, src, fan, calln, call, lane, wave);
        return;
    }

    {
        const int c = b - NCOMM, strip = c / 7, rg = c - strip * 7;
        const int kt0 = (rg * 128) / 7, kt1 = ((rg + 1) * 128) / 7;
        const rsrc_t rw = mk_rsrc(p.w, IN_BYTES);
        const int q = lane & 7, n4 = lane >> 3;
        if (wave == 7) { WAIT_VM(0); if (lane < 4) st_sys_g(rdflag, hword); }
        f32x4 v[3][8];
#pragma unroll
        for (int i = 0; i < 3; ++i) {
            const int kt = kt0 + wave + 8 * i;
            const u32 base = kt < kt1 ? ((u32)(kt * 64 + q * 8) * NLOC + (u32)(strip * 32 + n4 * 4)) * 4u : OOB;
#pragma unroll
            for (int jj = 0; jj < 8; ++jj) {
                v[i][jj] = __builtin_bit_cast(f32x4, __builtin_amdgcn_raw_buffer_load_b128(rw, base + (u32)jj * (NLOC * 4u), 0, 0));
                WAIT_VM(W_LD - 1);
            }
        }
        LAS u32* cm = (LAS u32*)(lds + L_RED + 256u);
        if (tid < 64) cm[tid] = 0u;
        WAIT_LGKM0(); BAR();
        u32 m[4] = {0u, 0u, 0u, 0u};
#pragma unroll
        for (int i = 0; i < 3; ++i)
#pragma unroll
            for (int jj = 0; jj < 8; ++jj)
#pragma unroll
                for (int k = 0; k < 4; ++k) m[k] = umax(m[k], absbits(v[i][jj][k]));
#pragma unroll
        for (int k = 0; k < 4; ++k) {
            m[k] = umax(m[k], (u32)__shfl_xor((int)m[k], 1)); m[k] = umax(m[k], (u32)__shfl_xor((int)m[k], 2)); m[k] = umax(m[k], (u32)__shfl_xor((int)m[k], 4));
        }
        if (q < 4) __hip_atomic_fetch_max(cm + n4 * 4 + q, q == 0 ? m[0] : q == 1 ? m[1] : q == 2 ? m[2] : m[3], __ATOMIC_RELAXED, __HIP_MEMORY_SCOPE_WORKGROUP);
        WAIT_LGKM0(); BAR();
        u64* wx = (u64*)(ws + OFF_WX) + (u32)strip * 256u;
        if (tid < 32) __hip_atomic_store(wx + rg * 32 + tid, ((u64)call << 32) | cm[tid], __ATOMIC_RELAXED, __HIP_MEMORY_SCOPE_AGENT);
        if (wave < 4) {
            const int gi = tid < 224 ? tid : 0;
            const u64 t0 = __builtin_amdgcn_s_memrealtime();
            u64 gv; u32 n = 0;
            for (;;) {
                gv = __hip_atomic_load(wx + gi, __ATOMIC_RELAXED, __HIP_MEMORY_SCOPE_AGENT);
                if (__all((u32)(gv >> 32) == call)) break;
                if ((++n & 15u) == 0u && give_up(ws, call, t0)) break;
                __builtin_amdgcn_s_sleep(1);
            }
            __hip_atomic_fetch_max(cm + 32 + (gi & 31), (u32)gv, __ATOMIC_RELAXED, __HIP_MEMORY_SCOPE_WORKGROUP);
        }
        WAIT_LGKM0(); BAR();
        const int ct = strip >> 2;
        if (rg == 0 && tid < 64) {
            const u32 nin = (u32)((strip & 3) * 32 + (tid & 31));
            const u32 ev = (u32)(exp_of(cm[32 + (tid & 31)]) - 141);
            char* t = tid < 32 ? ws + OFF_SB + (par * BSLOTS + (u32)ct) * 512u : p.ws[(1 - g) * 4 + (strip >> 3)] + OFF_SB + (par * BSLOTS + (u32)(8 + 2 * a + (ct & 1))) * 512u;
            st_sys(t + nin * 4u, ev);
        }
        const u32x4 mx = *(LAS u32x4*)(cm + 32 + n4 * 4);
        const u32 pbase = CTRL_BYTES + par * PAR_BYTES + (u32)(ASLOTS + ct) * PANEL;
#pragma unroll
        for (int k = 0; k < 4; ++k) {
            const float s = scale_of(exp_of(mx[k]));
            const u32 nin = (u32)((strip & 3) * 32 + n4 * 4 + k);
            const u32 ro = nin * 128u + (((u32)q ^ ((nin >> 1) & 7u)) << 4);
#pragma unroll
            for (int i = 0; i < 3; ++i) {
                const int kt = kt0 + wave + 8 * i;
                const u32 off = kt < kt1 ? pbase + (u32)kt * IMG + ro : OOB;
                const f32x4 lo = {v[i][0][k], v[i][1][k], v[i][2][k], v[i][3][k]}, hi = {v[i][4][k], v[i][5][k], v[i][6][k], v[i][7][k]};
                __builtin_amdgcn_raw_buffer_store_b128(cvt8(lo, hi, s), rws, off, 0, AUX_SC1);
                WAIT_VM(W_ST - 1);
            }
        }
        WAIT_VM(0); BAR();
        if (tid < 4 && (!(tid & 1) || rg == 6))
            st_sys((tid < 2 ? ws + OFF_WR + (u32)strip * 32u : p.ws[(1 - g) * 4 + (strip >> 3)] + OFF_WS + (u32)(a * 8 + (strip & 7)) * 32u) + (u32)(rg + (tid & 1)) * 4u, call);
        if (wave == 0) STAMP(ws, 2);
    }
    const int xcd = b & 7, q = (b >> 3) - 4;
    if (xcd >= 4 && q < 24) {
        const int r0 = 128 + (q * 4 + xcd - 4) * 4;
        x_rows(p, lds, r0, par, call, tid, wave);
        x_rows(p, lds, r0 + (r0 < 384 ? 384 : 512), par, call, tid, wave);
        if (wave == 0) STAMP(ws, 3);
    }
    const int qr = q >> 2, qc = q & 3, x4 = xcd & 3;
    const int early = (x4 >> 1) == (xcd < 4 ? ESWAP : 0) ? (qr < 6 ? a * 6 + qr : -1) : (qr < 3 ? ((a + 1 + qr) & 3) * 6 : 24 + 2 * (qr - 3));
    const int peer = ((a + 1 + (qr >> 1)) & 3) * 6;
    auto tile_of = [&](int it) {
        int slot = -1, ct = 4 * x4 + qc, rh = 0, ch = 0;
        if (it == 0) { if (xcd < 4) { slot = early; ct = 4 * (x4 & 1) + qc; } }
        else if (it == 1) { if (xcd < 4) { if (qr < 6) slot = peer + 1 + (qr & 1); } else { slot = early; ct = 8 + 4 * (x4 & 1) + qc; } }
        else if (it == 2) {
            if (qr == 6) { slot = 25 + (xcd & 6); ct = 8 + 4 * (xcd & 1) + qc; }
            else if (xcd >= 4) slot = peer + 3 + (qr & 1);
            else if (xcd < 2 && qr < 4) slot = 25 + 2 * qr;
        } else if (it == 3) {
            int pi = -1; ch = qc & 1;
            if (xcd < 2) { pi = qr - 4; rh = 1; ct = 8 + 2 * xcd + (qc >> 1); }
            else if (qr == 6) { pi = (xcd - 2) >> 1; rh = 1; ct = 12 + 2 * (xcd & 1) + (qc >> 1); }
            else if (xcd < 4) { pi = qr >> 1; ct = 4 * xcd + 2 * (qr & 1) + (qc >> 1); }
            else { pi = qr >> 1; rh = qr & 1; ct = 2 * x4 + (qc >> 1); }
            if (pi >= 0) slot = ((a + 1 + pi) & 3) * 6 + 5;
        }
        if (slot >= 24 && ct >= 8 && (slot - 24) >> 1 == (ct - 8) >> 1) { slot = 32 + (slot & 1); ct -= 8; }
        return Tile{slot, ct, rh, ch};
    };
    char* pend = nullptr;
    int it = 0;
    Tile tl = tile_of(0);
    while (tl.slot < 0) tl = tile_of(++it);
    for (bool first = true; it < 4; first = false) {
        int nt = it + 1;
        Tile nx = tile_of(nt);
        while (nx.slot < 0 && nt < 4) nx = tile_of(++nt);
        if (it < 3) gemm_tile<true>(p, lds, tl, first, nx, nt < 3, par, call, tid, lane, wave, 4 + it * 3, pend);
        else gemm_tile<false>(p, lds, tl, first, nx, false, par, call, tid, lane, wave, 4 + it * 3, pend);
        it = nt; tl = nx;
    }
    if (pend) { WAIT_VM(0); BAR(); if (tid < 2) st_sys(pend + (u32)tid * 4u, call); }
}
}

extern "C" void kernel_launch(int rank, int nranks, ncclComm_t comm,
                              hipStream_t stream,
                              void* const* d_in, const int* in_sizes, int n_in,
                              void* d_out, int out_size,
                              const int* in_dtypes, int out_dtype,
                              void* const* d_out_all,
                              void* const* d_ws_all, size_t ws_bytes) {
    (void)comm; (void)in_dtypes; (void)out_dtype; (void)d_out_all;
    if (nranks != 8 || (unsigned)rank >= 8u || n_in < 2 || ws_bytes < (size_t)agk::WS_NEED || in_sizes[0] != 1024 * 8192 ||
        in_sizes[1] != 8192 * 1024 || out_size != 8192 * 1024) {
        fprintf(stderr, "kernel_launch: unexpected problem shape\n");
        return;
    }
    static bool ready[8] = {false, false, false, false, false, false, false, false};
    if (!ready[rank]) {
        HIP_CHECK(hipFuncSetAttribute((const void*)agk::ag_gemm_kernel, hipFuncAttributeMaxDynamicSharedMemorySize, (int)agk::LDS_BYTES));
        ready[rank] = true;
    }
    agk::Params p;
    p.x = (const float*)d_in[0]; p.w = (const float*)d_in[1]; p.rank = rank; p.pad = 0;
    p.me = (char*)d_ws_all[rank]; p.out = (float*)d_out;
    for (int i = 0; i < 8; ++i) { p.ws[i] = (char*)d_ws_all[i]; p.peer[i] = (char*)d_ws_all[((rank ^ i) & 4) + ((rank + 1 + i) & 3)]; }
    hipLaunchKernelGGL(agk::ag_gemm_kernel, dim3(agk::NWG), dim3(512), agk::LDS_BYTES, stream, p);
    HIP_CHECK(hipGetLastError());
}
```

```cpp
#include <hip/hip_runtime.h>
#include <rccl/rccl.h>
#include <cstdio>

#define HIP_CHECK(expr) do { hipError_t _e = (expr); if (_e != hipSuccess) { \
    fprintf(stderr, "HIP error %s at %s:%d\n", hipGetErrorString(_e), __FILE__, __LINE__); \
    return; } } while (0)

namespace agk {
typedef unsigned u32;
typedef unsigned long long u64;
typedef u32 u32x4 __attribute__((ext_vector_type(4)));
typedef int i32x4 __attribute__((ext_vector_type(4)));
typedef float f32x4 __attribute__((ext_vector_type(4)));
typedef float f32x2 __attribute__((ext_vector_type(2)));
typedef _Float16 f16x2 __attribute__((ext_vector_type(2)));
typedef _Float16 f16x8 __attribute__((ext_vector_type(8)));
#define LAS __attribute__((address_space(3)))
typedef __amdgpu_buffer_rsrc_t rsrc_t;
typedef __attribute__((address_space(1))) char gchar;

constexpr int KDIM = 8192, NLOC = 1024, NKT = 128;
constexpr u32 IMG = 16384u;
constexpr u32 PANEL = NKT * IMG;
constexpr u32 UNIT = 131072u;
constexpr int ASLOTS = 34, BSLOTS = 16;
#define HU 3
#define HWIN 8
#define SCH 2
#define DRY 1
#define NF 4
#define NFL 2
#define RU0 68
#define HOP1 56
#define HOP2 64
#define LEAD 2
#define QSWAP 1
#define RMAP 0
#define FGALL 0
#define LASTP 1
#define ENDQ 1
#define FSLEEP 3
#define CTR7 1
#define ESWAP 1
#define HSTEP 1
#define HK 8
#define XDLY 30
#define HINT 1
#define DCUT 0
#define DELTA 3
constexpr u32 CTRL_BYTES = 1u << 20;
constexpr u32 RELAY = ASLOTS + BSLOTS;
constexpr u32 PAR_BYTES = (RELAY + 2u) * PANEL;
constexpr u32 OFF_YST = CTRL_BYTES + 2u * PAR_BYTES;
constexpr u32 WS_NEED = OFF_YST + 4u * 32u * UNIT;
constexpr u32 IN_BYTES = 1024u * 8192u * 4u;
constexpr u32 OFF_CTR = 0;
constexpr u32 OFF_TMO = 16384;
constexpr u32 OFF_CL = 20480;
constexpr u32 OFF_LF = 524288;
constexpr u32 OFF_RD = 655360;
constexpr u32 OFF_YR = 24576;
constexpr u32 OFF_RF = 28672;
constexpr u32 OFF_WR = 32768;
constexpr u32 OFF_WS = 33792;
constexpr u32 OFF_WX = 36864;
constexpr u32 OFF_SA = 131072;
constexpr u32 OFF_SB = 196608;
constexpr u32 OFF_DBG = 262144;
constexpr int NWG = 256, NCOMM = 32;
constexpr u32 NUNITS = 96u;
constexpr u32 OOB = 0x80000000u, NOFLAG = 0x40000000u;
constexpr u64 DEADLINE = 30000000ull;
constexpr u32 STAGE = 32768u;
constexpr u32 L_SNAP = 4u * STAGE;
constexpr u32 L_SC = L_SNAP + 512u;
constexpr u32 L_RED = L_SC + 1024u;
constexpr u32 LDS_BYTES = L_RED + 2048u;

struct Params {
    const float* x; char* me; int rank, pad;
    char* peer[8];
    const float* w; float* out;
    char* ws[8];
};

#define AUX_SC1 16
#define AUX_SYS 17
#define GL_AUX 16
#define W_LD 3
#define W_ST 2
#define X_LD 4
#define X_ST 2
#define WAIT_VM(n) asm volatile("s_waitcnt vmcnt(%0)" :: "n"(n) : "memory")
#define WAIT_LGKM0() do { asm volatile("" ::: "memory"); __builtin_amdgcn_s_waitcnt(15 | (7 << 4) | (0 << 8) | (3 << 14)); asm volatile("" ::: "memory"); } while (0)
#define BAR() do { asm volatile("" ::: "memory"); __builtin_amdgcn_s_barrier(); asm volatile("" ::: "memory"); } while (0)
#define WAIT_VM_TRACKED(n) do { asm volatile("" ::: "memory"); __builtin_amdgcn_s_waitcnt(((n) & 15) | (7 << 4) | (15 << 8) | (((n) >> 4) << 14)); asm volatile("" ::: "memory"); } while (0)

__device__ __forceinline__ rsrc_t mk_rsrc(const void* p, u32 bytes) { return __builtin_amdgcn_make_buffer_rsrc((void*)p, 0, bytes, 0x00020000); }
__device__ __forceinline__ u32 ld_agent(const void* p) { return __hip_atomic_load((const u32*)p, __ATOMIC_RELAXED, __HIP_MEMORY_SCOPE_AGENT); }
__device__ __forceinline__ u32 ld_sys(const void* p) { return __hip_atomic_load((const u32*)p, __ATOMIC_RELAXED, __HIP_MEMORY_SCOPE_SYSTEM); }
__device__ __forceinline__ void st_agent(void* p, u32 v) { __hip_atomic_store((u32*)p, v, __ATOMIC_RELAXED, __HIP_MEMORY_SCOPE_AGENT); }
__device__ __forceinline__ void st_sys_g(gchar* p, u32 v) { __hip_atomic_store((__attribute__((address_space(1))) u32*)p, v, __ATOMIC_RELAXED, __HIP_MEMORY_SCOPE_SYSTEM); }
__device__ __forceinline__ void st_sys(void* p, u32 v) { __hip_atomic_store((u32*)p, v, __ATOMIC_RELAXED, __HIP_MEMORY_SCOPE_SYSTEM); }

#define STAMP(ws, idx) do { } while (0)

__device__ __forceinline__ bool give_up(char* ws, u32 call, u64 t0) {
    const u32 t = __builtin_amdgcn_readfirstlane(ld_agent(ws + OFF_TMO));
    if (t == call || __builtin_amdgcn_s_memrealtime() - t0 > DEADLINE) { st_agent(ws + OFF_TMO, call); return true; }
    return false;
}
__device__ __forceinline__ void poll_ge(const char* p, u32 need, char* ws, u32 call) {
    const u64 t0 = __builtin_amdgcn_s_memrealtime();
    u32 n = 0;
    for (;;) {
        const u32 v = ld_sys(p);
        if (__all((int)(v - need) >= 0)) return;
        if ((++n & 15u) == 0u && give_up(ws, call, t0)) return;
        __builtin_amdgcn_s_sleep(2);
    }
}
__device__ __forceinline__ bool all4_ge(u32x4 v, u32 need) {
    return __all((int)(v[0] - need) >= 0 && (int)(v[1] - need) >= 0 && (int)(v[2] - need) >= 0 && (int)(v[3] - need) >= 0);
}
__device__ __forceinline__ void poll4_ge(rsrc_t rs, u32 off, u32 need, char* ws, u32 call) {
    const u64 t0 = __builtin_amdgcn_s_memrealtime();
    u32 n = 0;
    for (;;) {
        asm volatile("" ::: "memory");
        const u32x4 v = __builtin_amdgcn_raw_buffer_load_b128(rs, off, 0, AUX_SYS);
        if (all4_ge(v, need)) return;
        if ((++n & 15u) == 0u && give_up(ws, call, t0)) return;
        __builtin_amdgcn_s_sleep(1);
    }
}

__device__ __forceinline__ u32 umax(u32 a, u32 b) { return a > b ? a : b; }
__device__ __forceinline__ u32 absbits(float f) { return __float_as_uint(f) & 0x7fffffffu; }
__device__ __forceinline__ int exp_of(u32 amaxbits) { int e = (int)(amaxbits >> 23); return e < 15 ? 15 : e; }
__device__ __forceinline__ float scale_of(int e) { return __uint_as_float((u32)(268 - e) << 23); }
__device__ __forceinline__ u32 cvt2(float a, float b, float s) {
    const f32x2 t = (f32x2){a, b} * s;
    return __builtin_bit_cast(u32, __builtin_convertvector(t, f16x2));
}
__device__ __forceinline__ u32x4 cvt8(f32x4 a, f32x4 b, float s) {
    return (u32x4){cvt2(a[0], a[1], s), cvt2(a[2], a[3], s), cvt2(b[0], b[1], s), cvt2(b[2], b[3], s)};
}
__device__ __forceinline__ u32 amax4(u32 m, f32x4 v) {
    asm("v_max3_f32 %0, |%1|, |%2|, %0\n\tv_max3_f32 %0, |%3|, |%4|, %0" : "+v"(m) : "v"(v[0]), "v"(v[1]), "v"(v[2]), "v"(v[3]));
    return m;
}

__device__ __forceinline__ u32 wave_max(u32 m) {
    m = umax(m, (u32)__builtin_amdgcn_update_dpp(0, (int)m, 0xb1, 15, 15, true));
    m = umax(m, (u32)__builtin_amdgcn_update_dpp(0, (int)m, 0x4e, 15, 15, true));
    m = umax(m, (u32)__builtin_amdgcn_update_dpp(0, (int)m, 0x141, 15, 15, true));
    m = umax(m, (u32)__builtin_amdgcn_update_dpp(0, (int)m, 0x140, 15, 15, true));
    return umax(umax((u32)__builtin_amdgcn_readlane((int)m, 0), (u32)__builtin_amdgcn_readlane((int)m, 16)),
                umax((u32)__builtin_amdgcn_readlane((int)m, 32), (u32)__builtin_amdgcn_readlane((int)m, 48)));
}
__device__ __forceinline__ u32 row_word(u32 call, int e) { return call | ((u32)(e - 141) & 255u); }

__device__ __forceinline__ void x_rows(const Params& p, LAS char* lds, int row0, u32 par, u32 call, int tid, int wave) {
    constexpr int NR = 4;
    const int a = p.rank & 3;
    char* ws = p.me;
    const rsrc_t rx = mk_rsrc(p.x, IN_BYTES), rw = mk_rsrc(ws, WS_NEED);
    const u32 pi0 = (u32)(wave * 128 + (tid & 63));
    f32x4 v[NR][4];
#pragma unroll
    for (int r = 0; r < NR; ++r)
#pragma unroll
        for (int u = 0; u < 2; ++u) {
            const u32 off = (u32)(row0 + r) * (KDIM * 4u) + (pi0 + (u32)u * 64u) * 32u;
            v[r][2 * u] = __builtin_bit_cast(f32x4, __builtin_amdgcn_raw_buffer_load_b128(rx, off, 0, 0));
            v[r][2 * u + 1] = __builtin_bit_cast(f32x4, __builtin_amdgcn_raw_buffer_load_b128(rx, off + 16u, 0, 0));
            WAIT_VM(X_LD - 2);
        }
    const int rb = row0 >> 7;
    const u32 slot = rb < 6 ? (u32)(a * 6 + rb) : (u32)(32 + rb - 6);
    const u32 rslot = rb < 6 ? slot : (u32)(24 + 2 * a + rb - 6);
    char* hp[4];
#pragma unroll
    for (int i = 0; i < 4; ++i) hp[i] = rb < 6 ? p.peer[i] : p.peer[4 + i];
    char* fifth = rb < 6 ? p.peer[7] : ws;
    asm volatile("" : "+s"(hp[0]), "+s"(hp[1]), "+s"(hp[2]), "+s"(hp[3]), "+s"(fifth));
    gchar* user = (gchar*)((tid & 4) ? fifth : (tid & 2) ? ((tid & 1) ? hp[3] : hp[2]) : ((tid & 1) ? hp[1] : hp[0]));
    const u32 wr = (u32)(tid & 63) >> 3;
    const bool wsend = wr < (u32)NR && ((tid & 7) < 4 || ((tid & 7) == 4 && rb >= 3 && rb != 5));
    const u32 woff = OFF_SA + ((par * 64u + ((tid & 4) ? (rb < 6 ? (u32)(31 + rb) : slot) : rslot)) * 128u + (((u32)row0 + wr) & 127u)) * 4u;
    u32 word = 0;
    LAS u32* red = (LAS u32*)(lds + L_RED);
#pragma unroll
    for (int r = 0; r < NR; ++r) {
        u32 m = 0;
#pragma unroll
        for (int i = 0; i < 4; ++i) m = amax4(m, v[r][i]);
        m = wave_max(m);
        if ((tid & 63) == 0) red[r * 8 + wave] = m;
    }
    WAIT_LGKM0(); BAR();
#pragma unroll
    for (int r = 0; r < NR; ++r) {
        const u32x4 m0 = *(LAS u32x4*)(red + r * 8), m1 = *(LAS u32x4*)(red + r * 8 + 4);
        const u32 m = umax(umax(umax(m0[0], m0[1]), umax(m0[2], m0[3])), umax(umax(m1[0], m1[1]), umax(m1[2], m1[3])));
        const int e = exp_of(m); const float s = scale_of(e);
        const u32 rin = (u32)((row0 + r) & 127);
        if (wr == (u32)r) word = row_word(call, e);
        const u32 off = CTRL_BYTES + par * PAR_BYTES + rin * 128u + (pi0 >> 3) * IMG + (((pi0 & 7u) ^ ((rin >> 1) & 7u)) << 4);
        __builtin_amdgcn_raw_buffer_store_b128(cvt8(v[r][0], v[r][1], s), rw, off + slot * PANEL, 0, AUX_SC1);
        __builtin_amdgcn_raw_buffer_store_b128(cvt8(v[r][2], v[r][3], s), rw, off + slot * PANEL + 8u * IMG, 0, AUX_SC1);
        WAIT_VM(X_ST - 2);
    }
    WAIT_VM(0); BAR();
    if (wave == 0 && wsend) st_sys_g(user + woff, word);
}

template <class F>
__device__ __forceinline__ gchar* x_head(const Params& p, LAS char* lds, int row, u32 prev, u32& calln, u32& hword, int tid, int wave, F mid) {
    constexpr int NS = HU > 3 ? HU : 3;
    const int lane = tid & 63;
    char* ws = p.me;
    const rsrc_t rx = mk_rsrc(p.x, IN_BYTES), rw = mk_rsrc(ws, WS_NEED);
    LAS u32* red = (LAS u32*)(lds + L_RED);
    if (wave != 7) {
        if (tid < 4) red[tid] = 0u;
        WAIT_LGKM0(); BAR();
        __builtin_amdgcn_s_sleep(XDLY);
    }
    const u32 s0 = wave == 7 ? 0u : (u32)wave, sd = wave == 7 ? 1u : 7u, se = wave == 7 ? (u32)HU : 16u;
    f32x4 v[NS][2];
#pragma unroll
    for (int u = 0; u < NS; ++u) {
        const u32 set = s0 + (u32)u * sd;
        const u32 off = set < se ? (u32)row * (KDIM * 4u) + (set * 64u + (u32)lane) * 32u : OOB;
        v[u][0] = __builtin_bit_cast(f32x4, __builtin_amdgcn_raw_buffer_load_b128(rx, off, 0, 0));
        v[u][1] = __builtin_bit_cast(f32x4, __builtin_amdgcn_raw_buffer_load_b128(rx, off + 16u, 0, 0));
    }
    __builtin_amdgcn_sched_barrier(0);
    if (wave == 7) BAR();
    const bool major = row < 128;
    const int a = p.rank & 3;
    char* hp[4];
#pragma unroll
    for (int i = 0; i < 4; ++i) hp[i] = major ? p.peer[i] : p.peer[4 + i];
    const u32 slot = major ? (u32)(a * 6) : 32u, rslot = major ? slot : (u32)(24 + 2 * a), rin = (u32)row & 127u;
    u32 off = CTRL_BYTES + rin * 128u + ((u32)lane >> 3) * IMG + ((((u32)lane & 7u) ^ ((rin >> 1) & 7u)) << 4);
    u32 m = 0;
    gchar* user = (gchar*)((tid & 4) ? ws : (tid & 2) ? ((tid & 1) ? hp[3] : hp[2]) : ((tid & 1) ? hp[1] : hp[0]));
#define WORD() do { u32 t = 0; \
        if (lane == 0) t = __hip_atomic_fetch_add(red + 2, 1u, __ATOMIC_RELAXED, __HIP_MEMORY_SCOPE_WORKGROUP); \
        if (__builtin_amdgcn_readfirstlane((int)t) == 7 && lane < (major ? 4 : 5)) \
            st_sys_g(user + OFF_SA + (((calln & 1u) * 64u + (lane == 4 ? slot : rslot)) * 128u + rin) * 4u, \
                     row_word(calln << 8, exp_of(__hip_atomic_load(red, __ATOMIC_RELAXED, __HIP_MEMORY_SCOPE_WORKGROUP)))); } while (0)
    if (wave == 7) {
        rsrc_t hr[4];
#pragma unroll
        for (int i = 0; i < 4; ++i) hr[i] = mk_rsrc(hp[i], i == 3 && major ? 0u : WS_NEED);
        u32 o4[4];
#pragma unroll
        for (int i = 0; i < 4; ++i) o4[i] = (lane >> 4) == i ? off + rslot * PANEL : OOB;
        asm volatile("" : "+s"(hr[0]), "+s"(hr[1]), "+s"(hr[2]), "+s"(hr[3]), "+v"(o4[0]), "+v"(o4[1]), "+v"(o4[2]), "+v"(o4[3]), "+v"(m), "+v"(prev));
        calln = (u32)__builtin_amdgcn_readfirstlane(prev) + 1u;
        const u32 so = (calln & 1u) * PAR_BYTES;
#pragma unroll
        for (int u = 0; u < HU; ++u) m = amax4(amax4(m, v[u][0]), v[u][1]);
        m = wave_max(m);
        const int e = exp_of(m); const float s = scale_of(e);
        u32x4 h;
#pragma unroll
        for (int k = 0; k < 4 * HU; ++k) {
            if ((k & 3) == 0) h = cvt8(v[k >> 2][0], v[k >> 2][1], s);
            if (k == 2 && lane == 0) __hip_atomic_fetch_add(red + 1, 1u, __ATOMIC_RELAXED, __HIP_MEMORY_SCOPE_WORKGROUP);
#pragma unroll
            for (int i = 0; i < 4; ++i) {
                if (HSTEP ? i == 0 && 4 * k >= HWIN : 4 * k + i >= HWIN) WAIT_VM(HWIN - (HSTEP ? 4 : 1));
                __builtin_amdgcn_raw_buffer_store_b128(h, hr[i], o4[k & 3], so + (u32)(k >> 2) * 8u * IMG, AUX_SYS);
            }
            asm volatile("s_nop 1" : "+v"(h) :: "memory");
            if (k == HK) WORD();
        }
        hword = row_word(calln << 8, e);
        for (u32 n = 0; n < (1u << 24); ++n)
            if (__builtin_amdgcn_readfirstlane((int)__hip_atomic_load(red + 1, __ATOMIC_RELAXED, __HIP_MEMORY_SCOPE_WORKGROUP)) == 8) break;
    }
    else calln = (u32)__builtin_amdgcn_readfirstlane(prev) + 1u;
    const u32 par = calln & 1u;
    if (wave != 7) {
        mid();
        off += par * PAR_BYTES + slot * PANEL;
#pragma unroll
        for (int u = 0; u < 3; ++u) m = amax4(amax4(m, v[u][0]), v[u][1]);
        m = wave_max(m);
        if (lane == 0) {
            __hip_atomic_fetch_max(red, m, __ATOMIC_RELAXED, __HIP_MEMORY_SCOPE_WORKGROUP);
            __hip_atomic_fetch_add(red + 1, 1u, __ATOMIC_RELAXED, __HIP_MEMORY_SCOPE_WORKGROUP);
        }
        u64 mc;
        for (u32 n = 0; n < (1u << 24); ++n) {
            mc = __hip_atomic_load((LAS u64*)red, __ATOMIC_RELAXED, __HIP_MEMORY_SCOPE_WORKGROUP);
            if ((u32)__builtin_amdgcn_readfirstlane((int)(mc >> 32)) == 8u) break;
        }
        const int e = exp_of((u32)mc); const float s = scale_of(e);
#pragma unroll
        for (int u = 0; u < 3; ++u) {
            const u32 set = s0 + (u32)u * 7u;
            __builtin_amdgcn_raw_buffer_store_b128(cvt8(v[u][0], v[u][1], s), rw, set < 16u ? off + set * 8u * IMG : OOB, 0, AUX_SC1);
        }
        WAIT_VM(0);
        WORD();
    }
#undef WORD
    return user + OFF_RD + ((par * 8u + (u32)p.rank) * 128u + rin) * 4u;
}

__device__ __forceinline__ int slot_row(int rank, int slot) {
    const int g = rank >> 2;
    if (slot < 24) { const int i = slot / 6; return (g * 4 + i) * 1024 + (slot - i * 6) * 128; }
    return (slot < 32 ? (1 - g) * 4 + ((slot - 24) >> 1) : rank) * 1024 + 768 + (slot & 1) * 128;
}

struct Tile { int slot, ct, rh, ch; };
__device__ __forceinline__ void a_stream(int rank, int slot, int rh, int& a_src, u32& a_base, u32& a_sh) {
    const int g = rank >> 2, a = rank & 3;
    if (slot < 24) { const int i = slot / 6, a_rb = slot - i * 6; a_src = i == a ? -1 : g * 4 + i;
        a_base = a_rb == 0 ? 0u : a_rb < 3 ? 16u : a_rb < 5 ? 48u : 80u + (u32)rh * 8u; a_sh = a_rb == 0 ? 1u : 0u; }
    else { a_src = slot < 32 ? (1 - g) * 4 + ((slot - 24) >> 1) : -1; a_base = (slot & 1) ? 48u : 0u; a_sh = 1u; }
}
__device__ __forceinline__ u32 start_off(int rank, const Tile& t, bool big, u32 par, int lane, int wave) {
    int a_src; u32 a_base, a_sh;
    a_stream(rank, t.slot, t.rh, a_src, a_base, a_sh);
    const int b_src = t.ct < 8 ? -1 : (1 - (rank >> 2)) * 4 + ((t.ct - 8) >> 1);
    u32 o = OOB, m = 63u;
    if (wave == 0) { if (a_src >= 0) { o = OFF_LF + ((u32)a_src * NUNITS + (a_base ? a_base : (u32)HU)) * 128u; m = big ? 7u : 3u; } }
    else if (wave == 1) { if (a_src < 0) { o = OFF_SA + (par * 64u + (u32)t.slot) * 512u; m = 31u; } }
    else if (wave == 2) { if (b_src < 0) { o = OFF_WR + (u32)t.ct * 128u; m = 7u; } }
    else if (wave == 3) { if (a_src >= 0 && a_base == 0u) { o = OFF_RD + (par * 8u + (u32)a_src) * 512u; m = 31u; } }
    else if (b_src >= 0 && (!big || wave == 4)) { o = OFF_LF + ((u32)b_src * NUNITS + 16u) * 128u + (u32)(wave - 4) * 1024u; m = big ? 7u : 63u; }
    return o + ((u32)lane & m) * 16u;
}

template <int MODE>
__device__ __forceinline__ void gemm_tile(const Params& p, LAS char* lds, const Tile& tl, bool first, const Tile& nx, bool nbig, u32 par, u32 call, int tid, int lane, int wave, int stamp0, char*& pend) {
    constexpr bool BIG = MODE == 0;
    constexpr int NB = BIG ? 4 : 2, NC = BIG ? 32 : 16, BT = BIG ? 128 : 64, NR = MODE == 2 ? NF : MODE == 1 ? NFL : 0;
    const int rank = p.rank, g = rank >> 2, a = rank & 3;
    const int slot = tl.slot, ct = tl.ct, rh = tl.rh, ch = tl.ch;
    asm volatile("" : "+v"(lane), "+v"(tid));
    char* ws = p.me;
    const rsrc_t rws = mk_rsrc(ws, WS_NEED);
    int a_src; u32 a_base, a_sh;
    a_stream(rank, slot, rh, a_src, a_base, a_sh);
    const int b_src = ct < 8 ? -1 : (1 - g) * 4 + ((ct - 8) >> 1);
    const int src = lane < 32 ? a_src : (BIG ? b_src : -1);
    const u32 gsh = src < 0 ? 31u : lane < 32 && BIG ? a_sh : 0u;
    const char* gptr = ws + (src >= 0 ? OFF_LF + ((u32)src * NUNITS + (lane < 32 ? a_base : 16u)) * 128u + (u32)(lane & (BIG ? 31 : 15)) * 4u : OFF_CTR + blockIdx.x * 64u + 4u);
    const u32 gmin = src >= 0 && lane < 32 && a_base == 0u ? (u32)HU : 0u;
    const bool headt = BIG && a_src >= 0 && a_base == 0u;
#define GPTR(c) (gptr + (umax((u32)(c) >> gsh, gmin) << (BIG ? 7 : 6)))
#define PEND() do { if (pend && wave == 1 && lane < 2) st_sys(pend + (u32)lane * 4u, call); pend = nullptr; } while (0)
    LAS u32* okw = (LAS u32*)(lds + L_RED + 1024u);
    const u32 pb = CTRL_BYTES + par * PAR_BYTES;
    const int ks = BIG ? wave >> 2 : wave >> 1, wm = BIG ? (wave >> 1) & 1 : 0, wn = wave & 1;
    const u32 swz = (u32)((((ks & 1) * 4 + (lane >> 4)) ^ ((lane & 15) >> 1)) << 4);
    f16x8 bt[4 * NR + 1][2];
    WAIT_LGKM0(); BAR();
    u32 po = OOB;
    {
        const u32x4 k0 = *(const LAS u32x4*)okw, k1 = *(const LAS u32x4*)(okw + 4);
        if (first || __builtin_amdgcn_readfirstlane((int)(k0[0] | k0[1] | k0[2] | k0[3] | k1[0] | k1[1] | k1[2] | k1[3]))) {
            if (pend) { WAIT_VM(0); BAR(); PEND(); }
            po = start_off(rank, tl, BIG, par, lane, wave);
            if ((!NR || wave != 0) && __builtin_amdgcn_readfirstlane((int)po) >= 0) poll4_ge(rws, po, call, ws, call);
            WAIT_VM_TRACKED(0); BAR();
        }
    }
    if (!NR && wave == 0) STAMP(ws, stamp0);
    constexpr u32 RB = BT * 4u, PPR = BT / 4u, AREA = BT * RB;
    const u32 pc = (u32)lane & (PPR - 1u);
    const u32 col0 = BIG ? 0u : (u32)ch * 64u, row0 = BIG ? 0u : (u32)rh * 64u;
    const bool remote = BIG ? ct >= 8 : MODE == 2;
    const rsrc_t ro = mk_rsrc(remote ? (const void*)ws : (const void*)p.out, remote ? WS_NEED : IN_BYTES);
    const u32 ostr = remote ? 1024u : NLOC * 4u;
    const u32 obase = (remote ? OFF_YST + (u32)(((ct - 8) >> 1) * 32 + slot) * UNIT + (u32)(ct & 1) * 512u : ((u32)slot_row(rank, slot) * NLOC + (u32)ct * 128u) * 4u) + col0 * 4u + row0 * ostr + pc * 16u;
#define YFLAG() do { if (tid == 0) st_sys(p.ws[b_src] + OFF_YR + (u32)(a * 64 + rh * 32 + slot) * 16u + (u32)(ct & 1) * 8u + (u32)ch * 4u, call); } while (0)
    constexpr bool hint = MODE == 2;
    if (hint) {
        const u32x4 zero = {0u, 0u, 0u, 0u};
#pragma unroll
        for (u32 it = 0; it < 2u; ++it) __builtin_amdgcn_raw_buffer_store_b128(zero, ro, obase + ((u32)wave * (BT / 8u) + (it * 64u + (u32)lane) / PPR) * ostr, 0, AUX_SC1);
    }
    const u32 gA = pb + (u32)slot * PANEL + (BIG ? (u32)wave * 2048u : (u32)rh * 8192u + (u32)wave * 1024u);
    const u32 gB = pb + (u32)(ASLOTS + ct) * PANEL + (BIG ? (u32)wave * 2048u : (u32)ch * 8192u + (u32)wave * 1024u);
    const u32 l16 = (u32)lane * 16u;
    const u32 ldsw = (u32)wave * (BIG ? 2048u : 1024u);
    constexpr u32 GSTEP = BIG ? IMG : 2u * IMG, G2 = BIG ? 1024u : IMG, L2O = BIG ? 1024u : 8192u;
#define SCHED() __builtin_amdgcn_sched_barrier(0)
#define GLDS16(g, l) __builtin_amdgcn_raw_ptr_buffer_load_lds(rws, (LAS void*)(lds + (l)), 16, l16, (g), 0, GL_AUX)
#define LOADT(t, st) do { const u32 _o = (u32)(t) * GSTEP; \
        GLDS16(gA + _o, (st) * STAGE + ldsw); GLDS16(gA + _o + G2, (st) * STAGE + ldsw + L2O); \
        GLDS16(gB + _o, (st) * STAGE + IMG + ldsw); GLDS16(gB + _o + G2, (st) * STAGE + IMG + ldsw + L2O); } while (0)
#define SNAP(c) __builtin_amdgcn_global_load_lds((const u32*)((c) < NC ? GPTR(c) : ws + OFF_TMO), (LAS u32*)(lds + L_SNAP + ((u32)(c) & 1u) * 256u), 4, 0, AUX_SYS)
    const u32 swl = swz + (BIG ? 0u : (u32)(ks >> 1) * 8192u);
    const u32 ard = (u32)(wm * 64 + (lane & 15)) * 128u + swl, brd = IMG + (u32)(wn * (BT / 2) + (lane & 15)) * 128u + swl;
#define READF(fa, fb, st) READX(fa, fb, st, true)
#define READX(fa, fb, st, rb) do { _Pragma("unroll") for (int _i = 0; _i < 4; ++_i) { \
        if (_i < NB && (rb)) fb[_i] = *(const LAS f16x8*)(lds + (st) * STAGE + brd + _i * 2048u); \
        fa[_i] = *(const LAS f16x8*)(lds + (st) * STAGE + ard + _i * 2048u); } } while (0)
#define MMA(fa, fb) do { SCHED(); _Pragma("unroll") for (int _m = 0; _m < 4; ++_m) _Pragma("unroll") for (int _n = 0; _n < NB; ++_n) \
        acc[_m][_n] = __builtin_amdgcn_mfma_f32_16x16x32_f16(fb[_n], fa[_m], acc[_m][_n], 0, 0, 0); SCHED(); } while (0)
#define SB() __builtin_amdgcn_sched_barrier(0)
#define STEP(t, sl, na, nb, sr, fa, fb) STEPX(t, sl, na, nb, sr, fa, fb, true, true)
#define STEPX(t, sl, na, nb, sr, fa, fb, lb, rb) do { const u32 _o = (u32)(t) * GSTEP; \
        _Pragma("unroll") for (int _m = 0; _m < 4; ++_m) { \
            _Pragma("unroll") for (int _n = 0; _n < NB / 2; ++_n) acc[_m][_n] = __builtin_amdgcn_mfma_f32_16x16x32_f16(fb[_n], fa[_m], acc[_m][_n], 0, 0, 0); \
            SB(); \
            if (_m < 2) { _Pragma("unroll") for (int _i = 0; _i < 2; ++_i) { \
                if (_i < NB / 2 && (rb)) nb[_m * (NB / 2) + _i] = *(const LAS f16x8*)(lds + (sr) * STAGE + brd + (_m * (NB / 2) + _i) * 2048u); \
                na[_m * 2 + _i] = *(const LAS f16x8*)(lds + (sr) * STAGE + ard + (_m * 2 + _i) * 2048u); } } \
            if (_m < 2 || (lb)) GLDS16((_m < 2 ? gA : gB) + _o + (_m & 1) * G2, (sl) * STAGE + (_m >> 1) * IMG + ldsw + (_m & 1) * L2O); \
            SB(); \
            _Pragma("unroll") for (int _n = NB / 2; _n < NB; ++_n) acc[_m][_n] = __builtin_amdgcn_mfma_f32_16x16x32_f16(fb[_n], fa[_m], acc[_m][_n], 0, 0, 0); \
        } SB(); } while (0)
    f32x4 acc[4][NB];
#pragma unroll
    for (int m = 0; m < 4; ++m)
#pragma unroll
        for (int n = 0; n < NB; ++n) acc[m][n] = (f32x4){0.f, 0.f, 0.f, 0.f};
    f16x8 a0[4], b0[4], a1[4], b1[4];
#define VLOAD(v) do { const u32 _o = (u32)(4 * (NC - NR) + 2 * (v)) * GSTEP, _l = (u32)(((v) + 2 * NR) & 3) * STAGE + ldsw; \
        GLDS16(gB + _o, _l); GLDS16(gB + _o + G2, _l + L2O); GLDS16(gB + _o + GSTEP, _l + IMG); GLDS16(gB + _o + GSTEP + G2, _l + IMG + L2O); } while (0)
#define VREAD(v) do { _Pragma("unroll") for (int _i = 0; _i < 4; ++_i) \
        bt[NR ? 2 * (v) + (_i >> 1) : 0][_i & 1] = *(const LAS f16x8*)(lds + (u32)(((v) + 2 * NR) & 3) * STAGE + brd - (_i < 2 ? IMG : 0u) + (_i & 1) * 2048u); } while (0)
#define VSTEP(v) do { if (NR) { WAIT_VM(8); WAIT_LGKM0(); BAR(); VREAD(v); } } while (0)
    if (NR) {
        VLOAD(0); VLOAD(1); VLOAD(2); VLOAD(3);
        WAIT_VM(12); BAR();
        PEND();
        VREAD(0);
#pragma unroll
        for (int v = 4; v < 2 * NR; ++v) { VSTEP(v - 3); VLOAD(v); }
        if (wave == 0 && __builtin_amdgcn_readfirstlane((int)po) >= 0) { poll4_ge(rws, po, call, ws, call); WAIT_VM_TRACKED(0); }
        VSTEP(2 * NR - 3);
        if (wave == 0) STAMP(ws, stamp0);
    }
    if (wave >= 4) {
        const u32 so = (wave < 6 ? OFF_SA + (par * 64u + (u32)slot) * 512u : OFF_SB + (par * BSLOTS + (u32)ct) * 512u) + (u32)((wave & 1) * 64 + lane) * 4u;
        __builtin_amdgcn_global_load_lds((const u32*)(ws + so), (LAS u32*)(lds + L_SC + (u32)(wave - 4) * 256u), 4, 0, AUX_SYS);
    }
    else if (wave >= 2 && headt)
        __builtin_amdgcn_global_load_lds((const u32*)(ws + OFF_RD + (par * 8u + (u32)a_src) * 512u + (u32)((wave & 1) * 64 + lane) * 4u), (LAS u32*)(lds + L_RED + (u32)(wave - 2) * 256u), 4, 0, AUX_SYS);
    LOADT(0, 0u);
    if (wave == 0) SNAP(1);
    VSTEP(2 * NR - 2);
    LOADT(1, 1u);
    VSTEP(2 * NR - 1);
    LOADT(2, 2u);
    if (NR) { WAIT_VM(8); WAIT_LGKM0(); BAR(); LOADT(3, 3u); }
    else {
        LOADT(3, 3u);
        if (wave == 0) WAIT_VM(13); else WAIT_VM(12);
        BAR();
        PEND();
    }
    READF(a0, b0, 0u);
#define GATE(c) do { const u32 _v = *(const LAS u32*)(lds + L_SNAP + ((u32)(c) & 1u) * 256u + (u32)lane * 4u); \
        if (!__all((int)(_v - call) >= 0)) { if (wave == 0) { poll_ge(GPTR(c), call, ws, call); WAIT_VM(0); } BAR(); } } while (0)
    bool seen = false;
#define FP(i) do { __builtin_amdgcn_global_load_lds((const u32*)_gp, (LAS u32*)(lds + L_RED + (i) * 256u), 4, 0, AUX_SYS); __builtin_amdgcn_s_sleep(FSLEEP); } while (0)
#define FS(i) { FP(((i) + 3) & 3); WAIT_VM(3); u32 _w; \
            asm volatile("ds_read_b32 %0, %1 offset:%2\n\ts_waitcnt lgkmcnt(0)" : "=v"(_w) : "v"(_pa), "n"((i) * 256) : "memory"); \
            if (__all((int)(_w - call) >= 0)) break; }
#define FGATE(c) do { const u32 _v = *(const LAS u32*)(lds + L_SNAP + ((u32)(c) & 1u) * 256u + (u32)lane * 4u); \
        seen = false; \
        if (!__all((int)(_v - call) >= 0)) { if (wave == 0) { seen = true; \
            const char* _gp = GPTR(c); const u32 _pa = (u32)(size_t)(lds + L_RED) + (u32)lane * 4u; const u64 _t0 = __builtin_amdgcn_s_memrealtime(); \
            FP(0); FP(1); FP(2); \
            for (u32 _n = 1;; ++_n) { FS(0) FS(1) FS(2) FS(3) if ((_n & 15u) == 0u && give_up(ws, call, _t0)) break; } \
        } BAR(); } } while (0)
    for (int c = 0; c < NC - 1 - NR; ++c) {
        const int t = 4 * c;
        WAIT_VM(8); WAIT_LGKM0(); BAR();
        if (BIG || !FGALL) GATE(c + 1); else FGATE(c + 1);
        STEP(t + 4, 0u, a1, b1, 1u, a0, b0);
        WAIT_VM(8); WAIT_LGKM0(); BAR();
        if (wave == 0) SNAP(c + 2);
        STEP(t + 5, 1u, a0, b0, 2u, a1, b1);
        if (wave == 0) WAIT_VM(9); else WAIT_VM(8);
        WAIT_LGKM0(); BAR();
        STEP(t + 6, 2u, a1, b1, 3u, a0, b0);
        if (wave == 0) WAIT_VM(9); else WAIT_VM(8);
        WAIT_LGKM0(); BAR();
        STEP(t + 7, 3u, a0, b0, 0u, a1, b1);
        if (BIG && c == 2 * HU - 1 && headt) {
#pragma unroll
            for (int m = 0; m < 4; ++m) {
                const u32 r = (u32)(wm * 64 + m * 16 + (lane & 15)) * 4u;
                const int d = (int)*(const LAS signed char*)(lds + L_RED + r) - (int)*(const LAS signed char*)(lds + L_SC + r);
#pragma unroll
                for (int n = 0; n < NB; ++n)
#pragma unroll
                    for (int j = 0; j < 4; ++j) acc[m][n][j] = __builtin_amdgcn_ldexpf(acc[m][n][j], d);
            }
        }
    }
#define TS(i) do { } while (0)
#define FB(j, k, b) ((j) ? bt[(j) ? 4 * (j) - 4 + (k) : 0] : (b))
#pragma unroll
    for (int j = 0; j < NR; ++j) {
        const int c = NC - 1 - NR + j, t = 4 * c;
        if (j) WAIT_VM(4); else WAIT_VM(8);
        WAIT_LGKM0(); BAR();
        TS(2 * j);
        FGATE(c + 1);
        TS(2 * j + 1);
        STEPX(t + 4, 0u, a1, b1, 1u, a0, FB(j, 0, b0), false, !j);
        if (seen) { } else if (j) WAIT_VM(4); else WAIT_VM(6);
        WAIT_LGKM0(); BAR();
        if (wave == 0) SNAP(c + 2);
        STEPX(t + 5, 1u, a0, b0, 2u, a1, FB(j, 1, b1), false, !j);
        if (seen) { } else if (wave == 0) WAIT_VM(5); else WAIT_VM(4);
        WAIT_LGKM0(); BAR();
        STEPX(t + 6, 2u, a1, b1, 3u, a0, FB(j, 2, b0), false, !j);
        if (wave == 0) WAIT_VM(5); else WAIT_VM(4);
        WAIT_LGKM0(); BAR();
        STEPX(t + 7, 3u, a0, b0, 0u, a1, FB(j, 3, b1), false, false);
    }
#define DSTEP(k, n) do { if ((k) > HINT && hint && wave == 0) WAIT_VM((n) + 1); else WAIT_VM(n); WAIT_LGKM0(); BAR(); if ((k) == HINT && hint) YFLAG(); } while (0)
    TS(8);
    DSTEP(0, NR ? 4 : 8);
    TS(9);
    READX(a1, b1, 1u, !NR); MMA(a0, FB(NR, 0, b0));
    DSTEP(1, NR ? 2 : 4);
    READX(a0, b0, 2u, !NR); MMA(a1, FB(NR, 1, b1));
    DSTEP(2, 0);
    TS(10);
    READX(a1, b1, 3u, !NR); MMA(a0, FB(NR, 2, b0));
    WAIT_LGKM0(); BAR();
    if (HINT == 3 && hint) YFLAG();
    u32 npo = OOB; u32x4 nv = {0u, 0u, 0u, 0u};
    if (BIG && nx.slot >= 0) { npo = start_off(rank, nx, nbig, par, lane, wave); nv = __builtin_amdgcn_raw_buffer_load_b128(rws, npo, 0, AUX_SYS); }
    MMA(a1, FB(NR, 3, b1));
#undef DSTEP
#undef FB
#undef FGATE
#undef FS
#undef FP
    if (wave == 0) STAMP(ws, stamp0 + 1);
#pragma unroll
    for (int m = 0; m < 4; ++m)
#pragma unroll
        for (int n = 0; n < NB; ++n) {
            const u32 r = (u32)(wm * 64 + m * 16 + (lane & 15)), pc = (u32)(wn * (BT / 8) + n * 4 + (lane >> 4));
            *(LAS f32x4*)(lds + (u32)ks * AREA + r * RB + ((pc ^ (r & 7u)) << 4)) = acc[m][n];
        }
    WAIT_LGKM0(); BAR();
    const LAS int* sc = (const LAS int*)(lds + L_SC);
    const i32x4 eb = *(const LAS i32x4*)(sc + 128 + col0 + pc * 4u);
    const bool bad = *(const LAS u32*)(lds + L_SNAP) == call;
    if (BIG) { WAIT_VM_TRACKED(0); const bool ok = (int)npo < 0 || all4_ge(nv, call); if (lane == 0) okw[wave] = ok ? 0u : 1u; }
#pragma unroll
    for (u32 it = 0; it < (BIG ? 8u : 2u); ++it) {
        const u32 r = (u32)wave * (BT / 8u) + (it * 64u + (u32)lane) / PPR;
        const u32 lo = r * RB + ((pc ^ (r & 7u)) << 4);
        f32x4 s = *(const LAS f32x4*)(lds + lo) + *(const LAS f32x4*)(lds + AREA + lo);
        if (!BIG) s += *(const LAS f32x4*)(lds + 2u * AREA + lo) + *(const LAS f32x4*)(lds + 3u * AREA + lo);
        const int ea = (int)(signed char)sc[row0 + r];
#pragma unroll
        for (int j = 0; j < 4; ++j) s[j] = bad ? __builtin_nanf("") : __builtin_amdgcn_ldexpf(s[j], ea + eb[j]);
        u32x4 o = __builtin_bit_cast(u32x4, s);
        if (hint) {
#pragma unroll
            for (int j = 0; j < 4; ++j) o[j] = o[j] ? o[j] : 0x80000000u;
        }
        __builtin_amdgcn_raw_buffer_store_b128(o, ro, obase + r * ostr, 0, AUX_SC1);
    }
    if (remote && BIG) pend = p.ws[b_src] + OFF_YR + (u32)(a * 64 + slot) * 16u + (u32)(ct & 1) * 8u;
    if (wave == 0) STAMP(ws, stamp0 + 2);
#undef TS
#undef GLDS16
#undef SCHED
#undef SB
#undef STEP
#undef STEPX
#undef READX
#undef VLOAD
#undef VREAD
#undef VSTEP
#undef LOADT
#undef SNAP
#undef READF
#undef MMA
#undef GATE
#undef YFLAG
#undef PEND
#undef GPTR
}

constexpr u32 NSK = DELTA - (DCUT ? 1u : 0u);
constexpr u32 NPOS_INTRA = 96u - HU - NSK, NPOS_CROSS = 97u - HU + DELTA;
constexpr u32 FAN = 0x20000000u;
struct Desc { u32 src, dst, gate, flag; };
struct Stream {
    bool cross; int rank, g, a, j; u32 pb, sa;
    __device__ __forceinline__ int yslot(u32 k) const {
        if (k < 6u) return j * 6 + (int)k;
        if (k < 9u) return ((j + (int)k - 5) & 3) * 6;
        if (k < 12u) return 24 + 2 * ((a + (int)k - 8) & 3);
        if (k < 18u) return ((j + 1 + (((int)k - 12) >> 1)) & 3) * 6 + 1 + ((int)k & 1);
        if (k < 21u) return 25 + 2 * ((a + (int)k - 17) & 3);
        if (k < 27u) return ((j + 1 + (((int)k - 21) >> 1)) & 3) * 6 + 3 + (((int)k - 21) & 1);
        return ((j + 1 + (int)(k % 3u)) & 3) * 6 + 5;
    }
    __device__ __forceinline__ u32 single(u32 slot, u32 k) const { return pb + slot * PANEL + k * UNIT; }
    __device__ __forceinline__ u32 pair(u32 slot, u32 k) const { return pb + slot * PANEL + k * 65536u; }
    __device__ __forceinline__ u32 half(u32 slot, u32 h, u32 k) const { return pb + slot * PANEL + k * 2u * UNIT + h * 8192u; }
    __device__ __forceinline__ Desc at(u32 pos) const {
        Desc d;
        const u32 s = (u32)j * 6u, M8 = 7u << 20, PAIR = 1u << 28, HALF = 4u << 28;
        u32 u, fsrc = (u32)((cross ? 1 - g : g) * 4 + j);
        if (!cross) {
            if (pos >= NPOS_INTRA) pos = NPOS_INTRA - 1u;
            u = pos + HU;
            if (u >= RU0) u += NSK;
            if (u < 80u) {
                d.src = u < 16u ? single(s, u) : u < 48u ? pair(s + 1u, u - 16u) : pair(s + 3u, u - 48u);
                d.gate = sa + s * 512u + (u < 16u ? 0u : u < 48u ? 512u + PAIR + M8 : 1536u + PAIR + M8);
            } else {
                const u32 e = 95u - u, f = e - LEAD;
                const u32 h = e < LEAD || (e < 16u - LEAD && (f & 1u)) ? 1u : 0u;
                const u32 k = e < LEAD ? 7u - e : e >= 16u - LEAD ? 15u - e : h ? 7u - LEAD - (f >> 1) : 7u - (f >> 1);
                u = 80u + h * 8u + k;
                d.src = half(s + 5u, h, k);
                d.gate = sa + (s + 5u) * 512u + h * 256u + M8 + HALF;
            }
            d.dst = d.src;
            d.flag = OFF_LF + (fsrc * NUNITS + u) * 128u;
        } else {
            if (pos >= NPOS_CROSS) pos = NPOS_CROSS - 1u;
            pos += HU;
            const u32 hop = j == a ? HOP1 : HOP2;
            u = pos < hop ? pos : pos - DELTA;
            if (pos - hop < DELTA) {
                const u32 k = pos - hop;
                if (j == a) { d.src = pair(s + 3u, RU0 - 48u + k); d.dst = pair(RELAY, k); d.gate = sa + 34u * 512u + M8 + PAIR;
                              d.flag = FAN + OFF_RF + ((u32)a * 4u + k) * 128u; }
                else { d.src = pair(RELAY, k); d.dst = pair(s + 3u, RU0 - 48u + k); d.gate = OFF_RF + ((u32)j * 4u + k) * 128u + (5u << 28);
                       d.flag = OFF_LF + ((fsrc ^ 4u) * NUNITS + RU0 + k) * 128u; }
            } else if (u < 64u) {
                d.src = u < 16u ? single(32u, u) : u < 48u ? pair((u32)(ASLOTS + 2 * a), u - 16u) : single(33u, u - 48u);
                d.dst = u < 16u ? single((u32)(24 + 2 * j), u) : u < 48u ? pair((u32)(ASLOTS + 8 + 2 * j), u - 16u) : single((u32)(25 + 2 * j), u - 48u);
                d.gate = u < 16u ? sa + (u32)(24 + 2 * j) * 512u : u < 48u ? OFF_WS + (u32)j * 256u + (1u << 20) + (3u << 28) : sa + (u32)(25 + 2 * j) * 512u;
                d.flag = OFF_LF + (fsrc * NUNITS + u) * 128u;
            } else {
                const u32 k = u - 64u;
                const int ys = yslot(k);
                const u32 h = k >= 30u ? 1u : 0u;
                d.src = OFF_YST + (u32)(a * 32 + ys) * UNIT + h * 65536u;
                d.dst = ((u32)slot_row((int)fsrc, ys) + h * 64u) * (NLOC * 4u) + (u32)j * 1024u;
                d.gate = OFF_YR + ((u32)(j * 64 + ys) + h * 32u) * 16u + ((k >= 27u ? 10u : 2u) << 28);
                d.flag = NOFLAG;
            }
        }
        return d;
    }
};
struct Tables { Desc lo, hi; };
__device__ __forceinline__ u32 tab(u32 t0, u32 t1, u32 i) {
    const u32 x = (u32)__builtin_amdgcn_readlane((int)t0, (int)(i & 63u)), y = (u32)__builtin_amdgcn_readlane((int)t1, (int)(i & 63u));
    return i < 64u ? x : y;
}

#define PSLEEP0 10
#define PSLEEP 4
#define NACT 32
__device__ __forceinline__ void pull_stream(const Params& p, LAS char* lds, const Tables& T, bool cross, u32 cw, char* src, gchar* fan, u32 calln, u32 call, int lane, int wave) {
    const int hw = wave;
    char* ws = p.me;
    const rsrc_t rl = mk_rsrc(ws, WS_NEED), rs = mk_rsrc(src, WS_NEED), rout = mk_rsrc(p.out, IN_BYTES);
    LAS char* pl = lds + (u32)hw * 1024u;
    const u32 pla = (u32)(size_t)pl + (u32)lane * 4u;
    const u32 l16 = (u32)lane * 16u, l4r = (u32)(lane >> 5) * IMG + (l16 & 511u);
    const u32 l0 = lane == 0 ? 0u : OOB;
    const u32 co = OFF_CL + (u32)wave * 128u + l0;
    const u32 NPIECES = (cross ? NPOS_CROSS : NPOS_INTRA) * 32u;
    const u32 skip = !DCUT ? 0u : cross ? (u32)DCUT : 32u - DCUT;
    const u32 qcut = cross ? ((wave == 3 ? HOP1 : HOP2) + DELTA - HU) * 32u - DCUT : (RU0 - HU) * 32u;
    const u32 cbase = 2u * NACT - (calln - 1u) * (NPIECES - skip);
    u32x4 v0 = {0u, 0u, 0u, 0u}, v1 = v0, v2 = v0, v3;
    u32 q = OOB, so = OOB, ss = 1024u;
    u32 c0 = cw, c1 = cw + NACT;
    u32 pf = NOFLAG;
    bool claimed = false, dead = false;
    int cl = 0;
#define KIND(q) (tab(T.lo.gate, T.hi.gate, (q) >> 5) >> 28)
#define POFF(q, kd) (((kd) & 1u) ? (((q) >> 4) & 1u) * PANEL + ((q) & 15u) * 4096u : ((kd) & 4u) ? (((q) >> 1) & 15u) * IMG + ((q) & 1u) * 4096u : ((q) & 31u) * (((kd) & 8u) ? 2048u : (kd) ? 4096u : 512u))
#define GBASE(q) ({ const u32 _g = tab(T.lo.gate, T.hi.gate, (q) >> 5), _k = _g >> 28; \
        const u32 _ix = (((_k & 1u) ? ((q) >> 2) & 4u : 0u) + ((q) & (_k == 4u ? 1u : 3u))) << (_k == 3u ? 5 : 7); \
        (_g & 0xfffffu) + ((_k & 7u) == 2u ? 0u : _k == 5u ? ((q) & 28u) * 4u : _k == 0u ? ((q) & 31u) * 16u : _ix); })
#define GOFF(q) (GBASE(q) + (l16 & (tab(T.lo.gate, T.hi.gate, (q) >> 5) >> 16)))
#define PFLAG() st_sys_g((pf & FAN) ? fan + (pf - FAN) : (gchar*)ws + (pf >= NOFLAG ? OFF_CTR + cw * 64u + 8u + (u32)hw * 4u : pf), call)
#define PLD(u, o) v##u = __builtin_amdgcn_raw_buffer_load_b128(rs, (o) + (u) * ss, 0, AUX_SYS)
#define PSTA(u, o) do { asm volatile("" : "+v"(v##u)); \
        __builtin_amdgcn_raw_buffer_store_b128(v##u | (dead ? 0x7fc00000u : 0u), rd, o, 0, AUX_SC1); } while (0)
#define PST(u) PSTA(u, d + (u) * ds)
#define PSTAMP(i) do { } while (0)
#define GPOLL(i, sl) do { __builtin_amdgcn_global_load_lds((const u32*)gp, (LAS u32*)(pl + (i) * 256u), 4, 0, AUX_SYS); __builtin_amdgcn_s_sleep(sl); \
        if (DRY && c0 < 32u) __builtin_amdgcn_s_sleep(PSLEEP0 - PSLEEP); } while (0)
#define GSTEP(i, sl) { GPOLL(((i) + 3) & 3, sl); WAIT_VM_TRACKED(3); u32 _w; \
            asm volatile("ds_read_b32 %0, %1 offset:%2\n\ts_waitcnt lgkmcnt(0)" : "=v"(_w) : "v"(pla), "n"((i) * 256) : "memory"); \
            if (__all((int)(_w - call) >= 0)) break; }
#define WATCH(sl) do { \
        const char* gp = ws + GBASE(c0) + ((u32)lane * 4u & ((tab(T.lo.gate, T.hi.gate, c0 >> 5) >> 16 & 0x3f0u) | 12u)); \
        const u64 t0 = __builtin_amdgcn_s_memrealtime(); \
        GPOLL(0, sl); GPOLL(1, sl); GPOLL(2, sl); \
        for (u32 n = 1;; ++n) { \
            GSTEP(0, sl) GSTEP(1, sl) GSTEP(2, sl) GSTEP(3, sl) \
            if ((n & 15u) == 0u && give_up(ws, call, t0)) { dead = true; break; } \
        } } while (0)
    if (!DRY) WATCH(PSLEEP0);
    const u32 g0 = DRY ? call - 1u : call;
    u32x4 gv = {g0, g0, g0, g0};
    bool dry = DRY;
    u32 d = OOB, ds = 1024u, so2 = OOB, d2 = OOB; bool y = false, val = false;
#define VALID(u) __all(v##u[0] != 0u && v##u[1] != 0u && v##u[2] != 0u && v##u[3] != 0u)
#define RETRY(u) do { if (HINT >= 0 && val && !dead && !VALID(u)) { \
        const u64 t0 = __builtin_amdgcn_s_memrealtime(); u32 n = 0; \
        do { PLD(u, so); NRETRY; \
            if ((++n & 15u) == 0u && give_up(ws, call, t0)) dead = true; \
            WAIT_VM_TRACKED(0); \
        } while (!dead && !VALID(u)); } } while (0)
#define NRETRY do { } while (0)
    do {
        const rsrc_t rd = y ? rout : rl;
        WAIT_VM_TRACKED(5); PLD(2, so2); RETRY(0); PST(0);
        WAIT_VM_TRACKED(4); PLD(3, so2); PFLAG(); RETRY(1); PST(1);
        WAIT_VM_TRACKED(5);
        bool open = dead || all4_ge(gv, call);
        if (q == OOB && !open && !dry) { WATCH(PSLEEP); open = true; }
        dry = false;
        pf = q != OOB ? tab(T.lo.flag, T.hi.flag, q >> 5) + (q & 31u) * 4u : NOFLAG;
        if (claimed) { c1 = (u32)__builtin_amdgcn_readfirstlane(cl) + cbase; if (c1 >= qcut) c1 += skip; if (c1 > NPIECES + 127u) c1 = NPIECES + 127u; }
        claimed = open && c0 < NPIECES;
        q = claimed ? c0 : OOB; if (claimed) c0 = c1;
        const u32 qq = claimed ? q : 0u;
        const u32 kn = claimed ? KIND(qq) : 1u, po = POFF(qq, kn), lo = kn ? l16 : l4r;
        so = claimed ? tab(T.lo.src, T.hi.src, qq >> 5) + po + lo : OOB;
        ss = kn ? 1024u : 2u * IMG;
        y = (kn & 7u) == 2u;
        u32 dn = claimed ? tab(T.lo.dst, T.hi.dst, qq >> 5) + (y ? po * 4u : po) + lo : OOB;
        u32 go = GOFF(c0), ao = claimed ? co : OOB, so1 = so + ss, dx2 = d2 + 2u * ds, dx3 = d2 + 3u * ds;
        if (claimed) PSTAMP(q >> 5);
        asm volatile("" : "+v"(so), "+v"(dn), "+v"(go), "+v"(ao), "+v"(so1), "+v"(dx2), "+v"(dx3));
        WAIT_VM_TRACKED(4);
        v0 = __builtin_amdgcn_raw_buffer_load_b128(rs, so, 0, AUX_SYS);
        PSTA(2, dx2);
        WAIT_VM_TRACKED(4);
        PSTA(3, dx3);
        v1 = __builtin_amdgcn_raw_buffer_load_b128(rs, so1, 0, AUX_SYS);
        cl = __builtin_amdgcn_raw_ptr_buffer_atomic_add_i32(1, rl, ao, 0, AUX_SC1);
        gv = __builtin_amdgcn_raw_buffer_load_b128(rl, go, 0, AUX_SYS);
        d = dn; ds = y ? NLOC * 4u : ss;
        val = (kn & 8u) != 0u;
        so2 = val ? OOB : so; d2 = val ? OOB : dn;
    } while (c0 < NPIECES || (!LASTP && q != OOB));
    if (LASTP) {
        const rsrc_t rd = y ? rout : rl;
        PLD(2, so2); PLD(3, so2);
        WAIT_VM_TRACKED(7); RETRY(0); PST(0);
        WAIT_VM_TRACKED(5); PFLAG(); RETRY(1); PST(1);
        pf = q != OOB ? tab(T.lo.flag, T.hi.flag, q >> 5) + (q & 31u) * 4u : NOFLAG;
        WAIT_VM_TRACKED(4); PSTA(2, d2 + 2u * ds);
        WAIT_VM_TRACKED(4); PSTA(3, d2 + 3u * ds);
    }
    if (!ENDQ || pf < NOFLAG) { WAIT_VM_TRACKED(0); PFLAG(); }
    asm volatile("" :: "v"(gv), "v"(cl));
    asm volatile(".Lagk_end:");
    if (hw < 7) STAMP(ws, 2 + wave * 8 + 7);
#undef PSTAMP
#undef GOFF
#undef GBASE
#undef WATCH
#undef GSTEP
#undef GPOLL
#undef POFF
#undef KIND
#undef PST
#undef PSTA
#undef RETRY
#undef VALID
#undef NRETRY
#undef PLD
#undef PFLAG
}

__global__ void __launch_bounds__(512, 2) ag_gemm_kernel(Params p) {
    extern __shared__ __attribute__((aligned(16))) char smem[];
    LAS char* lds = (LAS char*)smem;
    const int tid = threadIdx.x, lane = tid & 63, wave = __builtin_amdgcn_readfirstlane(tid >> 6);
    const int b = blockIdx.x, rank = p.rank, g = rank >> 2, a = rank & 3;
    u64 pc; u32 len;
    asm volatile("s_getpc_b64 %0\n.Lagk_pc:\n\ts_mov_b32 %1, .Lagk_end-.Lagk_pc" : "=s"(pc), "=s"(len));
    const u32 code = __builtin_amdgcn_raw_buffer_load_b32(mk_rsrc((const void*)pc, (int)len > 0 ? len : 0u), (u32)(b >> 3) * 2048u + (u32)tid * 4u, 0, 0);
    char* ws = p.me;
    if (wave == 0) STAMP(ws, 0);
    const u32 prev = ld_agent(ws + OFF_CTR + (u32)b * 64u);
    u32 calln;
    const int lw = wave;
    const rsrc_t rws = mk_rsrc(ws, WS_NEED);

    Tables T = {};
    char* src = ws; gchar* fan = (gchar*)ws;
    u32 hword = 0;
    gchar* rdflag = x_head(p, lds, b < 128 ? b : 640 + b, prev, calln, hword, tid, wave, [&]() {
        if (b < NCOMM) {
            const u32 par = calln & 1u;
            Stream S;
            S.cross = lw >= 3; S.rank = rank; S.g = g; S.a = a; S.j = (a + lw + 1) & 3; S.pb = CTRL_BYTES + par * PAR_BYTES; S.sa = OFF_SA + par * 32768u;
            T.lo = S.at((u32)lane); T.hi = S.at((u32)lane + 64u);
            src = p.peer[lw == 3 ? 7 : lw];
            char *f0 = p.peer[4], *f1 = p.peer[5], *f2 = p.peer[6];
            asm volatile("" : "+s"(src), "+s"(f0), "+s"(f1), "+s"(f2));
            fan = (gchar*)(lane == 0 ? f0 : lane == 1 ? f1 : f2);
        }
    });
    asm volatile("" :: "v"(code));
    const u32 call = calln << 8, par = calln & 1u;
    if (wave == (CTR7 ? 7 : 0) && lane < 2) st_agent(ws + OFF_CTR + (u32)b * 64u + (u32)lane * 4u, lane ? call : calln);
    if (wave == 0 && b >= NCOMM) STAMP(ws, 1);
    if (b < NCOMM) {
        if (wave == 7) {
            WAIT_VM(0);
            if (lane < 4) st_sys_g(rdflag, hword);
        }
        else if (b < NACT) pull_stream(p, lds, T, wave >= 3, (u32)b, src, fan, calln, call, lane, wave);
        __builtin_amdgcn_endpgm();
    }

    {
        const int c = b - NCOMM, strip = c / 7, rg = c - strip * 7;
        const int kt0 = (rg * 128) / 7, kt1 = ((rg + 1) * 128) / 7;
        const rsrc_t rw = mk_rsrc(p.w, IN_BYTES);
        const int q = lane & 7, n4 = lane >> 3;
        if (wave == 7) { WAIT_VM(0); if (lane < 4) st_sys_g(rdflag, hword); }
        f32x4 v[3][8];
#pragma unroll
        for (int i = 0; i < 3; ++i) {
            const int kt = kt0 + wave + 8 * i;
            const u32 base = kt < kt1 ? ((u32)(kt * 64 + q * 8) * NLOC + (u32)(strip * 32 + n4 * 4)) * 4u : OOB;
#pragma unroll
            for (int jj = 0; jj < 8; ++jj) {
                v[i][jj] = __builtin_bit_cast(f32x4, __builtin_amdgcn_raw_buffer_load_b128(rw, base + (u32)jj * (NLOC * 4u), 0, 0));
                WAIT_VM(W_LD - 1);
            }
        }
        LAS u32* cm = (LAS u32*)(lds + L_RED + 256u);
        if (tid < 64) cm[tid] = 0u;
        WAIT_LGKM0(); BAR();
        u32 m[4] = {0u, 0u, 0u, 0u};
#pragma unroll
        for (int i = 0; i < 3; ++i)
#pragma unroll
            for (int jj = 0; jj < 8; ++jj)
#pragma unroll
                for (int k = 0; k < 4; ++k) m[k] = umax(m[k], absbits(v[i][jj][k]));
#pragma unroll
        for (int k = 0; k < 4; ++k) {
            m[k] = umax(m[k], (u32)__shfl_xor((int)m[k], 1)); m[k] = umax(m[k], (u32)__shfl_xor((int)m[k], 2)); m[k] = umax(m[k], (u32)__shfl_xor((int)m[k], 4));
        }
        if (q < 4) __hip_atomic_fetch_max(cm + n4 * 4 + q, q == 0 ? m[0] : q == 1 ? m[1] : q == 2 ? m[2] : m[3], __ATOMIC_RELAXED, __HIP_MEMORY_SCOPE_WORKGROUP);
        WAIT_LGKM0(); BAR();
        u64* wx = (u64*)(ws + OFF_WX) + (u32)strip * 256u;
        if (tid < 32) __hip_atomic_store(wx + rg * 32 + tid, ((u64)call << 32) | cm[tid], __ATOMIC_RELAXED, __HIP_MEMORY_SCOPE_AGENT);
        if (wave < 4) {
            const int gi = tid < 224 ? tid : 0;
            const u64 t0 = __builtin_amdgcn_s_memrealtime();
            u64 gv; u32 n = 0;
            for (;;) {
                gv = __hip_atomic_load(wx + gi, __ATOMIC_RELAXED, __HIP_MEMORY_SCOPE_AGENT);
                if (__all((u32)(gv >> 32) == call)) break;
                if ((++n & 15u) == 0u && give_up(ws, call, t0)) break;
                __builtin_amdgcn_s_sleep(1);
            }
            __hip_atomic_fetch_max(cm + 32 + (gi & 31), (u32)gv, __ATOMIC_RELAXED, __HIP_MEMORY_SCOPE_WORKGROUP);
        }
        WAIT_LGKM0(); BAR();
        const int ct = strip >> 2;
        if (rg == 0 && tid < 64) {
            const u32 nin = (u32)((strip & 3) * 32 + (tid & 31));
            const u32 ev = (u32)(exp_of(cm[32 + (tid & 31)]) - 141);
            char* t = tid < 32 ? ws + OFF_SB + (par * BSLOTS + (u32)ct) * 512u : p.ws[(1 - g) * 4 + (strip >> 3)] + OFF_SB + (par * BSLOTS + (u32)(8 + 2 * a + (ct & 1))) * 512u;
            st_sys(t + nin * 4u, ev);
        }
        const u32x4 mx = *(LAS u32x4*)(cm + 32 + n4 * 4);
        const u32 pbase = CTRL_BYTES + par * PAR_BYTES + (u32)(ASLOTS + ct) * PANEL;
#pragma unroll
        for (int k = 0; k < 4; ++k) {
            const float s = scale_of(exp_of(mx[k]));
            const u32 nin = (u32)((strip & 3) * 32 + n4 * 4 + k);
            const u32 ro = nin * 128u + (((u32)q ^ ((nin >> 1) & 7u)) << 4);
#pragma unroll
            for (int i = 0; i < 3; ++i) {
                const int kt = kt0 + wave + 8 * i;
                const u32 off = kt < kt1 ? pbase + (u32)kt * IMG + ro : OOB;
                const f32x4 lo = {v[i][0][k], v[i][1][k], v[i][2][k], v[i][3][k]}, hi = {v[i][4][k], v[i][5][k], v[i][6][k], v[i][7][k]};
                __builtin_amdgcn_raw_buffer_store_b128(cvt8(lo, hi, s), rws, off, 0, AUX_SC1);
                WAIT_VM(W_ST - 1);
            }
        }
        WAIT_VM(0); BAR();
        if (tid < 4 && (!(tid & 1) || rg == 6))
            st_sys((tid < 2 ? ws + OFF_WR + (u32)strip * 32u : p.ws[(1 - g) * 4 + (strip >> 3)] + OFF_WS + (u32)(a * 8 + (strip & 7)) * 32u) + (u32)(rg + (tid & 1)) * 4u, call);
        if (wave == 0) STAMP(ws, 2);
    }
    const int xcd = b & 7, q = (b >> 3) - 4;
    if (xcd >= 4 && q < 24) {
        const int r0 = 128 + (q * 4 + xcd - 4) * 4;
        x_rows(p, lds, r0, par, call, tid, wave);
        x_rows(p, lds, r0 + (r0 < 384 ? 384 : 512), par, call, tid, wave);
        if (wave == 0) STAMP(ws, 3);
    }
    const int qr = q >> 2, qc = q & 3, x4 = xcd & 3;
    const int early = (x4 >> 1) == (xcd < 4 ? ESWAP : 0) ? (qr < 6 ? a * 6 + qr : -1) : (qr < 3 ? ((a + 1 + qr) & 3) * 6 : 24 + 2 * (qr - 3));
    const int peer = ((a + 1 + (qr >> 1)) & 3) * 6;
    auto tile_of = [&](int it) {
        int slot = -1, ct = 4 * x4 + qc, rh = 0, ch = 0;
        if (it == 0) { if (xcd < 4) { slot = early; ct = 4 * (x4 & 1) + qc; } }
        else if (it == 1) {
            if (RMAP && qr == 6 && xcd >= 2 && xcd < 6) { slot = ((a + 2) & 3) * 6 + (xcd >> 1); ct = 4 * (xcd & 1) + qc; }
            else if (xcd < 4) { if (qr < 6 && !(RMAP && xcd < 2 && qr >> 1 == 1)) slot = peer + 1 + (qr & 1); }
            else { slot = early; ct = 8 + 4 * (x4 & 1) + qc; }
        }
        else if (it == 2) {
            if (qr == 6) { slot = 25 + (xcd & 6); ct = 8 + 4 * (xcd & 1) + qc; }
            else if (xcd >= 4) slot = peer + 3 + (qr & 1);
            else if (xcd < 2 && qr < 4) slot = 25 + 2 * qr;
        } else if (it == 3) {
            int pi = -1; ch = qc & 1;
            const int qs = !RMAP && QSWAP && xcd >= 4 && (qr & 5) == 4 ? qr ^ 2 : qr;
            if (RMAP && xcd < 2 && qr >> 1 == 1) { pi = qr - 1; rh = 1; ct = 12 + 2 * xcd + (qc >> 1); }
            else if (xcd < 2) { pi = qr - 4; rh = 1; ct = 8 + 2 * xcd + (qc >> 1); }
            else if (qs == 6) { pi = RMAP && xcd >= 4 ? -1 : (xcd - 2) >> 1; rh = 1; ct = 12 + 2 * (xcd & 1) + (qc >> 1); }
            else if (xcd < 4) { pi = qr >> 1; ct = 4 * xcd + 2 * (qr & 1) + (qc >> 1); }
            else { pi = qs >> 1; rh = qs & 1; ct = 2 * x4 + (qc >> 1); }
            if (pi >= 0) slot = ((a + 1 + pi) & 3) * 6 + 5;
        }
        if (slot >= 24 && ct >= 8 && (slot - 24) >> 1 == (ct - 8) >> 1) { slot = 32 + (slot & 1); ct -= 8; }
        return Tile{slot, ct, rh, ch};
    };
    char* pend = nullptr;
    int it = 0;
    Tile tl = tile_of(0);
    while (tl.slot < 0) tl = tile_of(++it);
    for (bool first = true; it < 3; first = false) {
        int nt = it + 1;
        Tile nx = tile_of(nt);
        while (nx.slot < 0 && nt < 3) nx = tile_of(++nt);
        gemm_tile<0>(p, lds, tl, first, nx, nt < 3, par, call, tid, lane, wave, 4 + it * 3, pend);
        it = nt; tl = nx;
    }
    if (tl.slot < 0) {
        if (pend) { WAIT_VM(0); BAR(); if (tid < 2) st_sys(pend + (u32)tid * 4u, call); }
        __builtin_amdgcn_endpgm();
    }
    if (tl.ct < 8) {
        gemm_tile<1>(p, lds, tl, false, tl, false, par, call, tid, lane, wave, 13, pend);
        __builtin_amdgcn_endpgm();
    }
    gemm_tile<2>(p, lds, tl, false, tl, false, par, call, tid, lane, wave, 13, pend);
}
}

extern "C" void kernel_launch(int rank, int nranks, ncclComm_t comm,
                              hipStream_t stream,
                              void* const* d_in, const int* in_sizes, int n_in,
                              void* d_out, int out_size,
                              const int* in_dtypes, int out_dtype,
                              void* const* d_out_all,
                              void* const* d_ws_all, size_t ws_bytes) {
    (void)comm; (void)in_dtypes; (void)out_dtype; (void)d_out_all;
    if (nranks != 8 || (unsigned)rank >= 8u || n_in < 2 || ws_bytes < (size_t)agk::WS_NEED || in_sizes[0] != 1024 * 8192 ||
        in_sizes[1] != 8192 * 1024 || out_size != 8192 * 1024) {
        fprintf(stderr, "kernel_launch: unexpected problem shape\n");
        return;
    }
    static bool ready[8] = {false, false, false, false, false, false, false, false};
    if (!ready[rank]) {
        HIP_CHECK(hipFuncSetAttribute((const void*)agk::ag_gemm_kernel, hipFuncAttributeMaxDynamicSharedMemorySize, (int)agk::LDS_BYTES));
        ready[rank] = true;
    }
    agk::Params p;
    p.x = (const float*)d_in[0]; p.w = (const float*)d_in[1]; p.rank = rank; p.pad = 0;
    p.me = (char*)d_ws_all[rank]; p.out = (float*)d_out;
    for (int i = 0; i < 8; ++i) { p.ws[i] = (char*)d_ws_all[i]; p.peer[i] = (char*)d_ws_all[((rank ^ i) & 4) + ((rank + 1 + i) & 3)]; }
    hipLaunchKernelGGL(agk::ag_gemm_kernel, dim3(agk::NWG), dim3(512), agk::LDS_BYTES, stream, p);
    HIP_CHECK(hipGetLastError());
}
```

```cpp
#include <hip/hip_runtime.h>
#include <rccl/rccl.h>
#include <cstdio>

#define HIP_CHECK(expr) do { hipError_t _e = (expr); if (_e != hipSuccess) { \
    fprintf(stderr, "HIP error %s at %s:%d\n", hipGetErrorString(_e), __FILE__, __LINE__); \
    return; } } while (0)

namespace agk {
typedef unsigned u32;
typedef unsigned long long u64;
typedef u32 u32x4 __attribute__((ext_vector_type(4)));
typedef int i32x4 __attribute__((ext_vector_type(4)));
typedef float f32x4 __attribute__((ext_vector_type(4)));
typedef float f32x2 __attribute__((ext_vector_type(2)));
typedef _Float16 f16x2 __attribute__((ext_vector_type(2)));
typedef _Float16 f16x8 __attribute__((ext_vector_type(8)));
#define LAS __attribute__((address_space(3)))
typedef __amdgpu_buffer_rsrc_t rsrc_t;
typedef __attribute__((address_space(1))) char gchar;

constexpr int KDIM = 8192, NLOC = 1024, NKT = 128;
constexpr u32 IMG = 16384u;
constexpr u32 PANEL = NKT * IMG;
constexpr u32 UNIT = 131072u;
constexpr int ASLOTS = 34, BSLOTS = 16;
#define HU 3
#define HWIN 8
#define SCH 2
#define DRY 1
#define NF 4
#define NFL 2
#define RU0 68
#define HOP1 56
#define HOP2 64
#define LEAD 2
#define QSWAP 1
#define RMAP 0
#define FGALL 0
#define LASTP 1
#define ENDQ 1
#define ROWU 6
#define R7C 1
#define EPW 1
#define EPS 0
#define EROT 1
#define EALL 0
#define LSL 0
#define LL 0
#define FSLEEP 3
#define CTR7 1
#define ESWAP 1
#define HSTEP 1
#define HK 8
#define XDLY 30
#define HINT 1
#define DCUT 0
#define DELTA 3
constexpr u32 CTRL_BYTES = 1u << 20;
constexpr u32 RELAY = ASLOTS + BSLOTS;
constexpr u32 PAR_BYTES = (RELAY + 2u) * PANEL;
constexpr u32 OFF_YST = CTRL_BYTES + 2u * PAR_BYTES;
constexpr u32 WS_NEED = OFF_YST + 4u * 32u * UNIT;
constexpr u32 IN_BYTES = 1024u * 8192u * 4u;
constexpr u32 OFF_CTR = 0;
constexpr u32 OFF_TMO = 16384;
constexpr u32 OFF_CL = 20480;
constexpr u32 OFF_LF = 524288;
constexpr u32 OFF_RD = 655360;
constexpr u32 OFF_YR = 24576;
constexpr u32 OFF_RF = 28672;
constexpr u32 OFF_WR = 32768;
constexpr u32 OFF_WS = 33792;
constexpr u32 OFF_WX = 36864;
constexpr u32 OFF_SA = 131072;
constexpr u32 OFF_SB = 196608;
constexpr u32 OFF_DBG = 262144;
constexpr int NWG = 256, NCOMM = 32;
constexpr u32 NUNITS = 96u;
constexpr u32 OOB = 0x80000000u, NOFLAG = 0x40000000u;
constexpr u64 DEADLINE = 30000000ull;
constexpr u32 STAGE = 32768u;
constexpr u32 L_SNAP = 4u * STAGE;
constexpr u32 L_SC = L_SNAP + 512u;
constexpr u32 L_RED = L_SC + 1024u;
constexpr u32 LDS_BYTES = L_RED + 2048u;

struct Params {
    const float* x; char* me; int rank, pad;
    char* peer[8];
    const float* w; float* out;
    char* ws[8];
};

#define AUX_SC1 16
#define AUX_SYS 17
#define GL_AUX 16
#define W_LD 2
#define W_LD0 1
#define W_N0 0
#define W_ST 2
#define X_LD 4
#define X_ST 2
#define WAIT_VM(n) asm volatile("s_waitcnt vmcnt(%0)" :: "n"(n) : "memory")
#define WAIT_LGKM0() do { asm volatile("" ::: "memory"); __builtin_amdgcn_s_waitcnt(15 | (7 << 4) | (0 << 8) | (3 << 14)); asm volatile("" ::: "memory"); } while (0)
#define BAR() do { asm volatile("" ::: "memory"); __builtin_amdgcn_s_barrier(); asm volatile("" ::: "memory"); } while (0)
#define WAIT_VM_TRACKED(n) do { asm volatile("" ::: "memory"); __builtin_amdgcn_s_waitcnt(((n) & 15) | (7 << 4) | (15 << 8) | (((n) >> 4) << 14)); asm volatile("" ::: "memory"); } while (0)

__device__ __forceinline__ rsrc_t mk_rsrc(const void* p, u32 bytes) { return __builtin_amdgcn_make_buffer_rsrc((void*)p, 0, bytes, 0x00020000); }
__device__ __forceinline__ u32 ld_agent(const void* p) { return __hip_atomic_load((const u32*)p, __ATOMIC_RELAXED, __HIP_MEMORY_SCOPE_AGENT); }
__device__ __forceinline__ u32 ld_sys(const void* p) { return __hip_atomic_load((const u32*)p, __ATOMIC_RELAXED, __HIP_MEMORY_SCOPE_SYSTEM); }
__device__ __forceinline__ void st_agent(void* p, u32 v) { __hip_atomic_store((u32*)p, v, __ATOMIC_RELAXED, __HIP_MEMORY_SCOPE_AGENT); }
__device__ __forceinline__ void st_sys_g(gchar* p, u32 v) { __hip_atomic_store((__attribute__((address_space(1))) u32*)p, v, __ATOMIC_RELAXED, __HIP_MEMORY_SCOPE_SYSTEM); }
__device__ __forceinline__ void st_sys(void* p, u32 v) { __hip_atomic_store((u32*)p, v, __ATOMIC_RELAXED, __HIP_MEMORY_SCOPE_SYSTEM); }

#define STAMP(ws, idx) do { } while (0)

__device__ __forceinline__ bool give_up(char* ws, u32 call, u64 t0) {
    const u32 t = __builtin_amdgcn_readfirstlane(ld_agent(ws + OFF_TMO));
    if (t == call || __builtin_amdgcn_s_memrealtime() - t0 > DEADLINE) { st_agent(ws + OFF_TMO, call); return true; }
    return false;
}
__device__ __forceinline__ void poll_ge(const char* p, u32 need, char* ws, u32 call) {
    const u64 t0 = __builtin_amdgcn_s_memrealtime();
    u32 n = 0;
    for (;;) {
        const u32 v = ld_sys(p);
        if (__all((int)(v - need) >= 0)) return;
        if ((++n & 15u) == 0u && give_up(ws, call, t0)) return;
        __builtin_amdgcn_s_sleep(2);
    }
}
__device__ __forceinline__ bool all4_ge(u32x4 v, u32 need) {
    return __all((int)(v[0] - need) >= 0 && (int)(v[1] - need) >= 0 && (int)(v[2] - need) >= 0 && (int)(v[3] - need) >= 0);
}
__device__ __forceinline__ void poll4_ge(rsrc_t rs, u32 off, u32 need, char* ws, u32 call) {
    const u64 t0 = __builtin_amdgcn_s_memrealtime();
    u32 n = 0;
    for (;;) {
        asm volatile("" ::: "memory");
        const u32x4 v = __builtin_amdgcn_raw_buffer_load_b128(rs, off, 0, AUX_SYS);
        if (all4_ge(v, need)) return;
        if ((++n & 15u) == 0u && give_up(ws, call, t0)) return;
        __builtin_amdgcn_s_sleep(1);
    }
}

__device__ __forceinline__ u32 umax(u32 a, u32 b) { return a > b ? a : b; }
__device__ __forceinline__ u32 absbits(float f) { return __float_as_uint(f) & 0x7fffffffu; }
__device__ __forceinline__ int exp_of(u32 amaxbits) { int e = (int)(amaxbits >> 23); return e < 15 ? 15 : e; }
__device__ __forceinline__ float scale_of(int e) { return __uint_as_float((u32)(268 - e) << 23); }
__device__ __forceinline__ u32 cvt2(float a, float b, float s) {
    const f32x2 t = (f32x2){a, b} * s;
    return __builtin_bit_cast(u32, __builtin_convertvector(t, f16x2));
}
__device__ __forceinline__ u32x4 cvt8(f32x4 a, f32x4 b, float s) {
    return (u32x4){cvt2(a[0], a[1], s), cvt2(a[2], a[3], s), cvt2(b[0], b[1], s), cvt2(b[2], b[3], s)};
}
__device__ __forceinline__ u32 amax4(u32 m, f32x4 v) {
    asm("v_max3_f32 %0, |%1|, |%2|, %0\n\tv_max3_f32 %0, |%3|, |%4|, %0" : "+v"(m) : "v"(v[0]), "v"(v[1]), "v"(v[2]), "v"(v[3]));
    return m;
}

__device__ __forceinline__ u32 wave_max(u32 m) {
    m = umax(m, (u32)__builtin_amdgcn_update_dpp(0, (int)m, 0xb1, 15, 15, true));
    m = umax(m, (u32)__builtin_amdgcn_update_dpp(0, (int)m, 0x4e, 15, 15, true));
    m = umax(m, (u32)__builtin_amdgcn_update_dpp(0, (int)m, 0x141, 15, 15, true));
    m = umax(m, (u32)__builtin_amdgcn_update_dpp(0, (int)m, 0x140, 15, 15, true));
    return umax(umax((u32)__builtin_amdgcn_readlane((int)m, 0), (u32)__builtin_amdgcn_readlane((int)m, 16)),
                umax((u32)__builtin_amdgcn_readlane((int)m, 32), (u32)__builtin_amdgcn_readlane((int)m, 48)));
}
__device__ __forceinline__ u32 row_word(u32 call, int e) { return call | ((u32)(e - 141) & 255u); }

__device__ __forceinline__ void x_rows(const Params& p, LAS char* lds, int row0, u32 par, u32 call, int tid, int wave) {
    constexpr int NR = 4;
    const int a = p.rank & 3;
    char* ws = p.me;
    const rsrc_t rx = mk_rsrc(p.x, IN_BYTES), rw = mk_rsrc(ws, WS_NEED);
    const u32 pi0 = (u32)(wave * 128 + (tid & 63));
    f32x4 v[NR][4];
#pragma unroll
    for (int r = 0; r < NR; ++r)
#pragma unroll
        for (int u = 0; u < 2; ++u) {
            const u32 off = (u32)(row0 + r) * (KDIM * 4u) + (pi0 + (u32)u * 64u) * 32u;
            v[r][2 * u] = __builtin_bit_cast(f32x4, __builtin_amdgcn_raw_buffer_load_b128(rx, off, 0, 0));
            v[r][2 * u + 1] = __builtin_bit_cast(f32x4, __builtin_amdgcn_raw_buffer_load_b128(rx, off + 16u, 0, 0));
            WAIT_VM(X_LD - 2);
        }
    const int rb = row0 >> 7;
    const u32 slot = rb < 6 ? (u32)(a * 6 + rb) : (u32)(32 + rb - 6);
    const u32 rslot = rb < 6 ? slot : (u32)(24 + 2 * a + rb - 6);
    char* hp[4];
#pragma unroll
    for (int i = 0; i < 4; ++i) hp[i] = rb < 6 ? p.peer[i] : p.peer[4 + i];
    char* fifth = rb < 6 ? p.peer[7] : ws;
    asm volatile("" : "+s"(hp[0]), "+s"(hp[1]), "+s"(hp[2]), "+s"(hp[3]), "+s"(fifth));
    gchar* user = (gchar*)((tid & 4) ? fifth : (tid & 2) ? ((tid & 1) ? hp[3] : hp[2]) : ((tid & 1) ? hp[1] : hp[0]));
    const u32 wr = (u32)(tid & 63) >> 3;
    const bool wsend = wr < (u32)NR && ((tid & 7) < 4 || ((tid & 7) == 4 && rb >= 3 && rb != 5));
    const u32 woff = OFF_SA + ((par * 64u + ((tid & 4) ? (rb < 6 ? (u32)(31 + rb) : slot) : rslot)) * 128u + (((u32)row0 + wr) & 127u)) * 4u;
    u32 word = 0;
    LAS u32* red = (LAS u32*)(lds + L_RED);
#pragma unroll
    for (int r = 0; r < NR; ++r) {
        u32 m = 0;
#pragma unroll
        for (int i = 0; i < 4; ++i) m = amax4(m, v[r][i]);
        m = wave_max(m);
        if ((tid & 63) == 0) red[r * 8 + wave] = m;
    }
    WAIT_LGKM0(); BAR();
#pragma unroll
    for (int r = 0; r < NR; ++r) {
        const u32x4 m0 = *(LAS u32x4*)(red + r * 8), m1 = *(LAS u32x4*)(red + r * 8 + 4);
        const u32 m = umax(umax(umax(m0[0], m0[1]), umax(m0[2], m0[3])), umax(umax(m1[0], m1[1]), umax(m1[2], m1[3])));
        const int e = exp_of(m); const float s = scale_of(e);
        const u32 rin = (u32)((row0 + r) & 127);
        if (wr == (u32)r) word = row_word(call, e);
        const u32 off = CTRL_BYTES + par * PAR_BYTES + rin * 128u + (pi0 >> 3) * IMG + (((pi0 & 7u) ^ ((rin >> 1) & 7u)) << 4);
        __builtin_amdgcn_raw_buffer_store_b128(cvt8(v[r][0], v[r][1], s), rw, off + slot * PANEL, 0, AUX_SC1);
        if (X_ST == 1) WAIT_VM(0);
        __builtin_amdgcn_raw_buffer_store_b128(cvt8(v[r][2], v[r][3], s), rw, off + slot * PANEL + 8u * IMG, 0, AUX_SC1);
        WAIT_VM(X_ST < 2 ? 0 : X_ST - 2);
    }
    WAIT_VM(0); BAR();
    if (wave == 0 && wsend) st_sys_g(user + woff, word);
}

template <class F>
__device__ __forceinline__ gchar* x_head(const Params& p, LAS char* lds, int row, u32 prev, u32& calln, u32& hword, int tid, int wave, F mid) {
    constexpr int NS = HU > 3 ? HU : 3;
    const int lane = tid & 63;
    char* ws = p.me;
    const rsrc_t rx = mk_rsrc(p.x, IN_BYTES), rw = mk_rsrc(ws, WS_NEED);
    LAS u32* red = (LAS u32*)(lds + L_RED);
    if (wave != 7) {
        if (tid < 4) red[tid] = 0u;
        WAIT_LGKM0(); BAR();
        __builtin_amdgcn_s_sleep(XDLY);
    }
    const u32 s0 = wave == 7 ? 0u : (u32)wave, sd = wave == 7 ? 1u : 7u, se = wave == 7 ? (u32)HU : 16u;
    f32x4 v[NS][2];
#pragma unroll
    for (int u = 0; u < NS; ++u) {
        const u32 set = s0 + (u32)u * sd;
        const u32 off = set < se ? (u32)row * (KDIM * 4u) + (set * 64u + (u32)lane) * 32u : OOB;
        v[u][0] = __builtin_bit_cast(f32x4, __builtin_amdgcn_raw_buffer_load_b128(rx, off, 0, 0));
        v[u][1] = __builtin_bit_cast(f32x4, __builtin_amdgcn_raw_buffer_load_b128(rx, off + 16u, 0, 0));
    }
    __builtin_amdgcn_sched_barrier(0);
    if (wave == 7) BAR();
    const bool major = row < 128;
    const int a = p.rank & 3;
    char* hp[4];
#pragma unroll
    for (int i = 0; i < 4; ++i) hp[i] = major ? p.peer[i] : p.peer[4 + i];
    const u32 slot = major ? (u32)(a * 6) : 32u, rslot = major ? slot : (u32)(24 + 2 * a), rin = (u32)row & 127u;
    u32 off = CTRL_BYTES + rin * 128u + ((u32)lane >> 3) * IMG + ((((u32)lane & 7u) ^ ((rin >> 1) & 7u)) << 4);
    u32 m = 0;
    gchar* user = (gchar*)((tid & 4) ? ws : (tid & 2) ? ((tid & 1) ? hp[3] : hp[2]) : ((tid & 1) ? hp[1] : hp[0]));
#define WORD() do { u32 t = 0; \
        if (lane == 0) t = __hip_atomic_fetch_add(red + 2, 1u, __ATOMIC_RELAXED, __HIP_MEMORY_SCOPE_WORKGROUP); \
        if (__builtin_amdgcn_readfirstlane((int)t) == 7 && lane < (major ? 4 : 5)) \
            st_sys_g(user + OFF_SA + (((calln & 1u) * 64u + (lane == 4 ? slot : rslot)) * 128u + rin) * 4u, \
                     row_word(calln << 8, exp_of(__hip_atomic_load(red, __ATOMIC_RELAXED, __HIP_MEMORY_SCOPE_WORKGROUP)))); } while (0)
    if (wave == 7) {
        rsrc_t hr[4];
#pragma unroll
        for (int i = 0; i < 4; ++i) hr[i] = mk_rsrc(hp[i], i == 3 && major ? 0u : WS_NEED);
        u32 o4[4];
#pragma unroll
        for (int i = 0; i < 4; ++i) o4[i] = (lane >> 4) == i ? off + rslot * PANEL : OOB;
        asm volatile("" : "+s"(hr[0]), "+s"(hr[1]), "+s"(hr[2]), "+s"(hr[3]), "+v"(o4[0]), "+v"(o4[1]), "+v"(o4[2]), "+v"(o4[3]), "+v"(m), "+v"(prev));
        calln = (u32)__builtin_amdgcn_readfirstlane(prev) + 1u;
        const u32 so = (calln & 1u) * PAR_BYTES;
#pragma unroll
        for (int u = 0; u < HU; ++u) m = amax4(amax4(m, v[u][0]), v[u][1]);
        m = wave_max(m);
        const int e = exp_of(m); const float s = scale_of(e);
        u32x4 h;
#pragma unroll
        for (int k = 0; k < 4 * HU; ++k) {
            if ((k & 3) == 0) h = cvt8(v[k >> 2][0], v[k >> 2][1], s);
            if (k == 2 && lane == 0) __hip_atomic_fetch_add(red + 1, 1u, __ATOMIC_RELAXED, __HIP_MEMORY_SCOPE_WORKGROUP);
#pragma unroll
            for (int i = 0; i < 4; ++i) {
                if (HSTEP ? i == 0 && 4 * k >= HWIN : 4 * k + i >= HWIN) WAIT_VM(HWIN - (HSTEP ? 4 : 1));
                __builtin_amdgcn_raw_buffer_store_b128(h, hr[i], o4[k & 3], so + (u32)(k >> 2) * 8u * IMG, AUX_SYS);
            }
            asm volatile("s_nop 1" : "+v"(h) :: "memory");
            if (k == HK) WORD();
        }
        hword = row_word(calln << 8, e);
        for (u32 n = 0; n < (1u << 24); ++n)
            if (__builtin_amdgcn_readfirstlane((int)__hip_atomic_load(red + 1, __ATOMIC_RELAXED, __HIP_MEMORY_SCOPE_WORKGROUP)) == 8) break;
    }
    else calln = (u32)__builtin_amdgcn_readfirstlane(prev) + 1u;
    const u32 par = calln & 1u;
    if (wave != 7) {
        mid();
        off += par * PAR_BYTES + slot * PANEL;
#pragma unroll
        for (int u = 0; u < 3; ++u) m = amax4(amax4(m, v[u][0]), v[u][1]);
        m = wave_max(m);
        if (lane == 0) {
            __hip_atomic_fetch_max(red, m, __ATOMIC_RELAXED, __HIP_MEMORY_SCOPE_WORKGROUP);
            __hip_atomic_fetch_add(red + 1, 1u, __ATOMIC_RELAXED, __HIP_MEMORY_SCOPE_WORKGROUP);
        }
        u64 mc;
        for (u32 n = 0; n < (1u << 24); ++n) {
            mc = __hip_atomic_load((LAS u64*)red, __ATOMIC_RELAXED, __HIP_MEMORY_SCOPE_WORKGROUP);
            if ((u32)__builtin_amdgcn_readfirstlane((int)(mc >> 32)) == 8u) break;
        }
        const int e = exp_of((u32)mc); const float s = scale_of(e);
#pragma unroll
        for (int u = 0; u < 3; ++u) {
            const u32 set = s0 + (u32)u * 7u;
            __builtin_amdgcn_raw_buffer_store_b128(cvt8(v[u][0], v[u][1], s), rw, set < 16u ? off + set * 8u * IMG : OOB, 0, AUX_SC1);
        }
        WAIT_VM(0);
        WORD();
    }
#undef WORD
    return user + OFF_RD + ((par * 8u + (u32)p.rank) * 128u + rin) * 4u;
}

__device__ __forceinline__ int slot_row(int rank, int slot) {
    const int g = rank >> 2;
    if (slot < 24) { const int i = slot / 6; return (g * 4 + i) * 1024 + (slot - i * 6) * 128; }
    return (slot < 32 ? (1 - g) * 4 + ((slot - 24) >> 1) : rank) * 1024 + 768 + (slot & 1) * 128;
}

struct Tile { int slot, ct, rh, ch; };
__device__ __forceinline__ void a_stream(int rank, int slot, int rh, int& a_src, u32& a_base, u32& a_sh) {
    const int g = rank >> 2, a = rank & 3;
    if (slot < 24) { const int i = slot / 6, a_rb = slot - i * 6; a_src = i == a ? -1 : g * 4 + i;
        a_base = a_rb == 0 ? 0u : a_rb < 3 ? 16u : a_rb < 5 ? 48u : 80u + (u32)rh * 8u; a_sh = a_rb == 0 ? 1u : 0u; }
    else { a_src = slot < 32 ? (1 - g) * 4 + ((slot - 24) >> 1) : -1; a_base = (slot & 1) ? 48u : 0u; a_sh = 1u; }
}
__device__ __forceinline__ u32 start_off(int rank, const Tile& t, bool big, u32 par, int lane, int wave) {
    int a_src; u32 a_base, a_sh;
    a_stream(rank, t.slot, t.rh, a_src, a_base, a_sh);
    const int b_src = t.ct < 8 ? -1 : (1 - (rank >> 2)) * 4 + ((t.ct - 8) >> 1);
    u32 o = OOB, m = 63u;
    if (wave == 0) { if (a_src >= 0) { o = OFF_LF + ((u32)a_src * NUNITS + (a_base ? a_base : (u32)HU)) * 128u; m = big ? 7u : 3u; } }
    else if (wave == 1) { if (a_src < 0) { o = OFF_SA + (par * 64u + (u32)t.slot) * 512u; m = 31u; } }
    else if (wave == 2) { if (b_src < 0) { o = OFF_WR + (u32)t.ct * 128u; m = 7u; } }
    else if (wave == 3) { if (a_src >= 0 && a_base == 0u) { o = OFF_RD + (par * 8u + (u32)a_src) * 512u; m = 31u; } }
    else if (b_src >= 0 && (!big || wave == 4)) { o = OFF_LF + ((u32)b_src * NUNITS + 16u) * 128u + (u32)(wave - 4) * 1024u; m = big ? 7u : 63u; }
    return o + ((u32)lane & m) * 16u;
}

template <int MODE>
__device__ __forceinline__ void gemm_tile(const Params& p, LAS char* lds, const Tile& tl, bool first, const Tile& nx, bool nbig, u32 par, u32 call, int tid, int lane, int wave, int stamp0, char*& pend) {
    constexpr bool BIG = MODE == 0;
    constexpr int NB = BIG ? 4 : 2, NC = BIG ? 32 : 16, BT = BIG ? 128 : 64, NR = MODE == 2 ? NF : MODE == 1 ? NFL : 0;
    const int rank = p.rank, g = rank >> 2, a = rank & 3;
    const int slot = tl.slot, ct = tl.ct, rh = tl.rh, ch = tl.ch;
    asm volatile("" : "+v"(lane), "+v"(tid));
    char* ws = p.me;
    const rsrc_t rws = mk_rsrc(ws, WS_NEED);
    int a_src; u32 a_base, a_sh;
    a_stream(rank, slot, rh, a_src, a_base, a_sh);
    const int b_src = ct < 8 ? -1 : (1 - g) * 4 + ((ct - 8) >> 1);
    const int src = lane < 32 ? a_src : (BIG ? b_src : -1);
    const u32 gsh = src < 0 ? 31u : lane < 32 && BIG ? a_sh : 0u;
    const char* gptr = ws + (src >= 0 ? OFF_LF + ((u32)src * NUNITS + (lane < 32 ? a_base : 16u)) * 128u + (u32)(lane & (BIG ? 31 : 15)) * 4u : OFF_CTR + blockIdx.x * 64u + 4u);
    const u32 gmin = src >= 0 && lane < 32 && a_base == 0u ? (u32)HU : 0u;
    const bool headt = BIG && a_src >= 0 && a_base == 0u;
#define GPTR(c) (gptr + (umax((u32)(c) >> gsh, gmin) << (BIG ? 7 : 6)))
#define PEND() do { if (pend && wave == 1 && lane < 2) st_sys(pend + (u32)lane * 4u, call); pend = nullptr; } while (0)
    LAS u32* okw = (LAS u32*)(lds + L_RED + 1024u);
    const u32 pb = CTRL_BYTES + par * PAR_BYTES;
    const int ks = BIG ? wave >> 2 : wave >> 1, wm = BIG ? (wave >> 1) & 1 : 0, wn = wave & 1;
    const u32 swz = (u32)((((ks & 1) * 4 + (lane >> 4)) ^ ((lane & 15) >> 1)) << 4);
    f16x8 bt[4 * NR + 1][2];
    WAIT_LGKM0(); BAR();
    u32 po = OOB;
    {
        const u32x4 k0 = *(const LAS u32x4*)okw, k1 = *(const LAS u32x4*)(okw + 4);
        if (first || __builtin_amdgcn_readfirstlane((int)(k0[0] | k0[1] | k0[2] | k0[3] | k1[0] | k1[1] | k1[2] | k1[3]))) {
            if (pend) { WAIT_VM(0); BAR(); PEND(); }
            po = start_off(rank, tl, BIG, par, lane, wave);
            if ((!NR || wave != 0) && __builtin_amdgcn_readfirstlane((int)po) >= 0) poll4_ge(rws, po, call, ws, call);
            WAIT_VM_TRACKED(0); BAR();
        }
    }
    if (!NR && wave == 0) STAMP(ws, stamp0);
    constexpr u32 RB = BT * 4u, PPR = BT / 4u, AREA = BT * RB;
    const u32 pc = (u32)lane & (PPR - 1u);
    const u32 col0 = BIG ? 0u : (u32)ch * 64u, row0 = BIG ? 0u : (u32)rh * 64u;
    const bool remote = BIG ? ct >= 8 : MODE == 2;
    const rsrc_t ro = mk_rsrc(remote ? (const void*)ws : (const void*)p.out, remote ? WS_NEED : IN_BYTES);
    const u32 ostr = remote ? 1024u : NLOC * 4u;
    const u32 obase = (remote ? OFF_YST + (u32)(((ct - 8) >> 1) * 32 + slot) * UNIT + (u32)(ct & 1) * 512u : ((u32)slot_row(rank, slot) * NLOC + (u32)ct * 128u) * 4u) + col0 * 4u + row0 * ostr + pc * 16u;
#define YFLAG() do { if (tid == 0) st_sys(p.ws[b_src] + OFF_YR + (u32)(a * 64 + rh * 32 + slot) * 16u + (u32)(ct & 1) * 8u + (u32)ch * 4u, call); } while (0)
    constexpr bool hint = MODE == 2;
    if (hint) {
        const u32x4 zero = {0u, 0u, 0u, 0u};
#pragma unroll
        for (u32 it = 0; it < 2u; ++it) __builtin_amdgcn_raw_buffer_store_b128(zero, ro, obase + ((u32)wave * (BT / 8u) + (it * 64u + (u32)lane) / PPR) * ostr, 0, AUX_SC1);
    }
    const u32 gA = pb + (u32)slot * PANEL + (BIG ? (u32)wave * 2048u : (u32)rh * 8192u + (u32)wave * 1024u);
    const u32 gB = pb + (u32)(ASLOTS + ct) * PANEL + (BIG ? (u32)wave * 2048u : (u32)ch * 8192u + (u32)wave * 1024u);
    const u32 l16 = (u32)lane * 16u;
    const u32 ldsw = (u32)wave * (BIG ? 2048u : 1024u);
    constexpr u32 GSTEP = BIG ? IMG : 2u * IMG, G2 = BIG ? 1024u : IMG, L2O = BIG ? 1024u : 8192u;
#define SCHED() __builtin_amdgcn_sched_barrier(0)
#define GLDS16(g, l) __builtin_amdgcn_raw_ptr_buffer_load_lds(rws, (LAS void*)(lds + (l)), 16, l16, (g), 0, GL_AUX)
#define LOADT(t, st) do { const u32 _o = (u32)(t) * GSTEP; \
        GLDS16(gA + _o, (st) * STAGE + ldsw); GLDS16(gA + _o + G2, (st) * STAGE + ldsw + L2O); \
        GLDS16(gB + _o, (st) * STAGE + IMG + ldsw); GLDS16(gB + _o + G2, (st) * STAGE + IMG + ldsw + L2O); } while (0)
#define SNAP(c) __builtin_amdgcn_global_load_lds((const u32*)((c) < NC ? GPTR(c) : ws + OFF_TMO), (LAS u32*)(lds + L_SNAP + ((u32)(c) & 1u) * 256u), 4, 0, AUX_SYS)
    const u32 swl = swz + (BIG ? 0u : (u32)(ks >> 1) * 8192u);
    const u32 ard = (u32)(wm * 64 + (lane & 15)) * 128u + swl, brd = IMG + (u32)(wn * (BT / 2) + (lane & 15)) * 128u + swl;
#define READF(fa, fb, st) READX(fa, fb, st, true)
#define READX(fa, fb, st, rb) do { _Pragma("unroll") for (int _i = 0; _i < 4; ++_i) { \
        if (_i < NB && (rb)) fb[_i] = *(const LAS f16x8*)(lds + (st) * STAGE + brd + _i * 2048u); \
        fa[_i] = *(const LAS f16x8*)(lds + (st) * STAGE + ard + _i * 2048u); } } while (0)
#define MMA(fa, fb) do { SCHED(); _Pragma("unroll") for (int _m = 0; _m < 4; ++_m) _Pragma("unroll") for (int _n = 0; _n < NB; ++_n) \
        acc[_m][_n] = __builtin_amdgcn_mfma_f32_16x16x32_f16(fb[_n], fa[_m], acc[_m][_n], 0, 0, 0); SCHED(); } while (0)
#define SB() __builtin_amdgcn_sched_barrier(0)
#define STEP(t, sl, na, nb, sr, fa, fb) STEPX(t, sl, na, nb, sr, fa, fb, true, true)
#define STEPX(t, sl, na, nb, sr, fa, fb, lb, rb) do { const u32 _o = (u32)(t) * GSTEP; \
        _Pragma("unroll") for (int _m = 0; _m < 4; ++_m) { \
            _Pragma("unroll") for (int _n = 0; _n < NB / 2; ++_n) acc[_m][_n] = __builtin_amdgcn_mfma_f32_16x16x32_f16(fb[_n], fa[_m], acc[_m][_n], 0, 0, 0); \
            SB(); \
            if (_m < 2) { _Pragma("unroll") for (int _i = 0; _i < 2; ++_i) { \
                if (_i < NB / 2 && (rb)) nb[_m * (NB / 2) + _i] = *(const LAS f16x8*)(lds + (sr) * STAGE + brd + (_m * (NB / 2) + _i) * 2048u); \
                na[_m * 2 + _i] = *(const LAS f16x8*)(lds + (sr) * STAGE + ard + (_m * 2 + _i) * 2048u); } } \
            if (_m < 2 || (lb)) GLDS16((_m < 2 ? gA : gB) + _o + (_m & 1) * G2, (sl) * STAGE + (_m >> 1) * IMG + ldsw + (_m & 1) * L2O); \
            SB(); \
            _Pragma("unroll") for (int _n = NB / 2; _n < NB; ++_n) acc[_m][_n] = __builtin_amdgcn_mfma_f32_16x16x32_f16(fb[_n], fa[_m], acc[_m][_n], 0, 0, 0); \
        } SB(); } while (0)
    f32x4 acc[4][NB];
#pragma unroll
    for (int m = 0; m < 4; ++m)
#pragma unroll
        for (int n = 0; n < NB; ++n) acc[m][n] = (f32x4){0.f, 0.f, 0.f, 0.f};
    f16x8 a0[4], b0[4], a1[4], b1[4];
#define VLOAD(v) do { const u32 _o = (u32)(4 * (NC - NR) + 2 * (v)) * GSTEP, _l = (u32)(((v) + 2 * NR) & 3) * STAGE + ldsw; \
        GLDS16(gB + _o, _l); GLDS16(gB + _o + G2, _l + L2O); GLDS16(gB + _o + GSTEP, _l + IMG); GLDS16(gB + _o + GSTEP + G2, _l + IMG + L2O); } while (0)
#define VREAD(v) do { _Pragma("unroll") for (int _i = 0; _i < 4; ++_i) \
        bt[NR ? 2 * (v) + (_i >> 1) : 0][_i & 1] = *(const LAS f16x8*)(lds + (u32)(((v) + 2 * NR) & 3) * STAGE + brd - (_i < 2 ? IMG : 0u) + (_i & 1) * 2048u); } while (0)
#define VSTEP(v) do { if (NR) { WAIT_VM(8); WAIT_LGKM0(); BAR(); VREAD(v); } } while (0)
    if (NR) {
        VLOAD(0); VLOAD(1); VLOAD(2); VLOAD(3);
        WAIT_VM(12); BAR();
        PEND();
        VREAD(0);
#pragma unroll
        for (int v = 4; v < 2 * NR; ++v) { VSTEP(v - 3); VLOAD(v); }
        if (wave == 0 && __builtin_amdgcn_readfirstlane((int)po) >= 0) { poll4_ge(rws, po, call, ws, call); WAIT_VM_TRACKED(0); }
        VSTEP(2 * NR - 3);
        if (wave == 0) STAMP(ws, stamp0);
    }
    if (wave >= 4) {
        const u32 so = (wave < 6 ? OFF_SA + (par * 64u + (u32)slot) * 512u : OFF_SB + (par * BSLOTS + (u32)ct) * 512u) + (u32)((wave & 1) * 64 + lane) * 4u;
        __builtin_amdgcn_global_load_lds((const u32*)(ws + so), (LAS u32*)(lds + L_SC + (u32)(wave - 4) * 256u), 4, 0, AUX_SYS);
    }
    else if (wave >= 2 && headt)
        __builtin_amdgcn_global_load_lds((const u32*)(ws + OFF_RD + (par * 8u + (u32)a_src) * 512u + (u32)((wave & 1) * 64 + lane) * 4u), (LAS u32*)(lds + L_RED + (u32)(wave - 2) * 256u), 4, 0, AUX_SYS);
    LOADT(0, 0u);
    if (wave == 0) SNAP(1);
    VSTEP(2 * NR - 2);
    LOADT(1, 1u);
    VSTEP(2 * NR - 1);
    LOADT(2, 2u);
    if (NR) { WAIT_VM(8); WAIT_LGKM0(); BAR(); LOADT(3, 3u); }
    else {
        LOADT(3, 3u);
        if (wave == 0) WAIT_VM(13); else WAIT_VM(12);
        BAR();
        PEND();
    }
    READF(a0, b0, 0u);
#define GATE(c) do { const u32 _v = *(const LAS u32*)(lds + L_SNAP + ((u32)(c) & 1u) * 256u + (u32)lane * 4u); \
        if (!__all((int)(_v - call) >= 0)) { if (wave == 0) { poll_ge(GPTR(c), call, ws, call); WAIT_VM(0); } BAR(); } } while (0)
    bool seen = false;
#define FP(i) do { __builtin_amdgcn_global_load_lds((const u32*)_gp, (LAS u32*)(lds + L_RED + (i) * 256u), 4, 0, AUX_SYS); __builtin_amdgcn_s_sleep(FSLEEP); } while (0)
#define FS(i) { FP(((i) + 3) & 3); WAIT_VM(3); u32 _w; \
            asm volatile("ds_read_b32 %0, %1 offset:%2\n\ts_waitcnt lgkmcnt(0)" : "=v"(_w) : "v"(_pa), "n"((i) * 256) : "memory"); \
            if (__all((int)(_w - call) >= 0)) break; }
#define FGATE(c) do { const u32 _v = *(const LAS u32*)(lds + L_SNAP + ((u32)(c) & 1u) * 256u + (u32)lane * 4u); \
        seen = false; \
        if (!__all((int)(_v - call) >= 0)) { if (wave == 0) { seen = true; \
            const char* _gp = GPTR(c); const u32 _pa = (u32)(size_t)(lds + L_RED) + (u32)lane * 4u; const u64 _t0 = __builtin_amdgcn_s_memrealtime(); \
            FP(0); FP(1); FP(2); \
            for (u32 _n = 1;; ++_n) { FS(0) FS(1) FS(2) FS(3) if ((_n & 15u) == 0u && give_up(ws, call, _t0)) break; } \
        } BAR(); } } while (0)
    for (int c = 0; c < NC - 1 - NR; ++c) {
        const int t = 4 * c;
        WAIT_VM(8); WAIT_LGKM0(); BAR();
        if (BIG || !FGALL) GATE(c + 1); else FGATE(c + 1);
        STEP(t + 4, 0u, a1, b1, 1u, a0, b0);
        WAIT_VM(8); WAIT_LGKM0(); BAR();
        if (wave == 0) SNAP(c + 2);
        STEP(t + 5, 1u, a0, b0, 2u, a1, b1);
        if (wave == 0) WAIT_VM(9); else WAIT_VM(8);
        WAIT_LGKM0(); BAR();
        STEP(t + 6, 2u, a1, b1, 3u, a0, b0);
        if (wave == 0) WAIT_VM(9); else WAIT_VM(8);
        WAIT_LGKM0(); BAR();
        STEP(t + 7, 3u, a0, b0, 0u, a1, b1);
        if (BIG && c == 2 * HU - 1 && headt) {
#pragma unroll
            for (int m = 0; m < 4; ++m) {
                const u32 r = (u32)(wm * 64 + m * 16 + (lane & 15)) * 4u;
                const int d = (int)*(const LAS signed char*)(lds + L_RED + r) - (int)*(const LAS signed char*)(lds + L_SC + r);
#pragma unroll
                for (int n = 0; n < NB; ++n)
#pragma unroll
                    for (int j = 0; j < 4; ++j) acc[m][n][j] = __builtin_amdgcn_ldexpf(acc[m][n][j], d);
            }
        }
    }
#define TS(i) do { } while (0)
#define FB(j, k, b) ((j) ? bt[(j) ? 4 * (j) - 4 + (k) : 0] : (b))
#pragma unroll
    for (int j = 0; j < NR; ++j) {
        const int c = NC - 1 - NR + j, t = 4 * c;
        if (j) WAIT_VM(4); else WAIT_VM(8);
        WAIT_LGKM0(); BAR();
        TS(2 * j);
        FGATE(c + 1);
        TS(2 * j + 1);
        STEPX(t + 4, 0u, a1, b1, 1u, a0, FB(j, 0, b0), false, !j);
        if (seen) { } else if (j) WAIT_VM(4); else WAIT_VM(6);
        WAIT_LGKM0(); BAR();
        if (wave == 0) SNAP(c + 2);
        STEPX(t + 5, 1u, a0, b0, 2u, a1, FB(j, 1, b1), false, !j);
        if (seen) { } else if (wave == 0) WAIT_VM(5); else WAIT_VM(4);
        WAIT_LGKM0(); BAR();
        STEPX(t + 6, 2u, a1, b1, 3u, a0, FB(j, 2, b0), false, !j);
        if (wave == 0) WAIT_VM(5); else WAIT_VM(4);
        WAIT_LGKM0(); BAR();
        STEPX(t + 7, 3u, a0, b0, 0u, a1, FB(j, 3, b1), false, false);
    }
#define DSTEP(k, n) do { if ((k) > HINT && hint && wave == 0) WAIT_VM((n) + 1); else WAIT_VM(n); WAIT_LGKM0(); BAR(); if ((k) == HINT && hint) YFLAG(); } while (0)
    TS(8);
    DSTEP(0, NR ? 4 : 8);
    TS(9);
    READX(a1, b1, 1u, !NR); MMA(a0, FB(NR, 0, b0));
    DSTEP(1, NR ? 2 : 4);
    READX(a0, b0, 2u, !NR); MMA(a1, FB(NR, 1, b1));
    DSTEP(2, 0);
    TS(10);
    READX(a1, b1, 3u, !NR); MMA(a0, FB(NR, 2, b0));
    WAIT_LGKM0(); BAR();
    if (HINT == 3 && hint) YFLAG();
    u32 npo = OOB; u32x4 nv = {0u, 0u, 0u, 0u};
    if (BIG && nx.slot >= 0) { npo = start_off(rank, nx, nbig, par, lane, wave); nv = __builtin_amdgcn_raw_buffer_load_b128(rws, npo, 0, AUX_SYS); }
    MMA(a1, FB(NR, 3, b1));
#undef DSTEP
#undef FB
#undef FGATE
#undef FS
#undef FP
    if (wave == 0) STAMP(ws, stamp0 + 1);
#pragma unroll
    for (int m = 0; m < 4; ++m)
#pragma unroll
        for (int n = 0; n < NB; ++n) {
            const u32 r = (u32)(wm * 64 + m * 16 + (lane & 15)), pc = (u32)(wn * (BT / 8) + n * 4 + (lane >> 4));
            *(LAS f32x4*)(lds + (u32)ks * AREA + r * RB + ((pc ^ (r & 7u)) << 4)) = acc[m][n];
        }
    WAIT_LGKM0(); BAR();
    if (MODE == 1 && LSL) __builtin_amdgcn_s_sleep(LSL);
    const LAS int* sc = (const LAS int*)(lds + L_SC);
    const i32x4 eb = *(const LAS i32x4*)(sc + 128 + col0 + pc * 4u);
    const bool bad = *(const LAS u32*)(lds + L_SNAP) == call;
    if (BIG) { WAIT_VM_TRACKED(0); const bool ok = (int)npo < 0 || all4_ge(nv, call); if (lane == 0) okw[wave] = ok ? 0u : 1u; }
#pragma unroll
    for (u32 it = 0; it < (BIG ? 8u : 2u); ++it) {
        const u32 r = (u32)wave * (BT / 8u) + ((BIG && EROT ? (it + (blockIdx.x >> 3)) & 7u : it) * 64u + (u32)lane) / PPR;
        const u32 lo = r * RB + ((pc ^ (r & 7u)) << 4);
        f32x4 s = *(const LAS f32x4*)(lds + lo) + *(const LAS f32x4*)(lds + AREA + lo);
        if (!BIG) s += *(const LAS f32x4*)(lds + 2u * AREA + lo) + *(const LAS f32x4*)(lds + 3u * AREA + lo);
        const int ea = (int)(signed char)sc[row0 + r];
#pragma unroll
        for (int j = 0; j < 4; ++j) s[j] = bad ? __builtin_nanf("") : __builtin_amdgcn_ldexpf(s[j], ea + eb[j]);
        u32x4 o = __builtin_bit_cast(u32x4, s);
        if (hint) {
#pragma unroll
            for (int j = 0; j < 4; ++j) o[j] = o[j] ? o[j] : 0x80000000u;
        }
        __builtin_amdgcn_raw_buffer_store_b128(o, ro, obase + r * ostr, 0, AUX_SC1);
        if (BIG && EPW && it < 7u && (first || EALL)) WAIT_VM(EPW ? EPW - 1 : 0);
        if (BIG && EPS && it < 7u && first) __builtin_amdgcn_s_sleep(EPS);
    }
    if (remote && BIG) pend = p.ws[b_src] + OFF_YR + (u32)(a * 64 + slot) * 16u + (u32)(ct & 1) * 8u;
    if (wave == 0) STAMP(ws, stamp0 + 2);
#undef TS
#undef GLDS16
#undef SCHED
#undef SB
#undef STEP
#undef STEPX
#undef READX
#undef VLOAD
#undef VREAD
#undef VSTEP
#undef LOADT
#undef SNAP
#undef READF
#undef MMA
#undef GATE
#undef YFLAG
#undef PEND
#undef GPTR
}

constexpr u32 NSK = DELTA - (DCUT ? 1u : 0u);
constexpr u32 NPOS_INTRA = 96u - HU - NSK, NPOS_CROSS = 97u - HU + DELTA;
constexpr u32 FAN = 0x20000000u;
struct Desc { u32 src, dst, gate, flag; };
struct Stream {
    bool cross; int rank, g, a, j; u32 pb, sa;
    __device__ __forceinline__ int yslot(u32 k) const {
        if (k < 6u) return j * 6 + (int)k;
        if (k < 9u) return ((j + (int)k - 5) & 3) * 6;
        if (k < 12u) return 24 + 2 * ((a + (int)k - 8) & 3);
        if (k < 18u) return ((j + 1 + (((int)k - 12) >> 1)) & 3) * 6 + 1 + ((int)k & 1);
        if (k < 21u) return 25 + 2 * ((a + (int)k - 17) & 3);
        if (k < 27u) return ((j + 1 + (((int)k - 21) >> 1)) & 3) * 6 + 3 + (((int)k - 21) & 1);
        return ((j + 1 + (int)(k % 3u)) & 3) * 6 + 5;
    }
    __device__ __forceinline__ u32 single(u32 slot, u32 k) const { return pb + slot * PANEL + k * UNIT; }
    __device__ __forceinline__ u32 pair(u32 slot, u32 k) const { return pb + slot * PANEL + k * 65536u; }
    __device__ __forceinline__ u32 half(u32 slot, u32 h, u32 k) const { return pb + slot * PANEL + k * 2u * UNIT + h * 8192u; }
    __device__ __forceinline__ Desc at(u32 pos) const {
        Desc d;
        const u32 s = (u32)j * 6u, M8 = 7u << 20, PAIR = 1u << 28, HALF = 4u << 28, ONE = M8 + (6u << 28);
        u32 u, fsrc = (u32)((cross ? 1 - g : g) * 4 + j);
        if (!cross) {
            if (pos >= NPOS_INTRA) pos = NPOS_INTRA - 1u;
            u = pos + HU;
            if (u >= RU0) u += NSK;
            if (u < 80u) {
                d.src = u < 16u ? single(s, u) : u < 48u ? pair(s + 1u, u - 16u) : pair(s + 3u, u - 48u);
                d.gate = sa + s * 512u + (u < ROWU ? 0u : u < 16u ? ONE : u < 48u ? 512u + PAIR + M8 : 1536u + PAIR + M8);
            } else {
                const u32 e = 95u - u, f = e - LEAD;
                const u32 h = e < LEAD || (e < 16u - LEAD && (f & 1u)) ? 1u : 0u;
                const u32 k = e < LEAD ? 7u - e : e >= 16u - LEAD ? 15u - e : h ? 7u - LEAD - (f >> 1) : 7u - (f >> 1);
                u = 80u + h * 8u + k;
                d.src = half(s + 5u, h, k);
                d.gate = sa + (s + 5u) * 512u + h * 256u + M8 + HALF;
            }
            d.dst = d.src;
            d.flag = OFF_LF + (fsrc * NUNITS + u) * 128u;
        } else {
            if (pos >= NPOS_CROSS) pos = NPOS_CROSS - 1u;
            pos += HU;
            const u32 hop = j == a ? HOP1 : HOP2;
            u = pos < hop ? pos : pos - DELTA;
            if (pos - hop < DELTA) {
                const u32 k = pos - hop;
                if (j == a) { d.src = pair(s + 3u, RU0 - 48u + k); d.dst = pair(RELAY, k); d.gate = sa + 34u * 512u + M8 + PAIR;
                              d.flag = FAN + OFF_RF + ((u32)a * 4u + k) * 128u; }
                else { d.src = pair(RELAY, k); d.dst = pair(s + 3u, RU0 - 48u + k); d.gate = OFF_RF + ((u32)j * 4u + k) * 128u + (5u << 28);
                       d.flag = OFF_LF + ((fsrc ^ 4u) * NUNITS + RU0 + k) * 128u; }
            } else if (u < 64u) {
                d.src = u < 16u ? single(32u, u) : u < 48u ? pair((u32)(ASLOTS + 2 * a), u - 16u) : single(33u, u - 48u);
                d.dst = u < 16u ? single((u32)(24 + 2 * j), u) : u < 48u ? pair((u32)(ASLOTS + 8 + 2 * j), u - 16u) : single((u32)(25 + 2 * j), u - 48u);
                d.gate = u < 16u ? sa + (u32)(24 + 2 * j) * 512u + (u < ROWU ? 0u : ONE) : u < 48u ? OFF_WS + (u32)j * 256u + (1u << 20) + (3u << 28) : sa + (u32)(25 + 2 * j) * 512u + (R7C ? ONE : 0u);
                d.flag = OFF_LF + (fsrc * NUNITS + u) * 128u;
            } else {
                const u32 k = u - 64u;
                const int ys = yslot(k);
                const u32 h = k >= 30u ? 1u : 0u;
                d.src = OFF_YST + (u32)(a * 32 + ys) * UNIT + h * 65536u;
                d.dst = ((u32)slot_row((int)fsrc, ys) + h * 64u) * (NLOC * 4u) + (u32)j * 1024u;
                d.gate = OFF_YR + ((u32)(j * 64 + ys) + h * 32u) * 16u + ((k >= 27u ? 10u : 2u) << 28);
                d.flag = NOFLAG;
            }
        }
        return d;
    }
};
struct Tables { Desc lo, hi; };
__device__ __forceinline__ u32 tab(u32 t0, u32 t1, u32 i) {
    const u32 x = (u32)__builtin_amdgcn_readlane((int)t0, (int)(i & 63u)), y = (u32)__builtin_amdgcn_readlane((int)t1, (int)(i & 63u));
    return i < 64u ? x : y;
}

#define PSLEEP0 10
#define PSLEEP 4
#define NACT 32
__device__ __forceinline__ void pull_stream(const Params& p, LAS char* lds, const Tables& T, bool cross, u32 cw, char* src, gchar* fan, u32 calln, u32 call, int lane, int wave) {
    const int hw = wave;
    char* ws = p.me;
    const rsrc_t rl = mk_rsrc(ws, WS_NEED), rs = mk_rsrc(src, WS_NEED), rout = mk_rsrc(p.out, IN_BYTES);
    LAS char* pl = lds + (u32)hw * 1024u;
    const u32 pla = (u32)(size_t)pl + (u32)lane * 4u;
    const u32 l16 = (u32)lane * 16u, l4r = (u32)(lane >> 5) * IMG + (l16 & 511u);
    const u32 l0 = lane == 0 ? 0u : OOB;
    const u32 co = OFF_CL + (u32)wave * 128u + l0;
    const u32 NPIECES = (cross ? NPOS_CROSS : NPOS_INTRA) * 32u;
    const u32 skip = !DCUT ? 0u : cross ? (u32)DCUT : 32u - DCUT;
    const u32 qcut = cross ? ((wave == 3 ? HOP1 : HOP2) + DELTA - HU) * 32u - DCUT : (RU0 - HU) * 32u;
    const u32 cbase = 2u * NACT - (calln - 1u) * (NPIECES - skip);
    u32x4 v0 = {0u, 0u, 0u, 0u}, v1 = v0, v2 = v0, v3;
    u32 q = OOB, so = OOB, ss = 1024u;
    u32 c0 = cw, c1 = cw + NACT;
    u32 pf = NOFLAG;
    bool claimed = false, dead = false;
    int cl = 0;
#define KIND(q) (tab(T.lo.gate, T.hi.gate, (q) >> 5) >> 28)
#define POFF(q, kd) (((kd) & 1u) ? (((q) >> 4) & 1u) * PANEL + ((q) & 15u) * 4096u : (kd) == 4u ? (((q) >> 1) & 15u) * IMG + ((q) & 1u) * 4096u : ((q) & 31u) * (((kd) & 8u) ? 2048u : (kd) ? 4096u : 512u))
#define GBASE(q) ({ const u32 _g = tab(T.lo.gate, T.hi.gate, (q) >> 5), _k = _g >> 28; \
        const u32 _ix = (((_k & 1u) ? ((q) >> 2) & 4u : 0u) + ((q) & (_k == 4u ? 1u : 3u))) << (_k == 3u ? 5 : 7); \
        (_g & 0xfffffu) + ((_k & 7u) == 2u ? 0u : _k == 5u ? ((q) & 28u) * 4u : _k == 0u ? ((q) & 31u) * 16u : _ix); })
#define GOFF(q) (GBASE(q) + (l16 & (tab(T.lo.gate, T.hi.gate, (q) >> 5) >> 16)))
#define PFLAG() st_sys_g((pf & FAN) ? fan + (pf - FAN) : (gchar*)ws + (pf >= NOFLAG ? OFF_CTR + cw * 64u + 8u + (u32)hw * 4u : pf), call)
#define PLD(u, o) v##u = __builtin_amdgcn_raw_buffer_load_b128(rs, (o) + (u) * ss, 0, AUX_SYS)
#define PSTA(u, o) do { asm volatile("" : "+v"(v##u)); \
        __builtin_amdgcn_raw_buffer_store_b128(v##u | (dead ? 0x7fc00000u : 0u), rd, o, 0, AUX_SC1); } while (0)
#define PST(u) PSTA(u, d + (u) * ds)
#define PSTAMP(i) do { } while (0)
#define GPOLL(i, sl) do { __builtin_amdgcn_global_load_lds((const u32*)gp, (LAS u32*)(pl + (i) * 256u), 4, 0, AUX_SYS); __builtin_amdgcn_s_sleep(sl); \
        if (DRY && c0 < 32u) __builtin_amdgcn_s_sleep(PSLEEP0 - PSLEEP); } while (0)
#define GSTEP(i, sl) { GPOLL(((i) + 3) & 3, sl); WAIT_VM_TRACKED(3); u32 _w; \
            asm volatile("ds_read_b32 %0, %1 offset:%2\n\ts_waitcnt lgkmcnt(0)" : "=v"(_w) : "v"(pla), "n"((i) * 256) : "memory"); \
            if (__all((int)(_w - call) >= 0)) break; }
#define WATCH(sl) do { \
        const char* gp = ws + GBASE(c0) + ((u32)lane * 4u & ((tab(T.lo.gate, T.hi.gate, c0 >> 5) >> 16 & 0x3f0u) | 12u)); \
        const u64 t0 = __builtin_amdgcn_s_memrealtime(); \
        GPOLL(0, sl); GPOLL(1, sl); GPOLL(2, sl); \
        for (u32 n = 1;; ++n) { \
            GSTEP(0, sl) GSTEP(1, sl) GSTEP(2, sl) GSTEP(3, sl) \
            if ((n & 15u) == 0u && give_up(ws, call, t0)) { dead = true; break; } \
        } } while (0)
    if (!DRY) WATCH(PSLEEP0);
    const u32 g0 = DRY ? call - 1u : call;
    u32x4 gv = {g0, g0, g0, g0};
    bool dry = DRY;
    u32 d = OOB, ds = 1024u, so2 = OOB, d2 = OOB; bool y = false, val = false;
#define VALID(u) __all(v##u[0] != 0u && v##u[1] != 0u && v##u[2] != 0u && v##u[3] != 0u)
#define RETRY(u) do { if (HINT >= 0 && val && !dead && !VALID(u)) { \
        const u64 t0 = __builtin_amdgcn_s_memrealtime(); u32 n = 0; \
        do { PLD(u, so); NRETRY; \
            if ((++n & 15u) == 0u && give_up(ws, call, t0)) dead = true; \
            WAIT_VM_TRACKED(0); \
        } while (!dead && !VALID(u)); } } while (0)
#define NRETRY do { } while (0)
    do {
        const rsrc_t rd = y ? rout : rl;
        WAIT_VM_TRACKED(5); PLD(2, so2); RETRY(0); PST(0);
        WAIT_VM_TRACKED(4); PLD(3, so2); PFLAG(); RETRY(1); PST(1);
        WAIT_VM_TRACKED(5);
        bool open = dead || all4_ge(gv, call);
        if (q == OOB && !open && !dry) { WATCH(PSLEEP); open = true; }
        dry = false;
        pf = q != OOB ? tab(T.lo.flag, T.hi.flag, q >> 5) + (q & 31u) * 4u : NOFLAG;
        if (claimed) { c1 = (u32)__builtin_amdgcn_readfirstlane(cl) + cbase; if (c1 >= qcut) c1 += skip; if (c1 > NPIECES + 127u) c1 = NPIECES + 127u; }
        claimed = open && c0 < NPIECES;
        q = claimed ? c0 : OOB; if (claimed) c0 = c1;
        const u32 qq = claimed ? q : 0u;
        const u32 kn = claimed ? KIND(qq) : 1u, po = POFF(qq, kn), lo = kn ? l16 : l4r;
        so = claimed ? tab(T.lo.src, T.hi.src, qq >> 5) + po + lo : OOB;
        ss = kn ? 1024u : 2u * IMG;
        y = (kn & 7u) == 2u;
        u32 dn = claimed ? tab(T.lo.dst, T.hi.dst, qq >> 5) + (y ? po * 4u : po) + lo : OOB;
        u32 go = GOFF(c0), ao = claimed ? co : OOB, so1 = so + ss, dx2 = d2 + 2u * ds, dx3 = d2 + 3u * ds;
        if (claimed) PSTAMP(q >> 5);
        asm volatile("" : "+v"(so), "+v"(dn), "+v"(go), "+v"(ao), "+v"(so1), "+v"(dx2), "+v"(dx3));
        WAIT_VM_TRACKED(4);
        v0 = __builtin_amdgcn_raw_buffer_load_b128(rs, so, 0, AUX_SYS);
        PSTA(2, dx2);
        WAIT_VM_TRACKED(4);
        PSTA(3, dx3);
        v1 = __builtin_amdgcn_raw_buffer_load_b128(rs, so1, 0, AUX_SYS);
        cl = __builtin_amdgcn_raw_ptr_buffer_atomic_add_i32(1, rl, ao, 0, AUX_SC1);
        gv = __builtin_amdgcn_raw_buffer_load_b128(rl, go, 0, AUX_SYS);
        d = dn; ds = y ? NLOC * 4u : ss;
        val = (kn & 8u) != 0u;
        so2 = val ? OOB : so; d2 = val ? OOB : dn;
    } while (c0 < NPIECES || (!LASTP && q != OOB));
    if (LASTP) {
        const rsrc_t rd = y ? rout : rl;
        PLD(2, so2); PLD(3, so2);
        WAIT_VM_TRACKED(7); RETRY(0); PST(0);
        WAIT_VM_TRACKED(5); PFLAG(); RETRY(1); PST(1);
        pf = q != OOB ? tab(T.lo.flag, T.hi.flag, q >> 5) + (q & 31u) * 4u : NOFLAG;
        WAIT_VM_TRACKED(4); PSTA(2, d2 + 2u * ds);
        WAIT_VM_TRACKED(4); PSTA(3, d2 + 3u * ds);
    }
    if (!ENDQ || pf < NOFLAG) { WAIT_VM_TRACKED(0); PFLAG(); }
    asm volatile("" :: "v"(gv), "v"(cl));
    asm volatile(".Lagk_end:");
    if (hw < 7) STAMP(ws, 2 + wave * 8 + 7);
#undef PSTAMP
#undef GOFF
#undef GBASE
#undef WATCH
#undef GSTEP
#undef GPOLL
#undef POFF
#undef KIND
#undef PST
#undef PSTA
#undef RETRY
#undef VALID
#undef NRETRY
#undef PLD
#undef PFLAG
}

__global__ void __launch_bounds__(512, 2) ag_gemm_kernel(Params p) {
    extern __shared__ __attribute__((aligned(16))) char smem[];
    LAS char* lds = (LAS char*)smem;
    const int tid = threadIdx.x, lane = tid & 63, wave = __builtin_amdgcn_readfirstlane(tid >> 6);
    const int b = blockIdx.x, rank = p.rank, g = rank >> 2, a = rank & 3;
    u64 pc; u32 len;
    asm volatile("s_getpc_b64 %0\n.Lagk_pc:\n\ts_mov_b32 %1, .Lagk_end-.Lagk_pc" : "=s"(pc), "=s"(len));
    const u32 code = __builtin_amdgcn_raw_buffer_load_b32(mk_rsrc((const void*)pc, (int)len > 0 ? len : 0u), (u32)(b >> 3) * 2048u + (u32)tid * 4u, 0, 0);
    char* ws = p.me;
    if (wave == 0) STAMP(ws, 0);
    const u32 prev = ld_agent(ws + OFF_CTR + (u32)b * 64u);
    u32 calln;
    const int lw = wave;
    const rsrc_t rws = mk_rsrc(ws, WS_NEED);

    Tables T = {};
    char* src = ws; gchar* fan = (gchar*)ws;
    u32 hword = 0;
    gchar* rdflag = x_head(p, lds, b < 128 ? b : 640 + b, prev, calln, hword, tid, wave, [&]() {
        if (b < NCOMM) {
            const u32 par = calln & 1u;
            Stream S;
            S.cross = lw >= 3; S.rank = rank; S.g = g; S.a = a; S.j = (a + lw + 1) & 3; S.pb = CTRL_BYTES + par * PAR_BYTES; S.sa = OFF_SA + par * 32768u;
            T.lo = S.at((u32)lane); T.hi = S.at((u32)lane + 64u);
            src = p.peer[lw == 3 ? 7 : lw];
            char *f0 = p.peer[4], *f1 = p.peer[5], *f2 = p.peer[6];
            asm volatile("" : "+s"(src), "+s"(f0), "+s"(f1), "+s"(f2));
            fan = (gchar*)(lane == 0 ? f0 : lane == 1 ? f1 : f2);
        }
    });
    asm volatile("" :: "v"(code));
    const u32 call = calln << 8, par = calln & 1u;
    if (wave == (CTR7 ? 7 : 0) && lane < 2) st_agent(ws + OFF_CTR + (u32)b * 64u + (u32)lane * 4u, lane ? call : calln);
    if (wave == 0 && b >= NCOMM) STAMP(ws, 1);
    if (b < NCOMM) {
        if (wave == 7) {
            WAIT_VM(0);
            if (lane < 4) st_sys_g(rdflag, hword);
        }
        else if (b < NACT) pull_stream(p, lds, T, wave >= 3, (u32)b, src, fan, calln, call, lane, wave);
        __builtin_amdgcn_endpgm();
    }

    {
        const int c = b - NCOMM, strip = c / 7, rg = c - strip * 7;
        const int kt0 = (rg * 128) / 7, kt1 = ((rg + 1) * 128) / 7;
        const rsrc_t rw = mk_rsrc(p.w, IN_BYTES);
        const int q = lane & 7, n4 = lane >> 3;
        if (wave == 7) { WAIT_VM(0); if (lane < 4) st_sys_g(rdflag, hword); }
        f32x4 v[3][8];
#pragma unroll
        for (int i = 0; i < 3; ++i) {
            const int kt = kt0 + wave + 8 * i;
            const u32 base = kt < kt1 ? ((u32)(kt * 64 + q * 8) * NLOC + (u32)(strip * 32 + n4 * 4)) * 4u : OOB;
#pragma unroll
            for (int jj = 0; jj < 8; ++jj) {
                v[i][jj] = __builtin_bit_cast(f32x4, __builtin_amdgcn_raw_buffer_load_b128(rw, base + (u32)jj * (NLOC * 4u), 0, 0));
                if (i * 8 + jj < W_N0) WAIT_VM(W_LD0 - 1); else WAIT_VM(W_LD - 1);
            }
        }
        LAS u32* cm = (LAS u32*)(lds + L_RED + 256u);
        if (tid < 64) cm[tid] = 0u;
        WAIT_LGKM0(); BAR();
        u32 m[4] = {0u, 0u, 0u, 0u};
#pragma unroll
        for (int i = 0; i < 3; ++i)
#pragma unroll
            for (int jj = 0; jj < 8; ++jj)
#pragma unroll
                for (int k = 0; k < 4; ++k) m[k] = umax(m[k], absbits(v[i][jj][k]));
#pragma unroll
        for (int k = 0; k < 4; ++k) {
            m[k] = umax(m[k], (u32)__shfl_xor((int)m[k], 1)); m[k] = umax(m[k], (u32)__shfl_xor((int)m[k], 2)); m[k] = umax(m[k], (u32)__shfl_xor((int)m[k], 4));
        }
        if (q < 4) __hip_atomic_fetch_max(cm + n4 * 4 + q, q == 0 ? m[0] : q == 1 ? m[1] : q == 2 ? m[2] : m[3], __ATOMIC_RELAXED, __HIP_MEMORY_SCOPE_WORKGROUP);
        WAIT_LGKM0(); BAR();
        u64* wx = (u64*)(ws + OFF_WX) + (u32)strip * 256u;
        if (tid < 32) __hip_atomic_store(wx + rg * 32 + tid, ((u64)call << 32) | cm[tid], __ATOMIC_RELAXED, __HIP_MEMORY_SCOPE_AGENT);
        if (wave < 4) {
            const int gi = tid < 224 ? tid : 0;
            const u64 t0 = __builtin_amdgcn_s_memrealtime();
            u64 gv; u32 n = 0;
            for (;;) {
                gv = __hip_atomic_load(wx + gi, __ATOMIC_RELAXED, __HIP_MEMORY_SCOPE_AGENT);
                if (__all((u32)(gv >> 32) == call)) break;
                if ((++n & 15u) == 0u && give_up(ws, call, t0)) break;
                __builtin_amdgcn_s_sleep(1);
            }
            __hip_atomic_fetch_max(cm + 32 + (gi & 31), (u32)gv, __ATOMIC_RELAXED, __HIP_MEMORY_SCOPE_WORKGROUP);
        }
        WAIT_LGKM0(); BAR();
        const int ct = strip >> 2;
        if (rg == 0 && tid < 64) {
            const u32 nin = (u32)((strip & 3) * 32 + (tid & 31));
            const u32 ev = (u32)(exp_of(cm[32 + (tid & 31)]) - 141);
            char* t = tid < 32 ? ws + OFF_SB + (par * BSLOTS + (u32)ct) * 512u : p.ws[(1 - g) * 4 + (strip >> 3)] + OFF_SB + (par * BSLOTS + (u32)(8 + 2 * a + (ct & 1))) * 512u;
            st_sys(t + nin * 4u, ev);
        }
        const u32x4 mx = *(LAS u32x4*)(cm + 32 + n4 * 4);
        const u32 pbase = CTRL_BYTES + par * PAR_BYTES + (u32)(ASLOTS + ct) * PANEL;
#pragma unroll
        for (int k = 0; k < 4; ++k) {
            const float s = scale_of(exp_of(mx[k]));
            const u32 nin = (u32)((strip & 3) * 32 + n4 * 4 + k);
            const u32 ro = nin * 128u + (((u32)q ^ ((nin >> 1) & 7u)) << 4);
#pragma unroll
            for (int i = 0; i < 3; ++i) {
                const int kt = kt0 + wave + 8 * i;
                const u32 off = kt < kt1 ? pbase + (u32)kt * IMG + ro : OOB;
                const f32x4 lo = {v[i][0][k], v[i][1][k], v[i][2][k], v[i][3][k]}, hi = {v[i][4][k], v[i][5][k], v[i][6][k], v[i][7][k]};
                __builtin_amdgcn_raw_buffer_store_b128(cvt8(lo, hi, s), rws, off, 0, AUX_SC1);
                WAIT_VM(W_ST - 1);
            }
        }
        WAIT_VM(0); BAR();
        if (tid < 4 && (!(tid & 1) || rg == 6))
            st_sys((tid < 2 ? ws + OFF_WR + (u32)strip * 32u : p.ws[(1 - g) * 4 + (strip >> 3)] + OFF_WS + (u32)(a * 8 + (strip & 7)) * 32u) + (u32)(rg + (tid & 1)) * 4u, call);
        if (wave == 0) STAMP(ws, 2);
    }
    const int xcd = b & 7, q = (b >> 3) - 4;
    if (xcd >= 4 && q < 24) {
        const int r0 = 128 + (q * 4 + xcd - 4) * 4;
        x_rows(p, lds, r0, par, call, tid, wave);
        x_rows(p, lds, r0 + (r0 < 384 ? 384 : 512), par, call, tid, wave);
        if (wave == 0) STAMP(ws, 3);
    }
    const int qr = q >> 2, qc = q & 3, x4 = xcd & 3;
    const int early = (x4 >> 1) == (xcd < 4 ? ESWAP : 0) ? (qr < 6 ? a * 6 + qr : -1) : (qr < 3 ? ((a + 1 + qr) & 3) * 6 : 24 + 2 * (qr - 3));
    const int peer = ((a + 1 + (qr >> 1)) & 3) * 6;
    auto tile_of = [&](int it) {
        int slot = -1, ct = 4 * x4 + qc, rh = 0, ch = 0;
        if (it == 0) { if (xcd < 4) { slot = early; ct = 4 * (x4 & 1) + qc; } }
        else if (it == 1) {
            if (RMAP && qr == 6 && xcd >= 2 && xcd < 6) { slot = ((a + 2) & 3) * 6 + (xcd >> 1); ct = 4 * (xcd & 1) + qc; }
            else if (xcd < 4) { if (qr < 6 && !(RMAP && xcd < 2 && qr >> 1 == 1)) slot = peer + 1 + (qr & 1); }
            else { slot = early; ct = 8 + 4 * (x4 & 1) + qc; }
        }
        else if (it == 2) {
            if (qr == 6) { slot = 25 + (xcd & 6); ct = 8 + 4 * (xcd & 1) + qc; }
            else if (xcd >= 4) slot = peer + 3 + (qr & 1);
            else if (xcd < 2 && qr < 4) slot = 25 + 2 * qr;
        } else if (it == 3) {
            int pi = -1; ch = qc & 1;
            const int qs = !RMAP && QSWAP && xcd >= 4 && (qr & 5) == 4 ? qr ^ 2 : qr;
            if (RMAP && xcd < 2 && qr >> 1 == 1) { pi = qr - 1; rh = 1; ct = 12 + 2 * xcd + (qc >> 1); }
            else if (xcd < 2) { pi = qr - 4; rh = 1; ct = 8 + 2 * xcd + (qc >> 1); }
            else if (qs == 6) { pi = RMAP && xcd >= 4 ? -1 : (xcd - 2) >> 1; rh = 1; ct = 12 + 2 * (xcd & 1) + (qc >> 1); }
            else if (xcd < 4) { pi = qr >> 1; ct = 4 * xcd + 2 * (qr & 1) + (qc >> 1); }
            else { pi = qs >> 1; rh = qs & 1; ct = 2 * x4 + (qc >> 1); }
            if (pi >= 0) slot = ((a + 1 + pi) & 3) * 6 + 5;
        }
        if (slot >= 24 && ct >= 8 && (slot - 24) >> 1 == (ct - 8) >> 1) { slot = 32 + (slot & 1); ct -= 8; }
        return Tile{slot, ct, rh, ch};
    };
    char* pend = nullptr;
    int it = 0;
    Tile tl = tile_of(0);
    while (tl.slot < 0) tl = tile_of(++it);
    for (bool first = true; it < 3; first = false) {
        int nt = it + 1;
        Tile nx = tile_of(nt);
        while (nx.slot < 0 && nt < 3) nx = tile_of(++nt);
        gemm_tile<0>(p, lds, tl, first, nx, nt < 3, par, call, tid, lane, wave, 4 + it * 3, pend);
        it = nt; tl = nx;
    }
    if (tl.slot < 0) {
        if (pend) { WAIT_VM(0); BAR(); if (tid < 2) st_sys(pend + (u32)tid * 4u, call); }
        __builtin_amdgcn_endpgm();
    }
    if (tl.ct < 8) {
        gemm_tile<1>(p, lds, tl, false, tl, false, par, call, tid, lane, wave, 13, pend);
        __builtin_amdgcn_endpgm();
    }
    gemm_tile<2>(p, lds, tl, false, tl, false, par, call, tid, lane, wave, 13, pend);
}
}

extern "C" void kernel_launch(int rank, int nranks, ncclComm_t comm,
                              hipStream_t stream,
                              void* const* d_in, const int* in_sizes, int n_in,
                              void* d_out, int out_size,
                              const int* in_dtypes, int out_dtype,
                              void* const* d_out_all,
                              void* const* d_ws_all, size_t ws_bytes) {
    (void)comm; (void)in_dtypes; (void)out_dtype; (void)d_out_all;
    if (nranks != 8 || (unsigned)rank >= 8u || n_in < 2 || ws_bytes < (size_t)agk::WS_NEED || in_sizes[0] != 1024 * 8192 ||
        in_sizes[1] != 8192 * 1024 || out_size != 8192 * 1024) {
        fprintf(stderr, "kernel_launch: unexpected problem shape\n");
        return;
    }
    static bool ready[8] = {false, false, false, false, false, false, false, false};
    if (!ready[rank]) {
        HIP_CHECK(hipFuncSetAttribute((const void*)agk::ag_gemm_kernel, hipFuncAttributeMaxDynamicSharedMemorySize, (int)agk::LDS_BYTES));
        ready[rank] = true;
    }
    agk::Params p;
    p.x = (const float*)d_in[0]; p.w = (const float*)d_in[1]; p.rank = rank; p.pad = 0;
    p.me = (char*)d_ws_all[rank]; p.out = (float*)d_out;
    for (int i = 0; i < 8; ++i) { p.ws[i] = (char*)d_ws_all[i]; p.peer[i] = (char*)d_ws_all[((rank ^ i) & 4) + ((rank + 1 + i) & 3)]; }
    hipLaunchKernelGGL(agk::ag_gemm_kernel, dim3(agk::NWG), dim3(512), agk::LDS_BYTES, stream, p);
    HIP_CHECK(hipGetLastError());
}
```

```cpp
#include <hip/hip_runtime.h>
#include <rccl/rccl.h>
#include <cstdio>

#define HIP_CHECK(expr) do { hipError_t _e = (expr); if (_e != hipSuccess) { \
    fprintf(stderr, "HIP error %s at %s:%d\n", hipGetErrorString(_e), __FILE__, __LINE__); \
    return; } } while (0)

namespace agk {
typedef unsigned u32;
typedef unsigned long long u64;
typedef u32 u32x4 __attribute__((ext_vector_type(4)));
typedef int i32x4 __attribute__((ext_vector_type(4)));
typedef float f32x4 __attribute__((ext_vector_type(4)));
typedef float f32x2 __attribute__((ext_vector_type(2)));
typedef _Float16 f16x2 __attribute__((ext_vector_type(2)));
typedef _Float16 f16x8 __attribute__((ext_vector_type(8)));
#define LAS __attribute__((address_space(3)))
typedef __amdgpu_buffer_rsrc_t rsrc_t;
typedef __attribute__((address_space(1))) char gchar;

constexpr int KDIM = 8192, NLOC = 1024, NKT = 128;
constexpr u32 IMG = 16384u;
constexpr u32 PANEL = NKT * IMG;
constexpr u32 UNIT = 131072u;
constexpr int ASLOTS = 34, BSLOTS = 16;
#define HU 3
#define HWIN 8
#define SCH 2
#define DRY 1
#define NF 4
#define NFL 2
#define RU0 68
#define HOP1 56
#define HOP2 64
#define LEAD 2
#define QSWAP 1
#define RMAP 0
#define FGALL 0
#define LASTP 1
#define ENDQ 1
#define ROWU 6
#define R7C 1
#define EPW 1
#define EPS 0
#define EROT 1
#define EALL 0
#define LSL 0
#define EFL 3
#define LROT 1
#define WEND 1
#define FSLEEP 3
#define CTR7 1
#define ESWAP 1
#define HSTEP 1
#define HK 8
#define XDLY 30
#define HINT 1
#define DCUT 0
#define DELTA 3
constexpr u32 CTRL_BYTES = 1u << 20;
constexpr u32 RELAY = ASLOTS + BSLOTS;
constexpr u32 PAR_BYTES = (RELAY + 2u) * PANEL;
constexpr u32 OFF_YST = CTRL_BYTES + 2u * PAR_BYTES;
constexpr u32 WS_NEED = OFF_YST + 4u * 32u * UNIT;
constexpr u32 IN_BYTES = 1024u * 8192u * 4u;
constexpr u32 OFF_CTR = 0;
constexpr u32 OFF_TMO = 16384;
constexpr u32 OFF_CL = 20480;
constexpr u32 OFF_LF = 524288;
constexpr u32 OFF_RD = 655360;
constexpr u32 OFF_YR = 24576;
constexpr u32 OFF_RF = 28672;
constexpr u32 OFF_WR = 32768;
constexpr u32 OFF_WS = 33792;
constexpr u32 OFF_WX = 36864;
constexpr u32 OFF_SA = 131072;
constexpr u32 OFF_SB = 196608;
constexpr u32 OFF_DBG = 262144;
constexpr int NWG = 256, NCOMM = 32;
constexpr u32 NUNITS = 96u;
constexpr u32 OOB = 0x80000000u, NOFLAG = 0x40000000u;
constexpr u64 DEADLINE = 30000000ull;
constexpr u32 STAGE = 32768u;
constexpr u32 L_SNAP = 4u * STAGE;
constexpr u32 L_SC = L_SNAP + 512u;
constexpr u32 L_RED = L_SC + 1024u;
constexpr u32 LDS_BYTES = L_RED + 2048u;

struct Params {
    const float* x; char* me; int rank, pad;
    char* peer[8];
    const float* w; float* out;
    char* ws[8];
};

#define AUX_SC1 16
#define AUX_SYS 17
#define GL_AUX 16
#define W_LD 2
#define W_LD0 1
#define W_N0 0
#define W_ST 2
#define X_LD 4
#define X_ST 2
#define WAIT_VM(n) asm volatile("s_waitcnt vmcnt(%0)" :: "n"(n) : "memory")
#define WAIT_LGKM0() do { asm volatile("" ::: "memory"); __builtin_amdgcn_s_waitcnt(15 | (7 << 4) | (0 << 8) | (3 << 14)); asm volatile("" ::: "memory"); } while (0)
#define BAR() do { asm volatile("" ::: "memory"); __builtin_amdgcn_s_barrier(); asm volatile("" ::: "memory"); } while (0)
#define WAIT_VM_TRACKED(n) do { asm volatile("" ::: "memory"); __builtin_amdgcn_s_waitcnt(((n) & 15) | (7 << 4) | (15 << 8) | (((n) >> 4) << 14)); asm volatile("" ::: "memory"); } while (0)

__device__ __forceinline__ rsrc_t mk_rsrc(const void* p, u32 bytes) { return __builtin_amdgcn_make_buffer_rsrc((void*)p, 0, bytes, 0x00020000); }
__device__ __forceinline__ u32 ld_agent(const void* p) { return __hip_atomic_load((const u32*)p, __ATOMIC_RELAXED, __HIP_MEMORY_SCOPE_AGENT); }
__device__ __forceinline__ u32 ld_sys(const void* p) { return __hip_atomic_load((const u32*)p, __ATOMIC_RELAXED, __HIP_MEMORY_SCOPE_SYSTEM); }
__device__ __forceinline__ void st_agent(void* p, u32 v) { __hip_atomic_store((u32*)p, v, __ATOMIC_RELAXED, __HIP_MEMORY_SCOPE_AGENT); }
__device__ __forceinline__ void st_sys_g(gchar* p, u32 v) { __hip_atomic_store((__attribute__((address_space(1))) u32*)p, v, __ATOMIC_RELAXED, __HIP_MEMORY_SCOPE_SYSTEM); }
__device__ __forceinline__ void st_sys(void* p, u32 v) { __hip_atomic_store((u32*)p, v, __ATOMIC_RELAXED, __HIP_MEMORY_SCOPE_SYSTEM); }

#define STAMP(ws, idx) do { } while (0)

__device__ __forceinline__ bool give_up(char* ws, u32 call, u64 t0) {
    const u32 t = __builtin_amdgcn_readfirstlane(ld_agent(ws + OFF_TMO));
    if (t == call || __builtin_amdgcn_s_memrealtime() - t0 > DEADLINE) { st_agent(ws + OFF_TMO, call); return true; }
    return false;
}
__device__ __forceinline__ void poll_ge(const char* p, u32 need, char* ws, u32 call) {
    const u64 t0 = __builtin_amdgcn_s_memrealtime();
    u32 n = 0;
    for (;;) {
        const u32 v = ld_sys(p);
        if (__all((int)(v - need) >= 0)) return;
        if ((++n & 15u) == 0u && give_up(ws, call, t0)) return;
        __builtin_amdgcn_s_sleep(2);
    }
}
__device__ __forceinline__ bool all4_ge(u32x4 v, u32 need) {
    return __all((int)(v[0] - need) >= 0 && (int)(v[1] - need) >= 0 && (int)(v[2] - need) >= 0 && (int)(v[3] - need) >= 0);
}
__device__ __forceinline__ void poll4_ge(rsrc_t rs, u32 off, u32 need, char* ws, u32 call) {
    const u64 t0 = __builtin_amdgcn_s_memrealtime();
    u32 n = 0;
    for (;;) {
        asm volatile("" ::: "memory");
        const u32x4 v = __builtin_amdgcn_raw_buffer_load_b128(rs, off, 0, AUX_SYS);
        if (all4_ge(v, need)) return;
        if ((++n & 15u) == 0u && give_up(ws, call, t0)) return;
        __builtin_amdgcn_s_sleep(1);
    }
}

__device__ __forceinline__ u32 umax(u32 a, u32 b) { return a > b ? a : b; }
__device__ __forceinline__ u32 absbits(float f) { return __float_as_uint(f) & 0x7fffffffu; }
__device__ __forceinline__ int exp_of(u32 amaxbits) { int e = (int)(amaxbits >> 23); return e < 15 ? 15 : e; }
__device__ __forceinline__ float scale_of(int e) { return __uint_as_float((u32)(268 - e) << 23); }
__device__ __forceinline__ u32 cvt2(float a, float b, float s) {
    const f32x2 t = (f32x2){a, b} * s;
    return __builtin_bit_cast(u32, __builtin_convertvector(t, f16x2));
}
__device__ __forceinline__ u32x4 cvt8(f32x4 a, f32x4 b, float s) {
    return (u32x4){cvt2(a[0], a[1], s), cvt2(a[2], a[3], s), cvt2(b[0], b[1], s), cvt2(b[2], b[3], s)};
}
__device__ __forceinline__ u32 amax4(u32 m, f32x4 v) {
    asm("v_max3_f32 %0, |%1|, |%2|, %0\n\tv_max3_f32 %0, |%3|, |%4|, %0" : "+v"(m) : "v"(v[0]), "v"(v[1]), "v"(v[2]), "v"(v[3]));
    return m;
}

__device__ __forceinline__ u32 wave_max(u32 m) {
    m = umax(m, (u32)__builtin_amdgcn_update_dpp(0, (int)m, 0xb1, 15, 15, true));
    m = umax(m, (u32)__builtin_amdgcn_update_dpp(0, (int)m, 0x4e, 15, 15, true));
    m = umax(m, (u32)__builtin_amdgcn_update_dpp(0, (int)m, 0x141, 15, 15, true));
    m = umax(m, (u32)__builtin_amdgcn_update_dpp(0, (int)m, 0x140, 15, 15, true));
    return umax(umax((u32)__builtin_amdgcn_readlane((int)m, 0), (u32)__builtin_amdgcn_readlane((int)m, 16)),
                umax((u32)__builtin_amdgcn_readlane((int)m, 32), (u32)__builtin_amdgcn_readlane((int)m, 48)));
}
__device__ __forceinline__ u32 row_word(u32 call, int e) { return call | ((u32)(e - 141) & 255u); }

__device__ __forceinline__ void x_rows(const Params& p, LAS char* lds, int row0, u32 par, u32 call, int tid, int wave) {
    constexpr int NR = 4;
    const int a = p.rank & 3;
    char* ws = p.me;
    const rsrc_t rx = mk_rsrc(p.x, IN_BYTES), rw = mk_rsrc(ws, WS_NEED);
    const u32 pi0 = (u32)(wave * 128 + (tid & 63));
    f32x4 v[NR][4];
#pragma unroll
    for (int r = 0; r < NR; ++r)
#pragma unroll
        for (int u = 0; u < 2; ++u) {
            const u32 off = (u32)(row0 + r) * (KDIM * 4u) + (pi0 + (u32)u * 64u) * 32u;
            v[r][2 * u] = __builtin_bit_cast(f32x4, __builtin_amdgcn_raw_buffer_load_b128(rx, off, 0, 0));
            v[r][2 * u + 1] = __builtin_bit_cast(f32x4, __builtin_amdgcn_raw_buffer_load_b128(rx, off + 16u, 0, 0));
            WAIT_VM(X_LD - 2);
        }
    const int rb = row0 >> 7;
    const u32 slot = rb < 6 ? (u32)(a * 6 + rb) : (u32)(32 + rb - 6);
    const u32 rslot = rb < 6 ? slot : (u32)(24 + 2 * a + rb - 6);
    char* hp[4];
#pragma unroll
    for (int i = 0; i < 4; ++i) hp[i] = rb < 6 ? p.peer[i] : p.peer[4 + i];
    char* fifth = rb < 6 ? p.peer[7] : ws;
    asm volatile("" : "+s"(hp[0]), "+s"(hp[1]), "+s"(hp[2]), "+s"(hp[3]), "+s"(fifth));
    gchar* user = (gchar*)((tid & 4) ? fifth : (tid & 2) ? ((tid & 1) ? hp[3] : hp[2]) : ((tid & 1) ? hp[1] : hp[0]));
    const u32 wr = (u32)(tid & 63) >> 3;
    const bool wsend = wr < (u32)NR && ((tid & 7) < 4 || ((tid & 7) == 4 && rb >= 3 && rb != 5));
    const u32 woff = OFF_SA + ((par * 64u + ((tid & 4) ? (rb < 6 ? (u32)(31 + rb) : slot) : rslot)) * 128u + (((u32)row0 + wr) & 127u)) * 4u;
    u32 word = 0;
    LAS u32* red = (LAS u32*)(lds + L_RED);
#pragma unroll
    for (int r = 0; r < NR; ++r) {
        u32 m = 0;
#pragma unroll
        for (int i = 0; i < 4; ++i) m = amax4(m, v[r][i]);
        m = wave_max(m);
        if ((tid & 63) == 0) red[r * 8 + wave] = m;
    }
    WAIT_LGKM0(); BAR();
#pragma unroll
    for (int r = 0; r < NR; ++r) {
        const u32x4 m0 = *(LAS u32x4*)(red + r * 8), m1 = *(LAS u32x4*)(red + r * 8 + 4);
        const u32 m = umax(umax(umax(m0[0], m0[1]), umax(m0[2], m0[3])), umax(umax(m1[0], m1[1]), umax(m1[2], m1[3])));
        const int e = exp_of(m); const float s = scale_of(e);
        const u32 rin = (u32)((row0 + r) & 127);
        if (wr == (u32)r) word = row_word(call, e);
        const u32 off = CTRL_BYTES + par * PAR_BYTES + rin * 128u + (pi0 >> 3) * IMG + (((pi0 & 7u) ^ ((rin >> 1) & 7u)) << 4);
        __builtin_amdgcn_raw_buffer_store_b128(cvt8(v[r][0], v[r][1], s), rw, off + slot * PANEL, 0, AUX_SC1);
        if (X_ST == 1) WAIT_VM(0);
        __builtin_amdgcn_raw_buffer_store_b128(cvt8(v[r][2], v[r][3], s), rw, off + slot * PANEL + 8u * IMG, 0, AUX_SC1);
        WAIT_VM(X_ST < 2 ? 0 : X_ST - 2);
    }
    WAIT_VM(0); BAR();
    if (wave == 0 && wsend) st_sys_g(user + woff, word);
}

template <class F>
__device__ __forceinline__ gchar* x_head(const Params& p, LAS char* lds, int row, u32 prev, u32& calln, u32& hword, int tid, int wave, F mid) {
    constexpr int NS = HU > 3 ? HU : 3;
    const int lane = tid & 63;
    char* ws = p.me;
    const rsrc_t rx = mk_rsrc(p.x, IN_BYTES), rw = mk_rsrc(ws, WS_NEED);
    LAS u32* red = (LAS u32*)(lds + L_RED);
    if (wave != 7) {
        if (tid < 4) red[tid] = 0u;
        WAIT_LGKM0(); BAR();
        __builtin_amdgcn_s_sleep(XDLY);
    }
    const u32 s0 = wave == 7 ? 0u : (u32)wave, sd = wave == 7 ? 1u : 7u, se = wave == 7 ? (u32)HU : 16u;
    f32x4 v[NS][2];
#pragma unroll
    for (int u = 0; u < NS; ++u) {
        const u32 set = s0 + (u32)u * sd;
        const u32 off = set < se ? (u32)row * (KDIM * 4u) + (set * 64u + (u32)lane) * 32u : OOB;
        v[u][0] = __builtin_bit_cast(f32x4, __builtin_amdgcn_raw_buffer_load_b128(rx, off, 0, 0));
        v[u][1] = __builtin_bit_cast(f32x4, __builtin_amdgcn_raw_buffer_load_b128(rx, off + 16u, 0, 0));
    }
    __builtin_amdgcn_sched_barrier(0);
    if (wave == 7) BAR();
    const bool major = row < 128;
    const int a = p.rank & 3;
    char* hp[4];
#pragma unroll
    for (int i = 0; i < 4; ++i) hp[i] = major ? p.peer[i] : p.peer[4 + i];
    const u32 slot = major ? (u32)(a * 6) : 32u, rslot = major ? slot : (u32)(24 + 2 * a), rin = (u32)row & 127u;
    u32 off = CTRL_BYTES + rin * 128u + ((u32)lane >> 3) * IMG + ((((u32)lane & 7u) ^ ((rin >> 1) & 7u)) << 4);
    u32 m = 0;
    gchar* user = (gchar*)((tid & 4) ? ws : (tid & 2) ? ((tid & 1) ? hp[3] : hp[2]) : ((tid & 1) ? hp[1] : hp[0]));
#define WORD() do { u32 t = 0; \
        if (lane == 0) t = __hip_atomic_fetch_add(red + 2, 1u, __ATOMIC_RELAXED, __HIP_MEMORY_SCOPE_WORKGROUP); \
        if (__builtin_amdgcn_readfirstlane((int)t) == 7 && lane < (major ? 4 : 5)) \
            st_sys_g(user + OFF_SA + (((calln & 1u) * 64u + (lane == 4 ? slot : rslot)) * 128u + rin) * 4u, \
                     row_word(calln << 8, exp_of(__hip_atomic_load(red, __ATOMIC_RELAXED, __HIP_MEMORY_SCOPE_WORKGROUP)))); } while (0)
    if (wave == 7) {
        rsrc_t hr[4];
#pragma unroll
        for (int i = 0; i < 4; ++i) hr[i] = mk_rsrc(hp[i], i == 3 && major ? 0u : WS_NEED);
        u32 o4[4];
#pragma unroll
        for (int i = 0; i < 4; ++i) o4[i] = (lane >> 4) == i ? off + rslot * PANEL : OOB;
        asm volatile("" : "+s"(hr[0]), "+s"(hr[1]), "+s"(hr[2]), "+s"(hr[3]), "+v"(o4[0]), "+v"(o4[1]), "+v"(o4[2]), "+v"(o4[3]), "+v"(m), "+v"(prev));
        calln = (u32)__builtin_amdgcn_readfirstlane(prev) + 1u;
        const u32 so = (calln & 1u) * PAR_BYTES;
#pragma unroll
        for (int u = 0; u < HU; ++u) m = amax4(amax4(m, v[u][0]), v[u][1]);
        m = wave_max(m);
        const int e = exp_of(m); const float s = scale_of(e);
        u32x4 h;
#pragma unroll
        for (int k = 0; k < 4 * HU; ++k) {
            if ((k & 3) == 0) h = cvt8(v[k >> 2][0], v[k >> 2][1], s);
            if (k == 2 && lane == 0) __hip_atomic_fetch_add(red + 1, 1u, __ATOMIC_RELAXED, __HIP_MEMORY_SCOPE_WORKGROUP);
#pragma unroll
            for (int i = 0; i < 4; ++i) {
                if (HSTEP ? i == 0 && 4 * k >= HWIN : 4 * k + i >= HWIN) WAIT_VM(HWIN - (HSTEP ? 4 : 1));
                __builtin_amdgcn_raw_buffer_store_b128(h, hr[i], o4[k & 3], so + (u32)(k >> 2) * 8u * IMG, AUX_SYS);
            }
            asm volatile("s_nop 1" : "+v"(h) :: "memory");
            if (k == HK) WORD();
        }
        hword = row_word(calln << 8, e);
        for (u32 n = 0; n < (1u << 24); ++n)
            if (__builtin_amdgcn_readfirstlane((int)__hip_atomic_load(red + 1, __ATOMIC_RELAXED, __HIP_MEMORY_SCOPE_WORKGROUP)) == 8) break;
    }
    else calln = (u32)__builtin_amdgcn_readfirstlane(prev) + 1u;
    const u32 par = calln & 1u;
    if (wave != 7) {
        mid();
        off += par * PAR_BYTES + slot * PANEL;
#pragma unroll
        for (int u = 0; u < 3; ++u) m = amax4(amax4(m, v[u][0]), v[u][1]);
        m = wave_max(m);
        if (lane == 0) {
            __hip_atomic_fetch_max(red, m, __ATOMIC_RELAXED, __HIP_MEMORY_SCOPE_WORKGROUP);
            __hip_atomic_fetch_add(red + 1, 1u, __ATOMIC_RELAXED, __HIP_MEMORY_SCOPE_WORKGROUP);
        }
        u64 mc;
        for (u32 n = 0; n < (1u << 24); ++n) {
            mc = __hip_atomic_load((LAS u64*)red, __ATOMIC_RELAXED, __HIP_MEMORY_SCOPE_WORKGROUP);
            if ((u32)__builtin_amdgcn_readfirstlane((int)(mc >> 32)) == 8u) break;
        }
        const int e = exp_of((u32)mc); const float s = scale_of(e);
#pragma unroll
        for (int u = 0; u < 3; ++u) {
            const u32 set = s0 + (u32)u * 7u;
            __builtin_amdgcn_raw_buffer_store_b128(cvt8(v[u][0], v[u][1], s), rw, set < 16u ? off + set * 8u * IMG : OOB, 0, AUX_SC1);
        }
        WAIT_VM(0);
        WORD();
    }
#undef WORD
    return user + OFF_RD + ((par * 8u + (u32)p.rank) * 128u + rin) * 4u;
}

__device__ __forceinline__ int slot_row(int rank, int slot) {
    const int g = rank >> 2;
    if (slot < 24) { const int i = slot / 6; return (g * 4 + i) * 1024 + (slot - i * 6) * 128; }
    return (slot < 32 ? (1 - g) * 4 + ((slot - 24) >> 1) : rank) * 1024 + 768 + (slot & 1) * 128;
}

struct Tile { int slot, ct, rh, ch; };
__device__ __forceinline__ void a_stream(int rank, int slot, int rh, int& a_src, u32& a_base, u32& a_sh) {
    const int g = rank >> 2, a = rank & 3;
    if (slot < 24) { const int i = slot / 6, a_rb = slot - i * 6; a_src = i == a ? -1 : g * 4 + i;
        a_base = a_rb == 0 ? 0u : a_rb < 3 ? 16u : a_rb < 5 ? 48u : 80u + (u32)rh * 8u; a_sh = a_rb == 0 ? 1u : 0u; }
    else { a_src = slot < 32 ? (1 - g) * 4 + ((slot - 24) >> 1) : -1; a_base = (slot & 1) ? 48u : 0u; a_sh = 1u; }
}
__device__ __forceinline__ u32 start_off(int rank, const Tile& t, bool big, u32 par, int lane, int wave) {
    int a_src; u32 a_base, a_sh;
    a_stream(rank, t.slot, t.rh, a_src, a_base, a_sh);
    const int b_src = t.ct < 8 ? -1 : (1 - (rank >> 2)) * 4 + ((t.ct - 8) >> 1);
    u32 o = OOB, m = 63u;
    if (wave == 0) { if (a_src >= 0) { o = OFF_LF + ((u32)a_src * NUNITS + (a_base ? a_base : (u32)HU)) * 128u; m = big ? 7u : 3u; } }
    else if (wave == 1) { if (a_src < 0) { o = OFF_SA + (par * 64u + (u32)t.slot) * 512u; m = 31u; } }
    else if (wave == 2) { if (b_src < 0) { o = OFF_WR + (u32)t.ct * 128u; m = 7u; } }
    else if (wave == 3) { if (a_src >= 0 && a_base == 0u) { o = OFF_RD + (par * 8u + (u32)a_src) * 512u; m = 31u; } }
    else if (b_src >= 0 && (!big || wave == 4)) { o = OFF_LF + ((u32)b_src * NUNITS + 16u) * 128u + (u32)(wave - 4) * 1024u; m = big ? 7u : 63u; }
    return o + ((u32)lane & m) * 16u;
}

template <int MODE>
__device__ __forceinline__ void gemm_tile(const Params& p, LAS char* lds, const Tile& tl, bool first, const Tile& nx, bool nbig, u32 par, u32 call, int tid, int lane, int wave, int stamp0, char*& pend) {
    constexpr bool BIG = MODE == 0;
    constexpr int NB = BIG ? 4 : 2, NC = BIG ? 32 : 16, BT = BIG ? 128 : 64, NR = MODE == 2 ? NF : MODE == 1 ? NFL : 0;
    const int rank = p.rank, g = rank >> 2, a = rank & 3;
    const int slot = tl.slot, ct = tl.ct, rh = tl.rh, ch = tl.ch;
    asm volatile("" : "+v"(lane), "+v"(tid));
    char* ws = p.me;
    const rsrc_t rws = mk_rsrc(ws, WS_NEED);
    int a_src; u32 a_base, a_sh;
    a_stream(rank, slot, rh, a_src, a_base, a_sh);
    const int b_src = ct < 8 ? -1 : (1 - g) * 4 + ((ct - 8) >> 1);
    const int src = lane < 32 ? a_src : (BIG ? b_src : -1);
    const u32 gsh = src < 0 ? 31u : lane < 32 && BIG ? a_sh : 0u;
    const char* gptr = ws + (src >= 0 ? OFF_LF + ((u32)src * NUNITS + (lane < 32 ? a_base : 16u)) * 128u + (u32)(lane & (BIG ? 31 : 15)) * 4u : OFF_CTR + blockIdx.x * 64u + 4u);
    const u32 gmin = src >= 0 && lane < 32 && a_base == 0u ? (u32)HU : 0u;
    const bool headt = BIG && a_src >= 0 && a_base == 0u;
#define GPTR(c) (gptr + (umax((u32)(c) >> gsh, gmin) << (BIG ? 7 : 6)))
#define PEND() do { if (pend && wave == 1 && lane < 2) st_sys(pend + (u32)lane * 4u, call); pend = nullptr; } while (0)
    LAS u32* okw = (LAS u32*)(lds + L_RED + 1024u);
    const u32 pb = CTRL_BYTES + par * PAR_BYTES;
    const int ks = BIG ? wave >> 2 : wave >> 1, wm = BIG ? (wave >> 1) & 1 : 0, wn = wave & 1;
    const u32 swz = (u32)((((ks & 1) * 4 + (lane >> 4)) ^ ((lane & 15) >> 1)) << 4);
    f16x8 bt[4 * NR + 1][2];
    WAIT_LGKM0(); BAR();
    u32 po = OOB;
    {
        const u32x4 k0 = *(const LAS u32x4*)okw, k1 = *(const LAS u32x4*)(okw + 4);
        if (first || __builtin_amdgcn_readfirstlane((int)(k0[0] | k0[1] | k0[2] | k0[3] | k1[0] | k1[1] | k1[2] | k1[3]))) {
            if (pend) { WAIT_VM(0); BAR(); PEND(); }
            po = start_off(rank, tl, BIG, par, lane, wave);
            if ((!NR || wave != 0) && __builtin_amdgcn_readfirstlane((int)po) >= 0) poll4_ge(rws, po, call, ws, call);
            WAIT_VM_TRACKED(0); BAR();
        }
    }
    if (!NR && wave == 0) STAMP(ws, stamp0);
    constexpr u32 RB = BT * 4u, PPR = BT / 4u, AREA = BT * RB;
    const u32 pc = (u32)lane & (PPR - 1u);
    const u32 col0 = BIG ? 0u : (u32)ch * 64u, row0 = BIG ? 0u : (u32)rh * 64u;
    const bool remote = BIG ? ct >= 8 : MODE == 2;
    const rsrc_t ro = mk_rsrc(remote ? (const void*)ws : (const void*)p.out, remote ? WS_NEED : IN_BYTES);
    const u32 ostr = remote ? 1024u : NLOC * 4u;
    const u32 obase = (remote ? OFF_YST + (u32)(((ct - 8) >> 1) * 32 + slot) * UNIT + (u32)(ct & 1) * 512u : ((u32)slot_row(rank, slot) * NLOC + (u32)ct * 128u) * 4u) + col0 * 4u + row0 * ostr + pc * 16u;
#define YFLAG() do { if (tid == 0) st_sys(p.ws[b_src] + OFF_YR + (u32)(a * 64 + rh * 32 + slot) * 16u + (u32)(ct & 1) * 8u + (u32)ch * 4u, call); } while (0)
    constexpr bool hint = MODE == 2;
    if (hint) {
        const u32x4 zero = {0u, 0u, 0u, 0u};
#pragma unroll
        for (u32 it = 0; it < 2u; ++it) __builtin_amdgcn_raw_buffer_store_b128(zero, ro, obase + ((u32)wave * (BT / 8u) + (it * 64u + (u32)lane) / PPR) * ostr, 0, AUX_SC1);
    }
    const u32 gA = pb + (u32)slot * PANEL + (BIG ? (u32)wave * 2048u : (u32)rh * 8192u + (u32)wave * 1024u);
    const u32 gB = pb + (u32)(ASLOTS + ct) * PANEL + (BIG ? (u32)wave * 2048u : (u32)ch * 8192u + (u32)wave * 1024u);
    const u32 l16 = (u32)lane * 16u;
    const u32 ldsw = (u32)wave * (BIG ? 2048u : 1024u);
    constexpr u32 GSTEP = BIG ? IMG : 2u * IMG, G2 = BIG ? 1024u : IMG, L2O = BIG ? 1024u : 8192u;
#define SCHED() __builtin_amdgcn_sched_barrier(0)
#define GLDS16(g, l) __builtin_amdgcn_raw_ptr_buffer_load_lds(rws, (LAS void*)(lds + (l)), 16, l16, (g), 0, GL_AUX)
#define LOADT(t, st) do { const u32 _o = (u32)(t) * GSTEP; \
        GLDS16(gA + _o, (st) * STAGE + ldsw); GLDS16(gA + _o + G2, (st) * STAGE + ldsw + L2O); \
        GLDS16(gB + _o, (st) * STAGE + IMG + ldsw); GLDS16(gB + _o + G2, (st) * STAGE + IMG + ldsw + L2O); } while (0)
#define SNAP(c) __builtin_amdgcn_global_load_lds((const u32*)((c) < NC ? GPTR(c) : ws + OFF_TMO), (LAS u32*)(lds + L_SNAP + ((u32)(c) & 1u) * 256u), 4, 0, AUX_SYS)
    const u32 swl = swz + (BIG ? 0u : (u32)(ks >> 1) * 8192u);
    const u32 ard = (u32)(wm * 64 + (lane & 15)) * 128u + swl, brd = IMG + (u32)(wn * (BT / 2) + (lane & 15)) * 128u + swl;
#define READF(fa, fb, st) READX(fa, fb, st, true)
#define READX(fa, fb, st, rb) do { _Pragma("unroll") for (int _i = 0; _i < 4; ++_i) { \
        if (_i < NB && (rb)) fb[_i] = *(const LAS f16x8*)(lds + (st) * STAGE + brd + _i * 2048u); \
        fa[_i] = *(const LAS f16x8*)(lds + (st) * STAGE + ard + _i * 2048u); } } while (0)
#define MMA(fa, fb) do { SCHED(); _Pragma("unroll") for (int _m = 0; _m < 4; ++_m) _Pragma("unroll") for (int _n = 0; _n < NB; ++_n) \
        acc[_m][_n] = __builtin_amdgcn_mfma_f32_16x16x32_f16(fb[_n], fa[_m], acc[_m][_n], 0, 0, 0); SCHED(); } while (0)
#define SB() __builtin_amdgcn_sched_barrier(0)
#define STEP(t, sl, na, nb, sr, fa, fb) STEPX(t, sl, na, nb, sr, fa, fb, true, true)
#define STEPX(t, sl, na, nb, sr, fa, fb, lb, rb) do { const u32 _o = (u32)(t) * GSTEP; \
        _Pragma("unroll") for (int _m = 0; _m < 4; ++_m) { \
            _Pragma("unroll") for (int _n = 0; _n < NB / 2; ++_n) acc[_m][_n] = __builtin_amdgcn_mfma_f32_16x16x32_f16(fb[_n], fa[_m], acc[_m][_n], 0, 0, 0); \
            SB(); \
            if (_m < 2) { _Pragma("unroll") for (int _i = 0; _i < 2; ++_i) { \
                if (_i < NB / 2 && (rb)) nb[_m * (NB / 2) + _i] = *(const LAS f16x8*)(lds + (sr) * STAGE + brd + (_m * (NB / 2) + _i) * 2048u); \
                na[_m * 2 + _i] = *(const LAS f16x8*)(lds + (sr) * STAGE + ard + (_m * 2 + _i) * 2048u); } } \
            if (_m < 2 || (lb)) GLDS16((_m < 2 ? gA : gB) + _o + (_m & 1) * G2, (sl) * STAGE + (_m >> 1) * IMG + ldsw + (_m & 1) * L2O); \
            SB(); \
            _Pragma("unroll") for (int _n = NB / 2; _n < NB; ++_n) acc[_m][_n] = __builtin_amdgcn_mfma_f32_16x16x32_f16(fb[_n], fa[_m], acc[_m][_n], 0, 0, 0); \
        } SB(); } while (0)
    f32x4 acc[4][NB];
#pragma unroll
    for (int m = 0; m < 4; ++m)
#pragma unroll
        for (int n = 0; n < NB; ++n) acc[m][n] = (f32x4){0.f, 0.f, 0.f, 0.f};
    f16x8 a0[4], b0[4], a1[4], b1[4];
#define VLOAD(v) do { const u32 _o = (u32)(4 * (NC - NR) + 2 * (v)) * GSTEP, _l = (u32)(((v) + 2 * NR) & 3) * STAGE + ldsw; \
        GLDS16(gB + _o, _l); GLDS16(gB + _o + G2, _l + L2O); GLDS16(gB + _o + GSTEP, _l + IMG); GLDS16(gB + _o + GSTEP + G2, _l + IMG + L2O); } while (0)
#define VREAD(v) do { _Pragma("unroll") for (int _i = 0; _i < 4; ++_i) \
        bt[NR ? 2 * (v) + (_i >> 1) : 0][_i & 1] = *(const LAS f16x8*)(lds + (u32)(((v) + 2 * NR) & 3) * STAGE + brd - (_i < 2 ? IMG : 0u) + (_i & 1) * 2048u); } while (0)
#define VSTEP(v) do { if (NR) { WAIT_VM(8); WAIT_LGKM0(); BAR(); VREAD(v); } } while (0)
    if (NR) {
        VLOAD(0); VLOAD(1); VLOAD(2); VLOAD(3);
        WAIT_VM(12); BAR();
        PEND();
        VREAD(0);
#pragma unroll
        for (int v = 4; v < 2 * NR; ++v) { VSTEP(v - 3); VLOAD(v); }
        if (wave == 0 && __builtin_amdgcn_readfirstlane((int)po) >= 0) { poll4_ge(rws, po, call, ws, call); WAIT_VM_TRACKED(0); }
        VSTEP(2 * NR - 3);
        if (wave == 0) STAMP(ws, stamp0);
    }
    if (wave >= 4) {
        const u32 so = (wave < 6 ? OFF_SA + (par * 64u + (u32)slot) * 512u : OFF_SB + (par * BSLOTS + (u32)ct) * 512u) + (u32)((wave & 1) * 64 + lane) * 4u;
        __builtin_amdgcn_global_load_lds((const u32*)(ws + so), (LAS u32*)(lds + L_SC + (u32)(wave - 4) * 256u), 4, 0, AUX_SYS);
    }
    else if (wave >= 2 && headt)
        __builtin_amdgcn_global_load_lds((const u32*)(ws + OFF_RD + (par * 8u + (u32)a_src) * 512u + (u32)((wave & 1) * 64 + lane) * 4u), (LAS u32*)(lds + L_RED + (u32)(wave - 2) * 256u), 4, 0, AUX_SYS);
    LOADT(0, 0u);
    if (wave == 0) SNAP(1);
    VSTEP(2 * NR - 2);
    LOADT(1, 1u);
    VSTEP(2 * NR - 1);
    LOADT(2, 2u);
    if (NR) { WAIT_VM(8); WAIT_LGKM0(); BAR(); LOADT(3, 3u); }
    else {
        LOADT(3, 3u);
        if (wave == 0) WAIT_VM(13); else WAIT_VM(12);
        BAR();
        PEND();
    }
    READF(a0, b0, 0u);
#define GATE(c) do { const u32 _v = *(const LAS u32*)(lds + L_SNAP + ((u32)(c) & 1u) * 256u + (u32)lane * 4u); \
        if (!__all((int)(_v - call) >= 0)) { if (wave == 0) { poll_ge(GPTR(c), call, ws, call); WAIT_VM(0); } BAR(); } } while (0)
    bool seen = false;
#define FP(i) do { __builtin_amdgcn_global_load_lds((const u32*)_gp, (LAS u32*)(lds + L_RED + (i) * 256u), 4, 0, AUX_SYS); __builtin_amdgcn_s_sleep(FSLEEP); } while (0)
#define FS(i) { FP(((i) + 3) & 3); WAIT_VM(3); u32 _w; \
            asm volatile("ds_read_b32 %0, %1 offset:%2\n\ts_waitcnt lgkmcnt(0)" : "=v"(_w) : "v"(_pa), "n"((i) * 256) : "memory"); \
            if (__all((int)(_w - call) >= 0)) break; }
#define FGATE(c) do { const u32 _v = *(const LAS u32*)(lds + L_SNAP + ((u32)(c) & 1u) * 256u + (u32)lane * 4u); \
        seen = false; \
        if (!__all((int)(_v - call) >= 0)) { if (wave == 0) { seen = true; \
            const char* _gp = GPTR(c); const u32 _pa = (u32)(size_t)(lds + L_RED) + (u32)lane * 4u; const u64 _t0 = __builtin_amdgcn_s_memrealtime(); \
            FP(0); FP(1); FP(2); \
            for (u32 _n = 1;; ++_n) { FS(0) FS(1) FS(2) FS(3) if ((_n & 15u) == 0u && give_up(ws, call, _t0)) break; } \
        } BAR(); } } while (0)
    for (int c = 0; c < NC - 1 - NR; ++c) {
        const int t = 4 * c;
        WAIT_VM(8); WAIT_LGKM0(); BAR();
        if (BIG || !FGALL) GATE(c + 1); else FGATE(c + 1);
        STEP(t + 4, 0u, a1, b1, 1u, a0, b0);
        WAIT_VM(8); WAIT_LGKM0(); BAR();
        if (wave == 0) SNAP(c + 2);
        STEP(t + 5, 1u, a0, b0, 2u, a1, b1);
        if (wave == 0) WAIT_VM(9); else WAIT_VM(8);
        WAIT_LGKM0(); BAR();
        STEP(t + 6, 2u, a1, b1, 3u, a0, b0);
        if (wave == 0) WAIT_VM(9); else WAIT_VM(8);
        WAIT_LGKM0(); BAR();
        STEP(t + 7, 3u, a0, b0, 0u, a1, b1);
        if (BIG && c == 2 * HU - 1 && headt) {
#pragma unroll
            for (int m = 0; m < 4; ++m) {
                const u32 r = (u32)(wm * 64 + m * 16 + (lane & 15)) * 4u;
                const int d = (int)*(const LAS signed char*)(lds + L_RED + r) - (int)*(const LAS signed char*)(lds + L_SC + r);
#pragma unroll
                for (int n = 0; n < NB; ++n)
#pragma unroll
                    for (int j = 0; j < 4; ++j) acc[m][n][j] = __builtin_amdgcn_ldexpf(acc[m][n][j], d);
            }
        }
    }
#define TS(i) do { } while (0)
#define FB(j, k, b) ((j) ? bt[(j) ? 4 * (j) - 4 + (k) : 0] : (b))
#pragma unroll
    for (int j = 0; j < NR; ++j) {
        const int c = NC - 1 - NR + j, t = 4 * c;
        if (j) WAIT_VM(4); else WAIT_VM(8);
        WAIT_LGKM0(); BAR();
        TS(2 * j);
        FGATE(c + 1);
        TS(2 * j + 1);
        STEPX(t + 4, 0u, a1, b1, 1u, a0, FB(j, 0, b0), false, !j);
        if (seen) { } else if (j) WAIT_VM(4); else WAIT_VM(6);
        WAIT_LGKM0(); BAR();
        if (wave == 0) SNAP(c + 2);
        STEPX(t + 5, 1u, a0, b0, 2u, a1, FB(j, 1, b1), false, !j);
        if (seen) { } else if (wave == 0) WAIT_VM(5); else WAIT_VM(4);
        WAIT_LGKM0(); BAR();
        STEPX(t + 6, 2u, a1, b1, 3u, a0, FB(j, 2, b0), false, !j);
        if (wave == 0) WAIT_VM(5); else WAIT_VM(4);
        WAIT_LGKM0(); BAR();
        STEPX(t + 7, 3u, a0, b0, 0u, a1, FB(j, 3, b1), false, false);
    }
#define DSTEP(k, n) do { if ((k) > HINT && hint && wave == 0) WAIT_VM((n) + 1); else WAIT_VM(n); WAIT_LGKM0(); BAR(); if ((k) == HINT && hint) YFLAG(); } while (0)
    TS(8);
    DSTEP(0, NR ? 4 : 8);
    TS(9);
    READX(a1, b1, 1u, !NR); MMA(a0, FB(NR, 0, b0));
    DSTEP(1, NR ? 2 : 4);
    READX(a0, b0, 2u, !NR); MMA(a1, FB(NR, 1, b1));
    DSTEP(2, 0);
    TS(10);
    READX(a1, b1, 3u, !NR); MMA(a0, FB(NR, 2, b0));
    WAIT_LGKM0(); BAR();
    if (HINT == 3 && hint) YFLAG();
    u32 npo = OOB; u32x4 nv = {0u, 0u, 0u, 0u};
    if (BIG && nx.slot >= 0) { npo = start_off(rank, nx, nbig, par, lane, wave); nv = __builtin_amdgcn_raw_buffer_load_b128(rws, npo, 0, AUX_SYS); }
    MMA(a1, FB(NR, 3, b1));
#undef DSTEP
#undef FB
#undef FGATE
#undef FS
#undef FP
    if (wave == 0) STAMP(ws, stamp0 + 1);
#pragma unroll
    for (int m = 0; m < 4; ++m)
#pragma unroll
        for (int n = 0; n < NB; ++n) {
            const u32 r = (u32)(wm * 64 + m * 16 + (lane & 15)), pc = (u32)(wn * (BT / 8) + n * 4 + (lane >> 4));
            *(LAS f32x4*)(lds + (u32)ks * AREA + r * RB + ((pc ^ (r & 7u)) << 4)) = acc[m][n];
        }
    WAIT_LGKM0(); BAR();
    if (MODE == 1 && LSL) __builtin_amdgcn_s_sleep(LSL);
    const LAS int* sc = (const LAS int*)(lds + L_SC);
    const i32x4 eb = *(const LAS i32x4*)(sc + 128 + col0 + pc * 4u);
    const bool bad = *(const LAS u32*)(lds + L_SNAP) == call;
    if (BIG) { WAIT_VM_TRACKED(0); const bool ok = (int)npo < 0 || all4_ge(nv, call); if (lane == 0) okw[wave] = ok ? 0u : 1u; }
#pragma unroll
    for (u32 it = 0; it < (BIG ? 8u : 2u); ++it) {
        const u32 r = (u32)wave * (BT / 8u) + ((BIG && EROT ? (it + (blockIdx.x >> 3)) & 7u : it) * 64u + (u32)lane) / PPR;
        const u32 lo = r * RB + ((pc ^ (r & 7u)) << 4);
        f32x4 s = *(const LAS f32x4*)(lds + lo) + *(const LAS f32x4*)(lds + AREA + lo);
        if (!BIG) s += *(const LAS f32x4*)(lds + 2u * AREA + lo) + *(const LAS f32x4*)(lds + 3u * AREA + lo);
        const int ea = (int)(signed char)sc[row0 + r];
#pragma unroll
        for (int j = 0; j < 4; ++j) s[j] = bad ? __builtin_nanf("") : __builtin_amdgcn_ldexpf(s[j], ea + eb[j]);
        u32x4 o = __builtin_bit_cast(u32x4, s);
        if (hint) {
#pragma unroll
            for (int j = 0; j < 4; ++j) o[j] = o[j] ? o[j] : 0x80000000u;
        }
        __builtin_amdgcn_raw_buffer_store_b128(o, ro, obase + r * ostr, 0, AUX_SC1);
        if (BIG && EPW && it < 7u && (first || EALL)) WAIT_VM(EPW ? EPW - 1 : 0);
        if (BIG && EPS && it < 7u && first) __builtin_amdgcn_s_sleep(EPS);
    }
    if (remote && BIG) pend = p.ws[b_src] + OFF_YR + (u32)(a * 64 + slot) * 16u + (u32)(ct & 1) * 8u;
    if (wave == 0) STAMP(ws, stamp0 + 2);
#undef TS
#undef GLDS16
#undef SCHED
#undef SB
#undef STEP
#undef STEPX
#undef READX
#undef VLOAD
#undef VREAD
#undef VSTEP
#undef LOADT
#undef SNAP
#undef READF
#undef MMA
#undef GATE
#undef YFLAG
#undef PEND
#undef GPTR
}

constexpr u32 NSK = DELTA - (DCUT ? 1u : 0u);
constexpr u32 NPOS_INTRA = 96u - HU - NSK, NPOS_CROSS = 97u - HU + DELTA;
constexpr u32 FAN = 0x20000000u;
struct Desc { u32 src, dst, gate, flag; };
struct Stream {
    bool cross; int rank, g, a, j; u32 pb, sa;
    __device__ __forceinline__ int yslot(u32 k) const {
        if (k < 6u) return j * 6 + (int)k;
        if (k < 9u) return ((j + (int)k - 5) & 3) * 6;
        if (k < 12u) return 24 + 2 * ((a + (int)k - 8) & 3);
        if (k < 18u) return ((j + 1 + (((int)k - 12) >> 1)) & 3) * 6 + 1 + ((int)k & 1);
        if (k < 21u) return 25 + 2 * ((a + (int)k - 17) & 3);
        if (k < 27u) return ((j + 1 + (((int)k - 21) >> 1)) & 3) * 6 + 3 + (((int)k - 21) & 1);
        return ((j + 1 + (int)(k % 3u)) & 3) * 6 + 5;
    }
    __device__ __forceinline__ u32 single(u32 slot, u32 k) const { return pb + slot * PANEL + k * UNIT; }
    __device__ __forceinline__ u32 pair(u32 slot, u32 k) const { return pb + slot * PANEL + k * 65536u; }
    __device__ __forceinline__ u32 half(u32 slot, u32 h, u32 k) const { return pb + slot * PANEL + k * 2u * UNIT + h * 8192u; }
    __device__ __forceinline__ Desc at(u32 pos) const {
        Desc d;
        const u32 s = (u32)j * 6u, M8 = 7u << 20, PAIR = 1u << 28, HALF = 4u << 28, ONE = M8 + (6u << 28);
        u32 u, fsrc = (u32)((cross ? 1 - g : g) * 4 + j);
        if (!cross) {
            if (pos >= NPOS_INTRA) pos = NPOS_INTRA - 1u;
            u = pos + HU;
            if (u >= RU0) u += NSK;
            if (u < 80u) {
                d.src = u < 16u ? single(s, u) : u < 48u ? pair(s + 1u, u - 16u) : pair(s + 3u, u - 48u);
                d.gate = sa + s * 512u + (u < ROWU ? 0u : u < 16u ? ONE : u < 48u ? 512u + PAIR + M8 : 1536u + PAIR + M8);
            } else {
                const u32 e = 95u - u, f = e - LEAD;
                const u32 h = e < LEAD || (e < 16u - LEAD && (f & 1u)) ? 1u : 0u;
                const u32 k = e < LEAD ? 7u - e : e >= 16u - LEAD ? 15u - e : h ? 7u - LEAD - (f >> 1) : 7u - (f >> 1);
                u = 80u + h * 8u + k;
                d.src = half(s + 5u, h, k);
                d.gate = sa + (s + 5u) * 512u + h * 256u + M8 + HALF;
            }
            d.dst = d.src;
            d.flag = OFF_LF + (fsrc * NUNITS + u) * 128u;
        } else {
            if (pos >= NPOS_CROSS) pos = NPOS_CROSS - 1u;
            pos += HU;
            const u32 hop = j == a ? HOP1 : HOP2;
            u = pos < hop ? pos : pos - DELTA;
            if (pos - hop < DELTA) {
                const u32 k = pos - hop;
                if (j == a) { d.src = pair(s + 3u, RU0 - 48u + k); d.dst = pair(RELAY, k); d.gate = sa + 34u * 512u + M8 + PAIR;
                              d.flag = FAN + OFF_RF + ((u32)a * 4u + k) * 128u; }
                else { d.src = pair(RELAY, k); d.dst = pair(s + 3u, RU0 - 48u + k); d.gate = OFF_RF + ((u32)j * 4u + k) * 128u + (5u << 28);
                       d.flag = OFF_LF + ((fsrc ^ 4u) * NUNITS + RU0 + k) * 128u; }
            } else if (u < 64u) {
                d.src = u < 16u ? single(32u, u) : u < 48u ? pair((u32)(ASLOTS + 2 * a), u - 16u) : single(33u, u - 48u);
                d.dst = u < 16u ? single((u32)(24 + 2 * j), u) : u < 48u ? pair((u32)(ASLOTS + 8 + 2 * j), u - 16u) : single((u32)(25 + 2 * j), u - 48u);
                d.gate = u < 16u ? sa + (u32)(24 + 2 * j) * 512u + (u < ROWU ? 0u : ONE) : u < 48u ? OFF_WS + (u32)j * 256u + (1u << 20) + (3u << 28) : sa + (u32)(25 + 2 * j) * 512u + (R7C ? ONE : 0u);
                d.flag = OFF_LF + (fsrc * NUNITS + u) * 128u;
            } else {
                const u32 k = u - 64u;
                const int ys = yslot(k);
                const u32 h = k >= 30u ? 1u : 0u;
                d.src = OFF_YST + (u32)(a * 32 + ys) * UNIT + h * 65536u;
                d.dst = ((u32)slot_row((int)fsrc, ys) + h * 64u) * (NLOC * 4u) + (u32)j * 1024u;
                d.gate = OFF_YR + ((u32)(j * 64 + ys) + h * 32u) * 16u + ((k >= 27u ? 10u : 2u) << 28);
                d.flag = NOFLAG;
            }
        }
        return d;
    }
};
struct Tables { Desc lo, hi; };
__device__ __forceinline__ u32 tab(u32 t0, u32 t1, u32 i) {
    const u32 x = (u32)__builtin_amdgcn_readlane((int)t0, (int)(i & 63u)), y = (u32)__builtin_amdgcn_readlane((int)t1, (int)(i & 63u));
    return i < 64u ? x : y;
}

#define PSLEEP0 10
#define PSLEEP 4
#define NACT 32
__device__ __forceinline__ void pull_stream(const Params& p, LAS char* lds, const Tables& T, bool cross, u32 cw, char* src, gchar* fan, u32 calln, u32 call, int lane, int wave, int hw) {
    char* ws = p.me;
    const rsrc_t rl = mk_rsrc(ws, WS_NEED), rs = mk_rsrc(src, WS_NEED), rout = mk_rsrc(p.out, IN_BYTES);
    LAS char* pl = lds + (u32)hw * 1024u;
    const u32 pla = (u32)(size_t)pl + (u32)lane * 4u;
    const u32 l16 = (u32)lane * 16u, l4r = (u32)(lane >> 5) * IMG + (l16 & 511u);
    const u32 l0 = lane == 0 ? 0u : OOB;
    const u32 co = OFF_CL + (u32)wave * 128u + l0;
    const u32 NPIECES = (cross ? NPOS_CROSS : NPOS_INTRA) * 32u;
    const u32 skip = !DCUT ? 0u : cross ? (u32)DCUT : 32u - DCUT;
    const u32 qcut = cross ? ((wave == 3 ? HOP1 : HOP2) + DELTA - HU) * 32u - DCUT : (RU0 - HU) * 32u;
    const u32 cbase = 2u * NACT - (calln - 1u) * (NPIECES - skip);
    u32x4 v0 = {0u, 0u, 0u, 0u}, v1 = v0, v2 = v0, v3;
    u32 q = OOB, so = OOB, ss = 1024u;
    u32 c0 = cw, c1 = cw + NACT;
    u32 pf = NOFLAG;
    bool claimed = false, dead = false;
    int cl = 0;
#define KIND(q) (tab(T.lo.gate, T.hi.gate, (q) >> 5) >> 28)
#define POFF(q, kd) (((kd) & 1u) ? (((q) >> 4) & 1u) * PANEL + ((q) & 15u) * 4096u : (kd) == 4u ? (((q) >> 1) & 15u) * IMG + ((q) & 1u) * 4096u : ((q) & 31u) * (((kd) & 8u) ? 2048u : (kd) ? 4096u : 512u))
#define GBASE(q) ({ const u32 _g = tab(T.lo.gate, T.hi.gate, (q) >> 5), _k = _g >> 28; \
        const u32 _ix = (((_k & 1u) ? ((q) >> 2) & 4u : 0u) + ((q) & (_k == 4u ? 1u : 3u))) << (_k == 3u ? 5 : 7); \
        (_g & 0xfffffu) + ((_k & 7u) == 2u ? 0u : _k == 5u ? ((q) & 28u) * 4u : _k == 0u ? ((q) & 31u) * 16u : _ix); })
#define GOFF(q) (GBASE(q) + (l16 & (tab(T.lo.gate, T.hi.gate, (q) >> 5) >> 16)))
#define PFLAG() st_sys_g((pf & FAN) ? fan + (pf - FAN) : (gchar*)ws + (pf >= NOFLAG ? OFF_CTR + cw * 64u + 8u + (u32)hw * 4u : pf), call)
#define PLD(u, o) v##u = __builtin_amdgcn_raw_buffer_load_b128(rs, (o) + (u) * ss, 0, AUX_SYS)
#define PSTA(u, o) do { asm volatile("" : "+v"(v##u)); \
        __builtin_amdgcn_raw_buffer_store_b128(v##u | (dead ? 0x7fc00000u : 0u), rd, o, 0, AUX_SC1); } while (0)
#define PST(u) PSTA(u, d + (u) * ds)
#define PSTAMP(i) do { } while (0)
#define GPOLL(i, sl) do { __builtin_amdgcn_global_load_lds((const u32*)gp, (LAS u32*)(pl + (i) * 256u), 4, 0, AUX_SYS); __builtin_amdgcn_s_sleep(sl); \
        if (DRY && c0 < 32u) __builtin_amdgcn_s_sleep(PSLEEP0 - PSLEEP); } while (0)
#define GSTEP(i, sl) { GPOLL(((i) + 3) & 3, sl); WAIT_VM_TRACKED(3); u32 _w; \
            asm volatile("ds_read_b32 %0, %1 offset:%2\n\ts_waitcnt lgkmcnt(0)" : "=v"(_w) : "v"(pla), "n"((i) * 256) : "memory"); \
            if (__all((int)(_w - call) >= 0)) break; }
#define WATCH(sl) do { \
        const char* gp = ws + GBASE(c0) + ((u32)lane * 4u & ((tab(T.lo.gate, T.hi.gate, c0 >> 5) >> 16 & 0x3f0u) | 12u)); \
        const u64 t0 = __builtin_amdgcn_s_memrealtime(); \
        GPOLL(0, sl); GPOLL(1, sl); GPOLL(2, sl); \
        for (u32 n = 1;; ++n) { \
            GSTEP(0, sl) GSTEP(1, sl) GSTEP(2, sl) GSTEP(3, sl) \
            if ((n & 15u) == 0u && give_up(ws, call, t0)) { dead = true; break; } \
        } } while (0)
    if (!DRY) WATCH(PSLEEP0);
    const u32 g0 = DRY ? call - 1u : call;
    u32x4 gv = {g0, g0, g0, g0};
    bool dry = DRY;
    u32 d = OOB, ds = 1024u, so2 = OOB, d2 = OOB; bool y = false, val = false;
#define VALID(u) __all(v##u[0] != 0u && v##u[1] != 0u && v##u[2] != 0u && v##u[3] != 0u)
#define RETRY(u) do { if (HINT >= 0 && val && !dead && !VALID(u)) { \
        const u64 t0 = __builtin_amdgcn_s_memrealtime(); u32 n = 0; \
        do { PLD(u, so); NRETRY; \
            if ((++n & 15u) == 0u && give_up(ws, call, t0)) dead = true; \
            WAIT_VM_TRACKED(0); \
        } while (!dead && !VALID(u)); } } while (0)
#define NRETRY do { } while (0)
    do {
        const rsrc_t rd = y ? rout : rl;
        WAIT_VM_TRACKED(5); PLD(2, so2); RETRY(0); PST(0);
        if (EFL & 1) { WAIT_VM_TRACKED(5); PFLAG(); WAIT_VM_TRACKED(5); PLD(3, so2); }
        else { WAIT_VM_TRACKED(4); PLD(3, so2); PFLAG(); }
        RETRY(1); PST(1);
        WAIT_VM_TRACKED(5);
        bool open = dead || all4_ge(gv, call);
        if (q == OOB && !open && !dry) { WATCH(PSLEEP); open = true; }
        dry = false;
        pf = q != OOB ? tab(T.lo.flag, T.hi.flag, q >> 5) + (q & 31u) * 4u : NOFLAG;
        if (claimed) { c1 = (u32)__builtin_amdgcn_readfirstlane(cl) + cbase; if (c1 >= qcut) c1 += skip; if (c1 > NPIECES + 127u) c1 = NPIECES + 127u; }
        claimed = open && c0 < NPIECES;
        q = claimed ? c0 : OOB; if (claimed) c0 = c1;
        const u32 qq = claimed ? q : 0u;
        const u32 kn = claimed ? KIND(qq) : 1u, po = POFF(qq, kn), lo = kn ? l16 : l4r;
        so = claimed ? tab(T.lo.src, T.hi.src, qq >> 5) + po + lo : OOB;
        ss = kn ? 1024u : 2u * IMG;
        y = (kn & 7u) == 2u;
        u32 dn = claimed ? tab(T.lo.dst, T.hi.dst, qq >> 5) + (y ? po * 4u : po) + lo : OOB;
        u32 go = GOFF(c0), ao = claimed ? co : OOB, so1 = so + ss, dx2 = d2 + 2u * ds, dx3 = d2 + 3u * ds;
        if (claimed) PSTAMP(q >> 5);
        asm volatile("" : "+v"(so), "+v"(dn), "+v"(go), "+v"(ao), "+v"(so1), "+v"(dx2), "+v"(dx3));
        WAIT_VM_TRACKED(4);
        v0 = __builtin_amdgcn_raw_buffer_load_b128(rs, so, 0, AUX_SYS);
        PSTA(2, dx2);
        WAIT_VM_TRACKED((EFL & 1) ? 3 : 4);
        PSTA(3, dx3);
        v1 = __builtin_amdgcn_raw_buffer_load_b128(rs, so1, 0, AUX_SYS);
        cl = __builtin_amdgcn_raw_ptr_buffer_atomic_add_i32(1, rl, ao, 0, AUX_SC1);
        gv = __builtin_amdgcn_raw_buffer_load_b128(rl, go, 0, AUX_SYS);
        d = dn; ds = y ? NLOC * 4u : ss;
        val = (kn & 8u) != 0u;
        so2 = val ? OOB : so; d2 = val ? OOB : dn;
    } while (c0 < NPIECES || (!LASTP && q != OOB));
    if (LASTP) {
        const rsrc_t rd = y ? rout : rl;
        PLD(2, so2); PLD(3, so2);
        WAIT_VM_TRACKED(7); RETRY(0); PST(0);
        if (EFL & 2) { WAIT_VM_TRACKED(6); PFLAG(); WAIT_VM_TRACKED(6); } else { WAIT_VM_TRACKED(5); PFLAG(); }
        RETRY(1); PST(1);
        pf = q != OOB ? tab(T.lo.flag, T.hi.flag, q >> 5) + (q & 31u) * 4u : NOFLAG;
        WAIT_VM_TRACKED(4); PSTA(2, d2 + 2u * ds);
        WAIT_VM_TRACKED(4); PSTA(3, d2 + 3u * ds);
    }
    if (!ENDQ || pf < NOFLAG) { WAIT_VM_TRACKED(0); PFLAG(); }
    asm volatile("" :: "v"(gv), "v"(cl));
    asm volatile(".Lagk_end:");
    if (hw < 7) STAMP(ws, 2 + wave * 8 + 7);
#undef PSTAMP
#undef GOFF
#undef GBASE
#undef WATCH
#undef GSTEP
#undef GPOLL
#undef POFF
#undef KIND
#undef PST
#undef PSTA
#undef RETRY
#undef VALID
#undef NRETRY
#undef PLD
#undef PFLAG
}

__global__ void __launch_bounds__(512, 2) ag_gemm_kernel(Params p) {
    extern __shared__ __attribute__((aligned(16))) char smem[];
    LAS char* lds = (LAS char*)smem;
    const int tid = threadIdx.x, lane = tid & 63, wave = __builtin_amdgcn_readfirstlane(tid >> 6);
    const int b = blockIdx.x, rank = p.rank, g = rank >> 2, a = rank & 3;
    u64 pc; u32 len;
    asm volatile("s_getpc_b64 %0\n.Lagk_pc:\n\ts_mov_b32 %1, .Lagk_end-.Lagk_pc" : "=s"(pc), "=s"(len));
    const u32 code = __builtin_amdgcn_raw_buffer_load_b32(mk_rsrc((const void*)pc, (int)len > 0 ? len : 0u), (u32)(b >> 3) * 2048u + (u32)tid * 4u, 0, 0);
    char* ws = p.me;
    if (wave == 0) STAMP(ws, 0);
    const u32 prev = ld_agent(ws + OFF_CTR + (u32)b * 64u);
    u32 calln;
    const int lw = LROT ? (wave + (b >> 3) * 2 + (b & 7)) % 7 : wave;
    const rsrc_t rws = mk_rsrc(ws, WS_NEED);

    Tables T = {};
    char* src = ws; gchar* fan = (gchar*)ws;
    u32 hword = 0;
    gchar* rdflag = x_head(p, lds, b < 128 ? b : 640 + b, prev, calln, hword, tid, wave, [&]() {
        if (b < NCOMM) {
            const u32 par = calln & 1u;
            Stream S;
            S.cross = lw >= 3; S.rank = rank; S.g = g; S.a = a; S.j = (a + lw + 1) & 3; S.pb = CTRL_BYTES + par * PAR_BYTES; S.sa = OFF_SA + par * 32768u;
            T.lo = S.at((u32)lane); T.hi = S.at((u32)lane + 64u);
            src = p.peer[lw == 3 ? 7 : lw];
            char *f0 = p.peer[4], *f1 = p.peer[5], *f2 = p.peer[6];
            asm volatile("" : "+s"(src), "+s"(f0), "+s"(f1), "+s"(f2));
            fan = (gchar*)(lane == 0 ? f0 : lane == 1 ? f1 : f2);
        }
    });
    asm volatile("" :: "v"(code));
    const u32 call = calln << 8, par = calln & 1u;
    if (wave == (CTR7 ? 7 : 0) && lane < 2) st_agent(ws + OFF_CTR + (u32)b * 64u + (u32)lane * 4u, lane ? call : calln);
    if (wave == 0 && b >= NCOMM) STAMP(ws, 1);
    if (b < NCOMM) {
        if (wave == 7) {
            WAIT_VM(0);
            if (lane < 4) st_sys_g(rdflag, hword);
        }
        else if (b < NACT) pull_stream(p, lds, T, lw >= 3, (u32)b, src, fan, calln, call, lane, lw, wave);
        __builtin_amdgcn_endpgm();
    }

    {
        const int c = b - NCOMM, strip = c / 7, rg = c - strip * 7;
        const int kt0 = (rg * 128) / 7, kt1 = ((rg + 1) * 128) / 7;
        const rsrc_t rw = mk_rsrc(p.w, IN_BYTES);
        const int q = lane & 7, n4 = lane >> 3;
        if (wave == 7) { WAIT_VM(0); if (lane < 4) st_sys_g(rdflag, hword); }
        f32x4 v[3][8];
#pragma unroll
        for (int i = 0; i < 3; ++i) {
            const int kt = kt0 + wave + 8 * i;
            const u32 base = kt < kt1 ? ((u32)(kt * 64 + q * 8) * NLOC + (u32)(strip * 32 + n4 * 4)) * 4u : OOB;
#pragma unroll
            for (int jj = 0; jj < 8; ++jj) {
                v[i][jj] = __builtin_bit_cast(f32x4, __builtin_amdgcn_raw_buffer_load_b128(rw, base + (u32)jj * (NLOC * 4u), 0, 0));
                if (i * 8 + jj < W_N0) WAIT_VM(W_LD0 - 1); else WAIT_VM(W_LD - 1);
            }
        }
        LAS u32* cm = (LAS u32*)(lds + L_RED + 256u);
        if (tid < 64) cm[tid] = 0u;
        WAIT_LGKM0(); BAR();
        u32 m[4] = {0u, 0u, 0u, 0u};
#pragma unroll
        for (int i = 0; i < 3; ++i)
#pragma unroll
            for (int jj = 0; jj < 8; ++jj)
#pragma unroll
                for (int k = 0; k < 4; ++k) m[k] = umax(m[k], absbits(v[i][jj][k]));
#pragma unroll
        for (int k = 0; k < 4; ++k) {
            m[k] = umax(m[k], (u32)__shfl_xor((int)m[k], 1)); m[k] = umax(m[k], (u32)__shfl_xor((int)m[k], 2)); m[k] = umax(m[k], (u32)__shfl_xor((int)m[k], 4));
        }
        if (q < 4) __hip_atomic_fetch_max(cm + n4 * 4 + q, q == 0 ? m[0] : q == 1 ? m[1] : q == 2 ? m[2] : m[3], __ATOMIC_RELAXED, __HIP_MEMORY_SCOPE_WORKGROUP);
        WAIT_LGKM0(); BAR();
        u64* wx = (u64*)(ws + OFF_WX) + (u32)strip * 256u;
        if (tid < 32) __hip_atomic_store(wx + rg * 32 + tid, ((u64)call << 32) | cm[tid], __ATOMIC_RELAXED, __HIP_MEMORY_SCOPE_AGENT);
        if (wave < 4) {
            const int gi = tid < 224 ? tid : 0;
            const u64 t0 = __builtin_amdgcn_s_memrealtime();
            u64 gv; u32 n = 0;
            for (;;) {
                gv = __hip_atomic_load(wx + gi, __ATOMIC_RELAXED, __HIP_MEMORY_SCOPE_AGENT);
                if (__all((u32)(gv >> 32) == call)) break;
                if ((++n & 15u) == 0u && give_up(ws, call, t0)) break;
                __builtin_amdgcn_s_sleep(1);
            }
            __hip_atomic_fetch_max(cm + 32 + (gi & 31), (u32)gv, __ATOMIC_RELAXED, __HIP_MEMORY_SCOPE_WORKGROUP);
        }
        WAIT_LGKM0(); BAR();
        const int ct = strip >> 2;
        if (rg == 0 && tid < 64) {
            const u32 nin = (u32)((strip & 3) * 32 + (tid & 31));
            const u32 ev = (u32)(exp_of(cm[32 + (tid & 31)]) - 141);
            char* t = tid < 32 ? ws + OFF_SB + (par * BSLOTS + (u32)ct) * 512u : p.ws[(1 - g) * 4 + (strip >> 3)] + OFF_SB + (par * BSLOTS + (u32)(8 + 2 * a + (ct & 1))) * 512u;
            st_sys(t + nin * 4u, ev);
        }
        const u32x4 mx = *(LAS u32x4*)(cm + 32 + n4 * 4);
        const u32 pbase = CTRL_BYTES + par * PAR_BYTES + (u32)(ASLOTS + ct) * PANEL;
#pragma unroll
        for (int k = 0; k < 4; ++k) {
            const float s = scale_of(exp_of(mx[k]));
            const u32 nin = (u32)((strip & 3) * 32 + n4 * 4 + k);
            const u32 ro = nin * 128u + (((u32)q ^ ((nin >> 1) & 7u)) << 4);
#pragma unroll
            for (int i = 0; i < 3; ++i) {
                const int kt = kt0 + wave + 8 * i;
                const u32 off = kt < kt1 ? pbase + (u32)kt * IMG + ro : OOB;
                const f32x4 lo = {v[i][0][k], v[i][1][k], v[i][2][k], v[i][3][k]}, hi = {v[i][4][k], v[i][5][k], v[i][6][k], v[i][7][k]};
                __builtin_amdgcn_raw_buffer_store_b128(cvt8(lo, hi, s), rws, off, 0, AUX_SC1);
                WAIT_VM(W_ST - 1);
            }
        }
        WAIT_VM(0); BAR();
        if (tid < 4 && (!(tid & 1) || rg == 6))
            st_sys((tid < 2 ? ws + OFF_WR + (u32)strip * 32u : p.ws[(1 - g) * 4 + (strip >> 3)] + OFF_WS + (u32)(a * 8 + (strip & 7)) * 32u) + (u32)(rg + (tid & 1)) * 4u, call);
        if (wave == 0) STAMP(ws, 2);
    }
    const int xcd = b & 7, q = (b >> 3) - 4;
    if (xcd >= 4 && q < 24) {
        const int r0 = 128 + (q * 4 + xcd - 4) * 4;
        x_rows(p, lds, r0, par, call, tid, wave);
        x_rows(p, lds, r0 + (r0 < 384 ? 384 : 512), par, call, tid, wave);
        if (wave == 0) STAMP(ws, 3);
    }
    const int qr = q >> 2, qc = q & 3, x4 = xcd & 3;
    const int early = (x4 >> 1) == (xcd < 4 ? ESWAP : 0) ? (qr < 6 ? a * 6 + qr : -1) : (qr < 3 ? ((a + 1 + qr) & 3) * 6 : 24 + 2 * (qr - 3));
    const int peer = ((a + 1 + (qr >> 1)) & 3) * 6;
    auto tile_of = [&](int it) {
        int slot = -1, ct = 4 * x4 + qc, rh = 0, ch = 0;
        if (it == 0) { if (xcd < 4) { slot = early; ct = 4 * (x4 & 1) + qc; } }
        else if (it == 1) {
            if (RMAP && qr == 6 && xcd >= 2 && xcd < 6) { slot = ((a + 2) & 3) * 6 + (xcd >> 1); ct = 4 * (xcd & 1) + qc; }
            else if (xcd < 4) { if (qr < 6 && !(RMAP && xcd < 2 && qr >> 1 == 1)) slot = peer + 1 + (qr & 1); }
            else { slot = early; ct = 8 + 4 * (x4 & 1) + qc; }
        }
        else if (it == 2) {
            if (qr == 6) { slot = 25 + (xcd & 6); ct = 8 + 4 * (xcd & 1) + qc; }
            else if (xcd >= 4) slot = peer + 3 + (qr & 1);
            else if (xcd < 2 && qr < 4) slot = 25 + 2 * qr;
        } else if (it == 3) {
            int pi = -1; ch = qc & 1;
            const int qs = !RMAP && QSWAP && xcd >= 4 && (qr & 5) == 4 ? qr ^ 2 : qr;
            if (RMAP && xcd < 2 && qr >> 1 == 1) { pi = qr - 1; rh = 1; ct = 12 + 2 * xcd + (qc >> 1); }
            else if (xcd < 2) { pi = qr - 4; rh = 1; ct = 8 + 2 * xcd + (qc >> 1); }
            else if (qs == 6) { pi = RMAP && xcd >= 4 ? -1 : (xcd - 2) >> 1; rh = 1; ct = 12 + 2 * (xcd & 1) + (qc >> 1); }
            else if (xcd < 4) { pi = qr >> 1; ct = 4 * xcd + 2 * (qr & 1) + (qc >> 1); }
            else { pi = qs >> 1; rh = qs & 1; ct = 2 * x4 + (qc >> 1); }
            if (pi >= 0) slot = ((a + 1 + pi) & 3) * 6 + 5;
        }
        if (slot >= 24 && ct >= 8 && (slot - 24) >> 1 == (ct - 8) >> 1) { slot = 32 + (slot & 1); ct -= 8; }
        return Tile{slot, ct, rh, ch};
    };
    char* pend = nullptr;
    int it = 0;
    Tile tl = tile_of(0);
    while (tl.slot < 0) tl = tile_of(++it);
    for (bool first = true; it < 3; first = false) {
        int nt = it + 1;
        Tile nx = tile_of(nt);
        while (nx.slot < 0 && nt < 3) nx = tile_of(++nt);
        gemm_tile<0>(p, lds, tl, first, nx, nt < 3, par, call, tid, lane, wave, 4 + it * 3, pend);
        it = nt; tl = nx;
    }
    if (tl.slot < 0) {
        if (pend) { WAIT_VM(0); BAR(); if (tid < 2) st_sys(pend + (u32)tid * 4u, call); }
        __builtin_amdgcn_endpgm();
    }
    if (tl.ct < 8) {
        gemm_tile<1>(p, lds, tl, false, tl, false, par, call, tid, lane, wave, 13, pend);
        __builtin_amdgcn_endpgm();
    }
    gemm_tile<2>(p, lds, tl, false, tl, false, par, call, tid, lane, wave, 13, pend);
}
}

extern "C" void kernel_launch(int rank, int nranks, ncclComm_t comm,
                              hipStream_t stream,
                              void* const* d_in, const int* in_sizes, int n_in,
                              void* d_out, int out_size,
                              const int* in_dtypes, int out_dtype,
                              void* const* d_out_all,
                              void* const* d_ws_all, size_t ws_bytes) {
    (void)comm; (void)in_dtypes; (void)out_dtype; (void)d_out_all;
    if (nranks != 8 || (unsigned)rank >= 8u || n_in < 2 || ws_bytes < (size_t)agk::WS_NEED || in_sizes[0] != 1024 * 8192 ||
        in_sizes[1] != 8192 * 1024 || out_size != 8192 * 1024) {
        fprintf(stderr, "kernel_launch: unexpected problem shape\n");
        return;
    }
    static bool ready[8] = {false, false, false, false, false, false, false, false};
    if (!ready[rank]) {
        HIP_CHECK(hipFuncSetAttribute((const void*)agk::ag_gemm_kernel, hipFuncAttributeMaxDynamicSharedMemorySize, (int)agk::LDS_BYTES));
        ready[rank] = true;
    }
    agk::Params p;
    p.x = (const float*)d_in[0]; p.w = (const float*)d_in[1]; p.rank = rank; p.pad = 0;
    const size_t wo = WEND ? (ws_bytes - agk::WS_NEED) & ~(size_t)0x1fffff : 0;
    p.me = (char*)d_ws_all[rank] + wo; p.out = (float*)d_out;
    for (int i = 0; i < 8; ++i) { p.ws[i] = (char*)d_ws_all[i] + wo; p.peer[i] = (char*)d_ws_all[((rank ^ i) & 4) + ((rank + 1 + i) & 3)] + wo; }
    hipLaunchKernelGGL(agk::ag_gemm_kernel, dim3(agk::NWG), dim3(512), agk::LDS_BYTES, stream, p);
    HIP_CHECK(hipGetLastError());
}
```

```cpp
#include <hip/hip_runtime.h>
#include <rccl/rccl.h>
#include <cstdio>

#define HIP_CHECK(expr) do { hipError_t _e = (expr); if (_e != hipSuccess) { \
    fprintf(stderr, "HIP error %s at %s:%d\n", hipGetErrorString(_e), __FILE__, __LINE__); \
    return; } } while (0)

namespace agk {
typedef unsigned u32;
typedef unsigned long long u64;
typedef u32 u32x4 __attribute__((ext_vector_type(4)));
typedef int i32x4 __attribute__((ext_vector_type(4)));
typedef float f32x4 __attribute__((ext_vector_type(4)));
typedef float f32x2 __attribute__((ext_vector_type(2)));
typedef _Float16 f16x2 __attribute__((ext_vector_type(2)));
typedef _Float16 f16x8 __attribute__((ext_vector_type(8)));
#define LAS __attribute__((address_space(3)))
typedef __amdgpu_buffer_rsrc_t rsrc_t;
typedef __attribute__((address_space(1))) char gchar;

constexpr int KDIM = 8192, NLOC = 1024, NKT = 128;
constexpr u32 IMG = 16384u;
constexpr u32 PANEL = NKT * IMG;
constexpr u32 UNIT = 131072u;
constexpr int ASLOTS = 34, BSLOTS = 16;
#define HU 3
#define HWIN 8
#define SCH 2
#define DRY 1
#define NF 4
#define NFL 2
#define RU0 68
#define HOP1 56
#define HOP2 64
#define LEAD 2
#define QSWAP 1
#define RMAP 0
#define FGALL 0
#define LASTP 1
#define ENDQ 1
#define ROWU 6
#define R7C 1
#define EPW 1
#define EPS 0
#define EROT 1
#define EALL 0
#define LSL 0
#define Y4 4
#define EFL 3
#define LROT 1
#define WEND 1
#define FSLEEP 3
#define CTR7 1
#define ESWAP 1
#define HSTEP 1
#define HK 8
#define XDLY 30
#define HINT 1
#define DCUT 0
#define DELTA 3
constexpr u32 CTRL_BYTES = 1u << 20;
constexpr u32 RELAY = ASLOTS + BSLOTS;
constexpr u32 PAR_BYTES = (RELAY + 2u) * PANEL;
constexpr u32 OFF_YST = CTRL_BYTES + 2u * PAR_BYTES;
constexpr u32 WS_NEED = OFF_YST + 4u * 32u * UNIT;
constexpr u32 IN_BYTES = 1024u * 8192u * 4u;
constexpr u32 OFF_CTR = 0;
constexpr u32 OFF_TMO = 16384;
constexpr u32 OFF_CL = 20480;
constexpr u32 OFF_LF = 524288;
constexpr u32 OFF_RD = 655360;
constexpr u32 OFF_YR = 24576;
constexpr u32 OFF_RF = 28672;
constexpr u32 OFF_WR = 32768;
constexpr u32 OFF_WS = 33792;
constexpr u32 OFF_WX = 36864;
constexpr u32 OFF_SA = 131072;
constexpr u32 OFF_SB = 196608;
constexpr u32 OFF_DBG = 262144;
constexpr int NWG = 256, NCOMM = 32;
constexpr u32 NUNITS = 96u;
constexpr u32 OOB = 0x80000000u, NOFLAG = 0x40000000u;
constexpr u64 DEADLINE = 30000000ull;
constexpr u32 STAGE = 32768u;
constexpr u32 L_SNAP = 4u * STAGE;
constexpr u32 L_SC = L_SNAP + 512u;
constexpr u32 L_RED = L_SC + 1024u;
constexpr u32 LDS_BYTES = L_RED + 2048u;

struct Params {
    const float* x; char* me; int rank, pad;
    char* peer[8];
    const float* w; float* out;
    char* ws[8];
};

#define AUX_SC1 16
#define AUX_SYS 17
#define GL_AUX 16
#define W_LD 2
#define W_LD0 1
#define W_N0 0
#define W_ST 2
#define X_LD 4
#define X_ST 2
#define WAIT_VM(n) asm volatile("s_waitcnt vmcnt(%0)" :: "n"(n) : "memory")
#define WAIT_LGKM0() do { asm volatile("" ::: "memory"); __builtin_amdgcn_s_waitcnt(15 | (7 << 4) | (0 << 8) | (3 << 14)); asm volatile("" ::: "memory"); } while (0)
#define BAR() do { asm volatile("" ::: "memory"); __builtin_amdgcn_s_barrier(); asm volatile("" ::: "memory"); } while (0)
#define WAIT_VM_TRACKED(n) do { asm volatile("" ::: "memory"); __builtin_amdgcn_s_waitcnt(((n) & 15) | (7 << 4) | (15 << 8) | (((n) >> 4) << 14)); asm volatile("" ::: "memory"); } while (0)

__device__ __forceinline__ rsrc_t mk_rsrc(const void* p, u32 bytes) { return __builtin_amdgcn_make_buffer_rsrc((void*)p, 0, bytes, 0x00020000); }
__device__ __forceinline__ u32 ld_agent(const void* p) { return __hip_atomic_load((const u32*)p, __ATOMIC_RELAXED, __HIP_MEMORY_SCOPE_AGENT); }
__device__ __forceinline__ u32 ld_sys(const void* p) { return __hip_atomic_load((const u32*)p, __ATOMIC_RELAXED, __HIP_MEMORY_SCOPE_SYSTEM); }
__device__ __forceinline__ void st_agent(void* p, u32 v) { __hip_atomic_store((u32*)p, v, __ATOMIC_RELAXED, __HIP_MEMORY_SCOPE_AGENT); }
__device__ __forceinline__ void st_sys_g(gchar* p, u32 v) { __hip_atomic_store((__attribute__((address_space(1))) u32*)p, v, __ATOMIC_RELAXED, __HIP_MEMORY_SCOPE_SYSTEM); }
__device__ __forceinline__ void st_sys(void* p, u32 v) { __hip_atomic_store((u32*)p, v, __ATOMIC_RELAXED, __HIP_MEMORY_SCOPE_SYSTEM); }

#define STAMP(ws, idx) do { } while (0)

__device__ __forceinline__ bool give_up(char* ws, u32 call, u64 t0) {
    const u32 t = __builtin_amdgcn_readfirstlane(ld_agent(ws + OFF_TMO));
    if (t == call || __builtin_amdgcn_s_memrealtime() - t0 > DEADLINE) { st_agent(ws + OFF_TMO, call); return true; }
    return false;
}
__device__ __forceinline__ void poll_ge(const char* p, u32 need, char* ws, u32 call) {
    const u64 t0 = __builtin_amdgcn_s_memrealtime();
    u32 n = 0;
    for (;;) {
        const u32 v = ld_sys(p);
        if (__all((int)(v - need) >= 0)) return;
        if ((++n & 15u) == 0u && give_up(ws, call, t0)) return;
        __builtin_amdgcn_s_sleep(2);
    }
}
__device__ __forceinline__ bool all4_ge(u32x4 v, u32 need) {
    return __all((int)(v[0] - need) >= 0 && (int)(v[1] - need) >= 0 && (int)(v[2] - need) >= 0 && (int)(v[3] - need) >= 0);
}
__device__ __forceinline__ void poll4_ge(rsrc_t rs, u32 off, u32 need, char* ws, u32 call) {
    const u64 t0 = __builtin_amdgcn_s_memrealtime();
    u32 n = 0;
    for (;;) {
        asm volatile("" ::: "memory");
        const u32x4 v = __builtin_amdgcn_raw_buffer_load_b128(rs, off, 0, AUX_SYS);
        if (all4_ge(v, need)) return;
        if ((++n & 15u) == 0u && give_up(ws, call, t0)) return;
        __builtin_amdgcn_s_sleep(1);
    }
}

__device__ __forceinline__ u32 umax(u32 a, u32 b) { return a > b ? a : b; }
__device__ __forceinline__ u32 absbits(float f) { return __float_as_uint(f) & 0x7fffffffu; }
__device__ __forceinline__ int exp_of(u32 amaxbits) { int e = (int)(amaxbits >> 23); return e < 15 ? 15 : e; }
__device__ __forceinline__ float scale_of(int e) { return __uint_as_float((u32)(268 - e) << 23); }
__device__ __forceinline__ u32 cvt2(float a, float b, float s) {
    const f32x2 t = (f32x2){a, b} * s;
    return __builtin_bit_cast(u32, __builtin_convertvector(t, f16x2));
}
__device__ __forceinline__ u32x4 cvt8(f32x4 a, f32x4 b, float s) {
    return (u32x4){cvt2(a[0], a[1], s), cvt2(a[2], a[3], s), cvt2(b[0], b[1], s), cvt2(b[2], b[3], s)};
}
__device__ __forceinline__ u32 amax4(u32 m, f32x4 v) {
    asm("v_max3_f32 %0, |%1|, |%2|, %0\n\tv_max3_f32 %0, |%3|, |%4|, %0" : "+v"(m) : "v"(v[0]), "v"(v[1]), "v"(v[2]), "v"(v[3]));
    return m;
}

__device__ __forceinline__ u32 wave_max(u32 m) {
    m = umax(m, (u32)__builtin_amdgcn_update_dpp(0, (int)m, 0xb1, 15, 15, true));
    m = umax(m, (u32)__builtin_amdgcn_update_dpp(0, (int)m, 0x4e, 15, 15, true));
    m = umax(m, (u32)__builtin_amdgcn_update_dpp(0, (int)m, 0x141, 15, 15, true));
    m = umax(m, (u32)__builtin_amdgcn_update_dpp(0, (int)m, 0x140, 15, 15, true));
    return umax(umax((u32)__builtin_amdgcn_readlane((int)m, 0), (u32)__builtin_amdgcn_readlane((int)m, 16)),
                umax((u32)__builtin_amdgcn_readlane((int)m, 32), (u32)__builtin_amdgcn_readlane((int)m, 48)));
}
__device__ __forceinline__ u32 row_word(u32 call, int e) { return call | ((u32)(e - 141) & 255u); }

__device__ __forceinline__ void x_rows(const Params& p, LAS char* lds, int row0, u32 par, u32 call, int tid, int wave) {
    constexpr int NR = 4;
    const int a = p.rank & 3;
    char* ws = p.me;
    const rsrc_t rx = mk_rsrc(p.x, IN_BYTES), rw = mk_rsrc(ws, WS_NEED);
    const u32 pi0 = (u32)(wave * 128 + (tid & 63));
    f32x4 v[NR][4];
#pragma unroll
    for (int r = 0; r < NR; ++r)
#pragma unroll
        for (int u = 0; u < 2; ++u) {
            const u32 off = (u32)(row0 + r) * (KDIM * 4u) + (pi0 + (u32)u * 64u) * 32u;
            v[r][2 * u] = __builtin_bit_cast(f32x4, __builtin_amdgcn_raw_buffer_load_b128(rx, off, 0, 0));
            v[r][2 * u + 1] = __builtin_bit_cast(f32x4, __builtin_amdgcn_raw_buffer_load_b128(rx, off + 16u, 0, 0));
            WAIT_VM(X_LD - 2);
        }
    const int rb = row0 >> 7;
    const u32 slot = rb < 6 ? (u32)(a * 6 + rb) : (u32)(32 + rb - 6);
    const u32 rslot = rb < 6 ? slot : (u32)(24 + 2 * a + rb - 6);
    char* hp[4];
#pragma unroll
    for (int i = 0; i < 4; ++i) hp[i] = rb < 6 ? p.peer[i] : p.peer[4 + i];
    char* fifth = rb < 6 ? p.peer[7] : ws;
    asm volatile("" : "+s"(hp[0]), "+s"(hp[1]), "+s"(hp[2]), "+s"(hp[3]), "+s"(fifth));
    gchar* user = (gchar*)((tid & 4) ? fifth : (tid & 2) ? ((tid & 1) ? hp[3] : hp[2]) : ((tid & 1) ? hp[1] : hp[0]));
    const u32 wr = (u32)(tid & 63) >> 3;
    const bool wsend = wr < (u32)NR && ((tid & 7) < 4 || ((tid & 7) == 4 && rb >= 3 && rb != 5));
    const u32 woff = OFF_SA + ((par * 64u + ((tid & 4) ? (rb < 6 ? (u32)(31 + rb) : slot) : rslot)) * 128u + (((u32)row0 + wr) & 127u)) * 4u;
    u32 word = 0;
    LAS u32* red = (LAS u32*)(lds + L_RED);
#pragma unroll
    for (int r = 0; r < NR; ++r) {
        u32 m = 0;
#pragma unroll
        for (int i = 0; i < 4; ++i) m = amax4(m, v[r][i]);
        m = wave_max(m);
        if ((tid & 63) == 0) red[r * 8 + wave] = m;
    }
    WAIT_LGKM0(); BAR();
#pragma unroll
    for (int r = 0; r < NR; ++r) {
        const u32x4 m0 = *(LAS u32x4*)(red + r * 8), m1 = *(LAS u32x4*)(red + r * 8 + 4);
        const u32 m = umax(umax(umax(m0[0], m0[1]), umax(m0[2], m0[3])), umax(umax(m1[0], m1[1]), umax(m1[2], m1[3])));
        const int e = exp_of(m); const float s = scale_of(e);
        const u32 rin = (u32)((row0 + r) & 127);
        if (wr == (u32)r) word = row_word(call, e);
        const u32 off = CTRL_BYTES + par * PAR_BYTES + rin * 128u + (pi0 >> 3) * IMG + (((pi0 & 7u) ^ ((rin >> 1) & 7u)) << 4);
        __builtin_amdgcn_raw_buffer_store_b128(cvt8(v[r][0], v[r][1], s), rw, off + slot * PANEL, 0, AUX_SC1);
        if (X_ST == 1) WAIT_VM(0);
        __builtin_amdgcn_raw_buffer_store_b128(cvt8(v[r][2], v[r][3], s), rw, off + slot * PANEL + 8u * IMG, 0, AUX_SC1);
        WAIT_VM(X_ST < 2 ? 0 : X_ST - 2);
    }
    WAIT_VM(0); BAR();
    if (wave == 0 && wsend) st_sys_g(user + woff, word);
}

template <class F>
__device__ __forceinline__ gchar* x_head(const Params& p, LAS char* lds, int row, u32 prev, u32& calln, u32& hword, int tid, int wave, F mid) {
    constexpr int NS = HU > 3 ? HU : 3;
    const int lane = tid & 63;
    char* ws = p.me;
    const rsrc_t rx = mk_rsrc(p.x, IN_BYTES), rw = mk_rsrc(ws, WS_NEED);
    LAS u32* red = (LAS u32*)(lds + L_RED);
    if (wave != 7) {
        if (tid < 4) red[tid] = 0u;
        WAIT_LGKM0(); BAR();
        __builtin_amdgcn_s_sleep(XDLY);
    }
    const u32 s0 = wave == 7 ? 0u : (u32)wave, sd = wave == 7 ? 1u : 7u, se = wave == 7 ? (u32)HU : 16u;
    f32x4 v[NS][2];
#pragma unroll
    for (int u = 0; u < NS; ++u) {
        const u32 set = s0 + (u32)u * sd;
        const u32 off = set < se ? (u32)row * (KDIM * 4u) + (set * 64u + (u32)lane) * 32u : OOB;
        v[u][0] = __builtin_bit_cast(f32x4, __builtin_amdgcn_raw_buffer_load_b128(rx, off, 0, 0));
        v[u][1] = __builtin_bit_cast(f32x4, __builtin_amdgcn_raw_buffer_load_b128(rx, off + 16u, 0, 0));
    }
    __builtin_amdgcn_sched_barrier(0);
    if (wave == 7) BAR();
    const bool major = row < 128;
    const int a = p.rank & 3;
    char* hp[4];
#pragma unroll
    for (int i = 0; i < 4; ++i) hp[i] = major ? p.peer[i] : p.peer[4 + i];
    const u32 slot = major ? (u32)(a * 6) : 32u, rslot = major ? slot : (u32)(24 + 2 * a), rin = (u32)row & 127u;
    u32 off = CTRL_BYTES + rin * 128u + ((u32)lane >> 3) * IMG + ((((u32)lane & 7u) ^ ((rin >> 1) & 7u)) << 4);
    u32 m = 0;
    gchar* user = (gchar*)((tid & 4) ? ws : (tid & 2) ? ((tid & 1) ? hp[3] : hp[2]) : ((tid & 1) ? hp[1] : hp[0]));
#define WORD() do { u32 t = 0; \
        if (lane == 0) t = __hip_atomic_fetch_add(red + 2, 1u, __ATOMIC_RELAXED, __HIP_MEMORY_SCOPE_WORKGROUP); \
        if (__builtin_amdgcn_readfirstlane((int)t) == 7 && lane < (major ? 4 : 5)) \
            st_sys_g(user + OFF_SA + (((calln & 1u) * 64u + (lane == 4 ? slot : rslot)) * 128u + rin) * 4u, \
                     row_word(calln << 8, exp_of(__hip_atomic_load(red, __ATOMIC_RELAXED, __HIP_MEMORY_SCOPE_WORKGROUP)))); } while (0)
    if (wave == 7) {
        rsrc_t hr[4];
#pragma unroll
        for (int i = 0; i < 4; ++i) hr[i] = mk_rsrc(hp[i], i == 3 && major ? 0u : WS_NEED);
        u32 o4[4];
#pragma unroll
        for (int i = 0; i < 4; ++i) o4[i] = (lane >> 4) == i ? off + rslot * PANEL : OOB;
        asm volatile("" : "+s"(hr[0]), "+s"(hr[1]), "+s"(hr[2]), "+s"(hr[3]), "+v"(o4[0]), "+v"(o4[1]), "+v"(o4[2]), "+v"(o4[3]), "+v"(m), "+v"(prev));
        calln = (u32)__builtin_amdgcn_readfirstlane(prev) + 1u;
        const u32 so = (calln & 1u) * PAR_BYTES;
#pragma unroll
        for (int u = 0; u < HU; ++u) m = amax4(amax4(m, v[u][0]), v[u][1]);
        m = wave_max(m);
        const int e = exp_of(m); const float s = scale_of(e);
        u32x4 h;
#pragma unroll
        for (int k = 0; k < 4 * HU; ++k) {
            if ((k & 3) == 0) h = cvt8(v[k >> 2][0], v[k >> 2][1], s);
            if (k == 2 && lane == 0) __hip_atomic_fetch_add(red + 1, 1u, __ATOMIC_RELAXED, __HIP_MEMORY_SCOPE_WORKGROUP);
#pragma unroll
            for (int i = 0; i < 4; ++i) {
                if (HSTEP ? i == 0 && 4 * k >= HWIN : 4 * k + i >= HWIN) WAIT_VM(HWIN - (HSTEP ? 4 : 1));
                __builtin_amdgcn_raw_buffer_store_b128(h, hr[i], o4[k & 3], so + (u32)(k >> 2) * 8u * IMG, AUX_SYS);
            }
            asm volatile("s_nop 1" : "+v"(h) :: "memory");
            if (k == HK) WORD();
        }
        hword = row_word(calln << 8, e);
        for (u32 n = 0; n < (1u << 24); ++n)
            if (__builtin_amdgcn_readfirstlane((int)__hip_atomic_load(red + 1, __ATOMIC_RELAXED, __HIP_MEMORY_SCOPE_WORKGROUP)) == 8) break;
    }
    else calln = (u32)__builtin_amdgcn_readfirstlane(prev) + 1u;
    const u32 par = calln & 1u;
    if (wave != 7) {
        mid();
        off += par * PAR_BYTES + slot * PANEL;
#pragma unroll
        for (int u = 0; u < 3; ++u) m = amax4(amax4(m, v[u][0]), v[u][1]);
        m = wave_max(m);
        if (lane == 0) {
            __hip_atomic_fetch_max(red, m, __ATOMIC_RELAXED, __HIP_MEMORY_SCOPE_WORKGROUP);
            __hip_atomic_fetch_add(red + 1, 1u, __ATOMIC_RELAXED, __HIP_MEMORY_SCOPE_WORKGROUP);
        }
        u64 mc;
        for (u32 n = 0; n < (1u << 24); ++n) {
            mc = __hip_atomic_load((LAS u64*)red, __ATOMIC_RELAXED, __HIP_MEMORY_SCOPE_WORKGROUP);
            if ((u32)__builtin_amdgcn_readfirstlane((int)(mc >> 32)) == 8u) break;
        }
        const int e = exp_of((u32)mc); const float s = scale_of(e);
#pragma unroll
        for (int u = 0; u < 3; ++u) {
            const u32 set = s0 + (u32)u * 7u;
            __builtin_amdgcn_raw_buffer_store_b128(cvt8(v[u][0], v[u][1], s), rw, set < 16u ? off + set * 8u * IMG : OOB, 0, AUX_SC1);
        }
        WAIT_VM(0);
        WORD();
    }
#undef WORD
    return user + OFF_RD + ((par * 8u + (u32)p.rank) * 128u + rin) * 4u;
}

__device__ __forceinline__ int slot_row(int rank, int slot) {
    const int g = rank >> 2;
    if (slot < 24) { const int i = slot / 6; return (g * 4 + i) * 1024 + (slot - i * 6) * 128; }
    return (slot < 32 ? (1 - g) * 4 + ((slot - 24) >> 1) : rank) * 1024 + 768 + (slot & 1) * 128;
}

struct Tile { int slot, ct, rh, ch; };
__device__ __forceinline__ void a_stream(int rank, int slot, int rh, int& a_src, u32& a_base, u32& a_sh) {
    const int g = rank >> 2, a = rank & 3;
    if (slot < 24) { const int i = slot / 6, a_rb = slot - i * 6; a_src = i == a ? -1 : g * 4 + i;
        a_base = a_rb == 0 ? 0u : a_rb < 3 ? 16u : a_rb < 5 ? 48u : 80u + (u32)rh * 8u; a_sh = a_rb == 0 ? 1u : 0u; }
    else { a_src = slot < 32 ? (1 - g) * 4 + ((slot - 24) >> 1) : -1; a_base = (slot & 1) ? 48u : 0u; a_sh = 1u; }
}
__device__ __forceinline__ u32 start_off(int rank, const Tile& t, bool big, u32 par, int lane, int wave) {
    int a_src; u32 a_base, a_sh;
    a_stream(rank, t.slot, t.rh, a_src, a_base, a_sh);
    const int b_src = t.ct < 8 ? -1 : (1 - (rank >> 2)) * 4 + ((t.ct - 8) >> 1);
    u32 o = OOB, m = 63u;
    if (wave == 0) { if (a_src >= 0) { o = OFF_LF + ((u32)a_src * NUNITS + (a_base ? a_base : (u32)HU)) * 128u; m = big ? 7u : 3u; } }
    else if (wave == 1) { if (a_src < 0) { o = OFF_SA + (par * 64u + (u32)t.slot) * 512u; m = 31u; } }
    else if (wave == 2) { if (b_src < 0) { o = OFF_WR + (u32)t.ct * 128u; m = 7u; } }
    else if (wave == 3) { if (a_src >= 0 && a_base == 0u) { o = OFF_RD + (par * 8u + (u32)a_src) * 512u; m = 31u; } }
    else if (b_src >= 0 && (!big || wave == 4)) { o = OFF_LF + ((u32)b_src * NUNITS + 16u) * 128u + (u32)(wave - 4) * 1024u; m = big ? 7u : 63u; }
    return o + ((u32)lane & m) * 16u;
}

template <int MODE>
__device__ __forceinline__ void gemm_tile(const Params& p, LAS char* lds, const Tile& tl, bool first, const Tile& nx, bool nbig, u32 par, u32 call, int tid, int lane, int wave, int stamp0, char*& pend) {
    constexpr bool BIG = MODE == 0;
    constexpr int NB = BIG ? 4 : 2, NC = BIG ? 32 : 16, BT = BIG ? 128 : 64, NR = MODE == 2 ? NF : MODE == 1 ? NFL : 0;
    const int rank = p.rank, g = rank >> 2, a = rank & 3;
    const int slot = tl.slot, ct = tl.ct, rh = tl.rh, ch = tl.ch;
    asm volatile("" : "+v"(lane), "+v"(tid));
    char* ws = p.me;
    const rsrc_t rws = mk_rsrc(ws, WS_NEED);
    int a_src; u32 a_base, a_sh;
    a_stream(rank, slot, rh, a_src, a_base, a_sh);
    const int b_src = ct < 8 ? -1 : (1 - g) * 4 + ((ct - 8) >> 1);
    const int src = lane < 32 ? a_src : (BIG ? b_src : -1);
    const u32 gsh = src < 0 ? 31u : lane < 32 && BIG ? a_sh : 0u;
    const char* gptr = ws + (src >= 0 ? OFF_LF + ((u32)src * NUNITS + (lane < 32 ? a_base : 16u)) * 128u + (u32)(lane & (BIG ? 31 : 15)) * 4u : OFF_CTR + blockIdx.x * 64u + 4u);
    const u32 gmin = src >= 0 && lane < 32 && a_base == 0u ? (u32)HU : 0u;
    const bool headt = BIG && a_src >= 0 && a_base == 0u;
#define GPTR(c) (gptr + (umax((u32)(c) >> gsh, gmin) << (BIG ? 7 : 6)))
#define PEND() do { if (pend && wave == 1 && lane < 2) st_sys(pend + (u32)lane * 4u, call); pend = nullptr; } while (0)
    LAS u32* okw = (LAS u32*)(lds + L_RED + 1024u);
    const u32 pb = CTRL_BYTES + par * PAR_BYTES;
    const int ks = BIG ? wave >> 2 : wave >> 1, wm = BIG ? (wave >> 1) & 1 : 0, wn = wave & 1;
    const u32 swz = (u32)((((ks & 1) * 4 + (lane >> 4)) ^ ((lane & 15) >> 1)) << 4);
    f16x8 bt[4 * NR + 1][2];
    WAIT_LGKM0(); BAR();
    u32 po = OOB;
    {
        const u32x4 k0 = *(const LAS u32x4*)okw, k1 = *(const LAS u32x4*)(okw + 4);
        if (first || __builtin_amdgcn_readfirstlane((int)(k0[0] | k0[1] | k0[2] | k0[3] | k1[0] | k1[1] | k1[2] | k1[3]))) {
            if (pend) { WAIT_VM(0); BAR(); PEND(); }
            po = start_off(rank, tl, BIG, par, lane, wave);
            if ((!NR || wave != 0) && __builtin_amdgcn_readfirstlane((int)po) >= 0) poll4_ge(rws, po, call, ws, call);
            WAIT_VM_TRACKED(0); BAR();
        }
    }
    if (!NR && wave == 0) STAMP(ws, stamp0);
    constexpr u32 RB = BT * 4u, PPR = BT / 4u, AREA = BT * RB;
    const u32 pc = (u32)lane & (PPR - 1u);
    const u32 col0 = BIG ? 0u : (u32)ch * 64u, row0 = BIG ? 0u : (u32)rh * 64u;
    const bool remote = BIG ? ct >= 8 : MODE == 2;
    const rsrc_t ro = mk_rsrc(remote ? (const void*)ws : (const void*)p.out, remote ? WS_NEED : IN_BYTES);
    const u32 ostr = remote ? 1024u : NLOC * 4u;
    const u32 obase = (remote ? OFF_YST + (u32)(((ct - 8) >> 1) * 32 + slot) * UNIT + (u32)(ct & 1) * 512u : ((u32)slot_row(rank, slot) * NLOC + (u32)ct * 128u) * 4u) + col0 * 4u + row0 * ostr + pc * 16u;
#define YFLAG() do { if (tid == 0) st_sys(p.ws[b_src] + OFF_YR + (u32)(a * 64 + rh * 32 + slot) * 16u + (u32)(ct & 1) * 8u + (u32)ch * 4u, call); } while (0)
    constexpr bool hint = MODE == 2;
    if (hint) {
        const u32x4 zero = {0u, 0u, 0u, 0u};
#pragma unroll
        for (u32 it = 0; it < 2u; ++it) __builtin_amdgcn_raw_buffer_store_b128(zero, ro, obase + ((u32)wave * (BT / 8u) + (it * 64u + (u32)lane) / PPR) * ostr, 0, AUX_SC1);
    }
    const u32 gA = pb + (u32)slot * PANEL + (BIG ? (u32)wave * 2048u : (u32)rh * 8192u + (u32)wave * 1024u);
    const u32 gB = pb + (u32)(ASLOTS + ct) * PANEL + (BIG ? (u32)wave * 2048u : (u32)ch * 8192u + (u32)wave * 1024u);
    const u32 l16 = (u32)lane * 16u;
    const u32 ldsw = (u32)wave * (BIG ? 2048u : 1024u);
    constexpr u32 GSTEP = BIG ? IMG : 2u * IMG, G2 = BIG ? 1024u : IMG, L2O = BIG ? 1024u : 8192u;
#define SCHED() __builtin_amdgcn_sched_barrier(0)
#define GLDS16(g, l) __builtin_amdgcn_raw_ptr_buffer_load_lds(rws, (LAS void*)(lds + (l)), 16, l16, (g), 0, GL_AUX)
#define LOADT(t, st) do { const u32 _o = (u32)(t) * GSTEP; \
        GLDS16(gA + _o, (st) * STAGE + ldsw); GLDS16(gA + _o + G2, (st) * STAGE + ldsw + L2O); \
        GLDS16(gB + _o, (st) * STAGE + IMG + ldsw); GLDS16(gB + _o + G2, (st) * STAGE + IMG + ldsw + L2O); } while (0)
#define SNAP(c) __builtin_amdgcn_global_load_lds((const u32*)((c) < NC ? GPTR(c) : ws + OFF_TMO), (LAS u32*)(lds + L_SNAP + ((u32)(c) & 1u) * 256u), 4, 0, AUX_SYS)
    const u32 swl = swz + (BIG ? 0u : (u32)(ks >> 1) * 8192u);
    const u32 ard = (u32)(wm * 64 + (lane & 15)) * 128u + swl, brd = IMG + (u32)(wn * (BT / 2) + (lane & 15)) * 128u + swl;
#define READF(fa, fb, st) READX(fa, fb, st, true)
#define READX(fa, fb, st, rb) do { _Pragma("unroll") for (int _i = 0; _i < 4; ++_i) { \
        if (_i < NB && (rb)) fb[_i] = *(const LAS f16x8*)(lds + (st) * STAGE + brd + _i * 2048u); \
        fa[_i] = *(const LAS f16x8*)(lds + (st) * STAGE + ard + _i * 2048u); } } while (0)
#define MMA(fa, fb) do { SCHED(); _Pragma("unroll") for (int _m = 0; _m < 4; ++_m) _Pragma("unroll") for (int _n = 0; _n < NB; ++_n) \
        acc[_m][_n] = __builtin_amdgcn_mfma_f32_16x16x32_f16(fb[_n], fa[_m], acc[_m][_n], 0, 0, 0); SCHED(); } while (0)
#define SB() __builtin_amdgcn_sched_barrier(0)
#define STEP(t, sl, na, nb, sr, fa, fb) STEPX(t, sl, na, nb, sr, fa, fb, true, true)
#define STEPX(t, sl, na, nb, sr, fa, fb, lb, rb) do { const u32 _o = (u32)(t) * GSTEP; \
        _Pragma("unroll") for (int _m = 0; _m < 4; ++_m) { \
            _Pragma("unroll") for (int _n = 0; _n < NB / 2; ++_n) acc[_m][_n] = __builtin_amdgcn_mfma_f32_16x16x32_f16(fb[_n], fa[_m], acc[_m][_n], 0, 0, 0); \
            SB(); \
            if (_m < 2) { _Pragma("unroll") for (int _i = 0; _i < 2; ++_i) { \
                if (_i < NB / 2 && (rb)) nb[_m * (NB / 2) + _i] = *(const LAS f16x8*)(lds + (sr) * STAGE + brd + (_m * (NB / 2) + _i) * 2048u); \
                na[_m * 2 + _i] = *(const LAS f16x8*)(lds + (sr) * STAGE + ard + (_m * 2 + _i) * 2048u); } } \
            if (_m < 2 || (lb)) GLDS16((_m < 2 ? gA : gB) + _o + (_m & 1) * G2, (sl) * STAGE + (_m >> 1) * IMG + ldsw + (_m & 1) * L2O); \
            SB(); \
            _Pragma("unroll") for (int _n = NB / 2; _n < NB; ++_n) acc[_m][_n] = __builtin_amdgcn_mfma_f32_16x16x32_f16(fb[_n], fa[_m], acc[_m][_n], 0, 0, 0); \
        } SB(); } while (0)
    f32x4 acc[4][NB];
#pragma unroll
    for (int m = 0; m < 4; ++m)
#pragma unroll
        for (int n = 0; n < NB; ++n) acc[m][n] = (f32x4){0.f, 0.f, 0.f, 0.f};
    f16x8 a0[4], b0[4], a1[4], b1[4];
#define VLOAD(v) do { const u32 _o = (u32)(4 * (NC - NR) + 2 * (v)) * GSTEP, _l = (u32)(((v) + 2 * NR) & 3) * STAGE + ldsw; \
        GLDS16(gB + _o, _l); GLDS16(gB + _o + G2, _l + L2O); GLDS16(gB + _o + GSTEP, _l + IMG); GLDS16(gB + _o + GSTEP + G2, _l + IMG + L2O); } while (0)
#define VREAD(v) do { _Pragma("unroll") for (int _i = 0; _i < 4; ++_i) \
        bt[NR ? 2 * (v) + (_i >> 1) : 0][_i & 1] = *(const LAS f16x8*)(lds + (u32)(((v) + 2 * NR) & 3) * STAGE + brd - (_i < 2 ? IMG : 0u) + (_i & 1) * 2048u); } while (0)
#define VSTEP(v) do { if (NR) { WAIT_VM(8); WAIT_LGKM0(); BAR(); VREAD(v); } } while (0)
    if (NR) {
        VLOAD(0); VLOAD(1); VLOAD(2); VLOAD(3);
        WAIT_VM(12); BAR();
        PEND();
        VREAD(0);
#pragma unroll
        for (int v = 4; v < 2 * NR; ++v) { VSTEP(v - 3); VLOAD(v); }
        if (wave == 0 && __builtin_amdgcn_readfirstlane((int)po) >= 0) { poll4_ge(rws, po, call, ws, call); WAIT_VM_TRACKED(0); }
        VSTEP(2 * NR - 3);
        if (wave == 0) STAMP(ws, stamp0);
    }
    if (wave >= 4) {
        const u32 so = (wave < 6 ? OFF_SA + (par * 64u + (u32)slot) * 512u : OFF_SB + (par * BSLOTS + (u32)ct) * 512u) + (u32)((wave & 1) * 64 + lane) * 4u;
        __builtin_amdgcn_global_load_lds((const u32*)(ws + so), (LAS u32*)(lds + L_SC + (u32)(wave - 4) * 256u), 4, 0, AUX_SYS);
    }
    else if (wave >= 2 && headt)
        __builtin_amdgcn_global_load_lds((const u32*)(ws + OFF_RD + (par * 8u + (u32)a_src) * 512u + (u32)((wave & 1) * 64 + lane) * 4u), (LAS u32*)(lds + L_RED + (u32)(wave - 2) * 256u), 4, 0, AUX_SYS);
    LOADT(0, 0u);
    if (wave == 0) SNAP(1);
    VSTEP(2 * NR - 2);
    LOADT(1, 1u);
    VSTEP(2 * NR - 1);
    LOADT(2, 2u);
    if (NR) { WAIT_VM(8); WAIT_LGKM0(); BAR(); LOADT(3, 3u); }
    else {
        LOADT(3, 3u);
        if (wave == 0) WAIT_VM(13); else WAIT_VM(12);
        BAR();
        PEND();
    }
    READF(a0, b0, 0u);
#define GATE(c) do { const u32 _v = *(const LAS u32*)(lds + L_SNAP + ((u32)(c) & 1u) * 256u + (u32)lane * 4u); \
        if (!__all((int)(_v - call) >= 0)) { if (wave == 0) { poll_ge(GPTR(c), call, ws, call); WAIT_VM(0); } BAR(); } } while (0)
    bool seen = false;
#define FP(i) do { __builtin_amdgcn_global_load_lds((const u32*)_gp, (LAS u32*)(lds + L_RED + (i) * 256u), 4, 0, AUX_SYS); __builtin_amdgcn_s_sleep(FSLEEP); } while (0)
#define FS(i) { FP(((i) + 3) & 3); WAIT_VM(3); u32 _w; \
            asm volatile("ds_read_b32 %0, %1 offset:%2\n\ts_waitcnt lgkmcnt(0)" : "=v"(_w) : "v"(_pa), "n"((i) * 256) : "memory"); \
            if (__all((int)(_w - call) >= 0)) break; }
#define FGATE(c) do { const u32 _v = *(const LAS u32*)(lds + L_SNAP + ((u32)(c) & 1u) * 256u + (u32)lane * 4u); \
        seen = false; \
        if (!__all((int)(_v - call) >= 0)) { if (wave == 0) { seen = true; \
            const char* _gp = GPTR(c); const u32 _pa = (u32)(size_t)(lds + L_RED) + (u32)lane * 4u; const u64 _t0 = __builtin_amdgcn_s_memrealtime(); \
            FP(0); FP(1); FP(2); \
            for (u32 _n = 1;; ++_n) { FS(0) FS(1) FS(2) FS(3) if ((_n & 15u) == 0u && give_up(ws, call, _t0)) break; } \
        } BAR(); } } while (0)
    for (int c = 0; c < NC - 1 - NR; ++c) {
        const int t = 4 * c;
        WAIT_VM(8); WAIT_LGKM0(); BAR();
        if (BIG || !FGALL) GATE(c + 1); else FGATE(c + 1);
        STEP(t + 4, 0u, a1, b1, 1u, a0, b0);
        WAIT_VM(8); WAIT_LGKM0(); BAR();
        if (wave == 0) SNAP(c + 2);
        STEP(t + 5, 1u, a0, b0, 2u, a1, b1);
        if (wave == 0) WAIT_VM(9); else WAIT_VM(8);
        WAIT_LGKM0(); BAR();
        STEP(t + 6, 2u, a1, b1, 3u, a0, b0);
        if (wave == 0) WAIT_VM(9); else WAIT_VM(8);
        WAIT_LGKM0(); BAR();
        STEP(t + 7, 3u, a0, b0, 0u, a1, b1);
        if (BIG && c == 2 * HU - 1 && headt) {
#pragma unroll
            for (int m = 0; m < 4; ++m) {
                const u32 r = (u32)(wm * 64 + m * 16 + (lane & 15)) * 4u;
                const int d = (int)*(const LAS signed char*)(lds + L_RED + r) - (int)*(const LAS signed char*)(lds + L_SC + r);
#pragma unroll
                for (int n = 0; n < NB; ++n)
#pragma unroll
                    for (int j = 0; j < 4; ++j) acc[m][n][j] = __builtin_amdgcn_ldexpf(acc[m][n][j], d);
            }
        }
    }
#define TS(i) do { } while (0)
#define FB(j, k, b) ((j) ? bt[(j) ? 4 * (j) - 4 + (k) : 0] : (b))
#pragma unroll
    for (int j = 0; j < NR; ++j) {
        const int c = NC - 1 - NR + j, t = 4 * c;
        if (j) WAIT_VM(4); else WAIT_VM(8);
        WAIT_LGKM0(); BAR();
        TS(2 * j);
        FGATE(c + 1);
        TS(2 * j + 1);
        STEPX(t + 4, 0u, a1, b1, 1u, a0, FB(j, 0, b0), false, !j);
        if (seen) { } else if (j) WAIT_VM(4); else WAIT_VM(6);
        WAIT_LGKM0(); BAR();
        if (wave == 0) SNAP(c + 2);
        STEPX(t + 5, 1u, a0, b0, 2u, a1, FB(j, 1, b1), false, !j);
        if (seen) { } else if (wave == 0) WAIT_VM(5); else WAIT_VM(4);
        WAIT_LGKM0(); BAR();
        STEPX(t + 6, 2u, a1, b1, 3u, a0, FB(j, 2, b0), false, !j);
        if (wave == 0) WAIT_VM(5); else WAIT_VM(4);
        WAIT_LGKM0(); BAR();
        STEPX(t + 7, 3u, a0, b0, 0u, a1, FB(j, 3, b1), false, false);
    }
#define DSTEP(k, n) do { if ((k) > HINT && hint && wave == 0) WAIT_VM((n) + 1); else WAIT_VM(n); WAIT_LGKM0(); BAR(); if ((k) == HINT && hint) YFLAG(); } while (0)
    TS(8);
    DSTEP(0, NR ? 4 : 8);
    TS(9);
    READX(a1, b1, 1u, !NR); MMA(a0, FB(NR, 0, b0));
    DSTEP(1, NR ? 2 : 4);
    READX(a0, b0, 2u, !NR); MMA(a1, FB(NR, 1, b1));
    DSTEP(2, 0);
    TS(10);
    READX(a1, b1, 3u, !NR); MMA(a0, FB(NR, 2, b0));
    WAIT_LGKM0(); BAR();
    if (HINT == 3 && hint) YFLAG();
    u32 npo = OOB; u32x4 nv = {0u, 0u, 0u, 0u};
    if (BIG && nx.slot >= 0) { npo = start_off(rank, nx, nbig, par, lane, wave); nv = __builtin_amdgcn_raw_buffer_load_b128(rws, npo, 0, AUX_SYS); }
    MMA(a1, FB(NR, 3, b1));
#undef DSTEP
#undef FB
#undef FGATE
#undef FS
#undef FP
    if (wave == 0) STAMP(ws, stamp0 + 1);
#pragma unroll
    for (int m = 0; m < 4; ++m)
#pragma unroll
        for (int n = 0; n < NB; ++n) {
            const u32 r = (u32)(wm * 64 + m * 16 + (lane & 15)), pc = (u32)(wn * (BT / 8) + n * 4 + (lane >> 4));
            *(LAS f32x4*)(lds + (u32)ks * AREA + r * RB + ((pc ^ (r & 7u)) << 4)) = acc[m][n];
        }
    WAIT_LGKM0(); BAR();
    if (MODE == 1 && LSL) __builtin_amdgcn_s_sleep(LSL);
    const LAS int* sc = (const LAS int*)(lds + L_SC);
    const i32x4 eb = *(const LAS i32x4*)(sc + 128 + col0 + pc * 4u);
    const bool bad = *(const LAS u32*)(lds + L_SNAP) == call;
    if (BIG) { WAIT_VM_TRACKED(0); const bool ok = (int)npo < 0 || all4_ge(nv, call); if (lane == 0) okw[wave] = ok ? 0u : 1u; }
#pragma unroll
    for (u32 it = 0; it < (BIG ? 8u : 2u); ++it) {
        const u32 r = (u32)wave * (BT / 8u) + ((BIG && EROT ? (it + (blockIdx.x >> 3)) & 7u : it) * 64u + (u32)lane) / PPR;
        const u32 lo = r * RB + ((pc ^ (r & 7u)) << 4);
        f32x4 s = *(const LAS f32x4*)(lds + lo) + *(const LAS f32x4*)(lds + AREA + lo);
        if (!BIG) s += *(const LAS f32x4*)(lds + 2u * AREA + lo) + *(const LAS f32x4*)(lds + 3u * AREA + lo);
        const int ea = (int)(signed char)sc[row0 + r];
#pragma unroll
        for (int j = 0; j < 4; ++j) s[j] = bad ? __builtin_nanf("") : __builtin_amdgcn_ldexpf(s[j], ea + eb[j]);
        u32x4 o = __builtin_bit_cast(u32x4, s);
        if (hint) {
#pragma unroll
            for (int j = 0; j < 4; ++j) o[j] = o[j] ? o[j] : 0x80000000u;
        }
        __builtin_amdgcn_raw_buffer_store_b128(o, ro, obase + r * ostr, 0, AUX_SC1);
        if (BIG && EPW && it < 7u && (first || EALL)) WAIT_VM(EPW ? EPW - 1 : 0);
        if (BIG && EPS && it < 7u && first) __builtin_amdgcn_s_sleep(EPS);
    }
    if (remote && BIG) pend = p.ws[b_src] + OFF_YR + (u32)(a * 64 + slot) * 16u + (u32)(ct & 1) * 8u;
    if (wave == 0) STAMP(ws, stamp0 + 2);
#undef TS
#undef GLDS16
#undef SCHED
#undef SB
#undef STEP
#undef STEPX
#undef READX
#undef VLOAD
#undef VREAD
#undef VSTEP
#undef LOADT
#undef SNAP
#undef READF
#undef MMA
#undef GATE
#undef YFLAG
#undef PEND
#undef GPTR
}

constexpr u32 NSK = DELTA - (DCUT ? 1u : 0u);
constexpr u32 NPOS_INTRA = 96u - HU - NSK, NPOS_CROSS = 97u - HU + DELTA;
constexpr u32 FAN = 0x20000000u;
struct Desc { u32 src, dst, gate, flag; };
struct Stream {
    bool cross; int rank, g, a, j; u32 pb, sa;
    __device__ __forceinline__ int yslot(u32 k) const {
        if (k < 6u) return j * 6 + (int)k;
        if (k < 9u) return ((j + (int)k - 5) & 3) * 6;
        if (k < 12u) return 24 + 2 * ((a + (int)k - 8) & 3);
        if (k < 18u) return ((j + 1 + (((int)k - 12) >> 1)) & 3) * 6 + 1 + ((int)k & 1);
        if (k < 21u) return 25 + 2 * ((a + (int)k - 17) & 3);
        if (k < 27u) return ((j + 1 + (((int)k - 21) >> 1)) & 3) * 6 + 3 + (((int)k - 21) & 1);
        return ((j + 1 + (int)(k % 3u)) & 3) * 6 + 5;
    }
    __device__ __forceinline__ u32 single(u32 slot, u32 k) const { return pb + slot * PANEL + k * UNIT; }
    __device__ __forceinline__ u32 pair(u32 slot, u32 k) const { return pb + slot * PANEL + k * 65536u; }
    __device__ __forceinline__ u32 half(u32 slot, u32 h, u32 k) const { return pb + slot * PANEL + k * 2u * UNIT + h * 8192u; }
    __device__ __forceinline__ Desc at(u32 pos) const {
        Desc d;
        const u32 s = (u32)j * 6u, M8 = 7u << 20, PAIR = 1u << 28, HALF = 4u << 28, ONE = M8 + (6u << 28);
        u32 u, fsrc = (u32)((cross ? 1 - g : g) * 4 + j);
        if (!cross) {
            if (pos >= NPOS_INTRA) pos = NPOS_INTRA - 1u;
            u = pos + HU;
            if (u >= RU0) u += NSK;
            if (u < 80u) {
                d.src = u < 16u ? single(s, u) : u < 48u ? pair(s + 1u, u - 16u) : pair(s + 3u, u - 48u);
                d.gate = sa + s * 512u + (u < ROWU ? 0u : u < 16u ? ONE : u < 48u ? 512u + PAIR + M8 : 1536u + PAIR + M8);
            } else {
                const u32 e = 95u - u, f = e - LEAD;
                const u32 h = e < LEAD || (e < 16u - LEAD && (f & 1u)) ? 1u : 0u;
                const u32 k = e < LEAD ? 7u - e : e >= 16u - LEAD ? 15u - e : h ? 7u - LEAD - (f >> 1) : 7u - (f >> 1);
                u = 80u + h * 8u + k;
                d.src = half(s + 5u, h, k);
                d.gate = sa + (s + 5u) * 512u + h * 256u + M8 + HALF;
            }
            d.dst = d.src;
            d.flag = OFF_LF + (fsrc * NUNITS + u) * 128u;
        } else {
            if (pos >= NPOS_CROSS) pos = NPOS_CROSS - 1u;
            pos += HU;
            const u32 hop = j == a ? HOP1 : HOP2;
            u = pos < hop ? pos : pos - DELTA;
            if (pos - hop < DELTA) {
                const u32 k = pos - hop;
                if (j == a) { d.src = pair(s + 3u, RU0 - 48u + k); d.dst = pair(RELAY, k); d.gate = sa + 34u * 512u + M8 + PAIR;
                              d.flag = FAN + OFF_RF + ((u32)a * 4u + k) * 128u; }
                else { d.src = pair(RELAY, k); d.dst = pair(s + 3u, RU0 - 48u + k); d.gate = OFF_RF + ((u32)j * 4u + k) * 128u + (5u << 28);
                       d.flag = OFF_LF + ((fsrc ^ 4u) * NUNITS + RU0 + k) * 128u; }
            } else if (u < 64u) {
                d.src = u < 16u ? single(32u, u) : u < 48u ? pair((u32)(ASLOTS + 2 * a), u - 16u) : single(33u, u - 48u);
                d.dst = u < 16u ? single((u32)(24 + 2 * j), u) : u < 48u ? pair((u32)(ASLOTS + 8 + 2 * j), u - 16u) : single((u32)(25 + 2 * j), u - 48u);
                d.gate = u < 16u ? sa + (u32)(24 + 2 * j) * 512u + (u < ROWU ? 0u : ONE) : u < 48u ? OFF_WS + (u32)j * 256u + (1u << 20) + (3u << 28) : sa + (u32)(25 + 2 * j) * 512u + (R7C ? ONE : 0u);
                d.flag = OFF_LF + (fsrc * NUNITS + u) * 128u;
            } else {
                const u32 k = u - 64u;
                const int ys = yslot(k);
                const u32 h = k >= 30u ? 1u : 0u;
                d.src = OFF_YST + (u32)(a * 32 + ys) * UNIT + h * 65536u;
                d.dst = ((u32)slot_row((int)fsrc, ys) + h * 64u) * (NLOC * 4u) + (u32)j * 1024u;
                d.gate = OFF_YR + ((u32)(j * 64 + ys) + h * 32u) * 16u + ((k >= 27u + Y4 ? 10u : 2u) << 28);
                d.flag = NOFLAG;
            }
        }
        return d;
    }
};
struct Tables { Desc lo, hi; };
__device__ __forceinline__ u32 tab(u32 t0, u32 t1, u32 i) {
    const u32 x = (u32)__builtin_amdgcn_readlane((int)t0, (int)(i & 63u)), y = (u32)__builtin_amdgcn_readlane((int)t1, (int)(i & 63u));
    return i < 64u ? x : y;
}

#define PSLEEP0 10
#define PSLEEP 4
#define NACT 32
__device__ __forceinline__ void pull_stream(const Params& p, LAS char* lds, const Tables& T, bool cross, u32 cw, char* src, gchar* fan, u32 calln, u32 call, int lane, int wave, int hw) {
    char* ws = p.me;
    const rsrc_t rl = mk_rsrc(ws, WS_NEED), rs = mk_rsrc(src, WS_NEED), rout = mk_rsrc(p.out, IN_BYTES);
    LAS char* pl = lds + (u32)hw * 1024u;
    const u32 pla = (u32)(size_t)pl + (u32)lane * 4u;
    const u32 l16 = (u32)lane * 16u, l4r = (u32)(lane >> 5) * IMG + (l16 & 511u);
    const u32 l0 = lane == 0 ? 0u : OOB;
    const u32 co = OFF_CL + (u32)wave * 128u + l0;
    const u32 NPIECES = (cross ? NPOS_CROSS : NPOS_INTRA) * 32u;
    const u32 skip = !DCUT ? 0u : cross ? (u32)DCUT : 32u - DCUT;
    const u32 qcut = cross ? ((wave == 3 ? HOP1 : HOP2) + DELTA - HU) * 32u - DCUT : (RU0 - HU) * 32u;
    const u32 qt = cross ? (NPOS_CROSS - 6u) * 32u : OOB;
    const u32 cbase = 2u * NACT - (calln - 1u) * (NPIECES - skip - (cross ? Y4 * 16u : 0u));
    u32x4 v0 = {0u, 0u, 0u, 0u}, v1 = v0, v2 = v0, v3;
    u32 q = OOB, so = OOB, ss = 1024u;
    u32 c0 = cw, c1 = cw + NACT;
    u32 pf = NOFLAG;
    bool claimed = false, dead = false;
    int cl = 0;
#define KIND(q) (tab(T.lo.gate, T.hi.gate, (q) >> 5) >> 28)
#define POFF(q, kd) (((kd) & 1u) ? (((q) >> 4) & 1u) * PANEL + ((q) & 15u) * 4096u : (kd) == 4u ? (((q) >> 1) & 15u) * IMG + ((q) & 1u) * 4096u : ((q) & 31u) * (((kd) & 8u) ? 2048u : (kd) ? 4096u : 512u))
#define GBASE(q) ({ const u32 _g = tab(T.lo.gate, T.hi.gate, (q) >> 5), _k = _g >> 28; \
        const u32 _ix = (((_k & 1u) ? ((q) >> 2) & 4u : 0u) + ((q) & (_k == 4u ? 1u : 3u))) << (_k == 3u ? 5 : 7); \
        (_g & 0xfffffu) + ((_k & 7u) == 2u ? 0u : _k == 5u ? ((q) & 28u) * 4u : _k == 0u ? ((q) & 31u) * 16u : _ix); })
#define GOFF(q) (GBASE(q) + (l16 & (tab(T.lo.gate, T.hi.gate, (q) >> 5) >> 16)))
#define PFLAG() st_sys_g((pf & FAN) ? fan + (pf - FAN) : (gchar*)ws + (pf >= NOFLAG ? OFF_CTR + cw * 64u + 8u + (u32)hw * 4u : pf), call)
#define PLD(u, o) v##u = __builtin_amdgcn_raw_buffer_load_b128(rs, (o) + (u) * ss, 0, AUX_SYS)
#define PSTA(u, o) do { asm volatile("" : "+v"(v##u)); \
        __builtin_amdgcn_raw_buffer_store_b128(v##u | (dead ? 0x7fc00000u : 0u), rd, o, 0, AUX_SC1); } while (0)
#define PST(u) PSTA(u, d + (u) * ds)
#define PSTAMP(i) do { } while (0)
#define GPOLL(i, sl) do { __builtin_amdgcn_global_load_lds((const u32*)gp, (LAS u32*)(pl + (i) * 256u), 4, 0, AUX_SYS); __builtin_amdgcn_s_sleep(sl); \
        if (DRY && c0 < 32u) __builtin_amdgcn_s_sleep(PSLEEP0 - PSLEEP); } while (0)
#define GSTEP(i, sl) { GPOLL(((i) + 3) & 3, sl); WAIT_VM_TRACKED(3); u32 _w; \
            asm volatile("ds_read_b32 %0, %1 offset:%2\n\ts_waitcnt lgkmcnt(0)" : "=v"(_w) : "v"(pla), "n"((i) * 256) : "memory"); \
            if (__all((int)(_w - call) >= 0)) break; }
#define WATCH(sl) do { \
        const char* gp = ws + GBASE(c0) + ((u32)lane * 4u & ((tab(T.lo.gate, T.hi.gate, c0 >> 5) >> 16 & 0x3f0u) | 12u)); \
        const u64 t0 = __builtin_amdgcn_s_memrealtime(); \
        GPOLL(0, sl); GPOLL(1, sl); GPOLL(2, sl); \
        for (u32 n = 1;; ++n) { \
            GSTEP(0, sl) GSTEP(1, sl) GSTEP(2, sl) GSTEP(3, sl) \
            if ((n & 15u) == 0u && give_up(ws, call, t0)) { dead = true; break; } \
        } } while (0)
    if (!DRY) WATCH(PSLEEP0);
    const u32 g0 = DRY ? call - 1u : call;
    u32x4 gv = {g0, g0, g0, g0};
    bool dry = DRY;
    u32 d = OOB, ds = 1024u, so2 = OOB, d2 = OOB; bool y = false, val = false, val4 = false; u32 sor = OOB;
#define VALID(u) __all(v##u[0] != 0u && v##u[1] != 0u && v##u[2] != 0u && v##u[3] != 0u)
#define RETRY(u) do { if (HINT >= 0 && ((u) < 2 ? val : Y4 && val4) && !dead && !VALID(u)) { \
        const u64 t0 = __builtin_amdgcn_s_memrealtime(); u32 n = 0; \
        do { PLD(u, sor); NRETRY; \
            if ((++n & 15u) == 0u && give_up(ws, call, t0)) dead = true; \
            WAIT_VM_TRACKED(0); \
        } while (!dead && !VALID(u)); } } while (0)
#define NRETRY do { } while (0)
    do {
        const rsrc_t rd = y ? rout : rl;
        WAIT_VM_TRACKED(5); PLD(2, so2); RETRY(0); PST(0);
        if (EFL & 1) { WAIT_VM_TRACKED(5); PFLAG(); WAIT_VM_TRACKED(5); PLD(3, so2); }
        else { WAIT_VM_TRACKED(4); PLD(3, so2); PFLAG(); }
        RETRY(1); PST(1);
        WAIT_VM_TRACKED(5);
        bool open = dead || all4_ge(gv, call);
        if (q == OOB && !open && !dry) { WATCH(PSLEEP); open = true; }
        dry = false;
        pf = q != OOB ? tab(T.lo.flag, T.hi.flag, q >> 5) + (q & 31u) * 4u : NOFLAG;
        if (claimed) { c1 = (u32)__builtin_amdgcn_readfirstlane(cl) + cbase; if (c1 >= qcut) c1 += skip;
            if (Y4 && c1 >= qt) c1 += (c1 - qt < Y4 * 16u ? c1 - qt : Y4 * 16u) & ~15u;
            if (c1 > NPIECES + 127u) c1 = NPIECES + 127u; }
        claimed = open && c0 < NPIECES;
        q = claimed ? c0 : OOB; if (claimed) c0 = c1;
        const u32 qq = claimed ? q : 0u;
        const u32 kn = claimed ? KIND(qq) : 1u, po = POFF(qq, kn), lo = kn ? l16 : l4r;
        so = claimed ? tab(T.lo.src, T.hi.src, qq >> 5) + po + lo : OOB;
        ss = kn ? 1024u : 2u * IMG;
        y = (kn & 7u) == 2u;
        u32 dn = claimed ? tab(T.lo.dst, T.hi.dst, qq >> 5) + (y ? po * 4u : po) + lo : OOB;
        u32 go = GOFF(c0), ao = claimed ? co : OOB, so1 = so + ss, dx2 = d2 + 2u * ds, dx3 = d2 + 3u * ds;
        if (claimed) PSTAMP(q >> 5);
        asm volatile("" : "+v"(so), "+v"(dn), "+v"(go), "+v"(ao), "+v"(so1), "+v"(dx2), "+v"(dx3));
        WAIT_VM_TRACKED(4);
        v0 = __builtin_amdgcn_raw_buffer_load_b128(rs, so, 0, AUX_SYS);
        RETRY(2); PSTA(2, dx2);
        WAIT_VM_TRACKED((EFL & 1) ? 3 : 4);
        RETRY(3); PSTA(3, dx3);
        v1 = __builtin_amdgcn_raw_buffer_load_b128(rs, so1, 0, AUX_SYS);
        cl = __builtin_amdgcn_raw_ptr_buffer_atomic_add_i32(1, rl, ao, 0, AUX_SC1);
        gv = __builtin_amdgcn_raw_buffer_load_b128(rl, go, 0, AUX_SYS);
        d = dn; ds = y ? NLOC * 4u : ss;
        val = claimed && qq >= qt; val4 = val && !(kn & 8u); sor = so;
        so2 = (kn & 8u) ? OOB : so; d2 = (kn & 8u) ? OOB : dn;
    } while (c0 < NPIECES || (!LASTP && q != OOB));
    if (LASTP) {
        const rsrc_t rd = y ? rout : rl;
        PLD(2, so2); PLD(3, so2);
        WAIT_VM_TRACKED(7); RETRY(0); PST(0);
        if (EFL & 2) { WAIT_VM_TRACKED(6); PFLAG(); WAIT_VM_TRACKED(6); } else { WAIT_VM_TRACKED(5); PFLAG(); }
        RETRY(1); PST(1);
        pf = q != OOB ? tab(T.lo.flag, T.hi.flag, q >> 5) + (q & 31u) * 4u : NOFLAG;
        WAIT_VM_TRACKED(4); RETRY(2); PSTA(2, d2 + 2u * ds);
        WAIT_VM_TRACKED(4); RETRY(3); PSTA(3, d2 + 3u * ds);
    }
    if (!ENDQ || pf < NOFLAG) { WAIT_VM_TRACKED(0); PFLAG(); }
    asm volatile("" :: "v"(gv), "v"(cl));
    asm volatile(".Lagk_end:");
    if (hw < 7) STAMP(ws, 2 + wave * 8 + 7);
#undef PSTAMP
#undef GOFF
#undef GBASE
#undef WATCH
#undef GSTEP
#undef GPOLL
#undef POFF
#undef KIND
#undef PST
#undef PSTA
#undef RETRY
#undef VALID
#undef NRETRY
#undef PLD
#undef PFLAG
}

__global__ void __launch_bounds__(512, 2) ag_gemm_kernel(Params p) {
    extern __shared__ __attribute__((aligned(16))) char smem[];
    LAS char* lds = (LAS char*)smem;
    const int tid = threadIdx.x, lane = tid & 63, wave = __builtin_amdgcn_readfirstlane(tid >> 6);
    const int b = blockIdx.x, rank = p.rank, g = rank >> 2, a = rank & 3;
    u64 pc; u32 len;
    asm volatile("s_getpc_b64 %0\n.Lagk_pc:\n\ts_mov_b32 %1, .Lagk_end-.Lagk_pc" : "=s"(pc), "=s"(len));
    const u32 code = __builtin_amdgcn_raw_buffer_load_b32(mk_rsrc((const void*)pc, (int)len > 0 ? len : 0u), (u32)(b >> 3) * 2048u + (u32)tid * 4u, 0, 0);
    char* ws = p.me;
    if (wave == 0) STAMP(ws, 0);
    const u32 prev = ld_agent(ws + OFF_CTR + (u32)b * 64u);
    u32 calln;
    const int lw = LROT ? (wave + (b >> 3) * 2 + (b & 7)) % 7 : wave;
    const rsrc_t rws = mk_rsrc(ws, WS_NEED);

    Tables T = {};
    char* src = ws; gchar* fan = (gchar*)ws;
    u32 hword = 0;
    gchar* rdflag = x_head(p, lds, b < 128 ? b : 640 + b, prev, calln, hword, tid, wave, [&]() {
        if (b < NCOMM) {
            const u32 par = calln & 1u;
            Stream S;
            S.cross = lw >= 3; S.rank = rank; S.g = g; S.a = a; S.j = (a + lw + 1) & 3; S.pb = CTRL_BYTES + par * PAR_BYTES; S.sa = OFF_SA + par * 32768u;
            T.lo = S.at((u32)lane); T.hi = S.at((u32)lane + 64u);
            src = p.peer[lw == 3 ? 7 : lw];
            char *f0 = p.peer[4], *f1 = p.peer[5], *f2 = p.peer[6];
            asm volatile("" : "+s"(src), "+s"(f0), "+s"(f1), "+s"(f2));
            fan = (gchar*)(lane == 0 ? f0 : lane == 1 ? f1 : f2);
        }
    });
    asm volatile("" :: "v"(code));
    const u32 call = calln << 8, par = calln & 1u;
    if (wave == (CTR7 ? 7 : 0) && lane < 2) st_agent(ws + OFF_CTR + (u32)b * 64u + (u32)lane * 4u, lane ? call : calln);
    if (wave == 0 && b >= NCOMM) STAMP(ws, 1);
    if (b < NCOMM) {
        if (wave == 7) {
            WAIT_VM(0);
            if (lane < 4) st_sys_g(rdflag, hword);
        }
        else if (b < NACT) pull_stream(p, lds, T, lw >= 3, (u32)b, src, fan, calln, call, lane, lw, wave);
        __builtin_amdgcn_endpgm();
    }

    {
        const int c = b - NCOMM, strip = c / 7, rg = c - strip * 7;
        const int kt0 = (rg * 128) / 7, kt1 = ((rg + 1) * 128) / 7;
        const rsrc_t rw = mk_rsrc(p.w, IN_BYTES);
        const int q = lane & 7, n4 = lane >> 3;
        if (wave == 7) { WAIT_VM(0); if (lane < 4) st_sys_g(rdflag, hword); }
        f32x4 v[3][8];
#pragma unroll
        for (int i = 0; i < 3; ++i) {
            const int kt = kt0 + wave + 8 * i;
            const u32 base = kt < kt1 ? ((u32)(kt * 64 + q * 8) * NLOC + (u32)(strip * 32 + n4 * 4)) * 4u : OOB;
#pragma unroll
            for (int jj = 0; jj < 8; ++jj) {
                v[i][jj] = __builtin_bit_cast(f32x4, __builtin_amdgcn_raw_buffer_load_b128(rw, base + (u32)jj * (NLOC * 4u), 0, 0));
                if (i * 8 + jj < W_N0) WAIT_VM(W_LD0 - 1); else WAIT_VM(W_LD - 1);
            }
        }
        LAS u32* cm = (LAS u32*)(lds + L_RED + 256u);
        if (tid < 64) cm[tid] = 0u;
        WAIT_LGKM0(); BAR();
        u32 m[4] = {0u, 0u, 0u, 0u};
#pragma unroll
        for (int i = 0; i < 3; ++i)
#pragma unroll
            for (int jj = 0; jj < 8; ++jj)
#pragma unroll
                for (int k = 0; k < 4; ++k) m[k] = umax(m[k], absbits(v[i][jj][k]));
#pragma unroll
        for (int k = 0; k < 4; ++k) {
            m[k] = umax(m[k], (u32)__shfl_xor((int)m[k], 1)); m[k] = umax(m[k], (u32)__shfl_xor((int)m[k], 2)); m[k] = umax(m[k], (u32)__shfl_xor((int)m[k], 4));
        }
        if (q < 4) __hip_atomic_fetch_max(cm + n4 * 4 + q, q == 0 ? m[0] : q == 1 ? m[1] : q == 2 ? m[2] : m[3], __ATOMIC_RELAXED, __HIP_MEMORY_SCOPE_WORKGROUP);
        WAIT_LGKM0(); BAR();
        u64* wx = (u64*)(ws + OFF_WX) + (u32)strip * 256u;
        if (tid < 32) __hip_atomic_store(wx + rg * 32 + tid, ((u64)call << 32) | cm[tid], __ATOMIC_RELAXED, __HIP_MEMORY_SCOPE_AGENT);
        if (wave < 4) {
            const int gi = tid < 224 ? tid : 0;
            const u64 t0 = __builtin_amdgcn_s_memrealtime();
            u64 gv; u32 n = 0;
            for (;;) {
                gv = __hip_atomic_load(wx + gi, __ATOMIC_RELAXED, __HIP_MEMORY_SCOPE_AGENT);
                if (__all((u32)(gv >> 32) == call)) break;
                if ((++n & 15u) == 0u && give_up(ws, call, t0)) break;
                __builtin_amdgcn_s_sleep(1);
            }
            __hip_atomic_fetch_max(cm + 32 + (gi & 31), (u32)gv, __ATOMIC_RELAXED, __HIP_MEMORY_SCOPE_WORKGROUP);
        }
        WAIT_LGKM0(); BAR();
        const int ct = strip >> 2;
        if (rg == 0 && tid < 64) {
            const u32 nin = (u32)((strip & 3) * 32 + (tid & 31));
            const u32 ev = (u32)(exp_of(cm[32 + (tid & 31)]) - 141);
            char* t = tid < 32 ? ws + OFF_SB + (par * BSLOTS + (u32)ct) * 512u : p.ws[(1 - g) * 4 + (strip >> 3)] + OFF_SB + (par * BSLOTS + (u32)(8 + 2 * a + (ct & 1))) * 512u;
            st_sys(t + nin * 4u, ev);
        }
        const u32x4 mx = *(LAS u32x4*)(cm + 32 + n4 * 4);
        const u32 pbase = CTRL_BYTES + par * PAR_BYTES + (u32)(ASLOTS + ct) * PANEL;
#pragma unroll
        for (int k = 0; k < 4; ++k) {
            const float s = scale_of(exp_of(mx[k]));
            const u32 nin = (u32)((strip & 3) * 32 + n4 * 4 + k);
            const u32 ro = nin * 128u + (((u32)q ^ ((nin >> 1) & 7u)) << 4);
#pragma unroll
            for (int i = 0; i < 3; ++i) {
                const int kt = kt0 + wave + 8 * i;
                const u32 off = kt < kt1 ? pbase + (u32)kt * IMG + ro : OOB;
                const f32x4 lo = {v[i][0][k], v[i][1][k], v[i][2][k], v[i][3][k]}, hi = {v[i][4][k], v[i][5][k], v[i][6][k], v[i][7][k]};
                __builtin_amdgcn_raw_buffer_store_b128(cvt8(lo, hi, s), rws, off, 0, AUX_SC1);
                WAIT_VM(W_ST - 1);
            }
        }
        WAIT_VM(0); BAR();
        if (tid < 4 && (!(tid & 1) || rg == 6))
            st_sys((tid < 2 ? ws + OFF_WR + (u32)strip * 32u : p.ws[(1 - g) * 4 + (strip >> 3)] + OFF_WS + (u32)(a * 8 + (strip & 7)) * 32u) + (u32)(rg + (tid & 1)) * 4u, call);
        if (wave == 0) STAMP(ws, 2);
    }
    const int xcd = b & 7, q = (b >> 3) - 4;
    if (xcd >= 4 && q < 24) {
        const int r0 = 128 + (q * 4 + xcd - 4) * 4;
        x_rows(p, lds, r0, par, call, tid, wave);
        x_rows(p, lds, r0 + (r0 < 384 ? 384 : 512), par, call, tid, wave);
        if (wave == 0) STAMP(ws, 3);
    }
    const int qr = q >> 2, qc = q & 3, x4 = xcd & 3;
    const int early = (x4 >> 1) == (xcd < 4 ? ESWAP : 0) ? (qr < 6 ? a * 6 + qr : -1) : (qr < 3 ? ((a + 1 + qr) & 3) * 6 : 24 + 2 * (qr - 3));
    const int peer = ((a + 1 + (qr >> 1)) & 3) * 6;
    auto tile_of = [&](int it) {
        int slot = -1, ct = 4 * x4 + qc, rh = 0, ch = 0;
        if (it == 0) { if (xcd < 4) { slot = early; ct = 4 * (x4 & 1) + qc; } }
        else if (it == 1) {
            if (RMAP && qr == 6 && xcd >= 2 && xcd < 6) { slot = ((a + 2) & 3) * 6 + (xcd >> 1); ct = 4 * (xcd & 1) + qc; }
            else if (xcd < 4) { if (qr < 6 && !(RMAP && xcd < 2 && qr >> 1 == 1)) slot = peer + 1 + (qr & 1); }
            else { slot = early; ct = 8 + 4 * (x4 & 1) + qc; }
        }
        else if (it == 2) {
            if (qr == 6) { slot = 25 + (xcd & 6); ct = 8 + 4 * (xcd & 1) + qc; }
            else if (xcd >= 4) slot = peer + 3 + (qr & 1);
            else if (xcd < 2 && qr < 4) slot = 25 + 2 * qr;
        } else if (it == 3) {
            int pi = -1; ch = qc & 1;
            const int qs = !RMAP && QSWAP && xcd >= 4 && (qr & 5) == 4 ? qr ^ 2 : qr;
            if (RMAP && xcd < 2 && qr >> 1 == 1) { pi = qr - 1; rh = 1; ct = 12 + 2 * xcd + (qc >> 1); }
            else if (xcd < 2) { pi = qr - 4; rh = 1; ct = 8 + 2 * xcd + (qc >> 1); }
            else if (qs == 6) { pi = RMAP && xcd >= 4 ? -1 : (xcd - 2) >> 1; rh = 1; ct = 12 + 2 * (xcd & 1) + (qc >> 1); }
            else if (xcd < 4) { pi = qr >> 1; ct = 4 * xcd + 2 * (qr & 1) + (qc >> 1); }
            else { pi = qs >> 1; rh = qs & 1; ct = 2 * x4 + (qc >> 1); }
            if (pi >= 0) slot = ((a + 1 + pi) & 3) * 6 + 5;
        }
        if (slot >= 24 && ct >= 8 && (slot - 24) >> 1 == (ct - 8) >> 1) { slot = 32 + (slot & 1); ct -= 8; }
        return Tile{slot, ct, rh, ch};
    };
    char* pend = nullptr;
    int it = 0;
    Tile tl = tile_of(0);
    while (tl.slot < 0) tl = tile_of(++it);
    for (bool first = true; it < 3; first = false) {
        int nt = it + 1;
        Tile nx = tile_of(nt);
        while (nx.slot < 0 && nt < 3) nx = tile_of(++nt);
        gemm_tile<0>(p, lds, tl, first, nx, nt < 3, par, call, tid, lane, wave, 4 + it * 3, pend);
        it = nt; tl = nx;
    }
    if (tl.slot < 0) {
        if (pend) { WAIT_VM(0); BAR(); if (tid < 2) st_sys(pend + (u32)tid * 4u, call); }
        __builtin_amdgcn_endpgm();
    }
    if (tl.ct < 8) {
        gemm_tile<1>(p, lds, tl, false, tl, false, par, call, tid, lane, wave, 13, pend);
        __builtin_amdgcn_endpgm();
    }
    gemm_tile<2>(p, lds, tl, false, tl, false, par, call, tid, lane, wave, 13, pend);
}
}

extern "C" void kernel_launch(int rank, int nranks, ncclComm_t comm,
                              hipStream_t stream,
                              void* const* d_in, const int* in_sizes, int n_in,
                              void* d_out, int out_size,
                              const int* in_dtypes, int out_dtype,
                              void* const* d_out_all,
                              void* const* d_ws_all, size_t ws_bytes) {
    (void)comm; (void)in_dtypes; (void)out_dtype; (void)d_out_all;
    if (nranks != 8 || (unsigned)rank >= 8u || n_in < 2 || ws_bytes < (size_t)agk::WS_NEED || in_sizes[0] != 1024 * 8192 ||
        in_sizes[1] != 8192 * 1024 || out_size != 8192 * 1024) {
        fprintf(stderr, "kernel_launch: unexpected problem shape\n");
        return;
    }
    static bool ready[8] = {false, false, false, false, false, false, false, false};
    if (!ready[rank]) {
        HIP_CHECK(hipFuncSetAttribute((const void*)agk::ag_gemm_kernel, hipFuncAttributeMaxDynamicSharedMemorySize, (int)agk::LDS_BYTES));
        ready[rank] = true;
    }
    agk::Params p;
    p.x = (const float*)d_in[0]; p.w = (const float*)d_in[1]; p.rank = rank; p.pad = 0;
    const size_t wo = WEND ? (ws_bytes - agk::WS_NEED) & ~(size_t)0x1fffff : 0;
    p.me = (char*)d_ws_all[rank] + wo; p.out = (float*)d_out;
    for (int i = 0; i < 8; ++i) { p.ws[i] = (char*)d_ws_all[i] + wo; p.peer[i] = (char*)d_ws_all[((rank ^ i) & 4) + ((rank + 1 + i) & 3)] + wo; }
    hipLaunchKernelGGL(agk::ag_gemm_kernel, dim3(agk::NWG), dim3(512), agk::LDS_BYTES, stream, p);
    HIP_CHECK(hipGetLastError());
}
```

```cpp
#include <hip/hip_runtime.h>
#include <rccl/rccl.h>
#include <cstdio>

#define HIP_CHECK(expr) do { hipError_t _e = (expr); if (_e != hipSuccess) { \
    fprintf(stderr, "HIP error %s at %s:%d\n", hipGetErrorString(_e), __FILE__, __LINE__); \
    return; } } while (0)

namespace agk {
typedef unsigned u32;
typedef unsigned long long u64;
typedef u32 u32x4 __attribute__((ext_vector_type(4)));
typedef int i32x4 __attribute__((ext_vector_type(4)));
typedef float f32x4 __attribute__((ext_vector_type(4)));
typedef float f32x2 __attribute__((ext_vector_type(2)));
typedef _Float16 f16x2 __attribute__((ext_vector_type(2)));
typedef _Float16 f16x8 __attribute__((ext_vector_type(8)));
#define LAS __attribute__((address_space(3)))
typedef __amdgpu_buffer_rsrc_t rsrc_t;
typedef __attribute__((address_space(1))) char gchar;

constexpr int KDIM = 8192, NLOC = 1024, NKT = 128;
constexpr u32 IMG = 16384u;
constexpr u32 PANEL = NKT * IMG;
constexpr u32 UNIT = 131072u;
constexpr int ASLOTS = 34, BSLOTS = 16;
#define HU 3
#define HWIN 8
#define SCH 2
#define DRY 1
#define NF 4
#define NFL 2
#define RU0 68
#define HOP1 56
#define HOP2 64
#define LEAD 2
#define QSWAP 1
#define RMAP 0
#define FGALL 0
#define LASTP 1
#define ENDQ 1
#define ROWU 6
#define R7C 1
#define EPW 1
#define EPS 0
#define EROT 1
#define EALL 0
#define LSL 0
#define Y4 4
#define XPACE 0
#define XSTAG 8
#define EFL 3
#define LROT 1
#define WEND 1
#define FSLEEP 3
#define CTR7 1
#define ESWAP 1
#define HSTEP 1
#define HK 8
#define XDLY 8
#define HINT 1
#define DCUT 0
#define DELTA 3
constexpr u32 CTRL_BYTES = 1u << 20;
constexpr u32 RELAY = ASLOTS + BSLOTS;
constexpr u32 PAR_BYTES = (RELAY + 2u) * PANEL;
constexpr u32 OFF_YST = CTRL_BYTES + 2u * PAR_BYTES;
constexpr u32 WS_NEED = OFF_YST + 4u * 32u * UNIT;
constexpr u32 IN_BYTES = 1024u * 8192u * 4u;
constexpr u32 OFF_CTR = 0;
constexpr u32 OFF_TMO = 16384;
constexpr u32 OFF_CL = 20480;
constexpr u32 OFF_LF = 524288;
constexpr u32 OFF_RD = 655360;
constexpr u32 OFF_YR = 24576;
constexpr u32 OFF_RF = 28672;
constexpr u32 OFF_WR = 32768;
constexpr u32 OFF_WS = 33792;
constexpr u32 OFF_WX = 36864;
constexpr u32 OFF_SA = 131072;
constexpr u32 OFF_SB = 196608;
constexpr u32 OFF_DBG = 262144;
constexpr int NWG = 256, NCOMM = 32;
constexpr u32 NUNITS = 96u;
constexpr u32 OOB = 0x80000000u, NOFLAG = 0x40000000u;
constexpr u64 DEADLINE = 100000000ull;
constexpr u32 STAGE = 32768u;
constexpr u32 L_SNAP = 4u * STAGE;
constexpr u32 L_SC = L_SNAP + 512u;
constexpr u32 L_RED = L_SC + 1024u;
constexpr u32 LDS_BYTES = L_RED + 2048u;

struct Params {
    const float* x; char* me; int rank, pad;
    char* peer[8];
    const float* w; float* out;
    char* ws[8];
};

#define AUX_SC1 16
#define AUX_SYS 17
#define GL_AUX 16
#define W_LD 2
#define W_LD0 1
#define W_N0 0
#define W_ST 2
#define X_LD 4
#define X_ST 2
#define WAIT_VM(n) asm volatile("s_waitcnt vmcnt(%0)" :: "n"(n) : "memory")
#define WAIT_LGKM0() do { asm volatile("" ::: "memory"); __builtin_amdgcn_s_waitcnt(15 | (7 << 4) | (0 << 8) | (3 << 14)); asm volatile("" ::: "memory"); } while (0)
#define BAR() do { asm volatile("" ::: "memory"); __builtin_amdgcn_s_barrier(); asm volatile("" ::: "memory"); } while (0)
#define WAIT_VM_TRACKED(n) do { asm volatile("" ::: "memory"); __builtin_amdgcn_s_waitcnt(((n) & 15) | (7 << 4) | (15 << 8) | (((n) >> 4) << 14)); asm volatile("" ::: "memory"); } while (0)

__device__ __forceinline__ rsrc_t mk_rsrc(const void* p, u32 bytes) { return __builtin_amdgcn_make_buffer_rsrc((void*)p, 0, bytes, 0x00020000); }
__device__ __forceinline__ u32 ld_agent(const void* p) { return __hip_atomic_load((const u32*)p, __ATOMIC_RELAXED, __HIP_MEMORY_SCOPE_AGENT); }
__device__ __forceinline__ u32 ld_sys(const void* p) { return __hip_atomic_load((const u32*)p, __ATOMIC_RELAXED, __HIP_MEMORY_SCOPE_SYSTEM); }
__device__ __forceinline__ void st_agent(void* p, u32 v) { __hip_atomic_store((u32*)p, v, __ATOMIC_RELAXED, __HIP_MEMORY_SCOPE_AGENT); }
__device__ __forceinline__ void st_sys_g(gchar* p, u32 v) { __hip_atomic_store((__attribute__((address_space(1))) u32*)p, v, __ATOMIC_RELAXED, __HIP_MEMORY_SCOPE_SYSTEM); }
__device__ __forceinline__ void st_sys(void* p, u32 v) { __hip_atomic_store((u32*)p, v, __ATOMIC_RELAXED, __HIP_MEMORY_SCOPE_SYSTEM); }

#define STAMP(ws, idx) do { } while (0)

__device__ __forceinline__ bool give_up(char* ws, u32 call, u64 t0) {
    const u32 t = __builtin_amdgcn_readfirstlane(ld_agent(ws + OFF_TMO));
    if (t == call || __builtin_amdgcn_s_memrealtime() - t0 > DEADLINE) { st_agent(ws + OFF_TMO, call); return true; }
    return false;
}
__device__ __forceinline__ void poll_ge(const char* p, u32 need, char* ws, u32 call) {
    const u64 t0 = __builtin_amdgcn_s_memrealtime();
    u32 n = 0;
    for (;;) {
        const u32 v = ld_sys(p);
        if (__all((int)(v - need) >= 0)) return;
        if ((++n & 15u) == 0u && give_up(ws, call, t0)) return;
        __builtin_amdgcn_s_sleep(2);
    }
}
__device__ __forceinline__ bool all4_ge(u32x4 v, u32 need) {
    return __all((int)(v[0] - need) >= 0 && (int)(v[1] - need) >= 0 && (int)(v[2] - need) >= 0 && (int)(v[3] - need) >= 0);
}
__device__ __forceinline__ void poll4_ge(rsrc_t rs, u32 off, u32 need, char* ws, u32 call) {
    const u64 t0 = __builtin_amdgcn_s_memrealtime();
    u32 n = 0;
    for (;;) {
        asm volatile("" ::: "memory");
        const u32x4 v = __builtin_amdgcn_raw_buffer_load_b128(rs, off, 0, AUX_SYS);
        if (all4_ge(v, need)) return;
        if ((++n & 15u) == 0u && give_up(ws, call, t0)) return;
        __builtin_amdgcn_s_sleep(1);
    }
}

__device__ __forceinline__ u32 umax(u32 a, u32 b) { return a > b ? a : b; }
__device__ __forceinline__ u32 absbits(float f) { return __float_as_uint(f) & 0x7fffffffu; }
__device__ __forceinline__ int exp_of(u32 amaxbits) { int e = (int)(amaxbits >> 23); return e < 15 ? 15 : e; }
__device__ __forceinline__ float scale_of(int e) { return __uint_as_float((u32)(268 - e) << 23); }
__device__ __forceinline__ u32 cvt2(float a, float b, float s) {
    const f32x2 t = (f32x2){a, b} * s;
    return __builtin_bit_cast(u32, __builtin_convertvector(t, f16x2));
}
__device__ __forceinline__ u32x4 cvt8(f32x4 a, f32x4 b, float s) {
    return (u32x4){cvt2(a[0], a[1], s), cvt2(a[2], a[3], s), cvt2(b[0], b[1], s), cvt2(b[2], b[3], s)};
}
__device__ __forceinline__ u32 amax4(u32 m, f32x4 v) {
    asm("v_max3_f32 %0, |%1|, |%2|, %0\n\tv_max3_f32 %0, |%3|, |%4|, %0" : "+v"(m) : "v"(v[0]), "v"(v[1]), "v"(v[2]), "v"(v[3]));
    return m;
}

__device__ __forceinline__ u32 wave_max(u32 m) {
    m = umax(m, (u32)__builtin_amdgcn_update_dpp(0, (int)m, 0xb1, 15, 15, true));
    m = umax(m, (u32)__builtin_amdgcn_update_dpp(0, (int)m, 0x4e, 15, 15, true));
    m = umax(m, (u32)__builtin_amdgcn_update_dpp(0, (int)m, 0x141, 15, 15, true));
    m = umax(m, (u32)__builtin_amdgcn_update_dpp(0, (int)m, 0x140, 15, 15, true));
    return umax(umax((u32)__builtin_amdgcn_readlane((int)m, 0), (u32)__builtin_amdgcn_readlane((int)m, 16)),
                umax((u32)__builtin_amdgcn_readlane((int)m, 32), (u32)__builtin_amdgcn_readlane((int)m, 48)));
}
__device__ __forceinline__ u32 row_word(u32 call, int e) { return call | ((u32)(e - 141) & 255u); }

__device__ __forceinline__ void x_rows(const Params& p, LAS char* lds, int row0, u32 par, u32 call, int tid, int wave) {
    constexpr int NR = 4;
    const int a = p.rank & 3;
    char* ws = p.me;
    const rsrc_t rx = mk_rsrc(p.x, IN_BYTES), rw = mk_rsrc(ws, WS_NEED);
    const u32 pi0 = (u32)(wave * 128 + (tid & 63));
    f32x4 v[NR][4];
#pragma unroll
    for (int r = 0; r < NR; ++r)
#pragma unroll
        for (int u = 0; u < 2; ++u) {
            const u32 off = (u32)(row0 + r) * (KDIM * 4u) + (pi0 + (u32)u * 64u) * 32u;
            v[r][2 * u] = __builtin_bit_cast(f32x4, __builtin_amdgcn_raw_buffer_load_b128(rx, off, 0, 0));
            v[r][2 * u + 1] = __builtin_bit_cast(f32x4, __builtin_amdgcn_raw_buffer_load_b128(rx, off + 16u, 0, 0));
            WAIT_VM(X_LD - 2);
        }
    const int rb = row0 >> 7;
    const u32 slot = rb < 6 ? (u32)(a * 6 + rb) : (u32)(32 + rb - 6);
    const u32 rslot = rb < 6 ? slot : (u32)(24 + 2 * a + rb - 6);
    char* hp[4];
#pragma unroll
    for (int i = 0; i < 4; ++i) hp[i] = rb < 6 ? p.peer[i] : p.peer[4 + i];
    char* fifth = rb < 6 ? p.peer[7] : ws;
    asm volatile("" : "+s"(hp[0]), "+s"(hp[1]), "+s"(hp[2]), "+s"(hp[3]), "+s"(fifth));
    gchar* user = (gchar*)((tid & 4) ? fifth : (tid & 2) ? ((tid & 1) ? hp[3] : hp[2]) : ((tid & 1) ? hp[1] : hp[0]));
    const u32 wr = (u32)(tid & 63) >> 3;
    const bool wsend = wr < (u32)NR && ((tid & 7) < 4 || ((tid & 7) == 4 && rb >= 3 && rb != 5));
    const u32 woff = OFF_SA + ((par * 64u + ((tid & 4) ? (rb < 6 ? (u32)(31 + rb) : slot) : rslot)) * 128u + (((u32)row0 + wr) & 127u)) * 4u;
    u32 word = 0;
    LAS u32* red = (LAS u32*)(lds + L_RED);
#pragma unroll
    for (int r = 0; r < NR; ++r) {
        u32 m = 0;
#pragma unroll
        for (int i = 0; i < 4; ++i) m = amax4(m, v[r][i]);
        m = wave_max(m);
        if ((tid & 63) == 0) red[r * 8 + wave] = m;
    }
    WAIT_LGKM0(); BAR();
#pragma unroll
    for (int r = 0; r < NR; ++r) {
        const u32x4 m0 = *(LAS u32x4*)(red + r * 8), m1 = *(LAS u32x4*)(red + r * 8 + 4);
        const u32 m = umax(umax(umax(m0[0], m0[1]), umax(m0[2], m0[3])), umax(umax(m1[0], m1[1]), umax(m1[2], m1[3])));
        const int e = exp_of(m); const float s = scale_of(e);
        const u32 rin = (u32)((row0 + r) & 127);
        if (wr == (u32)r) word = row_word(call, e);
        const u32 off = CTRL_BYTES + par * PAR_BYTES + rin * 128u + (pi0 >> 3) * IMG + (((pi0 & 7u) ^ ((rin >> 1) & 7u)) << 4);
        __builtin_amdgcn_raw_buffer_store_b128(cvt8(v[r][0], v[r][1], s), rw, off + slot * PANEL, 0, AUX_SC1);
        if (X_ST == 1) WAIT_VM(0);
        __builtin_amdgcn_raw_buffer_store_b128(cvt8(v[r][2], v[r][3], s), rw, off + slot * PANEL + 8u * IMG, 0, AUX_SC1);
        WAIT_VM(X_ST < 2 ? 0 : X_ST - 2);
    }
    WAIT_VM(0); BAR();
    if (wave == 0 && wsend) st_sys_g(user + woff, word);
}

template <class F>
__device__ __forceinline__ gchar* x_head(const Params& p, LAS char* lds, int row, u32 prev, u32& calln, u32& hword, int tid, int wave, F mid) {
    constexpr int NS = HU > 3 ? HU : 3;
    const int lane = tid & 63;
    char* ws = p.me;
    const rsrc_t rx = mk_rsrc(p.x, IN_BYTES), rw = mk_rsrc(ws, WS_NEED);
    LAS u32* red = (LAS u32*)(lds + L_RED);
    if (wave != 7) {
        if (tid < 4) red[tid] = 0u;
        WAIT_LGKM0(); BAR();
        __builtin_amdgcn_s_sleep(XDLY);
        if (XSTAG) for (int i = 0; i < wave; ++i) __builtin_amdgcn_s_sleep(XSTAG);
    }
    const u32 s0 = wave == 7 ? 0u : (u32)wave, sd = wave == 7 ? 1u : 7u, se = wave == 7 ? (u32)HU : 16u;
    f32x4 v[NS][2];
#pragma unroll
    for (int u = 0; u < NS; ++u) {
        const u32 set = s0 + (u32)u * sd;
        const u32 off = set < se ? (u32)row * (KDIM * 4u) + (set * 64u + (u32)lane) * 32u : OOB;
        v[u][0] = __builtin_bit_cast(f32x4, __builtin_amdgcn_raw_buffer_load_b128(rx, off, 0, 0));
        v[u][1] = __builtin_bit_cast(f32x4, __builtin_amdgcn_raw_buffer_load_b128(rx, off + 16u, 0, 0));
        if (XPACE && u < NS - 1 && wave != 7) __builtin_amdgcn_s_sleep(XPACE);
    }
    __builtin_amdgcn_sched_barrier(0);
    if (wave == 7) BAR();
    const bool major = row < 128;
    const int a = p.rank & 3;
    char* hp[4];
#pragma unroll
    for (int i = 0; i < 4; ++i) hp[i] = major ? p.peer[i] : p.peer[4 + i];
    const u32 slot = major ? (u32)(a * 6) : 32u, rslot = major ? slot : (u32)(24 + 2 * a), rin = (u32)row & 127u;
    u32 off = CTRL_BYTES + rin * 128u + ((u32)lane >> 3) * IMG + ((((u32)lane & 7u) ^ ((rin >> 1) & 7u)) << 4);
    u32 m = 0;
    gchar* user = (gchar*)((tid & 4) ? ws : (tid & 2) ? ((tid & 1) ? hp[3] : hp[2]) : ((tid & 1) ? hp[1] : hp[0]));
#define WORD() do { u32 t = 0; \
        if (lane == 0) t = __hip_atomic_fetch_add(red + 2, 1u, __ATOMIC_RELAXED, __HIP_MEMORY_SCOPE_WORKGROUP); \
        if (__builtin_amdgcn_readfirstlane((int)t) == 7 && lane < (major ? 4 : 5)) \
            st_sys_g(user + OFF_SA + (((calln & 1u) * 64u + (lane == 4 ? slot : rslot)) * 128u + rin) * 4u, \
                     row_word(calln << 8, exp_of(__hip_atomic_load(red, __ATOMIC_RELAXED, __HIP_MEMORY_SCOPE_WORKGROUP)))); } while (0)
    if (wave == 7) {
        rsrc_t hr[4];
#pragma unroll
        for (int i = 0; i < 4; ++i) hr[i] = mk_rsrc(hp[i], i == 3 && major ? 0u : WS_NEED);
        u32 o4[4];
#pragma unroll
        for (int i = 0; i < 4; ++i) o4[i] = (lane >> 4) == i ? off + rslot * PANEL : OOB;
        asm volatile("" : "+s"(hr[0]), "+s"(hr[1]), "+s"(hr[2]), "+s"(hr[3]), "+v"(o4[0]), "+v"(o4[1]), "+v"(o4[2]), "+v"(o4[3]), "+v"(m), "+v"(prev));
        calln = (u32)__builtin_amdgcn_readfirstlane(prev) + 1u;
        const u32 so = (calln & 1u) * PAR_BYTES;
#pragma unroll
        for (int u = 0; u < HU; ++u) m = amax4(amax4(m, v[u][0]), v[u][1]);
        m = wave_max(m);
        const int e = exp_of(m); const float s = scale_of(e);
        u32x4 h;
#pragma unroll
        for (int k = 0; k < 4 * HU; ++k) {
            if ((k & 3) == 0) h = cvt8(v[k >> 2][0], v[k >> 2][1], s);
            if (k == 2 && lane == 0) __hip_atomic_fetch_add(red + 1, 1u, __ATOMIC_RELAXED, __HIP_MEMORY_SCOPE_WORKGROUP);
#pragma unroll
            for (int i = 0; i < 4; ++i) {
                if (HSTEP ? i == 0 && 4 * k >= HWIN : 4 * k + i >= HWIN) WAIT_VM(HWIN - (HSTEP ? 4 : 1));
                __builtin_amdgcn_raw_buffer_store_b128(h, hr[i], o4[k & 3], so + (u32)(k >> 2) * 8u * IMG, AUX_SYS);
            }
            asm volatile("s_nop 1" : "+v"(h) :: "memory");
            if (k == HK) WORD();
        }
        hword = row_word(calln << 8, e);
        for (u32 n = 0; n < (1u << 24); ++n)
            if (__builtin_amdgcn_readfirstlane((int)__hip_atomic_load(red + 1, __ATOMIC_RELAXED, __HIP_MEMORY_SCOPE_WORKGROUP)) == 8) break;
    }
    else calln = (u32)__builtin_amdgcn_readfirstlane(prev) + 1u;
    const u32 par = calln & 1u;
    if (wave != 7) {
        mid();
        off += par * PAR_BYTES + slot * PANEL;
#pragma unroll
        for (int u = 0; u < 3; ++u) m = amax4(amax4(m, v[u][0]), v[u][1]);
        m = wave_max(m);
        if (lane == 0) {
            __hip_atomic_fetch_max(red, m, __ATOMIC_RELAXED, __HIP_MEMORY_SCOPE_WORKGROUP);
            __hip_atomic_fetch_add(red + 1, 1u, __ATOMIC_RELAXED, __HIP_MEMORY_SCOPE_WORKGROUP);
        }
        u64 mc;
        for (u32 n = 0; n < (1u << 24); ++n) {
            mc = __hip_atomic_load((LAS u64*)red, __ATOMIC_RELAXED, __HIP_MEMORY_SCOPE_WORKGROUP);
            if ((u32)__builtin_amdgcn_readfirstlane((int)(mc >> 32)) == 8u) break;
        }
        const int e = exp_of((u32)mc); const float s = scale_of(e);
#pragma unroll
        for (int u = 0; u < 3; ++u) {
            const u32 set = s0 + (u32)u * 7u;
            __builtin_amdgcn_raw_buffer_store_b128(cvt8(v[u][0], v[u][1], s), rw, set < 16u ? off + set * 8u * IMG : OOB, 0, AUX_SC1);
        }
        WAIT_VM(0);
        WORD();
    }
#undef WORD
    return user + OFF_RD + ((par * 8u + (u32)p.rank) * 128u + rin) * 4u;
}

__device__ __forceinline__ int slot_row(int rank, int slot) {
    const int g = rank >> 2;
    if (slot < 24) { const int i = slot / 6; return (g * 4 + i) * 1024 + (slot - i * 6) * 128; }
    return (slot < 32 ? (1 - g) * 4 + ((slot - 24) >> 1) : rank) * 1024 + 768 + (slot & 1) * 128;
}

struct Tile { int slot, ct, rh, ch; };
__device__ __forceinline__ void a_stream(int rank, int slot, int rh, int& a_src, u32& a_base, u32& a_sh) {
    const int g = rank >> 2, a = rank & 3;
    if (slot < 24) { const int i = slot / 6, a_rb = slot - i * 6; a_src = i == a ? -1 : g * 4 + i;
        a_base = a_rb == 0 ? 0u : a_rb < 3 ? 16u : a_rb < 5 ? 48u : 80u + (u32)rh * 8u; a_sh = a_rb == 0 ? 1u : 0u; }
    else { a_src = slot < 32 ? (1 - g) * 4 + ((slot - 24) >> 1) : -1; a_base = (slot & 1) ? 48u : 0u; a_sh = 1u; }
}
__device__ __forceinline__ u32 start_off(int rank, const Tile& t, bool big, u32 par, int lane, int wave) {
    int a_src; u32 a_base, a_sh;
    a_stream(rank, t.slot, t.rh, a_src, a_base, a_sh);
    const int b_src = t.ct < 8 ? -1 : (1 - (rank >> 2)) * 4 + ((t.ct - 8) >> 1);
    u32 o = OOB, m = 63u;
    if (wave == 0) { if (a_src >= 0) { o = OFF_LF + ((u32)a_src * NUNITS + (a_base ? a_base : (u32)HU)) * 128u; m = big ? 7u : 3u; } }
    else if (wave == 1) { if (a_src < 0) { o = OFF_SA + (par * 64u + (u32)t.slot) * 512u; m = 31u; } }
    else if (wave == 2) { if (b_src < 0) { o = OFF_WR + (u32)t.ct * 128u; m = 7u; } }
    else if (wave == 3) { if (a_src >= 0 && a_base == 0u) { o = OFF_RD + (par * 8u + (u32)a_src) * 512u; m = 31u; } }
    else if (b_src >= 0 && (!big || wave == 4)) { o = OFF_LF + ((u32)b_src * NUNITS + 16u) * 128u + (u32)(wave - 4) * 1024u; m = big ? 7u : 63u; }
    return o + ((u32)lane & m) * 16u;
}

template <int MODE>
__device__ __forceinline__ void gemm_tile(const Params& p, LAS char* lds, const Tile& tl, bool first, const Tile& nx, bool nbig, u32 par, u32 call, int tid, int lane, int wave, int stamp0, char*& pend) {
    constexpr bool BIG = MODE == 0;
    constexpr int NB = BIG ? 4 : 2, NC = BIG ? 32 : 16, BT = BIG ? 128 : 64, NR = MODE == 2 ? NF : MODE == 1 ? NFL : 0;
    const int rank = p.rank, g = rank >> 2, a = rank & 3;
    const int slot = tl.slot, ct = tl.ct, rh = tl.rh, ch = tl.ch;
    asm volatile("" : "+v"(lane), "+v"(tid));
    char* ws = p.me;
    const rsrc_t rws = mk_rsrc(ws, WS_NEED);
    int a_src; u32 a_base, a_sh;
    a_stream(rank, slot, rh, a_src, a_base, a_sh);
    const int b_src = ct < 8 ? -1 : (1 - g) * 4 + ((ct - 8) >> 1);
    const int src = lane < 32 ? a_src : (BIG ? b_src : -1);
    const u32 gsh = src < 0 ? 31u : lane < 32 && BIG ? a_sh : 0u;
    const char* gptr = ws + (src >= 0 ? OFF_LF + ((u32)src * NUNITS + (lane < 32 ? a_base : 16u)) * 128u + (u32)(lane & (BIG ? 31 : 15)) * 4u : OFF_CTR + blockIdx.x * 64u + 4u);
    const u32 gmin = src >= 0 && lane < 32 && a_base == 0u ? (u32)HU : 0u;
    const bool headt = BIG && a_src >= 0 && a_base == 0u;
#define GPTR(c) (gptr + (umax((u32)(c) >> gsh, gmin) << (BIG ? 7 : 6)))
#define PEND() do { if (pend && wave == 1 && lane < 2) st_sys(pend + (u32)lane * 4u, call); pend = nullptr; } while (0)
    LAS u32* okw = (LAS u32*)(lds + L_RED + 1024u);
    const u32 pb = CTRL_BYTES + par * PAR_BYTES;
    const int ks = BIG ? wave >> 2 : wave >> 1, wm = BIG ? (wave >> 1) & 1 : 0, wn = wave & 1;
    const u32 swz = (u32)((((ks & 1) * 4 + (lane >> 4)) ^ ((lane & 15) >> 1)) << 4);
    f16x8 bt[4 * NR + 1][2];
    WAIT_LGKM0(); BAR();
    u32 po = OOB;
    {
        const u32x4 k0 = *(const LAS u32x4*)okw, k1 = *(const LAS u32x4*)(okw + 4);
        if (first || __builtin_amdgcn_readfirstlane((int)(k0[0] | k0[1] | k0[2] | k0[3] | k1[0] | k1[1] | k1[2] | k1[3]))) {
            if (pend) { WAIT_VM(0); BAR(); PEND(); }
            po = start_off(rank, tl, BIG, par, lane, wave);
            if ((!NR || wave != 0) && __builtin_amdgcn_readfirstlane((int)po) >= 0) poll4_ge(rws, po, call, ws, call);
            WAIT_VM_TRACKED(0); BAR();
        }
    }
    if (!NR && wave == 0) STAMP(ws, stamp0);
    constexpr u32 RB = BT * 4u, PPR = BT / 4u, AREA = BT * RB;
    const u32 pc = (u32)lane & (PPR - 1u);
    const u32 col0 = BIG ? 0u : (u32)ch * 64u, row0 = BIG ? 0u : (u32)rh * 64u;
    const bool remote = BIG ? ct >= 8 : MODE == 2;
    const rsrc_t ro = mk_rsrc(remote ? (const void*)ws : (const void*)p.out, remote ? WS_NEED : IN_BYTES);
    const u32 ostr = remote ? 1024u : NLOC * 4u;
    const u32 obase = (remote ? OFF_YST + (u32)(((ct - 8) >> 1) * 32 + slot) * UNIT + (u32)(ct & 1) * 512u : ((u32)slot_row(rank, slot) * NLOC + (u32)ct * 128u) * 4u) + col0 * 4u + row0 * ostr + pc * 16u;
#define YFLAG() do { if (tid == 0) st_sys(p.ws[b_src] + OFF_YR + (u32)(a * 64 + rh * 32 + slot) * 16u + (u32)(ct & 1) * 8u + (u32)ch * 4u, call); } while (0)
    constexpr bool hint = MODE == 2;
    if (hint) {
        const u32x4 zero = {0u, 0u, 0u, 0u};
#pragma unroll
        for (u32 it = 0; it < 2u; ++it) __builtin_amdgcn_raw_buffer_store_b128(zero, ro, obase + ((u32)wave * (BT / 8u) + (it * 64u + (u32)lane) / PPR) * ostr, 0, AUX_SC1);
    }
    const u32 gA = pb + (u32)slot * PANEL + (BIG ? (u32)wave * 2048u : (u32)rh * 8192u + (u32)wave * 1024u);
    const u32 gB = pb + (u32)(ASLOTS + ct) * PANEL + (BIG ? (u32)wave * 2048u : (u32)ch * 8192u + (u32)wave * 1024u);
    const u32 l16 = (u32)lane * 16u;
    const u32 ldsw = (u32)wave * (BIG ? 2048u : 1024u);
    constexpr u32 GSTEP = BIG ? IMG : 2u * IMG, G2 = BIG ? 1024u : IMG, L2O = BIG ? 1024u : 8192u;
#define SCHED() __builtin_amdgcn_sched_barrier(0)
#define GLDS16(g, l) __builtin_amdgcn_raw_ptr_buffer_load_lds(rws, (LAS void*)(lds + (l)), 16, l16, (g), 0, GL_AUX)
#define LOADT(t, st) do { const u32 _o = (u32)(t) * GSTEP; \
        GLDS16(gA + _o, (st) * STAGE + ldsw); GLDS16(gA + _o + G2, (st) * STAGE + ldsw + L2O); \
        GLDS16(gB + _o, (st) * STAGE + IMG + ldsw); GLDS16(gB + _o + G2, (st) * STAGE + IMG + ldsw + L2O); } while (0)
#define SNAP(c) __builtin_amdgcn_global_load_lds((const u32*)((c) < NC ? GPTR(c) : ws + OFF_TMO), (LAS u32*)(lds + L_SNAP + ((u32)(c) & 1u) * 256u), 4, 0, AUX_SYS)
    const u32 swl = swz + (BIG ? 0u : (u32)(ks >> 1) * 8192u);
    const u32 ard = (u32)(wm * 64 + (lane & 15)) * 128u + swl, brd = IMG + (u32)(wn * (BT / 2) + (lane & 15)) * 128u + swl;
#define READF(fa, fb, st) READX(fa, fb, st, true)
#define READX(fa, fb, st, rb) do { _Pragma("unroll") for (int _i = 0; _i < 4; ++_i) { \
        if (_i < NB && (rb)) fb[_i] = *(const LAS f16x8*)(lds + (st) * STAGE + brd + _i * 2048u); \
        fa[_i] = *(const LAS f16x8*)(lds + (st) * STAGE + ard + _i * 2048u); } } while (0)
#define MMA(fa, fb) do { SCHED(); _Pragma("unroll") for (int _m = 0; _m < 4; ++_m) _Pragma("unroll") for (int _n = 0; _n < NB; ++_n) \
        acc[_m][_n] = __builtin_amdgcn_mfma_f32_16x16x32_f16(fb[_n], fa[_m], acc[_m][_n], 0, 0, 0); SCHED(); } while (0)
#define SB() __builtin_amdgcn_sched_barrier(0)
#define STEP(t, sl, na, nb, sr, fa, fb) STEPX(t, sl, na, nb, sr, fa, fb, true, true)
#define STEPX(t, sl, na, nb, sr, fa, fb, lb, rb) do { const u32 _o = (u32)(t) * GSTEP; \
        _Pragma("unroll") for (int _m = 0; _m < 4; ++_m) { \
            _Pragma("unroll") for (int _n = 0; _n < NB / 2; ++_n) acc[_m][_n] = __builtin_amdgcn_mfma_f32_16x16x32_f16(fb[_n], fa[_m], acc[_m][_n], 0, 0, 0); \
            SB(); \
            if (_m < 2) { _Pragma("unroll") for (int _i = 0; _i < 2; ++_i) { \
                if (_i < NB / 2 && (rb)) nb[_m * (NB / 2) + _i] = *(const LAS f16x8*)(lds + (sr) * STAGE + brd + (_m * (NB / 2) + _i) * 2048u); \
                na[_m * 2 + _i] = *(const LAS f16x8*)(lds + (sr) * STAGE + ard + (_m * 2 + _i) * 2048u); } } \
            if (_m < 2 || (lb)) GLDS16((_m < 2 ? gA : gB) + _o + (_m & 1) * G2, (sl) * STAGE + (_m >> 1) * IMG + ldsw + (_m & 1) * L2O); \
            SB(); \
            _Pragma("unroll") for (int _n = NB / 2; _n < NB; ++_n) acc[_m][_n] = __builtin_amdgcn_mfma_f32_16x16x32_f16(fb[_n], fa[_m], acc[_m][_n], 0, 0, 0); \
        } SB(); } while (0)
    f32x4 acc[4][NB];
#pragma unroll
    for (int m = 0; m < 4; ++m)
#pragma unroll
        for (int n = 0; n < NB; ++n) acc[m][n] = (f32x4){0.f, 0.f, 0.f, 0.f};
    f16x8 a0[4], b0[4], a1[4], b1[4];
#define VLOAD(v) do { const u32 _o = (u32)(4 * (NC - NR) + 2 * (v)) * GSTEP, _l = (u32)(((v) + 2 * NR) & 3) * STAGE + ldsw; \
        GLDS16(gB + _o, _l); GLDS16(gB + _o + G2, _l + L2O); GLDS16(gB + _o + GSTEP, _l + IMG); GLDS16(gB + _o + GSTEP + G2, _l + IMG + L2O); } while (0)
#define VREAD(v) do { _Pragma("unroll") for (int _i = 0; _i < 4; ++_i) \
        bt[NR ? 2 * (v) + (_i >> 1) : 0][_i & 1] = *(const LAS f16x8*)(lds + (u32)(((v) + 2 * NR) & 3) * STAGE + brd - (_i < 2 ? IMG : 0u) + (_i & 1) * 2048u); } while (0)
#define VSTEP(v) do { if (NR) { WAIT_VM(8); WAIT_LGKM0(); BAR(); VREAD(v); } } while (0)
    if (NR) {
        VLOAD(0); VLOAD(1); VLOAD(2); VLOAD(3);
        WAIT_VM(12); BAR();
        PEND();
        VREAD(0);
#pragma unroll
        for (int v = 4; v < 2 * NR; ++v) { VSTEP(v - 3); VLOAD(v); }
        if (wave == 0 && __builtin_amdgcn_readfirstlane((int)po) >= 0) { poll4_ge(rws, po, call, ws, call); WAIT_VM_TRACKED(0); }
        VSTEP(2 * NR - 3);
        if (wave == 0) STAMP(ws, stamp0);
    }
    if (wave >= 4) {
        const u32 so = (wave < 6 ? OFF_SA + (par * 64u + (u32)slot) * 512u : OFF_SB + (par * BSLOTS + (u32)ct) * 512u) + (u32)((wave & 1) * 64 + lane) * 4u;
        __builtin_amdgcn_global_load_lds((const u32*)(ws + so), (LAS u32*)(lds + L_SC + (u32)(wave - 4) * 256u), 4, 0, AUX_SYS);
    }
    else if (wave >= 2 && headt)
        __builtin_amdgcn_global_load_lds((const u32*)(ws + OFF_RD + (par * 8u + (u32)a_src) * 512u + (u32)((wave & 1) * 64 + lane) * 4u), (LAS u32*)(lds + L_RED + (u32)(wave - 2) * 256u), 4, 0, AUX_SYS);
    LOADT(0, 0u);
    if (wave == 0) SNAP(1);
    VSTEP(2 * NR - 2);
    LOADT(1, 1u);
    VSTEP(2 * NR - 1);
    LOADT(2, 2u);
    if (NR) { WAIT_VM(8); WAIT_LGKM0(); BAR(); LOADT(3, 3u); }
    else {
        LOADT(3, 3u);
        if (wave == 0) WAIT_VM(13); else WAIT_VM(12);
        BAR();
        PEND();
    }
    READF(a0, b0, 0u);
#define GATE(c) do { const u32 _v = *(const LAS u32*)(lds + L_SNAP + ((u32)(c) & 1u) * 256u + (u32)lane * 4u); \
        if (!__all((int)(_v - call) >= 0)) { if (wave == 0) { poll_ge(GPTR(c), call, ws, call); WAIT_VM(0); } BAR(); } } while (0)
    bool seen = false;
#define FP(i) do { __builtin_amdgcn_global_load_lds((const u32*)_gp, (LAS u32*)(lds + L_RED + (i) * 256u), 4, 0, AUX_SYS); __builtin_amdgcn_s_sleep(FSLEEP); } while (0)
#define FS(i) { FP(((i) + 3) & 3); WAIT_VM(3); u32 _w; \
            asm volatile("ds_read_b32 %0, %1 offset:%2\n\ts_waitcnt lgkmcnt(0)" : "=v"(_w) : "v"(_pa), "n"((i) * 256) : "memory"); \
            if (__all((int)(_w - call) >= 0)) break; }
#define FGATE(c) do { const u32 _v = *(const LAS u32*)(lds + L_SNAP + ((u32)(c) & 1u) * 256u + (u32)lane * 4u); \
        seen = false; \
        if (!__all((int)(_v - call) >= 0)) { if (wave == 0) { seen = true; \
            const char* _gp = GPTR(c); const u32 _pa = (u32)(size_t)(lds + L_RED) + (u32)lane * 4u; const u64 _t0 = __builtin_amdgcn_s_memrealtime(); \
            FP(0); FP(1); FP(2); \
            for (u32 _n = 1;; ++_n) { FS(0) FS(1) FS(2) FS(3) if ((_n & 15u) == 0u && give_up(ws, call, _t0)) break; } \
        } BAR(); } } while (0)
    for (int c = 0; c < NC - 1 - NR; ++c) {
        const int t = 4 * c;
        WAIT_VM(8); WAIT_LGKM0(); BAR();
        if (BIG || !FGALL) GATE(c + 1); else FGATE(c + 1);
        STEP(t + 4, 0u, a1, b1, 1u, a0, b0);
        WAIT_VM(8); WAIT_LGKM0(); BAR();
        if (wave == 0) SNAP(c + 2);
        STEP(t + 5, 1u, a0, b0, 2u, a1, b1);
        if (wave == 0) WAIT_VM(9); else WAIT_VM(8);
        WAIT_LGKM0(); BAR();
        STEP(t + 6, 2u, a1, b1, 3u, a0, b0);
        if (wave == 0) WAIT_VM(9); else WAIT_VM(8);
        WAIT_LGKM0(); BAR();
        STEP(t + 7, 3u, a0, b0, 0u, a1, b1);
        if (BIG && c == 2 * HU - 1 && headt) {
#pragma unroll
            for (int m = 0; m < 4; ++m) {
                const u32 r = (u32)(wm * 64 + m * 16 + (lane & 15)) * 4u;
                const int d = (int)*(const LAS signed char*)(lds + L_RED + r) - (int)*(const LAS signed char*)(lds + L_SC + r);
#pragma unroll
                for (int n = 0; n < NB; ++n)
#pragma unroll
                    for (int j = 0; j < 4; ++j) acc[m][n][j] = __builtin_amdgcn_ldexpf(acc[m][n][j], d);
            }
        }
    }
#define TS(i) do { } while (0)
#define FB(j, k, b) ((j) ? bt[(j) ? 4 * (j) - 4 + (k) : 0] : (b))
#pragma unroll
    for (int j = 0; j < NR; ++j) {
        const int c = NC - 1 - NR + j, t = 4 * c;
        if (j) WAIT_VM(4); else WAIT_VM(8);
        WAIT_LGKM0(); BAR();
        TS(2 * j);
        FGATE(c + 1);
        TS(2 * j + 1);
        STEPX(t + 4, 0u, a1, b1, 1u, a0, FB(j, 0, b0), false, !j);
        if (seen) { } else if (j) WAIT_VM(4); else WAIT_VM(6);
        WAIT_LGKM0(); BAR();
        if (wave == 0) SNAP(c + 2);
        STEPX(t + 5, 1u, a0, b0, 2u, a1, FB(j, 1, b1), false, !j);
        if (seen) { } else if (wave == 0) WAIT_VM(5); else WAIT_VM(4);
        WAIT_LGKM0(); BAR();
        STEPX(t + 6, 2u, a1, b1, 3u, a0, FB(j, 2, b0), false, !j);
        if (wave == 0) WAIT_VM(5); else WAIT_VM(4);
        WAIT_LGKM0(); BAR();
        STEPX(t + 7, 3u, a0, b0, 0u, a1, FB(j, 3, b1), false, false);
    }
#define DSTEP(k, n) do { if ((k) > HINT && hint && wave == 0) WAIT_VM((n) + 1); else WAIT_VM(n); WAIT_LGKM0(); BAR(); if ((k) == HINT && hint) YFLAG(); } while (0)
    TS(8);
    DSTEP(0, NR ? 4 : 8);
    TS(9);
    READX(a1, b1, 1u, !NR); MMA(a0, FB(NR, 0, b0));
    DSTEP(1, NR ? 2 : 4);
    READX(a0, b0, 2u, !NR); MMA(a1, FB(NR, 1, b1));
    DSTEP(2, 0);
    TS(10);
    READX(a1, b1, 3u, !NR); MMA(a0, FB(NR, 2, b0));
    WAIT_LGKM0(); BAR();
    if (HINT == 3 && hint) YFLAG();
    u32 npo = OOB; u32x4 nv = {0u, 0u, 0u, 0u};
    if (BIG && nx.slot >= 0) { npo = start_off(rank, nx, nbig, par, lane, wave); nv = __builtin_amdgcn_raw_buffer_load_b128(rws, npo, 0, AUX_SYS); }
    MMA(a1, FB(NR, 3, b1));
#undef DSTEP
#undef FB
#undef FGATE
#undef FS
#undef FP
    if (wave == 0) STAMP(ws, stamp0 + 1);
#pragma unroll
    for (int m = 0; m < 4; ++m)
#pragma unroll
        for (int n = 0; n < NB; ++n) {
            const u32 r = (u32)(wm * 64 + m * 16 + (lane & 15)), pc = (u32)(wn * (BT / 8) + n * 4 + (lane >> 4));
            *(LAS f32x4*)(lds + (u32)ks * AREA + r * RB + ((pc ^ (r & 7u)) << 4)) = acc[m][n];
        }
    WAIT_LGKM0(); BAR();
    if (MODE == 1 && LSL) __builtin_amdgcn_s_sleep(LSL);
    const LAS int* sc = (const LAS int*)(lds + L_SC);
    const i32x4 eb = *(const LAS i32x4*)(sc + 128 + col0 + pc * 4u);
    const bool bad = *(const LAS u32*)(lds + L_SNAP) == call;
    if (BIG) { WAIT_VM_TRACKED(0); const bool ok = (int)npo < 0 || all4_ge(nv, call); if (lane == 0) okw[wave] = ok ? 0u : 1u; }
#pragma unroll
    for (u32 it = 0; it < (BIG ? 8u : 2u); ++it) {
        const u32 r = (u32)wave * (BT / 8u) + ((BIG && EROT ? (it + (blockIdx.x >> 3)) & 7u : it) * 64u + (u32)lane) / PPR;
        const u32 lo = r * RB + ((pc ^ (r & 7u)) << 4);
        f32x4 s = *(const LAS f32x4*)(lds + lo) + *(const LAS f32x4*)(lds + AREA + lo);
        if (!BIG) s += *(const LAS f32x4*)(lds + 2u * AREA + lo) + *(const LAS f32x4*)(lds + 3u * AREA + lo);
        const int ea = (int)(signed char)sc[row0 + r];
#pragma unroll
        for (int j = 0; j < 4; ++j) s[j] = bad ? __builtin_nanf("") : __builtin_amdgcn_ldexpf(s[j], ea + eb[j]);
        u32x4 o = __builtin_bit_cast(u32x4, s);
        if (hint) {
#pragma unroll
            for (int j = 0; j < 4; ++j) o[j] = o[j] ? o[j] : 0x80000000u;
        }
        __builtin_amdgcn_raw_buffer_store_b128(o, ro, obase + r * ostr, 0, AUX_SC1);
        if (BIG && EPW && it < 7u && (first || EALL)) WAIT_VM(EPW ? EPW - 1 : 0);
        if (BIG && EPS && it < 7u && first) __builtin_amdgcn_s_sleep(EPS);
    }
    if (remote && BIG) pend = p.ws[b_src] + OFF_YR + (u32)(a * 64 + slot) * 16u + (u32)(ct & 1) * 8u;
    if (wave == 0) STAMP(ws, stamp0 + 2);
#undef TS
#undef GLDS16
#undef SCHED
#undef SB
#undef STEP
#undef STEPX
#undef READX
#undef VLOAD
#undef VREAD
#undef VSTEP
#undef LOADT
#undef SNAP
#undef READF
#undef MMA
#undef GATE
#undef YFLAG
#undef PEND
#undef GPTR
}

constexpr u32 NSK = DELTA - (DCUT ? 1u : 0u);
constexpr u32 NPOS_INTRA = 96u - HU - NSK, NPOS_CROSS = 97u - HU + DELTA;
constexpr u32 FAN = 0x20000000u;
struct Desc { u32 src, dst, gate, flag; };
struct Stream {
    bool cross; int rank, g, a, j; u32 pb, sa;
    __device__ __forceinline__ int yslot(u32 k) const {
        if (k < 6u) return j * 6 + (int)k;
        if (k < 9u) return ((j + (int)k - 5) & 3) * 6;
        if (k < 12u) return 24 + 2 * ((a + (int)k - 8) & 3);
        if (k < 18u) return ((j + 1 + (((int)k - 12) >> 1)) & 3) * 6 + 1 + ((int)k & 1);
        if (k < 21u) return 25 + 2 * ((a + (int)k - 17) & 3);
        if (k < 27u) return ((j + 1 + (((int)k - 21) >> 1)) & 3) * 6 + 3 + (((int)k - 21) & 1);
        return ((j + 1 + (int)(k % 3u)) & 3) * 6 + 5;
    }
    __device__ __forceinline__ u32 single(u32 slot, u32 k) const { return pb + slot * PANEL + k * UNIT; }
    __device__ __forceinline__ u32 pair(u32 slot, u32 k) const { return pb + slot * PANEL + k * 65536u; }
    __device__ __forceinline__ u32 half(u32 slot, u32 h, u32 k) const { return pb + slot * PANEL + k * 2u * UNIT + h * 8192u; }
    __device__ __forceinline__ Desc at(u32 pos) const {
        Desc d;
        const u32 s = (u32)j * 6u, M8 = 7u << 20, PAIR = 1u << 28, HALF = 4u << 28, ONE = M8 + (6u << 28);
        u32 u, fsrc = (u32)((cross ? 1 - g : g) * 4 + j);
        if (!cross) {
            if (pos >= NPOS_INTRA) pos = NPOS_INTRA - 1u;
            u = pos + HU;
            if (u >= RU0) u += NSK;
            if (u < 80u) {
                d.src = u < 16u ? single(s, u) : u < 48u ? pair(s + 1u, u - 16u) : pair(s + 3u, u - 48u);
                d.gate = sa + s * 512u + (u < ROWU ? 0u : u < 16u ? ONE : u < 48u ? 512u + PAIR + M8 : 1536u + PAIR + M8);
            } else {
                const u32 e = 95u - u, f = e - LEAD;
                const u32 h = e < LEAD || (e < 16u - LEAD && (f & 1u)) ? 1u : 0u;
                const u32 k = e < LEAD ? 7u - e : e >= 16u - LEAD ? 15u - e : h ? 7u - LEAD - (f >> 1) : 7u - (f >> 1);
                u = 80u + h * 8u + k;
                d.src = half(s + 5u, h, k);
                d.gate = sa + (s + 5u) * 512u + h * 256u + M8 + HALF;
            }
            d.dst = d.src;
            d.flag = OFF_LF + (fsrc * NUNITS + u) * 128u;
        } else {
            if (pos >= NPOS_CROSS) pos = NPOS_CROSS - 1u;
            pos += HU;
            const u32 hop = j == a ? HOP1 : HOP2;
            u = pos < hop ? pos : pos - DELTA;
            if (pos - hop < DELTA) {
                const u32 k = pos - hop;
                if (j == a) { d.src = pair(s + 3u, RU0 - 48u + k); d.dst = pair(RELAY, k); d.gate = sa + 34u * 512u + M8 + PAIR;
                              d.flag = FAN + OFF_RF + ((u32)a * 4u + k) * 128u; }
                else { d.src = pair(RELAY, k); d.dst = pair(s + 3u, RU0 - 48u + k); d.gate = OFF_RF + ((u32)j * 4u + k) * 128u + (5u << 28);
                       d.flag = OFF_LF + ((fsrc ^ 4u) * NUNITS + RU0 + k) * 128u; }
            } else if (u < 64u) {
                d.src = u < 16u ? single(32u, u) : u < 48u ? pair((u32)(ASLOTS + 2 * a), u - 16u) : single(33u, u - 48u);
                d.dst = u < 16u ? single((u32)(24 + 2 * j), u) : u < 48u ? pair((u32)(ASLOTS + 8 + 2 * j), u - 16u) : single((u32)(25 + 2 * j), u - 48u);
                d.gate = u < 16u ? sa + (u32)(24 + 2 * j) * 512u + (u < ROWU ? 0u : ONE) : u < 48u ? OFF_WS + (u32)j * 256u + (1u << 20) + (3u << 28) : sa + (u32)(25 + 2 * j) * 512u + (R7C ? ONE : 0u);
                d.flag = OFF_LF + (fsrc * NUNITS + u) * 128u;
            } else {
                const u32 k = u - 64u;
                const int ys = yslot(k);
                const u32 h = k >= 30u ? 1u : 0u;
                d.src = OFF_YST + (u32)(a * 32 + ys) * UNIT + h * 65536u;
                d.dst = ((u32)slot_row((int)fsrc, ys) + h * 64u) * (NLOC * 4u) + (u32)j * 1024u;
                d.gate = OFF_YR + ((u32)(j * 64 + ys) + h * 32u) * 16u + ((k >= 27u + Y4 ? 10u : 2u) << 28);
                d.flag = NOFLAG;
            }
        }
        return d;
    }
};
struct Tables { Desc lo, hi; };
__device__ __forceinline__ u32 tab(u32 t0, u32 t1, u32 i) {
    const u32 x = (u32)__builtin_amdgcn_readlane((int)t0, (int)(i & 63u)), y = (u32)__builtin_amdgcn_readlane((int)t1, (int)(i & 63u));
    return i < 64u ? x : y;
}

#define PSLEEP0 10
#define PSLEEP 4
#define NACT 32
__device__ __forceinline__ void pull_stream(const Params& p, LAS char* lds, const Tables& T, bool cross, u32 cw, char* src, gchar* fan, u32 calln, u32 call, int lane, int wave, int hw) {
    char* ws = p.me;
    const rsrc_t rl = mk_rsrc(ws, WS_NEED), rs = mk_rsrc(src, WS_NEED), rout = mk_rsrc(p.out, IN_BYTES);
    LAS char* pl = lds + (u32)hw * 1024u;
    const u32 pla = (u32)(size_t)pl + (u32)lane * 4u;
    const u32 l16 = (u32)lane * 16u, l4r = (u32)(lane >> 5) * IMG + (l16 & 511u);
    const u32 l0 = lane == 0 ? 0u : OOB;
    const u32 co = OFF_CL + (u32)wave * 128u + l0;
    const u32 NPIECES = (cross ? NPOS_CROSS : NPOS_INTRA) * 32u;
    const u32 skip = !DCUT ? 0u : cross ? (u32)DCUT : 32u - DCUT;
    const u32 qcut = cross ? ((wave == 3 ? HOP1 : HOP2) + DELTA - HU) * 32u - DCUT : (RU0 - HU) * 32u;
    const u32 qt = cross ? (NPOS_CROSS - 6u) * 32u : OOB;
    const u32 cbase = 2u * NACT - (calln - 1u) * (NPIECES - skip - (cross ? Y4 * 16u : 0u));
    u32x4 v0 = {0u, 0u, 0u, 0u}, v1 = v0, v2 = v0, v3;
    u32 q = OOB, so = OOB, ss = 1024u;
    u32 c0 = cw, c1 = cw + NACT;
    u32 pf = NOFLAG;
    bool claimed = false, dead = false;
    int cl = 0;
#define KIND(q) (tab(T.lo.gate, T.hi.gate, (q) >> 5) >> 28)
#define POFF(q, kd) (((kd) & 1u) ? (((q) >> 4) & 1u) * PANEL + ((q) & 15u) * 4096u : (kd) == 4u ? (((q) >> 1) & 15u) * IMG + ((q) & 1u) * 4096u : ((q) & 31u) * (((kd) & 8u) ? 2048u : (kd) ? 4096u : 512u))
#define GBASE(q) ({ const u32 _g = tab(T.lo.gate, T.hi.gate, (q) >> 5), _k = _g >> 28; \
        const u32 _ix = (((_k & 1u) ? ((q) >> 2) & 4u : 0u) + ((q) & (_k == 4u ? 1u : 3u))) << (_k == 3u ? 5 : 7); \
        (_g & 0xfffffu) + ((_k & 7u) == 2u ? 0u : _k == 5u ? ((q) & 28u) * 4u : _k == 0u ? ((q) & 31u) * 16u : _ix); })
#define GOFF(q) (GBASE(q) + (l16 & (tab(T.lo.gate, T.hi.gate, (q) >> 5) >> 16)))
#define PFLAG() st_sys_g((pf & FAN) ? fan + (pf - FAN) : (gchar*)ws + (pf >= NOFLAG ? OFF_CTR + cw * 64u + 8u + (u32)hw * 4u : pf), call)
#define PLD(u, o) v##u = __builtin_amdgcn_raw_buffer_load_b128(rs, (o) + (u) * ss, 0, AUX_SYS)
#define PSTA(u, o) do { asm volatile("" : "+v"(v##u)); \
        __builtin_amdgcn_raw_buffer_store_b128(v##u | (dead ? 0x7fc00000u : 0u), rd, o, 0, AUX_SC1); } while (0)
#define PST(u) PSTA(u, d + (u) * ds)
#define PSTAMP(i) do { } while (0)
#define GPOLL(i, sl) do { __builtin_amdgcn_global_load_lds((const u32*)gp, (LAS u32*)(pl + (i) * 256u), 4, 0, AUX_SYS); __builtin_amdgcn_s_sleep(sl); \
        if (DRY && c0 < 32u) __builtin_amdgcn_s_sleep(PSLEEP0 - PSLEEP); } while (0)
#define GSTEP(i, sl) { GPOLL(((i) + 3) & 3, sl); WAIT_VM_TRACKED(3); u32 _w; \
            asm volatile("ds_read_b32 %0, %1 offset:%2\n\ts_waitcnt lgkmcnt(0)" : "=v"(_w) : "v"(pla), "n"((i) * 256) : "memory"); \
            if (__all((int)(_w - call) >= 0)) break; }
#define WATCH(sl) do { \
        const char* gp = ws + GBASE(c0) + ((u32)lane * 4u & ((tab(T.lo.gate, T.hi.gate, c0 >> 5) >> 16 & 0x3f0u) | 12u)); \
        const u64 t0 = __builtin_amdgcn_s_memrealtime(); \
        GPOLL(0, sl); GPOLL(1, sl); GPOLL(2, sl); \
        for (u32 n = 1;; ++n) { \
            GSTEP(0, sl) GSTEP(1, sl) GSTEP(2, sl) GSTEP(3, sl) \
            if ((n & 15u) == 0u && give_up(ws, call, t0)) { dead = true; break; } \
        } } while (0)
    if (!DRY) WATCH(PSLEEP0);
    const u32 g0 = DRY ? call - 1u : call;
    u32x4 gv = {g0, g0, g0, g0};
    bool dry = DRY;
    u32 d = OOB, ds = 1024u, so2 = OOB, d2 = OOB; bool y = false, val = false, val4 = false; u32 sor = OOB;
#define VALID(u) __all(v##u[0] != 0u && v##u[1] != 0u && v##u[2] != 0u && v##u[3] != 0u)
#define RETRY(u) do { if (HINT >= 0 && ((u) < 2 ? val : Y4 && val4) && !dead && !VALID(u)) { \
        const u64 t0 = __builtin_amdgcn_s_memrealtime(); u32 n = 0; \
        do { PLD(u, sor); NRETRY; \
            if ((++n & 15u) == 0u && give_up(ws, call, t0)) dead = true; \
            WAIT_VM_TRACKED(0); \
        } while (!dead && !VALID(u)); } } while (0)
#define NRETRY do { } while (0)
    do {
        const rsrc_t rd = y ? rout : rl;
        WAIT_VM_TRACKED(5); PLD(2, so2); RETRY(0); PST(0);
        if (EFL & 1) { WAIT_VM_TRACKED(5); PFLAG(); WAIT_VM_TRACKED(5); PLD(3, so2); }
        else { WAIT_VM_TRACKED(4); PLD(3, so2); PFLAG(); }
        RETRY(1); PST(1);
        WAIT_VM_TRACKED(5);
        bool open = dead || all4_ge(gv, call);
        if (q == OOB && !open && !dry) { WATCH(PSLEEP); open = true; }
        dry = false;
        pf = q != OOB ? tab(T.lo.flag, T.hi.flag, q >> 5) + (q & 31u) * 4u : NOFLAG;
        if (claimed) { c1 = (u32)__builtin_amdgcn_readfirstlane(cl) + cbase; if (c1 >= qcut) c1 += skip;
            if (Y4 && c1 >= qt) c1 += (c1 - qt < Y4 * 16u ? c1 - qt : Y4 * 16u) & ~15u;
            if (c1 > NPIECES + 127u) c1 = NPIECES + 127u; }
        claimed = open && c0 < NPIECES;
        q = claimed ? c0 : OOB; if (claimed) c0 = c1;
        const u32 qq = claimed ? q : 0u;
        const u32 kn = claimed ? KIND(qq) : 1u, po = POFF(qq, kn), lo = kn ? l16 : l4r;
        so = claimed ? tab(T.lo.src, T.hi.src, qq >> 5) + po + lo : OOB;
        ss = kn ? 1024u : 2u * IMG;
        y = (kn & 7u) == 2u;
        u32 dn = claimed ? tab(T.lo.dst, T.hi.dst, qq >> 5) + (y ? po * 4u : po) + lo : OOB;
        u32 go = GOFF(c0), ao = claimed ? co : OOB, so1 = so + ss, dx2 = d2 + 2u * ds, dx3 = d2 + 3u * ds;
        if (claimed) PSTAMP(q >> 5);
        asm volatile("" : "+v"(so), "+v"(dn), "+v"(go), "+v"(ao), "+v"(so1), "+v"(dx2), "+v"(dx3));
        WAIT_VM_TRACKED(4);
        v0 = __builtin_amdgcn_raw_buffer_load_b128(rs, so, 0, AUX_SYS);
        RETRY(2); PSTA(2, dx2);
        WAIT_VM_TRACKED((EFL & 1) ? 3 : 4);
        RETRY(3); PSTA(3, dx3);
        v1 = __builtin_amdgcn_raw_buffer_load_b128(rs, so1, 0, AUX_SYS);
        cl = __builtin_amdgcn_raw_ptr_buffer_atomic_add_i32(1, rl, ao, 0, AUX_SC1);
        gv = __builtin_amdgcn_raw_buffer_load_b128(rl, go, 0, AUX_SYS);
        d = dn; ds = y ? NLOC * 4u : ss;
        val = claimed && qq >= qt; val4 = val && !(kn & 8u); sor = so;
        so2 = (kn & 8u) ? OOB : so; d2 = (kn & 8u) ? OOB : dn;
    } while (c0 < NPIECES || (!LASTP && q != OOB));
    if (LASTP) {
        const rsrc_t rd = y ? rout : rl;
        PLD(2, so2); PLD(3, so2);
        WAIT_VM_TRACKED(7); RETRY(0); PST(0);
        if (EFL & 2) { WAIT_VM_TRACKED(6); PFLAG(); WAIT_VM_TRACKED(6); } else { WAIT_VM_TRACKED(5); PFLAG(); }
        RETRY(1); PST(1);
        pf = q != OOB ? tab(T.lo.flag, T.hi.flag, q >> 5) + (q & 31u) * 4u : NOFLAG;
        WAIT_VM_TRACKED(4); RETRY(2); PSTA(2, d2 + 2u * ds);
        WAIT_VM_TRACKED(4); RETRY(3); PSTA(3, d2 + 3u * ds);
    }
    if (!ENDQ || pf < NOFLAG) { WAIT_VM_TRACKED(0); PFLAG(); }
    asm volatile("" :: "v"(gv), "v"(cl));
    asm volatile(".Lagk_end:");
    if (hw < 7) STAMP(ws, 2 + wave * 8 + 7);
#undef PSTAMP
#undef GOFF
#undef GBASE
#undef WATCH
#undef GSTEP
#undef GPOLL
#undef POFF
#undef KIND
#undef PST
#undef PSTA
#undef RETRY
#undef VALID
#undef NRETRY
#undef PLD
#undef PFLAG
}

__global__ void __launch_bounds__(512, 2) ag_gemm_kernel(Params p) {
    extern __shared__ __attribute__((aligned(16))) char smem[];
    LAS char* lds = (LAS char*)smem;
    const int tid = threadIdx.x, lane = tid & 63, wave = __builtin_amdgcn_readfirstlane(tid >> 6);
    const int b = blockIdx.x, rank = p.rank, g = rank >> 2, a = rank & 3;
    u64 pc; u32 len;
    asm volatile("s_getpc_b64 %0\n.Lagk_pc:\n\ts_mov_b32 %1, .Lagk_end-.Lagk_pc" : "=s"(pc), "=s"(len));
    const u32 code = __builtin_amdgcn_raw_buffer_load_b32(mk_rsrc((const void*)pc, (int)len > 0 ? len : 0u), (u32)(b >> 3) * 2048u + (u32)tid * 4u, 0, 0);
    char* ws = p.me;
    if (wave == 0) STAMP(ws, 0);
    const u32 prev = ld_agent(ws + OFF_CTR + (u32)b * 64u);
    u32 calln;
    const int lw = LROT ? (wave + (b >> 3) * 2 + (b & 7)) % 7 : wave;
    const rsrc_t rws = mk_rsrc(ws, WS_NEED);

    Tables T = {};
    char* src = ws; gchar* fan = (gchar*)ws;
    u32 hword = 0;
    gchar* rdflag = x_head(p, lds, b < 128 ? b : 640 + b, prev, calln, hword, tid, wave, [&]() {
        if (b < NCOMM) {
            const u32 par = calln & 1u;
            Stream S;
            S.cross = lw >= 3; S.rank = rank; S.g = g; S.a = a; S.j = (a + lw + 1) & 3; S.pb = CTRL_BYTES + par * PAR_BYTES; S.sa = OFF_SA + par * 32768u;
            T.lo = S.at((u32)lane); T.hi = S.at((u32)lane + 64u);
            src = p.peer[lw == 3 ? 7 : lw];
            char *f0 = p.peer[4], *f1 = p.peer[5], *f2 = p.peer[6];
            asm volatile("" : "+s"(src), "+s"(f0), "+s"(f1), "+s"(f2));
            fan = (gchar*)(lane == 0 ? f0 : lane == 1 ? f1 : f2);
        }
    });
    asm volatile("" :: "v"(code));
    const u32 call = calln << 8, par = calln & 1u;
    if (wave == (CTR7 ? 7 : 0) && lane < 2) st_agent(ws + OFF_CTR + (u32)b * 64u + (u32)lane * 4u, lane ? call : calln);
    if (wave == 0 && b >= NCOMM) STAMP(ws, 1);
    if (b < NCOMM) {
        if (wave == 7) {
            WAIT_VM(0);
            if (lane < 4) st_sys_g(rdflag, hword);
        }
        else if (b < NACT) pull_stream(p, lds, T, lw >= 3, (u32)b, src, fan, calln, call, lane, lw, wave);
        __builtin_amdgcn_endpgm();
    }

    {
        const int c = b - NCOMM, strip = c / 7, rg = c - strip * 7;
        const int kt0 = (rg * 128) / 7, kt1 = ((rg + 1) * 128) / 7;
        const rsrc_t rw = mk_rsrc(p.w, IN_BYTES);
        const int q = lane & 7, n4 = lane >> 3;
        if (wave == 7) { WAIT_VM(0); if (lane < 4) st_sys_g(rdflag, hword); }
        f32x4 v[3][8];
#pragma unroll
        for (int i = 0; i < 3; ++i) {
            const int kt = kt0 + wave + 8 * i;
            const u32 base = kt < kt1 ? ((u32)(kt * 64 + q * 8) * NLOC + (u32)(strip * 32 + n4 * 4)) * 4u : OOB;
#pragma unroll
            for (int jj = 0; jj < 8; ++jj) {
                v[i][jj] = __builtin_bit_cast(f32x4, __builtin_amdgcn_raw_buffer_load_b128(rw, base + (u32)jj * (NLOC * 4u), 0, 0));
                if (i * 8 + jj < W_N0) WAIT_VM(W_LD0 - 1); else WAIT_VM(W_LD - 1);
            }
        }
        LAS u32* cm = (LAS u32*)(lds + L_RED + 256u);
        if (tid < 64) cm[tid] = 0u;
        WAIT_LGKM0(); BAR();
        u32 m[4] = {0u, 0u, 0u, 0u};
#pragma unroll
        for (int i = 0; i < 3; ++i)
#pragma unroll
            for (int jj = 0; jj < 8; ++jj)
#pragma unroll
                for (int k = 0; k < 4; ++k) m[k] = umax(m[k], absbits(v[i][jj][k]));
#pragma unroll
        for (int k = 0; k < 4; ++k) {
            m[k] = umax(m[k], (u32)__shfl_xor((int)m[k], 1)); m[k] = umax(m[k], (u32)__shfl_xor((int)m[k], 2)); m[k] = umax(m[k], (u32)__shfl_xor((int)m[k], 4));
        }
        if (q < 4) __hip_atomic_fetch_max(cm + n4 * 4 + q, q == 0 ? m[0] : q == 1 ? m[1] : q == 2 ? m[2] : m[3], __ATOMIC_RELAXED, __HIP_MEMORY_SCOPE_WORKGROUP);
        WAIT_LGKM0(); BAR();
        u64* wx = (u64*)(ws + OFF_WX) + (u32)strip * 256u;
        if (tid < 32) __hip_atomic_store(wx + rg * 32 + tid, ((u64)call << 32) | cm[tid], __ATOMIC_RELAXED, __HIP_MEMORY_SCOPE_AGENT);
        if (wave < 4) {
            const int gi = tid < 224 ? tid : 0;
            const u64 t0 = __builtin_amdgcn_s_memrealtime();
            u64 gv; u32 n = 0;
            for (;;) {
                gv = __hip_atomic_load(wx + gi, __ATOMIC_RELAXED, __HIP_MEMORY_SCOPE_AGENT);
                if (__all((u32)(gv >> 32) == call)) break;
                if ((++n & 15u) == 0u && give_up(ws, call, t0)) break;
                __builtin_amdgcn_s_sleep(1);
            }
            __hip_atomic_fetch_max(cm + 32 + (gi & 31), (u32)gv, __ATOMIC_RELAXED, __HIP_MEMORY_SCOPE_WORKGROUP);
        }
        WAIT_LGKM0(); BAR();
        const int ct = strip >> 2;
        if (rg == 0 && tid < 64) {
            const u32 nin = (u32)((strip & 3) * 32 + (tid & 31));
            const u32 ev = (u32)(exp_of(cm[32 + (tid & 31)]) - 141);
            char* t = tid < 32 ? ws + OFF_SB + (par * BSLOTS + (u32)ct) * 512u : p.ws[(1 - g) * 4 + (strip >> 3)] + OFF_SB + (par * BSLOTS + (u32)(8 + 2 * a + (ct & 1))) * 512u;
            st_sys(t + nin * 4u, ev);
        }
        const u32x4 mx = *(LAS u32x4*)(cm + 32 + n4 * 4);
        const u32 pbase = CTRL_BYTES + par * PAR_BYTES + (u32)(ASLOTS + ct) * PANEL;
#pragma unroll
        for (int k = 0; k < 4; ++k) {
            const float s = scale_of(exp_of(mx[k]));
            const u32 nin = (u32)((strip & 3) * 32 + n4 * 4 + k);
            const u32 ro = nin * 128u + (((u32)q ^ ((nin >> 1) & 7u)) << 4);
#pragma unroll
            for (int i = 0; i < 3; ++i) {
                const int kt = kt0 + wave + 8 * i;
                const u32 off = kt < kt1 ? pbase + (u32)kt * IMG + ro : OOB;
                const f32x4 lo = {v[i][0][k], v[i][1][k], v[i][2][k], v[i][3][k]}, hi = {v[i][4][k], v[i][5][k], v[i][6][k], v[i][7][k]};
                __builtin_amdgcn_raw_buffer_store_b128(cvt8(lo, hi, s), rws, off, 0, AUX_SC1);
                WAIT_VM(W_ST - 1);
            }
        }
        WAIT_VM(0); BAR();
        if (tid < 4 && (!(tid & 1) || rg == 6))
            st_sys((tid < 2 ? ws + OFF_WR + (u32)strip * 32u : p.ws[(1 - g) * 4 + (strip >> 3)] + OFF_WS + (u32)(a * 8 + (strip & 7)) * 32u) + (u32)(rg + (tid & 1)) * 4u, call);
        if (wave == 0) STAMP(ws, 2);
    }
    const int xcd = b & 7, q = (b >> 3) - 4;
    if (xcd >= 4 && q < 24) {
        const int r0 = 128 + (q * 4 + xcd - 4) * 4;
        x_rows(p, lds, r0, par, call, tid, wave);
        x_rows(p, lds, r0 + (r0 < 384 ? 384 : 512), par, call, tid, wave);
        if (wave == 0) STAMP(ws, 3);
    }
    const int qr = q >> 2, qc = q & 3, x4 = xcd & 3;
    const int early = (x4 >> 1) == (xcd < 4 ? ESWAP : 0) ? (qr < 6 ? a * 6 + qr : -1) : (qr < 3 ? ((a + 1 + qr) & 3) * 6 : 24 + 2 * (qr - 3));
    const int peer = ((a + 1 + (qr >> 1)) & 3) * 6;
    auto tile_of = [&](int it) {
        int slot = -1, ct = 4 * x4 + qc, rh = 0, ch = 0;
        if (it == 0) { if (xcd < 4) { slot = early; ct = 4 * (x4 & 1) + qc; } }
        else if (it == 1) {
            if (RMAP && qr == 6 && xcd >= 2 && xcd < 6) { slot = ((a + 2) & 3) * 6 + (xcd >> 1); ct = 4 * (xcd & 1) + qc; }
            else if (xcd < 4) { if (qr < 6 && !(RMAP && xcd < 2 && qr >> 1 == 1)) slot = peer + 1 + (qr & 1); }
            else { slot = early; ct = 8 + 4 * (x4 & 1) + qc; }
        }
        else if (it == 2) {
            if (qr == 6) { slot = 25 + (xcd & 6); ct = 8 + 4 * (xcd & 1) + qc; }
            else if (xcd >= 4) slot = peer + 3 + (qr & 1);
            else if (xcd < 2 && qr < 4) slot = 25 + 2 * qr;
        } else if (it == 3) {
            int pi = -1; ch = qc & 1;
            const int qs = !RMAP && QSWAP && xcd >= 4 && (qr & 5) == 4 ? qr ^ 2 : qr;
            if (RMAP && xcd < 2 && qr >> 1 == 1) { pi = qr - 1; rh = 1; ct = 12 + 2 * xcd + (qc >> 1); }
            else if (xcd < 2) { pi = qr - 4; rh = 1; ct = 8 + 2 * xcd + (qc >> 1); }
            else if (qs == 6) { pi = RMAP && xcd >= 4 ? -1 : (xcd - 2) >> 1; rh = 1; ct = 12 + 2 * (xcd & 1) + (qc >> 1); }
            else if (xcd < 4) { pi = qr >> 1; ct = 4 * xcd + 2 * (qr & 1) + (qc >> 1); }
            else { pi = qs >> 1; rh = qs & 1; ct = 2 * x4 + (qc >> 1); }
            if (pi >= 0) slot = ((a + 1 + pi) & 3) * 6 + 5;
        }
        if (slot >= 24 && ct >= 8 && (slot - 24) >> 1 == (ct - 8) >> 1) { slot = 32 + (slot & 1); ct -= 8; }
        return Tile{slot, ct, rh, ch};
    };
    char* pend = nullptr;
    int it = 0;
    Tile tl = tile_of(0);
    while (tl.slot < 0) tl = tile_of(++it);
    for (bool first = true; it < 3; first = false) {
        int nt = it + 1;
        Tile nx = tile_of(nt);
        while (nx.slot < 0 && nt < 3) nx = tile_of(++nt);
        gemm_tile<0>(p, lds, tl, first, nx, nt < 3, par, call, tid, lane, wave, 4 + it * 3, pend);
        it = nt; tl = nx;
    }
    if (tl.slot < 0) {
        if (pend) { WAIT_VM(0); BAR(); if (tid < 2) st_sys(pend + (u32)tid * 4u, call); }
        __builtin_amdgcn_endpgm();
    }
    if (tl.ct < 8) {
        gemm_tile<1>(p, lds, tl, false, tl, false, par, call, tid, lane, wave, 13, pend);
        __builtin_amdgcn_endpgm();
    }
    gemm_tile<2>(p, lds, tl, false, tl, false, par, call, tid, lane, wave, 13, pend);
}
}

extern "C" void kernel_launch(int rank, int nranks, ncclComm_t comm,
                              hipStream_t stream,
                              void* const* d_in, const int* in_sizes, int n_in,
                              void* d_out, int out_size,
                              const int* in_dtypes, int out_dtype,
                              void* const* d_out_all,
                              void* const* d_ws_all, size_t ws_bytes) {
    (void)comm; (void)in_dtypes; (void)out_dtype; (void)d_out_all;
    if (nranks != 8 || (unsigned)rank >= 8u || n_in < 2 || ws_bytes < (size_t)agk::WS_NEED || in_sizes[0] != 1024 * 8192 ||
        in_sizes[1] != 8192 * 1024 || out_size != 8192 * 1024) {
        fprintf(stderr, "kernel_launch: unexpected problem shape\n");
        return;
    }
    static bool ready[8] = {false, false, false, false, false, false, false, false};
    if (!ready[rank]) {
        HIP_CHECK(hipFuncSetAttribute((const void*)agk::ag_gemm_kernel, hipFuncAttributeMaxDynamicSharedMemorySize, (int)agk::LDS_BYTES));
        ready[rank] = true;
    }
    agk::Params p;
    p.x = (const float*)d_in[0]; p.w = (const float*)d_in[1]; p.rank = rank; p.pad = 0;
    const size_t wo = WEND ? (ws_bytes - agk::WS_NEED) & ~(size_t)0x1fffff : 0;
    p.me = (char*)d_ws_all[rank] + wo; p.out = (float*)d_out;
    for (int i = 0; i < 8; ++i) { p.ws[i] = (char*)d_ws_all[i] + wo; p.peer[i] = (char*)d_ws_all[((rank ^ i) & 4) + ((rank + 1 + i) & 3)] + wo; }
    hipLaunchKernelGGL(agk::ag_gemm_kernel, dim3(agk::NWG), dim3(512), agk::LDS_BYTES, stream, p);
    HIP_CHECK(hipGetLastError());
}
```
